```python
import numpy as np

kernel_name = "gpu_gemm_ar_m4096_k4096_n4096_bf16_relu"


ABSMAX_RTOL = 0.008

COMPUTE_DTYPE = "bfloat16"
EPILOGUE = "relu"
IN_QUANT = None
OUT_QUANT = None
IN_DTYPES = {"x": "bfloat16", "w_mat": "bfloat16"}
WORLD_SIZE = 8

def _to_bf16(x):
    u = np.ascontiguousarray(x, dtype=np.float32).view(np.uint32)
    return ((u + 0x7FFF + ((u >> 16) & 1)) & 0xFFFF0000).view(np.float32)

def setup_inputs():

    x = _to_bf16(np.random.randn(4096, 4096).astype(np.float32))
    w_mat = _to_bf16((np.random.randn(4096, 4096) / np.sqrt(4096)).astype(np.float32))
    return {"x": x, "w_mat": w_mat}

def _epilogue(y):
    return np.maximum(y, 0.0).astype(np.float32)

def reference(x, w_mat):
    y = (x @ w_mat).astype(np.float32)
    return _epilogue(y)
```

```cpp
#include <hip/hip_runtime.h>
#include <rccl/rccl.h>
#include <cstdio>
#include <cstdint>

#define HIP_CHECK(expr) do { hipError_t _e = (expr); if (_e != hipSuccess) { \
    fprintf(stderr, "HIP error %s at %s:%d\n", hipGetErrorString(_e), __FILE__, __LINE__); \
    return; } } while (0)

#define COMPRESS 0
#define NWL 16
#define DIAG 0

typedef unsigned int u32;
typedef unsigned long long u64;
typedef u32 u32x4 __attribute__((ext_vector_type(4)));
typedef u32 u32x2 __attribute__((ext_vector_type(2)));
typedef float f32x4 __attribute__((ext_vector_type(4)));
typedef short bf16x8 __attribute__((ext_vector_type(8)));
typedef __bf16 bf16v2 __attribute__((ext_vector_type(2)));
typedef float f32x2 __attribute__((ext_vector_type(2)));
#define LAS __attribute__((address_space(3)))

constexpr int NR = 8;
constexpr int MM = 4096, NN = 4096, KS = 512;
constexpr u32 BLK = 16384;
constexpr u32 OFF_XA = 0;
constexpr u32 OFF_WB = 8u << 20;
constexpr u32 OFF_PART = 16u << 20;
constexpr u32 CSTR = 9216;
constexpr u32 OFF_CMP = 20u << 20;
constexpr u32 OFF_FLAGS = 60u << 20;
constexpr u32 OFF_FX = OFF_FLAGS;
constexpr u32 OFF_FW = OFF_FX + 2048;
constexpr u32 OFF_FP = OFF_FW + 2048;
constexpr u32 OFF_FC = OFF_FP + 512 * 64;
constexpr u32 OFF_EP = OFF_FC + 4096 * 64;
constexpr u32 OFF_TMO = OFF_EP + 4096 * 4;
constexpr u32 OFF_DIAG = OFF_TMO + 4096;
constexpr u32 WS_USED = OFF_DIAG + (1u << 20);
constexpr u32 OUT_BYTES = (u32)MM * NN * 2;
constexpr u32 IN_BYTES = (u32)MM * KS * 2;

constexpr int NPWG = NR * NWL / 4;
constexpr int NCWG = 256;
constexpr int GRID = NPWG + NCWG;
constexpr int LDS_BYTES = 65536;
constexpr u64 DEADLINE = 100000000ull;
constexpr int AUX_SYS = 17;
constexpr int AUX_SC1 = 16;
static_assert(NPWG % 8 == 0, "pusher workgroups must be a multiple of 8");
static_assert(GRID * 4 <= 4096, "epoch words");

struct Params {
    const void* x;
    const void* w;
    void* ws[NR];
    void* out[NR];
    int rank;
};

typedef __amdgpu_buffer_rsrc_t rsrc_t;
__device__ __forceinline__ rsrc_t mk_rsrc(const void* p, u32 bytes) {
    return __builtin_amdgcn_make_buffer_rsrc((void*)p, (short)0, (int)bytes, 0x00020000);
}
__device__ __forceinline__ u32 rfl(u32 v) { return (u32)__builtin_amdgcn_readfirstlane((int)v); }
__device__ __forceinline__ void* pick(void* const (&t)[NR], u32 idx) {
    void* r = t[0];
#pragma unroll
    for (u32 i = 1; i < NR; ++i) r = (idx == i) ? t[i] : r;
    return r;
}
__device__ __forceinline__ u32 glob_blk(u32 g, u32 b, u32 l) {
    return g == 0 ? b * 16 + l : ((l >> 3) * 2 + b) * 8 + (l & 7);
}
__device__ __forceinline__ u32 own_half(u32 g, u32 G) { return g == 0 ? G >> 4 : (G >> 3) & 1; }
__device__ __forceinline__ u32 own_loc(u32 g, u32 G) { return g == 0 ? G & 15 : (G >> 4) * 8 + (G & 7); }
__device__ __forceinline__ u32 perm8(u32 t) { return (t >> 1) + (t & 1) * 4; }
__device__ __forceinline__ u32 perm16(u32 t) { return (t >> 1) + (t & 1) * 8; }
__device__ __forceinline__ u32 row_lr(u32 j, u32 fin) { return (j >> 2) * 8 + fin * 4 + (j & 3); }

__device__ __forceinline__ u32 ld_flag(const void* ws, u32 off) {
    return __hip_atomic_load((const u32*)((const char*)ws + off), __ATOMIC_RELAXED, __HIP_MEMORY_SCOPE_SYSTEM);
}
__device__ __forceinline__ void st_flag(void* ws, u32 off, u32 v) {
    __hip_atomic_store((u32*)((char*)ws + off), v, __ATOMIC_RELAXED, __HIP_MEMORY_SCOPE_SYSTEM);
}
#define WAIT_VM(n) asm volatile("s_waitcnt vmcnt(" #n ")" ::: "memory")
#define WAIT_LGKM0() asm volatile("s_waitcnt lgkmcnt(0)" ::: "memory")
#define CBAR() asm volatile("" ::: "memory")

__device__ __forceinline__ bool give_up(void* myws, u32 call, u64 t0) {
    u32* tmo = (u32*)((char*)myws + OFF_TMO);
    bool late = __builtin_amdgcn_s_memrealtime() - t0 > DEADLINE;
    if (late || rfl(__hip_atomic_load(tmo, __ATOMIC_RELAXED, __HIP_MEMORY_SCOPE_AGENT)) == call) {
        __hip_atomic_store(tmo, call, __ATOMIC_RELAXED, __HIP_MEMORY_SCOPE_AGENT);
        return true;
    }
    return false;
}
__device__ __forceinline__ bool timed_out(void* myws, u32 call) {
    return rfl(__hip_atomic_load((u32*)((char*)myws + OFF_TMO), __ATOMIC_RELAXED, __HIP_MEMORY_SCOPE_AGENT)) == call;
}
__device__ __forceinline__ void wait_flag(void* myws, u32 off, u32 call) {
    if (rfl(ld_flag(myws, off)) >= call) return;
    u64 t0 = __builtin_amdgcn_s_memrealtime();
    for (u32 it = 1;; ++it) {
        __builtin_amdgcn_s_sleep(4);
        if (rfl(ld_flag(myws, off)) >= call) return;
        if ((it & 63) == 0 && give_up(myws, call, t0)) return;
    }
}

__device__ __forceinline__ u32 pack2(float a, float b) {
    f32x2 v = {a, b};
    return __builtin_bit_cast(u32, __builtin_convertvector(v, bf16v2));
}
__device__ __forceinline__ float bf_lo(u32 v) { return __builtin_bit_cast(float, v << 16); }
__device__ __forceinline__ float bf_hi(u32 v) { return __builtin_bit_cast(float, v & 0xffff0000u); }
__device__ __forceinline__ float relu(float y) { return y < 0.f ? 0.f : y; }

#define STAMP(slot) do {} while (0)

__global__ void __launch_bounds__(256, 2) fused_gemm_ar(Params P) {
    extern __shared__ __attribute__((aligned(16))) char smem[];
    const u32 lane = threadIdx.x & 63;
    const u32 wave = rfl(threadIdx.x >> 6);
    const u32 rank = (u32)P.rank & 7;
    const u32 g = rank >> 2, bi = (rank >> 1) & 1, bj = rank & 1;
    void* myws = pick(P.ws, rank);

    u32* ep = (u32*)((char*)myws + OFF_EP) + (blockIdx.x * 4 + wave);
    const u32 call = rfl(__hip_atomic_load(ep, __ATOMIC_RELAXED, __HIP_MEMORY_SCOPE_AGENT)) + 1;
    if (lane == 0) __hip_atomic_store(ep, call, __ATOMIC_RELAXED, __HIP_MEMORY_SCOPE_AGENT);
    STAMP(0);

    if (blockIdx.x < NPWG) {
        const u32 b = blockIdx.x;
        const u32 di = wave + 4 * ((b >> 3) & 1);
        const u32 slot = (b & 7) + 8 * (b >> 4);
        const u32 q = (rank + di) & 7;
        const u32 gq = q >> 2, qi = (q >> 1) & 1, qj = q & 1;
        const u32 nk = (gq == g) ? 5 : 3;
        const u32 kc0 = (gq == g) ? 0 : 5;
        const u32 ks0 = (gq == g) ? (rank & 3) * 5 : 20 + (rank & 3) * 3;
        const u32 nblk = 32 * nk;
        void* qws = pick(P.ws, q);
        const rsrc_t rq = mk_rsrc(qws, WS_USED);
        const rsrc_t rx = mk_rsrc(P.x, IN_BYTES);
        const rsrc_t rw = mk_rsrc(P.w, IN_BYTES);
        const u32 lox = (lane & 15) * 1024 + (lane >> 4) * 16;
        const u32 low = (lane >> 4) * 65536 + (lane & 15) * 16;
        char* tl = smem + wave * BLK;

        auto desc = [&](u32 n, u32& isw, u32& l, u32& kk) {
            u32 e = (nk == 5) ? n / 5 : n / 3;
            kk = n - e * nk;
            if (e < 24) {
                u32 t = e / 3, r = e - t * 3;
                if (r == 0) { isw = 0; l = row_lr(perm8(t), 1 - gq); }
                else { isw = 1; l = perm16(t * 2 + r - 1); }
            } else { isw = 0; l = row_lr(perm8(e - 24), gq); }
        };
        auto load_blk = [&](u32 n, u32x4 (&v)[16]) {
            u32 isw, l, kk; desc(n, isw, l, kk);
            if (isw) {
                const u32 base = (kc0 + kk) * 64 * 8192 + glob_blk(gq, qj, l) * 256;
#pragma unroll
                for (int i = 0; i < 16; ++i)
                    v[i] = __builtin_amdgcn_raw_buffer_load_b128(rw, low, base + (i & 7) * 8192 + (i >> 3) * 262144, 0);
            } else {
                const u32 base = glob_blk(gq, qi, l) * 131072 + (kc0 + kk) * 128;
#pragma unroll
                for (int i = 0; i < 16; ++i)
                    v[i] = __builtin_amdgcn_raw_buffer_load_b128(rx, lox, base + (i & 7) * 16384 + (i >> 3) * 64, 0);
            }
        };
        auto store_blk = [&](u32 n, u32x4 (&v)[16]) {
            u32 isw, l, kk; desc(n, isw, l, kk);
            if (isw) {
                const u32 wo = ((lane & 15) >> 2) * 2048 + ((lane >> 4) * 16 + (lane & 3) * 4) * 16;
#pragma unroll
                for (int h = 0; h < 2; ++h) {
#pragma unroll
                    for (int c = 0; c < 8; ++c) {
                        u32x4 o;
#pragma unroll
                        for (int t = 0; t < 4; ++t)
                            o[t] = __builtin_amdgcn_perm(v[h * 8 + 2 * t + 1][c >> 1], v[h * 8 + 2 * t][c >> 1],
                                                         (c & 1) ? 0x07060302u : 0x05040100u);
                        *(u32x4*)(tl + wo + h * 8192 + (c >> 2) * 1024 + (c & 3) * 16) = o;
                    }
                }
                WAIT_LGKM0();
#pragma unroll
                for (int i = 0; i < 16; ++i) v[i] = *(const u32x4*)(tl + i * 1024 + lane * 16);
                WAIT_LGKM0();
            }
            const u32 dst = (isw ? OFF_WB : OFF_XA) + (l * 32 + ks0 + kk) * BLK;
#pragma unroll
            for (int i = 0; i < 16; ++i)
                __builtin_amdgcn_raw_buffer_store_b128(v[i], rq, lane * 16, dst + i * 1024, AUX_SYS);
        };
        auto flag_blk = [&](u32 n) {
            u32 isw, l, kk; desc(n, isw, l, kk);
            if (lane == 0) st_flag(qws, (isw ? OFF_FW : OFF_FX) + (l * 32 + ks0 + kk) * 4, call);
        };

        u32x4 va[16], vb[16];
        u32 n = slot;
        load_blk(n, va);
        for (; n < nblk; n += 2 * NWL) {
            CBAR();
            load_blk(n + NWL, vb);
            CBAR();
            store_blk(n, va);
            WAIT_VM(16);
            if (n >= NWL) flag_blk(n - NWL);
            CBAR();
            if (n + 2 * NWL < nblk) load_blk(n + 2 * NWL, va);
            CBAR();
            store_blk(n + NWL, vb);
            WAIT_VM(16);
            flag_blk(n);
        }
        WAIT_VM(0);
        flag_blk(n - NWL);
        STAMP(1);
        return;
    }

    const u32 c = blockIdx.x - NPWG;
    const u32 xcd = c & 7, ci = c >> 3;
    const u32 keep = (ci >> 4) & 1;
    const u32 jt = (xcd >> 1) + 4 * ((ci >> 3) & 1);
    const u32 ct = (ci & 7) * 2 + (xcd & 1);
    const u32 j = perm8(jt);
    const u32 lc = perm16(ct);
    const u32 lr = row_lr(j, keep ? g : 1 - g);
    const u32 wm = wave >> 1, wn = wave & 1;
    const rsrc_t rme = mk_rsrc(myws, WS_USED);

    f32x4 acc[4][4];
#pragma unroll
    for (int a = 0; a < 4; ++a)
#pragma unroll
        for (int b2 = 0; b2 < 4; ++b2) acc[a][b2] = f32x4{0.f, 0.f, 0.f, 0.f};

    {
        const u32 xbase = OFF_XA + lr * 32 * BLK + wave * 2048;
        const u32 wbase = OFF_WB + lc * 32 * BLK + wave * 2048;
        const u32 foff = (lane < 32) ? OFF_FX + (lr * 32 + lane) * 4 : OFF_FW + (lc * 32 + lane - 32) * 4;
        const u32 own0 = (rank & 3) * 5;
        u32 ready = 0;
        auto kstep = [&](u32 s) -> u32 {
            if (s < 5) return own0 + s;
            if (s < 17) return 20 + (s - 5);
            u32 t = own0 + 5 + (s - 17);
            return t >= 20 ? t - 20 : t;
        };
        auto issue = [&](u32 h) {
            const u32 hh = h > 63 ? 63 : h;
            const u32 ks = kstep(hh >> 1);
            if (!((ready >> ks) & 1)) {
                u64 t0 = __builtin_amdgcn_s_memrealtime();
                for (u32 it = 1;; ++it) {
                    u64 m = __ballot(ld_flag(myws, foff) >= call);
                    ready = (u32)m & (u32)(m >> 32);
                    if ((ready >> ks) & 1) break;
                    __builtin_amdgcn_s_sleep(4);
                    if ((it & 63) == 0 && give_up(myws, call, t0)) { ready = 0xffffffffu; break; }
                }
            }
            const u32 so = ks * BLK + (hh & 1) * 8192;
            LAS char* d = (LAS char*)smem + (h & 3) * 16384 + wave * 2048;
            __builtin_amdgcn_raw_ptr_buffer_load_lds(rme, d, 16, lane * 16, xbase + so, 0, AUX_SC1);
            __builtin_amdgcn_raw_ptr_buffer_load_lds(rme, d + 1024, 16, lane * 16, xbase + so + 1024, 0, AUX_SC1);
            __builtin_amdgcn_raw_ptr_buffer_load_lds(rme, d + 8192, 16, lane * 16, wbase + so, 0, AUX_SC1);
            __builtin_amdgcn_raw_ptr_buffer_load_lds(rme, d + 9216, 16, lane * 16, wbase + so + 1024, 0, AUX_SC1);
        };
        issue(0); issue(1); issue(2);
        const char* xa = smem + wm * 4096 + lane * 16;
        const char* wb = smem + 8192 + wn * 4096 + lane * 16;
        for (u32 h = 0; h < 64; ++h) {
            WAIT_VM(8);
            __builtin_amdgcn_s_barrier();
            CBAR();
            issue(h + 3);
            CBAR();
            const u32 so = (h & 3) * 16384;
            bf16x8 xf[4], wf[4];
#pragma unroll
            for (int i = 0; i < 4; ++i) {
                xf[i] = *(const bf16x8*)(xa + so + i * 1024);
                wf[i] = *(const bf16x8*)(wb + so + i * 1024);
            }
#pragma unroll
            for (int a = 0; a < 4; ++a)
#pragma unroll
                for (int b2 = 0; b2 < 4; ++b2)
                    acc[a][b2] = __builtin_amdgcn_mfma_f32_16x16x32_bf16(wf[b2], xf[a], acc[a][b2], 0, 0, 0);
            WAIT_LGKM0();
        }
        WAIT_VM(0);
        __builtin_amdgcn_s_barrier();
        CBAR();
    }
    STAMP(1);

    const u32 RB = glob_blk(g, bi, lr), CB = glob_blk(g, bj, lc);
    if (!keep) {
        const u32 og = 1 - g;
        const u32 pr = og * 4 + own_half(og, RB) * 2 + own_half(og, CB);
        const u32 plr = own_loc(og, RB), plc = own_loc(og, CB);
        const u32 ptile = ((plr >> 3) * 4 + (plr & 3)) * 16 + plc;
        void* pws = pick(P.ws, pr);
        const rsrc_t rp = mk_rsrc(pws, WS_USED);
        const u32 dst = OFF_PART + (ptile * 4 + wave) * 8192;
#pragma unroll
        for (int a = 0; a < 4; ++a)
#pragma unroll
            for (int u = 0; u < 2; ++u) {
                u32x4 o;
                o[0] = pack2(acc[a][2 * u][0], acc[a][2 * u][1]);
                o[1] = pack2(acc[a][2 * u][2], acc[a][2 * u][3]);
                o[2] = pack2(acc[a][2 * u + 1][0], acc[a][2 * u + 1][1]);
                o[3] = pack2(acc[a][2 * u + 1][2], acc[a][2 * u + 1][3]);
                __builtin_amdgcn_raw_buffer_store_b128(o, rp, lane * 16, dst + (a * 2 + u) * 1024, AUX_SYS);
            }
        WAIT_VM(0);
        if (lane == 0) st_flag(pws, OFF_FP + (ptile * 4 + wave) * 64, call);
        STAMP(2);

        const u32 s = jt * 16 + ct;
        const rsrc_t ro = mk_rsrc(pick(P.out, rank), OUT_BYTES);
        for (u32 n = 0; n < 7; ++n) {
            const u32 idx = n * 128 + s;
            const u32 to = idx / 7, pi = idx - to * 7;
            const u32 p = (rank + 1 + pi) & 7;
            const u32 tj = perm8(to >> 4), tc = perm16(to & 15);
            const u32 tile = tj * 16 + tc;
            wait_flag(myws, OFF_FC + ((p * 128 + tile) * 4 + wave) * 64, call);
            const u32 gp = p >> 2;
            const u32 tRB = glob_blk(gp, (p >> 1) & 1, row_lr(tj, gp)), tCB = glob_blk(gp, p & 1, tc);
            const u32 obase = (tRB * 128 + wm * 64) * (NN * 2) + (tCB * 128 + wn * 64) * 2;
            const u32 olane = (lane & 15) * (NN * 2) + (lane >> 4) * 16;
            if (timed_out(myws, call)) {
                const u32x4 nanv = {0x7fc07fc0u, 0x7fc07fc0u, 0x7fc07fc0u, 0x7fc07fc0u};
                __builtin_amdgcn_raw_buffer_store_b128(nanv, ro, olane, obase, 0);
            }
            STAMP(3 + n);
        }
        return;
    }

    const u32 tile = j * 16 + lc;
    wait_flag(myws, OFF_FP + (tile * 4 + wave) * 64, call);
    STAMP(2);
    u32x4 o[8];
    {
        const u32 src = OFF_PART + (tile * 4 + wave) * 8192;
        u32x4 pv[8];
#pragma unroll
        for (int i = 0; i < 8; ++i) pv[i] = __builtin_amdgcn_raw_buffer_load_b128(rme, lane * 16, src + i * 1024, AUX_SC1);
        const bool bad = timed_out(myws, call);
#pragma unroll
        for (int a = 0; a < 4; ++a)
#pragma unroll
            for (int u = 0; u < 2; ++u) {
                const u32x4 pq = pv[a * 2 + u];
                u32x4 r;
                r[0] = pack2(relu(acc[a][2 * u][0] + bf_lo(pq[0])), relu(acc[a][2 * u][1] + bf_hi(pq[0])));
                r[1] = pack2(relu(acc[a][2 * u][2] + bf_lo(pq[1])), relu(acc[a][2 * u][3] + bf_hi(pq[1])));
                r[2] = pack2(relu(acc[a][2 * u + 1][0] + bf_lo(pq[2])), relu(acc[a][2 * u + 1][1] + bf_hi(pq[2])));
                r[3] = pack2(relu(acc[a][2 * u + 1][2] + bf_lo(pq[3])), relu(acc[a][2 * u + 1][3] + bf_hi(pq[3])));
                if (bad) r = u32x4{0x7fc07fc0u, 0x7fc07fc0u, 0x7fc07fc0u, 0x7fc07fc0u};
                o[a * 2 + u] = r;
            }
    }
    const u32 obase = (RB * 128 + wm * 64) * (NN * 2) + (CB * 128 + wn * 64) * 2;
    const u32 olane = (lane & 15) * (NN * 2) + (lane >> 4) * 16;
    for (u32 d = 0; d < 8; ++d) {
        const u32 p = (rank + 1 + ((d + c) % 7) + (d == 7 ? 7 - ((d + c) % 7) : 0)) & 7;
        const rsrc_t ro = mk_rsrc(pick(P.out, p), OUT_BYTES);
#pragma unroll
        for (int a = 0; a < 4; ++a)
#pragma unroll
            for (int u = 0; u < 2; ++u)
                __builtin_amdgcn_raw_buffer_store_b128(o[a * 2 + u], ro, olane, obase + a * 16 * (NN * 2) + u * 64, AUX_SYS);
    }
    WAIT_VM(0);
    if (lane < 7) {
        const u32 p = (rank + 1 + lane) & 7;
        void* t = P.ws[0];
#pragma unroll
        for (u32 i = 1; i < NR; ++i) t = (p == i) ? P.ws[i] : t;
        st_flag(t, OFF_FC + ((rank * 128 + tile) * 4 + wave) * 64, call);
    }
    STAMP(3);
}

extern "C" void kernel_launch(int rank, int nranks, ncclComm_t comm,
                              hipStream_t stream,
                              void* const* d_in, const int* in_sizes, int n_in,
                              void* d_out, int out_size,
                              const int* in_dtypes, int out_dtype,
                              void* const* d_out_all,
                              void* const* d_ws_all, size_t ws_bytes) {
    if (nranks != NR || n_in < 2 || ws_bytes < WS_USED || (unsigned)rank >= (unsigned)NR ||
        in_sizes[0] != MM * KS || in_sizes[1] != KS * NN || out_size != MM * NN) {
        fprintf(stderr, "unexpected problem shape\n");
        return;
    }
    Params P;
    P.x = d_in[0];
    P.w = d_in[1];
    for (int i = 0; i < NR; ++i) { P.ws[i] = d_ws_all[i]; P.out[i] = d_out_all[i]; }
    P.rank = rank;
    hipLaunchKernelGGL(fused_gemm_ar, dim3(GRID), dim3(256), LDS_BYTES, stream, P);
    HIP_CHECK(hipGetLastError());
}
```

```cpp
#include <hip/hip_runtime.h>
#include <rccl/rccl.h>
#include <cstdio>
#include <cstdint>

#define HIP_CHECK(expr) do { hipError_t _e = (expr); if (_e != hipSuccess) { \
    fprintf(stderr, "HIP error %s at %s:%d\n", hipGetErrorString(_e), __FILE__, __LINE__); \
    return; } } while (0)

#define NW2 0
#define JOIN 128
#define EWG 16
#define XWAVES 4
#define DIAG 0
#define LAT_K 17.0
#define LAT_S 30.0
#define ENTRY_ORDER "w0s0w1s1w2s2w3s3w4w5k0s4k1s5k2w6k3w7s6s7w8w9k4k5wAwBwCwDwEwFk6k7"
#define OUT_AUX 17
#define XSLEEP 16
#define PSLEEP 8

typedef unsigned int u32;
typedef unsigned long long u64;
typedef u32 u32x4 __attribute__((ext_vector_type(4)));
typedef u32 u32x2 __attribute__((ext_vector_type(2)));
typedef float f32x4 __attribute__((ext_vector_type(4)));
typedef short bf16x8 __attribute__((ext_vector_type(8)));
typedef __bf16 bf16v2 __attribute__((ext_vector_type(2)));
typedef float f32x2 __attribute__((ext_vector_type(2)));
#define LAS __attribute__((address_space(3)))

constexpr int NR = 8;
constexpr int MM = 4096, NN = 4096, KS = 512;
constexpr u32 BLK = 16384;
constexpr u32 OFF_XA = 0;
constexpr u32 OFF_WB = 8u << 20;
constexpr u32 OFF_PART = 16u << 20;
constexpr u32 OFF_SPART = 20u << 20;
constexpr u32 CSTR = 9216;
constexpr u32 CMAX = 8704;
constexpr u32 OFF_CMP = 24u << 20;
constexpr u32 OFF_STG = 60u << 20;
constexpr u32 OFF_FLAGS = 68u << 20;
constexpr u32 OFF_FX = OFF_FLAGS;
constexpr u32 OFF_FW = OFF_FX + 4096;
constexpr u32 OFF_FP = OFF_FW + 4096;
constexpr u32 OFF_LP = OFF_FP + 4096;
constexpr u32 OFF_FC = OFF_LP + 4096;
constexpr u32 OFF_LS = OFF_FC + 36864;
constexpr u32 OFF_CTR = OFF_LS + 8192;
constexpr u32 OFF_EP = OFF_CTR + 4096;
constexpr u32 OFF_TMO = OFF_EP + 8192;
constexpr u32 OFF_ALL = OFF_TMO + 128;
constexpr u32 OFF_DIAG = OFF_TMO + 4096;
constexpr u32 WS_USED = OFF_DIAG + (1u << 20);
constexpr u32 OUT_BYTES = (u32)MM * NN * 2;
constexpr u32 IN_BYTES = (u32)MM * KS * 2;

constexpr int NPWG = 7 * EWG;
constexpr int NWL = EWG + NW2;
constexpr int NCWG = EWG == 8 ? 192 : 128;
constexpr int GRID = 256;
constexpr u32 NXW = NCWG * XWAVES + (GRID - NPWG - NCWG) * 8;
constexpr u32 NSTREAM = 7 * 512;
constexpr u32 LDS_CTL = DIAG ? 1024 : 256;
constexpr int LDS_BYTES = LDS_CTL + 4 * 32768;
constexpr u64 DEADLINE = 100000000ull;
constexpr int AUX_SYS = 17;
constexpr int AUX_SC1 = 16;
static_assert(GRID * 8 * 4 <= 8192, "epoch words");
static_assert(OFF_CMP + 8u * 512u * CSTR <= OFF_STG, "layout");

struct Params {
    const void* x;
    const void* w;
    void* ws[NR];
    void* out;
    int rank;
    u64 entry[4];
    unsigned short tiles[NCWG];
    u32 ord[64];
};

typedef __amdgpu_buffer_rsrc_t rsrc_t;
__device__ __forceinline__ rsrc_t mk_rsrc(const void* p, u32 bytes) {
    return __builtin_amdgcn_make_buffer_rsrc((void*)p, (short)0, (int)bytes, 0x00020000);
}
__device__ __forceinline__ u32 rfl(u32 v) { return (u32)__builtin_amdgcn_readfirstlane((int)v); }
__device__ __forceinline__ void* pick(void* const (&t)[NR], u32 idx) {
    void* r = t[0];
#pragma unroll
    for (u32 i = 1; i < NR; ++i) r = (idx == i) ? t[i] : r;
    return r;
}
__device__ __forceinline__ u32 glob_blk(u32 g, u32 b, u32 l) {
    return g == 0 ? b * 16 + l : ((l >> 3) * 2 + b) * 8 + (l & 7);
}
__device__ __forceinline__ u32 perm8(u32 t) { return (t >> 1) + (t & 1) * 4; }
__device__ __forceinline__ u32 perm16(u32 t) { return (t >> 1) + (t & 1) * 8; }
__device__ __forceinline__ u32 row_lr(u32 j, u32 fin) { return (j >> 2) * 8 + fin * 4 + (j & 3); }

__device__ __forceinline__ u32 part_idx(u32 a, u32 b) { return b >= 4 ? b * 4 + a : (b > a ? b * b + a : a * a + a + b); }
__device__ __forceinline__ void part_ab(u32 m, u32& a, u32& b) {
    const u32 sh = m >= 9 ? 3 : m >= 4 ? 2 : m >= 1 ? 1 : 0, o = m - sh * sh;
    a = o < sh ? o : sh;
    b = o < sh ? sh : o - sh;
    if (m >= 16) { a = m & 3; b = m >> 2; }
}
__device__ __forceinline__ void fetch_unit(const void* x, const void* w, u32 su, u32 lane, char* wl, u32x4* v) {
    const u32 isw = su >> 9, GB = (su >> 4) & 31, kc = (su >> 1) & 7, hf = su & 1;
    if (isw) {
        const rsrc_t rw = mk_rsrc(w, IN_BYTES);
        const u32 vo = (lane >> 3) * 65536 + (lane & 7) * 16, so = kc * 64 * 8192 + GB * 256 + hf * 128;
#pragma unroll
        for (int i = 0; i < 8; ++i) v[i] = __builtin_amdgcn_raw_buffer_load_b128(rw, vo, so + i * 8192, 0);
        const u32 wo = (lane >> 5) * 4096 + ((lane & 7) >> 2) * 2048 + (((lane >> 3) & 3) * 16 + (lane & 3) * 4) * 16;
#pragma unroll
        for (int c = 0; c < 8; ++c) {
            u32x4 o;
#pragma unroll
            for (int t = 0; t < 4; ++t)
                o[t] = __builtin_amdgcn_perm(v[2 * t + 1][c >> 1], v[2 * t][c >> 1], (c & 1) ? 0x07060302u : 0x05040100u);
            *(u32x4*)(wl + wo + (c >> 2) * 1024 + (c & 3) * 16) = o;
        }
    } else {
        const rsrc_t rx = mk_rsrc(x, IN_BYTES);
        const u32 vo = (lane >> 3) * 1024 + (lane & 7) * 16, so = GB * 131072 + hf * 65536 + kc * 128;
#pragma unroll
        for (int i = 0; i < 8; ++i) v[i] = __builtin_amdgcn_raw_buffer_load_b128(rx, vo, so + i * 8192, 0);
        const u32 wo = ((lane & 7) >> 2) * 4096 + ((lane & 3) * 16 + (lane >> 3)) * 16;
#pragma unroll
        for (int i = 0; i < 8; ++i) *(u32x4*)(wl + wo + (i >> 1) * 1024 + (i & 1) * 128) = v[i];
    }
    asm volatile("s_waitcnt lgkmcnt(0)" ::: "memory");
#pragma unroll
    for (int i = 0; i < 8; ++i) v[i] = *(const u32x4*)(wl + i * 1024 + lane * 16);
    asm volatile("s_waitcnt lgkmcnt(0)" ::: "memory");
}
__device__ __forceinline__ void st128(u32x4 v, rsrc_t r, u32 voff, u32 soff, int) {
    __builtin_amdgcn_raw_buffer_store_b128(v, r, voff + soff, 0, AUX_SYS);
}
__device__ __forceinline__ void st_out(u32x4 v, rsrc_t r, u32 voff, u32 soff) {
    __builtin_amdgcn_raw_buffer_store_b128(v, r, voff + soff, 0, OUT_AUX);
}
__device__ __forceinline__ void st_flag(rsrc_t r, u32 l0, u32 off, u32 v) {
    __builtin_amdgcn_raw_buffer_store_b32(v, r, l0, off, AUX_SYS);
}
__device__ __forceinline__ void st_flag64(rsrc_t r, u32 l0, u32 off, u32 call, u32 len) {
    __builtin_amdgcn_raw_buffer_store_b64(u32x2{len, call}, r, l0, off, AUX_SYS);
}
__device__ __forceinline__ u64 ld_flag64(const void* ws, u32 off) {
    return __hip_atomic_load((const u64*)((const char*)ws + off), __ATOMIC_RELAXED, __HIP_MEMORY_SCOPE_SYSTEM);
}
#define CBAR() asm volatile("" ::: "memory")
#define WAIT_VM(n) asm volatile("s_waitcnt vmcnt(" #n ")" ::: "memory")
#define WAIT_VM0() do { WAIT_VM(0); __builtin_amdgcn_s_waitcnt(0x0F70); CBAR(); } while (0)
#define WAIT_LGKM0() asm volatile("s_waitcnt lgkmcnt(0)" ::: "memory")

__device__ __forceinline__ bool give_up(void* myws, u32 call, u64 t0) {
    u32* tmo = (u32*)((char*)myws + OFF_TMO);
    bool late = __builtin_amdgcn_s_memrealtime() - t0 > DEADLINE;
    if (late || rfl(__hip_atomic_load(tmo, __ATOMIC_RELAXED, __HIP_MEMORY_SCOPE_AGENT)) == call) {
        __hip_atomic_store(tmo, call, __ATOMIC_RELAXED, __HIP_MEMORY_SCOPE_AGENT);
        return true;
    }
    return false;
}
__device__ __forceinline__ u32 ld_tmo(void* myws) {
    return __hip_atomic_load((u32*)((char*)myws + OFF_TMO), __ATOMIC_RELAXED, __HIP_MEMORY_SCOPE_AGENT);
}
template <int SLP>
__device__ __forceinline__ u32 wait_flag64(void* myws, u32 off, u32 call) {
    u64 t0 = 0;
    for (u32 it = 0;; ++it) {
        u64 v = ld_flag64(myws, off);
        if (rfl((u32)(v >> 32)) >= call) return rfl((u32)v);
        if (it == 0) t0 = __builtin_amdgcn_s_memrealtime();
        if ((it & 63) == 63 && give_up(myws, call, t0)) return 0;
        __builtin_amdgcn_s_sleep(SLP);
    }
}
__device__ __forceinline__ u32 ticket(rsrc_t r, u32 l0, u32 idx) {
    return (u32)__builtin_amdgcn_raw_ptr_buffer_atomic_add_i32(1, r, l0, OFF_CTR + idx * 128, 0);
}
__device__ __forceinline__ u32 mbcnt64(u64 m) {
    return __builtin_amdgcn_mbcnt_hi((u32)(m >> 32), __builtin_amdgcn_mbcnt_lo((u32)m, 0u));
}
__device__ __forceinline__ u32 wave_excl(u32 cnt, u32& tot) {
    u32 pre = 0; tot = 0;
#pragma unroll
    for (int b = 0; b < 7; ++b) {
        const u64 m = __ballot((cnt >> b) & 1);
        pre += mbcnt64(m) << b;
        tot += (u32)__builtin_popcountll(m) << b;
    }
    return pre;
}
__device__ __forceinline__ u32 pack2(float a, float b) {
    f32x2 v = {a, b};
    return __builtin_bit_cast(u32, __builtin_convertvector(v, bf16v2));
}
__device__ __forceinline__ float bf_lo(u32 v) { return __builtin_bit_cast(float, v << 16); }
__device__ __forceinline__ float bf_hi(u32 v) { return __builtin_bit_cast(float, v & 0xffff0000u); }

#define STAMP(slot) do {} while (0)

__global__ void __launch_bounds__(512, 2) fused_gemm_ar(Params P) {
    extern __shared__ __attribute__((aligned(16))) char smem[];
    const u32 lane = threadIdx.x & 63;
    const u32 wave = rfl(threadIdx.x >> 6);
    const u32 rank = (u32)P.rank & 7;
    const u32 g = rank >> 2, bi = (rank >> 1) & 1, bj = rank & 1;
    void* myws = pick(P.ws, rank);
    const rsrc_t rme = mk_rsrc(myws, WS_USED);
    char* wl = smem + LDS_CTL + wave * 16384;

    const u32 l0 = lane == 0 ? 0u : 0x80000000u;
    const u32 epo = OFF_EP + (blockIdx.x * 8 + wave) * 4;
    const u32 epv = __builtin_amdgcn_raw_buffer_load_b32(rme, 0, epo, AUX_SC1);
    *(LAS u32*)smem = 0u;
    STAMP(0);

    if (blockIdx.x < NPWG) {
        const u32 b = blockIdx.x;
        const u32 di = 1 + ((b >> 3) + (b & 7)) % 7;
        const u32 eslot = (((b & 7) + 64 - di * NW2) & 7) + (b >= 56 ? 8 : 0);
        const u32 q = (rank + di) & 7;
        if (wave * EWG + eslot >= NWL) { st_flag(rme, l0, epo, epv + 1); return; }
        const u32 gq = q >> 2, qi = (q >> 1) & 1, qj = q & 1;
        const rsrc_t rq = mk_rsrc(pick(P.ws, q), WS_USED);
        const u32 nk2 = (gq == g) ? 10 : 6;
        const u32 kh0 = (gq == g) ? 0 : 10;
        const u32 hs0 = (gq == g) ? (rank & 3) * 10 : 40 + (rank & 3) * 6;
        const u32 T1 = 32 * nk2;
        const u32 T = T1 + 256;
        u32 call;
        auto desc = [&](u32 n, u32& su, u32& du) {
            const u32 e = (nk2 == 10) ? n / 10 : n / 6;
            const u32 kk = n - e * nk2;
            const u64 ew = (e & 16) ? ((e & 8) ? P.entry[3] : P.entry[2]) : ((e & 8) ? P.entry[1] : P.entry[0]);
            const u32 eb = (u32)(ew >> ((e & 7) * 8));
            const u32 kind = (eb >> 4) & 3, idx = eb & 15;
            const bool isw = kind == 0;
            const u32 l = isw ? perm16(idx) : row_lr(perm8(idx & 7), kind == 1 ? 1 - gq : gq);
            su = (isw ? 512 : 0) + glob_blk(gq, isw ? qj : qi, l) * 16 + kh0 + kk;
            du = (isw ? 1024 : 0) + l * 64 + hs0 + kk;
        };
        const u32 lco = OFF_FC + rank * 4096 - T1 * 16;
        const u32 lpo = OFF_LP + (q & 3) * 1024;
        u32x4 v[18];
        auto load_t = [&](u32 n, u32 len, u32 lenb) {
            u32 su, du; desc(n, su, du);
            const bool pair = n >= T1 && n < 1024;
            const u32 so = n < T1 ? OFF_STG + su * 8192
                                  : (pair ? OFF_CMP + (rank * 512 + (n - T1) * 2) * CSTR : OFF_SPART + ((q & 3) * 128 + n - 1024) * 8192);
            const u32 sb = so + (pair ? CSTR : 4096);
#pragma unroll
            for (int i = 0; i < 18; ++i)
                if ((i % 9) * 1024u < (i < 9 ? len : lenb))
                    v[i] = __builtin_amdgcn_raw_buffer_load_b128(rme, lane * 16, (i < 9 ? so : sb) + (i % 9) * 1024, AUX_SC1);
        };
        auto store_t = [&](u32 n, u32 len, u32 lenb) -> u32 {
            u32 dst, db, fo;
            if (n < T1) {
                u32 su, du; desc(n, su, du);
                dst = OFF_XA + du * 8192; db = dst + 4096; fo = OFF_FX + du * 4;
            } else if (n < 1024) {
                dst = OFF_CMP + (rank * 512 + (n - T1) * 2) * CSTR; db = dst + CSTR; fo = lco + n * 16;
            } else {
                u32 a, bb; part_ab((n - 1024) >> 2, a, bb);
                const u32 pu = ((bi * 4 + a) * 16 + bj * 8 + bb) * 4 + (n & 3);
                dst = OFF_PART + pu * 8192; db = dst + 4096; fo = OFF_FP + pu * 8;
            }
#pragma unroll
            for (int i = 0; i < 18; ++i)
                if ((i % 9) < 5 || (i % 9) * 1024u < (i < 9 ? len : lenb))
                    st128(v[i], rq, ((i % 9) * 1024u + lane * 16 < (i < 9 ? len : lenb)) ? lane * 16 : 0x80000000u,
                          (i < 9 ? dst : db) + (i % 9) * 1024, AUX_SYS);
            return fo;
        };
        auto flag_t = [&](u32 n, u32 fo, u32 len, u32 lenb) {
            st_flag(rq, n < T1 ? l0 : 0x80000000u, fo, call);
            __builtin_amdgcn_raw_buffer_store_b64(u32x2{lane == 0 ? len : lenb, call}, rq,
                                                  (n >= T1 && lane < (n < 1024 ? 2u : n < 2048 ? 1u : 0u)) ? lane * 8 : 0x80000000u, fo, AUX_SYS);
        };
        u32 n0 = eslot;
        u32 pn = (gq == g || wave) ? 128 : n0;
        u32 fprev = 0, nprev = n0, lprev = 4096, lprevb = 4096, len0 = 4096, len0b = 4096;
        if (wave == 0) {
            u32x4 v2[8];
            u32 su, du;
            desc(n0, su, du); fetch_unit(P.x, P.w, su, lane, wl, v);
            call = rfl(epv) + 1;
            CBAR();
            fprev = OFF_FX + du * 4;
#pragma unroll
            for (int i = 0; i < 8; ++i) st128(v[i], rq, lane * 16, OFF_XA + du * 8192 + i * 1024, AUX_SYS);
            CBAR();
            n0 += EWG;
            desc(n0, su, du); fetch_unit(P.x, P.w, su, lane, wl, v2);
            st_flag(rme, l0, epo, call);
            WAIT_VM0();
#pragma unroll
            for (int i = 0; i < 4; ++i) { v[i] = v2[i]; v[9 + i] = v2[4 + i]; }
        } else {
            call = rfl(epv) + 1;
            st_flag(rme, l0, epo, call);
            n0 = nprev = 4096; len0 = len0b = 0;
        }
        const u32 tbase = (call - 1) * (T - EWG + NW2) - 2 * EWG;
        if (wave) {
            u64 t0 = __builtin_amdgcn_s_memrealtime();
            for (u32 it = 0;; ++it) {
                if (rfl(__builtin_amdgcn_raw_buffer_load_b32(rme, 0, OFF_CTR + di * 128, AUX_SYS)) - tbase + JOIN >= T) break;
                if ((it & 63) == 63 && give_up(myws, call, t0)) break;
                __builtin_amdgcn_s_sleep(32);
            }
            WAIT_VM0();
        }
        u32 n1 = 0, wb = T1;
        bool have = false;
        for (;;) {
            const u32 t2v = ticket(rme, have ? 0x80000000u : l0, di);
            const u32 pf = __builtin_amdgcn_raw_buffer_load_b32(rme, lane == 0 ? lpo + (pn & 127) * 8 + 4 : OFF_ALL, 0, AUX_SYS);
            const u32x4 wf = __builtin_amdgcn_raw_buffer_load_b128(rme, lane * 16, lco + wb * 16, AUX_SYS);
            CBAR();
            flag_t(nprev, fprev, lprev, lprevb);
            CBAR();
            const u32 fo = store_t(n0, len0, len0b);
            CBAR();
            WAIT_VM(12); __builtin_amdgcn_s_waitcnt(0x0F7C); CBAR();
            if (!have) { n1 = rfl(t2v) - tbase; have = true; }
            if (n1 >= T && pn >= 128) {
                WAIT_VM0();
                flag_t(n0, fo, len0, len0b);
                break;
            }
            u32 nx = 0, len1 = 4096, len1b = 4096;
            if (pn < 128 && rfl(pf) >= call) nx = 1;
            else if (n1 < T1) nx = (u32)__builtin_amdgcn_readlane((int)pf, 1) >= call ? 2 : 0;
            else if (n1 < T && n1 - wb < 64) {
                const int d = (int)(n1 - wb);
                if ((u32)__builtin_amdgcn_readlane((int)wf[1], d) >= call && (u32)__builtin_amdgcn_readlane((int)wf[3], d) >= call) {
                    nx = 2;
                    len1 = (u32)__builtin_amdgcn_readlane((int)wf[0], d);
                    len1b = (u32)__builtin_amdgcn_readlane((int)wf[2], d);
                }
            }
            nprev = n0; fprev = fo; lprev = len0; lprevb = len0b;
            if (nx == 0) {
                WAIT_VM0();
                flag_t(n0, fo, len0, len0b);
                nprev = 4096;
                const u32 lo = n1 < T1 ? OFF_ALL - 4 : (n1 < T ? lco + n1 * 16 + (lane & 1) * 8 : OFF_TMO + 64);
                const u32 o2 = (lane == 0) ? (pn < 128 ? lpo + pn * 8 : OFF_TMO + 64) : lo;
                u64 t0 = 0;
                for (u32 it = 0;; ++it) {
                    const u32x2 f2 = __builtin_amdgcn_raw_buffer_load_b64(rme, o2, 0, AUX_SYS);
                    if (rfl(f2[1]) >= call) { nx = 1; break; }
                    if ((u32)__builtin_amdgcn_readlane((int)f2[1], 1) >= call && (u32)__builtin_amdgcn_readlane((int)f2[1], 2) >= call) {
                        nx = 2;
                        if (n1 >= T1) { len1 = (u32)__builtin_amdgcn_readlane((int)f2[0], 2); len1b = (u32)__builtin_amdgcn_readlane((int)f2[0], 1); }
                        break;
                    }
                    if (it == 0) t0 = __builtin_amdgcn_s_memrealtime();
                    if ((it & 63) == 63 && give_up(myws, call, t0)) { nx = pn < 128 ? 1 : 2; len1 = 0; len1b = 0; break; }
                    __builtin_amdgcn_s_sleep(PSLEEP);
                }
                CBAR();
            }
            len0 = len1 > CMAX ? CMAX : len1;
            len0b = len1b > CMAX ? CMAX : len1b;
            if (nx == 1) { n0 = 1024 + pn; pn += EWG; wb = n1; }
            else { n0 = n1; wb = n1 + 1; have = false; }
            wb = wb < T1 ? T1 : (wb > T ? T : wb);
            load_t(n0, len0, len0b);
            WAIT_VM0();
        }
        STAMP(3);
        return;
    }

    const u32 ordv = P.ord[lane];
    {
        const u32 su = (blockIdx.x - NPWG) + (GRID - NPWG) * wave;
        if (su < 1024) {
            const u32 isw = su >> 9, GB = (su >> 4) & 31, kc = (su >> 1) & 7, hf = su & 1;
            u32x4 v[8];
            fetch_unit(P.x, P.w, su, lane, wl, v);
            const u32 half = g == 0 ? GB >> 4 : (GB >> 3) & 1;
            const u32 loc = g == 0 ? GB & 15 : (GB >> 4) * 8 + (GB & 7);
            const bool mine = kc < 5 && half == (isw ? bj : bi);
            const u32 du = isw * 1024 + loc * 64 + (rank & 3) * 10 + kc * 2 + hf;
#pragma unroll
            for (int i = 0; i < 8; ++i) st128(v[i], rme, lane * 16, OFF_STG + su * 8192 + i * 1024, AUX_SYS);
#pragma unroll
            for (int i = 0; i < 8; ++i) st128(v[i], rme, mine ? lane * 16 : 0x80000000u, OFF_XA + du * 8192 + i * 1024, AUX_SYS);
            WAIT_VM0();
            st_flag(rme, l0, OFF_LS + su * 4, rfl(epv) + 1);
            st_flag(rme, mine ? l0 : 0x80000000u, OFF_FX + du * 4, rfl(epv) + 1);
        }
        STAMP(10);
        if (blockIdx.x == GRID - 1 && wave == 7) {
            const u32 call = rfl(epv) + 1;
            u64 t0 = 0;
            for (u32 it = 0;; ++it) {
                bool ok = true;
#pragma unroll
                for (int i = 0; i < 4; ++i) {
                    const u32x4 f = __builtin_amdgcn_raw_buffer_load_b128(rme, lane * 16, OFF_LS + i * 1024, AUX_SYS);
                    ok = ok && f[0] >= call && f[1] >= call && f[2] >= call && f[3] >= call;
                }
                if (__ballot(ok) == ~0ull) break;
                if (it == 0) t0 = __builtin_amdgcn_s_memrealtime();
                if ((it & 63) == 63 && give_up(myws, call, t0)) break;
            }
            st_flag(rme, l0, OFF_ALL, call);
        }
    }

    const u32 call = rfl(epv) + 1;
    const u32 wm = (wave >> 1) & 1, wn = wave & 1;
    const u32 olane = (lane & 15) * (NN * 2) + (lane >> 4) * 16;
    const rsrc_t ro = mk_rsrc(P.out, OUT_BYTES);

    st_flag(rme, l0, epo, call);
    if (blockIdx.x < NPWG + NCWG) {
        const u32 c = blockIdx.x - NPWG;
        const u32 tl2 = P.tiles[c];
        volatile LAS u32* sel = (volatile LAS u32*)(smem + 16);

        for (u32 pass = 0; pass < ((tl2 >> 8) == (tl2 & 255) ? 1u : 2u); ++pass) {
            const u32 tb = tl2 >> (pass * 8);
            const u32 keep = (tb >> 7) & 1, jt = (tb >> 4) & 7, ct = tb & 15;
            const u32 j = perm8(jt);
            const u32 lc = perm16(ct);
            const u32 lr = row_lr(j, keep ? g : 1 - g);

            if (wave >= 4) {
                const u32 lw = wave - 4;
                const u32 xbase = OFF_XA + lr * 32 * BLK + lw * 4096;
                const u32 wbase = OFF_WB + lc * 32 * BLK + lw * 4096;
                const u32 fo = (lane < 32) ? OFF_FX + (lr * 32 + lane) * 8 : OFF_FW + (lc * 32 + lane - 32) * 8;
                u32 pend = 0, taken = 0;
                for (int s = -4; s < 32; ++s) {
                    if (lw == 0 && s < 28) {
                        if (pend == 0) {
                            u64 t0 = 0;
                            for (u32 it = 0;; ++it) {
                                const u64 fv = ld_flag64(myws, fo);
                                const u64 m = __ballot((u32)fv >= call && (u32)(fv >> 32) >= call);
                                pend = (u32)m & (u32)(m >> 32) & ~taken;
                                if (pend) break;
                                if (it == 0) t0 = __builtin_amdgcn_s_memrealtime();
                                if ((it & 63) == 63 && give_up(myws, call, t0)) { pend = ~taken; break; }
                                __builtin_amdgcn_s_sleep(PSLEEP);
                            }
                        }
                        const u32 ks = (u32)__builtin_ctz(pend);
                        pend &= pend - 1;
                        taken |= 1u << ks;
                        sel[s & 1] = ks;
                    }
                    if (s < 29) WAIT_VM(16); else if (s == 29) WAIT_VM(8); else WAIT_VM(0);
                    WAIT_LGKM0();
                    __builtin_amdgcn_s_barrier();
                    CBAR();
                    if (s < 28) {
                        const u32 so = (rfl(sel[s & 1]) & 31) * BLK;
                        LAS char* d = (LAS char*)smem + LDS_CTL + (s & 3) * 32768 + lw * 4096;
#pragma unroll
                        for (int i = 0; i < 4; ++i)
                            __builtin_amdgcn_raw_ptr_buffer_load_lds(rme, d + i * 1024, 16, lane * 16, xbase + so + i * 1024, 0, AUX_SC1);
#pragma unroll
                        for (int i = 0; i < 4; ++i)
                            __builtin_amdgcn_raw_ptr_buffer_load_lds(rme, d + 16384 + i * 1024, 16, lane * 16, wbase + so + i * 1024, 0, AUX_SC1);
                    }
                    CBAR();
                }
                continue;
            }
            f32x4 acc[4][4];
#pragma unroll
            for (int a = 0; a < 4; ++a)
#pragma unroll
                for (int b2 = 0; b2 < 4; ++b2) acc[a][b2] = f32x4{0.f, 0.f, 0.f, 0.f};
            {
                const char* xa = smem + LDS_CTL + wm * 8192 + lane * 16;
                const char* wb = smem + LDS_CTL + 16384 + wn * 8192 + lane * 16;
                bf16x8 xA[8], wA[8], xB[8], wB[8];
                auto rd = [&](bf16x8 (&xf)[8], bf16x8 (&wf)[8], u32 slot) {
                    const u32 so = (slot & 3) * 32768;
#pragma unroll
                    for (int i = 0; i < 8; ++i) {
                        wf[i] = *(const bf16x8*)(wb + so + i * 1024);
                        xf[i] = *(const bf16x8*)(xa + so + i * 1024);
                    }
                };
                auto mm = [&](bf16x8 (&xf)[8], bf16x8 (&wf)[8]) {
#pragma unroll
                    for (int h = 0; h < 2; ++h)
#pragma unroll
                        for (int a = 0; a < 4; ++a)
#pragma unroll
                            for (int b2 = 0; b2 < 4; ++b2)
                                acc[a][b2] = __builtin_amdgcn_mfma_f32_16x16x32_bf16(wf[h * 4 + b2], xf[h * 4 + a], acc[a][b2], 0, 0, 0);
                };
                auto sched = [&]() {
#pragma unroll
                    for (int i = 0; i < 16; ++i) {
                        __builtin_amdgcn_sched_group_barrier(0x100, 1, 0);
                        __builtin_amdgcn_sched_group_barrier(0x008, 2, 0);
                    }
                };
#define BAR() do { __builtin_amdgcn_s_waitcnt(0xC07F); CBAR(); __builtin_amdgcn_s_barrier(); CBAR(); __builtin_amdgcn_sched_barrier(0); } while (0)
                BAR(); BAR(); BAR();
                BAR();
                rd(xA, wA, 0);
                for (u32 s = 0; s < 32; s += 2) {
                    BAR();
                    rd(xB, wB, s + 1);
                    mm(xA, wA);
                    sched();
                    __builtin_amdgcn_sched_barrier(0);
                    BAR();
                    rd(xA, wA, s + 2);
                    mm(xB, wB);
                    sched();
                    __builtin_amdgcn_sched_barrier(0);
                }
                WAIT_LGKM0();
            }
            STAMP(1 + 3 * pass);

            if (!keep) {
                const u32 pu = (((jt & 1) * 2 + (ct & 1)) * 32 + part_idx(jt >> 1, ct >> 1)) * 4 + wave;
                const u32 dst = OFF_SPART + pu * 8192;
#pragma unroll
                for (int a = 0; a < 4; ++a)
#pragma unroll
                    for (int u = 0; u < 2; ++u) {
                        u32x4 o;
                        o[0] = pack2(acc[a][2 * u][0], acc[a][2 * u][1]);
                        o[1] = pack2(acc[a][2 * u][2], acc[a][2 * u][3]);
                        o[2] = pack2(acc[a][2 * u + 1][0], acc[a][2 * u + 1][1]);
                        o[3] = pack2(acc[a][2 * u + 1][2], acc[a][2 * u + 1][3]);
                        st128(o, rme, lane * 16, dst + (a * 2 + u) * 1024, AUX_SYS);
                    }
                WAIT_VM(0);
                st_flag64(rme, l0, OFF_LP + pu * 8, call, 8192);
                STAMP(3 + 3 * pass);
            } else {
                const u32 pu = (j * 16 + lc) * 4 + wave;
                wait_flag64<PSLEEP>(myws, OFF_FP + pu * 8, call);
                STAMP(2 + 3 * pass);
                u32x4 o[8];
                u32 bits[2] = {0u, 0u};
                {
                    u32x4 pv[8];
#pragma unroll
                    for (int i = 0; i < 8; ++i) pv[i] = __builtin_amdgcn_raw_buffer_load_b128(rme, lane * 16, OFF_PART + pu * 8192 + i * 1024, AUX_SC1);
                    const bool bad = rfl(ld_tmo(myws)) == call;
#pragma unroll
                    for (int gi = 0; gi < 8; ++gi) {
                        float y[8];
#pragma unroll
                        for (int k = 0; k < 8; ++k) {
                            const u32 pq = pv[gi][k >> 1];
                            float t = acc[gi >> 1][(gi & 1) * 2 + (k >> 2)][k & 3] + ((k & 1) ? bf_hi(pq) : bf_lo(pq));
                            if (bad) t = __builtin_nanf("");
                            const bool nz = !(t <= 0.f);
                            y[k] = nz ? t : 0.f;
                            bits[gi >> 2] |= nz ? (1u << ((gi & 3) * 8 + k)) : 0u;
                        }
                        o[gi] = u32x4{pack2(y[0], y[1]), pack2(y[2], y[3]), pack2(y[4], y[5]), pack2(y[6], y[7])};
                    }
                }
                u32 tot;
                u32 addr = (u32)(LDS_CTL + wave * 16384 + 512) + 2 * wave_excl(__builtin_popcount(bits[0]) + __builtin_popcount(bits[1]), tot);
                *(u32x2*)(wl + lane * 8) = u32x2{bits[0], bits[1]};
#pragma unroll
                for (int gi = 0; gi < 8; ++gi)
#pragma unroll
                    for (int k = 0; k < 8; ++k) {
                        const u32 t = (bits[gi >> 2] >> ((gi & 3) * 8 + k)) & 1u;
                        const u32 d = o[gi][k >> 1];
                        *(LAS unsigned short*)((LAS char*)smem + __umul24(addr, t)) = (unsigned short)((k & 1) ? d >> 16 : d);
                        addr += 2 * t;
                    }
                WAIT_LGKM0();
                const u32 len = (512 + tot * 2 + 63) & ~63u;
                const u32 tl = jt * 16 + ct;
                const u32 sidx = rank * 512 + ((((u32)__builtin_amdgcn_readlane((int)ordv, (int)(tl >> 2))) >> ((tl & 3) * 8)) & 127) * 4 + wave;
                const u32 so = OFF_CMP + sidx * CSTR;
#pragma unroll
                for (int i = 0; i < 9; ++i)
                    if (i * 1024u < len) {
                        const u32x4 v = *(const u32x4*)(wl + i * 1024 + lane * 16);
                        st128(v, rme, lane * 16, so + i * 1024, AUX_SYS);
                    }
                WAIT_VM(0);
                st_flag64(rme, l0, OFF_FC + sidx * 8, call, len);
                STAMP(3 + 3 * pass);
                const u32 obase = (glob_blk(g, bi, lr) * 128 + wm * 64) * (NN * 2) + (glob_blk(g, bj, lc) * 128 + wn * 64) * 2;
#pragma unroll
                for (int gi = 0; gi < 8; ++gi)
                    st_out(o[gi], ro, olane, obase + (gi >> 1) * 16 * (NN * 2) + (gi & 1) * 64);
                WAIT_LGKM0();
            }
        }
    }

    if (wave >= 8 - XWAVES || blockIdx.x >= NPWG + NCWG) {
        const u32 tbase = (call - 1) * (NSTREAM + NXW);
        STAMP(7);
        for (;;) {
            const u32 idx = rfl(ticket(rme, l0, 8)) - tbase;
            if (idx >= NSTREAM) break;
            const u32 n3 = idx / 7, pi = idx - n3 * 7;
            const u32 p = (rank + 1 + pi) & 7;
            const u32 sidx = p * 512 + n3;
            const u32 len = wait_flag64<XSLEEP>(myws, OFF_FC + sidx * 8, call);
            STAMP(8);
            const u32 so = OFF_CMP + sidx * CSTR;
#pragma unroll
            for (int i = 0; i < 9; ++i)
                if (i * 1024u < len)
                    __builtin_amdgcn_raw_ptr_buffer_load_lds(rme, (LAS char*)wl + i * 1024, 16, lane * 16, so + i * 1024, 0, AUX_SC1);
            const u32 tm = ld_tmo(myws);
            const u32 to = n3 >> 2, sw = n3 & 3;
            const u32 tl = ((u32)__builtin_amdgcn_readlane((int)ordv, (int)(32 + (to >> 2)))) >> ((to & 3) * 8);
            const u32 tj = perm8((tl >> 4) & 7), tc = perm16(tl & 15);
            const u32 gp = p >> 2;
            const u32 tRB = glob_blk(gp, (p >> 1) & 1, row_lr(tj, gp)), tCB = glob_blk(gp, p & 1, tc);
            const u32 obase = (tRB * 128 + (sw >> 1) * 64) * (NN * 2) + (tCB * 128 + (sw & 1) * 64) * 2;
            WAIT_VM(0);
            const bool bad = rfl(tm) == call;
            const u32x2 bits = *(const u32x2*)(wl + lane * 8);
            u32 tot;
            u32 addr = (u32)(LDS_CTL + wave * 16384 + 512) + 2 * wave_excl(__builtin_popcount(bits[0]) + __builtin_popcount(bits[1]), tot);
#pragma unroll
            for (int gi = 0; gi < 8; ++gi) {
                u32 e[8];
#pragma unroll
                for (int k = 0; k < 8; ++k) {
                    const u32 t = (bits[gi >> 2] >> ((gi & 3) * 8 + k)) & 1u;
                    e[k] = *(const LAS unsigned short*)((LAS char*)smem + __umul24(addr, t));
                    addr += 2 * t;
                }
                u32x4 r = {e[0] | (e[1] << 16), e[2] | (e[3] << 16), e[4] | (e[5] << 16), e[6] | (e[7] << 16)};
                if (bad) r = u32x4{0x7fc07fc0u, 0x7fc07fc0u, 0x7fc07fc0u, 0x7fc07fc0u};
                st_out(r, ro, olane, obase + (gi >> 1) * 16 * (NN * 2) + (gi & 1) * 64);
            }
            WAIT_LGKM0();
            STAMP(9);
        }
    }
}

static void build_orders(Params& P) {
    const char* eo = ENTRY_ORDER;
    double t[3][16];
    for (int i = 0; i < 4; ++i) P.entry[i] = 0;
    for (int e = 0; e < 32; ++e) {
        const int kind = eo[2 * e] == 'w' ? 0 : eo[2 * e] == 's' ? 1 : 2;
        const char ch = eo[2 * e + 1];
        const int idx = ch <= '9' ? ch - '0' : ch - 'A' + 10;
        P.entry[e >> 3] |= (u64)(kind << 4 | idx) << ((e & 7) * 8);
        t[kind][idx] = (e + 1) * 1.445;
    }
    auto mx = [](double a, double b) { return a > b ? a : b; };
    double av[128];
    unsigned char* ord = (unsigned char*)P.ord;
    for (int i = 0; i < 128; ++i) {
        const int jt = i >> 4, ct = i & 15;
        av[i] = mx(mx(t[2][jt], t[0][ct]) + LAT_K, mx(mx(t[1][jt & ~1], t[1][jt | 1]), mx(t[0][ct & ~1], t[0][ct | 1])) + LAT_S) + 1e-3 * i;
    }
    for (int i = 0; i < 128; ++i) {
        int pos = 0;
        for (int k = 0; k < 128; ++k) pos += av[k] < av[i];
        ord[i] = (unsigned char)pos;
        ord[128 + pos] = (unsigned char)i;
    }
    const int W = NCWG / 8, D = 32 - W;
    for (int x = 0; x < 8; ++x) {
        int id[32]; double key[32];
        for (int i = 0; i < 32; ++i) {
            const int keep = i >> 4, jt = (x >> 1) + 4 * ((i >> 3) & 1), ct = (i & 7) * 2 + (x & 1);
            id[i] = keep << 7 | jt << 4 | ct;
            key[i] = (keep ? av[jt * 16 + ct] - LAT_K : mx(t[1][jt], t[0][ct])) + 1e-3 * i;
        }
        int sorted[32];
        for (int i = 0; i < 32; ++i) {
            int pos = 0;
            for (int k = 0; k < 32; ++k) pos += key[k] < key[i];
            sorted[pos] = id[i];
        }
        for (int i = 0; i < W; ++i) P.tiles[i * 8 + x] = (unsigned short)(sorted[i] | sorted[i < D ? W + i : i] << 8);
    }
}

extern "C" void kernel_launch(int rank, int nranks, ncclComm_t comm,
                              hipStream_t stream,
                              void* const* d_in, const int* in_sizes, int n_in,
                              void* d_out, int out_size,
                              const int* in_dtypes, int out_dtype,
                              void* const* d_out_all,
                              void* const* d_ws_all, size_t ws_bytes) {
    if (nranks != NR || n_in < 2 || ws_bytes < WS_USED || (unsigned)rank >= (unsigned)NR ||
        in_sizes[0] != MM * KS || in_sizes[1] != KS * NN || out_size != MM * NN) {
        fprintf(stderr, "unexpected problem shape\n");
        return;
    }
    Params P;
    P.x = d_in[0];
    P.w = d_in[1];
    for (int i = 0; i < NR; ++i) P.ws[i] = d_ws_all[i];
    P.out = d_out;
    P.rank = rank;
    build_orders(P);
    HIP_CHECK(hipFuncSetAttribute((const void*)fused_gemm_ar, hipFuncAttributeMaxDynamicSharedMemorySize, LDS_BYTES));
    hipLaunchKernelGGL(fused_gemm_ar, dim3(GRID), dim3(512), LDS_BYTES, stream, P);
    HIP_CHECK(hipGetLastError());
}
```

```cpp
#include <hip/hip_runtime.h>
#include <rccl/rccl.h>
#include <cstdio>
#include <cstdint>

#define HIP_CHECK(expr) do { hipError_t _e = (expr); if (_e != hipSuccess) { \
    fprintf(stderr, "HIP error %s at %s:%d\n", hipGetErrorString(_e), __FILE__, __LINE__); \
    return; } } while (0)

#define NFET 3
#define NPF 2
#define WIN 7
#define EWG 8
#define XWAVES 4
#define DIAG 0
#define LAT_K 12.0
#define LAT_S 19.0
#define ORD 1
#define ENTRY_ORDER "w0s0w1s1w2s2w3s3k0w4w5k1s4k2s5k3w6w7k4k5s6s7w8w9wAwBk6k7wCwDwEwF"
#define OUT_AUX 17
#define XSLEEP 4
#define PSLEEP 8

typedef unsigned int u32;
typedef unsigned long long u64;
typedef u32 u32x4 __attribute__((ext_vector_type(4)));
typedef u32 u32x2 __attribute__((ext_vector_type(2)));
typedef float f32x4 __attribute__((ext_vector_type(4)));
typedef short bf16x8 __attribute__((ext_vector_type(8)));
typedef __bf16 bf16v2 __attribute__((ext_vector_type(2)));
typedef float f32x2 __attribute__((ext_vector_type(2)));
#define LAS __attribute__((address_space(3)))

constexpr int NR = 8;
constexpr int MM = 4096, NN = 4096, KS = 512;
constexpr u32 BLK = 16384;
constexpr u32 OFF_XA = 0;
constexpr u32 OFF_WB = 8u << 20;
constexpr u32 OFF_PART = 16u << 20;
constexpr u32 OFF_SPART = 20u << 20;
constexpr u32 CSTR = 9216;
constexpr u32 CMAX = 8704;
constexpr u32 OFF_CMP = 24u << 20;
constexpr u32 OFF_FLAGS = 60u << 20;
constexpr u32 OFF_FX = OFF_FLAGS;
constexpr u32 OFF_FW = OFF_FX + 4096;
constexpr u32 OFF_FP = OFF_FW + 4096;
constexpr u32 OFF_LP = OFF_FP + 4096;
constexpr u32 OFF_FC = OFF_LP + 4096;
constexpr u32 OFF_CTR = OFF_FC + 36864;
constexpr u32 OFF_EP = OFF_CTR + 4096;
constexpr u32 OFF_TMO = OFF_EP + 8192;
constexpr u32 OFF_DIAG = OFF_TMO + 4096;
constexpr u32 WS_USED = OFF_DIAG + (1u << 20);
constexpr u32 OUT_BYTES = (u32)MM * NN * 2;
constexpr u32 IN_BYTES = (u32)MM * KS * 2;

constexpr int NPWG = 7 * EWG;
constexpr int NWL = EWG * NFET;
constexpr int NSL = NFET + NPF;
constexpr int NCWG = EWG == 8 ? 192 : 128;
constexpr int GRID = 256;
constexpr u32 NXW = NCWG * XWAVES + (GRID - NPWG - NCWG) * 8;
constexpr u32 LDS_CTL = DIAG ? 1024 : 512;
constexpr int LDS_BYTES = LDS_CTL + 4 * 32768;
constexpr u64 DEADLINE = 100000000ull;
constexpr int AUX_SYS = 17;
constexpr int AUX_SC1 = 16;
static_assert(GRID * 8 * 4 <= 8192, "epoch words");
static_assert(OFF_CMP + 8u * 512u * CSTR <= OFF_FLAGS, "layout");

struct Params {
    const void* x;
    const void* w;
    void* ws[NR];
    void* out;
    int rank;
    u64 entry[4];
    unsigned short tiles[NCWG];
    u32 ord[64];
};

typedef __amdgpu_buffer_rsrc_t rsrc_t;
__device__ __forceinline__ rsrc_t mk_rsrc(const void* p, u32 bytes) {
    return __builtin_amdgcn_make_buffer_rsrc((void*)p, (short)0, (int)bytes, 0x00020000);
}
__device__ __forceinline__ u32 rfl(u32 v) { return (u32)__builtin_amdgcn_readfirstlane((int)v); }
__device__ __forceinline__ void* pick(void* const (&t)[NR], u32 idx) {
    void* r = t[0];
#pragma unroll
    for (u32 i = 1; i < NR; ++i) r = (idx == i) ? t[i] : r;
    return r;
}
__device__ __forceinline__ u32 glob_blk(u32 g, u32 b, u32 l) {
    return g == 0 ? b * 16 + l : ((l >> 3) * 2 + b) * 8 + (l & 7);
}
__device__ __forceinline__ u32 perm8(u32 t) { return (t >> 1) + (t & 1) * 4; }
__device__ __forceinline__ u32 perm16(u32 t) { return (t >> 1) + (t & 1) * 8; }
__device__ __forceinline__ u32 row_lr(u32 j, u32 fin) { return (j >> 2) * 8 + fin * 4 + (j & 3); }

__device__ __forceinline__ u32 part_idx(u32 a, u32 b) { return b >= 4 ? b * 4 + a : (b > a ? b * b + a : a * a + a + b); }
__device__ __forceinline__ void part_ab(u32 m, u32& a, u32& b) {
    const u32 sh = m >= 9 ? 3 : m >= 4 ? 2 : m >= 1 ? 1 : 0, o = m - sh * sh;
    a = o < sh ? o : sh;
    b = o < sh ? sh : o - sh;
    if (m >= 16) { a = m & 3; b = m >> 2; }
}
template <bool READ_BACK>
__device__ __forceinline__ void fetch_unit(const void* x, const void* w, u32 su, u32 lane, char* wl, u32x4* v) {
    const u32 isw = su >> 9, GB = (su >> 4) & 31, kc = (su >> 1) & 7, hf = su & 1;
    if (isw) {
        const rsrc_t rw = mk_rsrc(w, IN_BYTES);
        const u32 vo = (lane >> 3) * 65536 + (lane & 7) * 16, so = kc * 64 * 8192 + GB * 256 + hf * 128;
#pragma unroll
        for (int i = 0; i < 8; ++i) v[i] = __builtin_amdgcn_raw_buffer_load_b128(rw, vo, so + i * 8192, 0);
        const u32 wo = (lane >> 5) * 4096 + ((lane & 7) >> 2) * 2048 + (((lane >> 3) & 3) * 16 + (lane & 3) * 4) * 16;
#pragma unroll
        for (int c = 0; c < 8; ++c) {
            u32x4 o;
#pragma unroll
            for (int t = 0; t < 4; ++t)
                o[t] = __builtin_amdgcn_perm(v[2 * t + 1][c >> 1], v[2 * t][c >> 1], (c & 1) ? 0x07060302u : 0x05040100u);
            *(u32x4*)(wl + wo + (c >> 2) * 1024 + (c & 3) * 16) = o;
        }
    } else {
        const rsrc_t rx = mk_rsrc(x, IN_BYTES);
        const u32 vo = (lane >> 3) * 1024 + (lane & 7) * 16, so = GB * 131072 + hf * 65536 + kc * 128;
#pragma unroll
        for (int i = 0; i < 8; ++i) v[i] = __builtin_amdgcn_raw_buffer_load_b128(rx, vo, so + i * 8192, 0);
        const u32 wo = ((lane & 7) >> 2) * 4096 + ((lane & 3) * 16 + (lane >> 3)) * 16;
#pragma unroll
        for (int i = 0; i < 8; ++i) *(u32x4*)(wl + wo + (i >> 1) * 1024 + (i & 1) * 128) = v[i];
    }
    asm volatile("s_waitcnt lgkmcnt(0)" ::: "memory");
    if (READ_BACK) {
#pragma unroll
        for (int i = 0; i < 8; ++i) v[i] = *(const u32x4*)(wl + i * 1024 + lane * 16);
        asm volatile("s_waitcnt lgkmcnt(0)" ::: "memory");
    }
}
__device__ __forceinline__ void st128(u32x4 v, rsrc_t r, u32 voff, u32 soff, int) {
    __builtin_amdgcn_raw_buffer_store_b128(v, r, voff + soff, 0, AUX_SYS);
}
__device__ __forceinline__ void st_out(u32x4 v, rsrc_t r, u32 voff, u32 soff) {
    __builtin_amdgcn_raw_buffer_store_b128(v, r, voff + soff, 0, OUT_AUX);
}
__device__ __forceinline__ void st_flag(rsrc_t r, u32 l0, u32 off, u32 v) {
    __builtin_amdgcn_raw_buffer_store_b32(v, r, l0, off, AUX_SYS);
}
__device__ __forceinline__ void st_flag64(rsrc_t r, u32 l0, u32 off, u32 call, u32 len) {
    __builtin_amdgcn_raw_buffer_store_b64(u32x2{len, call}, r, l0, off, AUX_SYS);
}
__device__ __forceinline__ u64 ld_flag64(const void* ws, u32 off) {
    return __hip_atomic_load((const u64*)((const char*)ws + off), __ATOMIC_RELAXED, __HIP_MEMORY_SCOPE_SYSTEM);
}
#define CBAR() asm volatile("" ::: "memory")
#define WAIT_VM(n) asm volatile("s_waitcnt vmcnt(" #n ")" ::: "memory")
#define WAIT_VM0() do { WAIT_VM(0); __builtin_amdgcn_s_waitcnt(0x0F70); CBAR(); } while (0)
#define WAIT_LGKM0() asm volatile("s_waitcnt lgkmcnt(0)" ::: "memory")

__device__ __forceinline__ bool give_up(void* myws, u32 call, u64 t0) {
    u32* tmo = (u32*)((char*)myws + OFF_TMO);
    bool late = __builtin_amdgcn_s_memrealtime() - t0 > DEADLINE;
    if (late || rfl(__hip_atomic_load(tmo, __ATOMIC_RELAXED, __HIP_MEMORY_SCOPE_AGENT)) == call) {
        __hip_atomic_store(tmo, call, __ATOMIC_RELAXED, __HIP_MEMORY_SCOPE_AGENT);
        return true;
    }
    return false;
}
__device__ __forceinline__ u32 ld_tmo(void* myws) {
    return __hip_atomic_load((u32*)((char*)myws + OFF_TMO), __ATOMIC_RELAXED, __HIP_MEMORY_SCOPE_AGENT);
}
template <int SLP>
__device__ __forceinline__ u32 wait_flag64(void* myws, u32 off, u32 call) {
    u64 t0 = 0;
    for (u32 it = 0;; ++it) {
        u64 v = ld_flag64(myws, off);
        if (rfl((u32)(v >> 32)) >= call) return rfl((u32)v);
        if (it == 0) t0 = __builtin_amdgcn_s_memrealtime();
        if ((it & 63) == 63 && give_up(myws, call, t0)) return 0;
        __builtin_amdgcn_s_sleep(SLP);
    }
}
__device__ __forceinline__ u32 ticket(rsrc_t r, u32 l0, u32 idx) {
    return (u32)__builtin_amdgcn_raw_ptr_buffer_atomic_add_i32(1, r, l0, OFF_CTR + idx * 128, 0);
}
__device__ __forceinline__ u32 mbcnt64(u64 m) {
    return __builtin_amdgcn_mbcnt_hi((u32)(m >> 32), __builtin_amdgcn_mbcnt_lo((u32)m, 0u));
}
__device__ __forceinline__ u32 wave_excl(u32 cnt, u32& tot) {
    u32 pre = 0; tot = 0;
#pragma unroll
    for (int b = 0; b < 7; ++b) {
        const u64 m = __ballot((cnt >> b) & 1);
        pre += mbcnt64(m) << b;
        tot += (u32)__builtin_popcountll(m) << b;
    }
    return pre;
}
__device__ __forceinline__ u32 pack2(float a, float b) {
    f32x2 v = {a, b};
    return __builtin_bit_cast(u32, __builtin_convertvector(v, bf16v2));
}
__device__ __forceinline__ float bf_lo(u32 v) { return __builtin_bit_cast(float, v << 16); }
__device__ __forceinline__ float bf_hi(u32 v) { return __builtin_bit_cast(float, v & 0xffff0000u); }

#define STAMP(slot) do {} while (0)

__global__ void __launch_bounds__(512, 2) fused_gemm_ar(Params P) {
    extern __shared__ __attribute__((aligned(16))) char smem[];
    const u32 lane = threadIdx.x & 63;
    const u32 wave = rfl(threadIdx.x >> 6);
    const u32 rank = (u32)P.rank & 7;
    const u32 g = rank >> 2, bi = (rank >> 1) & 1, bj = rank & 1;
    void* myws = pick(P.ws, rank);
    const rsrc_t rme = mk_rsrc(myws, WS_USED);
    char* wl = smem + LDS_CTL + wave * 16384;

    const u32 l0 = lane == 0 ? 0u : 0x80000000u;
    const u32 epo = OFF_EP + (blockIdx.x * 8 + wave) * 4;
    const u32 epv = __builtin_amdgcn_raw_buffer_load_b32(rme, 0, epo, AUX_SC1);
    *(LAS u32*)smem = 0u;
    STAMP(0);

    if (blockIdx.x < NPWG) {
        const u32 b = blockIdx.x;
        const u32 di = 1 + ((b >> 3) + (b & 7)) % 7;
        const u32 q = (rank + di) & 7;
        volatile LAS u32* ctl = (volatile LAS u32*)(smem + 32);
        if (wave == 0 && lane < 8) ctl[lane * 8] = 0;
        WAIT_LGKM0();
        __builtin_amdgcn_s_barrier();
        CBAR();
        STAMP(5);
        if (wave > NSL) { st_flag(rme, l0, epo, epv + 1); return; }
        if (wave == 0) {
            const rsrc_t rq = mk_rsrc(pick(P.ws, q), WS_USED);
            const u32 call = rfl(epv) + 1;
            st_flag(rme, l0, epo, call);
            WAIT_VM0();
            u32 pk = 3, pfo = 0, pla = 0, plb = 0;
            auto flush = [&]() {
                if (pk == 0) st_flag(rq, l0, pfo, call);
                else __builtin_amdgcn_raw_buffer_store_b64(u32x2{lane == 0 ? pla : plb, call}, rq, lane < pk ? lane * 8 : 0x80000000u, pfo, AUX_SYS);
                pk = 3;
            };
            u32 rr = 0;
            u64 t0 = __builtin_amdgcn_s_memrealtime();
            for (u32 it = 0;; ++it) {
                const u32 stv = ctl[(lane & 7) * 8];
                const u32 rdy = (u32)__ballot(stv == 1) & ((1u << NSL) - 1);
                if (rdy == 0) {
                    if (pk != 3) { WAIT_VM(0); flush(); }
                    if (((u32)__ballot(stv == 2) & ((1u << NSL) - 1)) == (1u << NSL) - 1) break;
                    if ((it & 1023) == 1023 && __builtin_amdgcn_s_memrealtime() - t0 > 2 * DEADLINE) break;
                    __builtin_amdgcn_s_sleep(1);
                    continue;
                }
                u32 sl;
                if (rdy >> NFET) sl = NFET + (u32)__builtin_ctz(rdy >> NFET);
                else {
                    sl = rr + (u32)__builtin_ctz((rdy | rdy << NFET) >> rr);
                    sl = sl >= NFET ? sl - NFET : sl;
                    rr = sl + 1 >= NFET ? 0 : sl + 1;
                }
                const u32 dv = ctl[sl * 8 + (lane & 7)];
                const u32 dA = (u32)__builtin_amdgcn_readlane((int)dv, 1), dB = (u32)__builtin_amdgcn_readlane((int)dv, 2);
                const u32 wA = (u32)__builtin_amdgcn_readlane((int)dv, 3), wB = (u32)__builtin_amdgcn_readlane((int)dv, 4);
                const u32 lA = wA & 0xffffu, lB = wB & 0xffffu;
                const u32 lds_b = (u32)__builtin_amdgcn_readlane((int)dv, 5);
                const u32 fo = (u32)__builtin_amdgcn_readlane((int)dv, 6), kind = (u32)__builtin_amdgcn_readlane((int)dv, 7);
                const char* sp = smem + LDS_CTL + sl * 18432 + lane * 16;
                const u32 nA = (lA + 1023) >> 10, n = nA + ((lB + 1023) >> 10);
                u32x4 cur = *(const u32x4*)(sp + (nA ? 0 : lds_b));
                for (u32 j = 0; j < n; ++j) {
                    const u32 jn = j + 1;
                    const u32x4 nxt = *(const u32x4*)(sp + (jn < nA ? jn * 1024 : lds_b + (jn - nA) * 1024));
                    const u32 o = j < nA ? j * 1024 : (j - nA) * 1024;
                    asm volatile("s_waitcnt vmcnt(%0)" :: "n"(WIN - 1) : "memory");
                    if (j == WIN - 1 && pk != 3) flush();
                    st128(cur, rq, (o + lane * 16 < (j < nA ? lA : lB)) ? lane * 16 : 0x80000000u, (j < nA ? dA : dB) + o, AUX_SYS);
                    cur = nxt;
                }
                WAIT_LGKM0();
                ctl[sl * 8] = 0;
                if (pk != 3) { WAIT_VM(0); flush(); }
                pk = kind; pfo = fo; pla = wA; plb = wB;
            }
            STAMP(3);
            return;
        }
        const u32 sl = wave - 1;
        const u32 gq = q >> 2, qi = (q >> 1) & 1, qj = q & 1;
        const u32 nk2 = (gq == g) ? 10 : 6;
        const u32 kh0 = (gq == g) ? 0 : 10;
        const u32 hs0 = (gq == g) ? (rank & 3) * 10 : 40 + (rank & 3) * 6;
        const u32 T1 = 32 * nk2;
        const u32 T = T1 + 256;
        const u32 lco = OFF_FC + rank * 4096 - T1 * 16;
        const u32 lpo = OFF_LP + (q & 3) * 1024;
        char* sp = smem + LDS_CTL + sl * 18432;
        const bool isp = sl >= NFET;
        const u32 wgi = (b & 7) + (b >= 56 ? 8 : 0);
        u32 n0 = isp ? 1024 + (sl - NFET) * EWG + wgi : sl * EWG + wgi;
        const u32 nend = isp ? ((gq == g) ? 0 : 1024 + 128) : T;
        u32 lA = 4096, lB = 4096, call = 0, tbase = 0;
        if (isp) {
            call = rfl(epv) + 1;
            st_flag(rme, l0, epo, call);
        }
        for (;;) {
            const bool last = n0 >= nend;
            bool ready = last || n0 < T1;
            const u32 o2 = isp ? lpo + ((n0 - 1024) & 127) * 8 : lco + (n0 < T ? n0 : T) * 16 + (lane & 1) * 8;
            u64 t0 = 0;
            bool dead = false;
            for (u32 it = 0;; ++it) {
                if (!ready) {
                    const u32x2 f2 = __builtin_amdgcn_raw_buffer_load_b64(rme, o2, 0, AUX_SYS);
                    ready = rfl(f2[1]) >= call && (u32)__builtin_amdgcn_readlane((int)f2[1], 1) >= call;
                    if (!isp) { lA = rfl(f2[0]); lB = (u32)__builtin_amdgcn_readlane((int)f2[0], 1); }
                }
                if (ready && rfl(ctl[sl * 8]) == 0) break;
                if (it == 0) t0 = __builtin_amdgcn_s_memrealtime();
                if ((it & 63) == 63 && give_up(myws, call, t0)) {
                    if (!ready) { ready = true; if (!isp) lA = lB = 0; }
                    if (__builtin_amdgcn_s_memrealtime() - t0 > 2 * DEADLINE) { dead = true; break; }
                }
                if (ready) __builtin_amdgcn_s_sleep(1); else __builtin_amdgcn_s_sleep(PSLEEP);
            }
            CBAR();
            if (last || dead) break;
            if ((lA & 0xffffu) > CMAX) lA = 0;
            if ((lB & 0xffffu) > CMAX) lB = 0;
            const u32 t2v = ticket(rme, isp ? 0x80000000u : l0, di);
            u32 so, dA, fo, kind;
            if (n0 < T1) {
                const u32 e = (nk2 == 10) ? n0 / 10 : n0 / 6;
                const u32 kk = n0 - e * nk2;
                const u32 eb = (u32)(P.entry[(e >> 3) & 3] >> ((e & 7) * 8));
                const u32 ek = (eb >> 4) & 3, idx = eb & 15;
                const bool isw = ek == 0;
                const u32 l = isw ? perm16(idx) : row_lr(perm8(idx & 7), ek == 1 ? 1 - gq : gq);
                const u32 su = (isw ? 512 : 0) + glob_blk(gq, isw ? qj : qi, l) * 16 + kh0 + kk;
                const u32 du = (isw ? 1024 : 0) + l * 64 + hs0 + kk;
                so = 0; dA = OFF_XA + du * 8192; fo = OFF_FX + du * 4; kind = 0;
                u32x4 v[8];
                fetch_unit<false>(P.x, P.w, su, lane, sp, v);
            } else if (n0 < 1024) {
                so = dA = OFF_CMP + (rank * 512 + (n0 - T1) * 2) * CSTR; fo = lco + n0 * 16; kind = 2;
            } else {
                u32 a, bb; part_ab((n0 - 1024) >> 2, a, bb);
                const u32 pu = ((bi * 4 + a) * 16 + bj * 8 + bb) * 4 + (n0 & 3);
                so = OFF_SPART + ((q & 3) * 128 + n0 - 1024) * 8192; dA = OFF_PART + pu * 8192; fo = OFF_FP + pu * 8; kind = 1;
            }
            const u32 ob = kind == 2 ? CSTR : 4096;
            if (kind != 0) {
                const u32 nA = ((lA & 0xffffu) + 1023) >> 10, n = nA + (((lB & 0xffffu) + 1023) >> 10);
#pragma unroll 1
                for (u32 j = 0; j < n; ++j) {
                    const u32 o = j < nA ? j * 1024 : ob + (j - nA) * 1024;
                    __builtin_amdgcn_raw_ptr_buffer_load_lds(rme, (LAS char*)sp + o, 16, lane * 16, so + o, 0, AUX_SC1);
                }
            }
            *(volatile LAS u32x4*)(ctl + sl * 8) = u32x4{0u, dA, dA + ob, lA};
            *(volatile LAS u32x4*)(ctl + sl * 8 + 4) = u32x4{lB, ob, fo, kind};
            WAIT_VM0();
            ctl[sl * 8] = 1;
            CBAR();
            if (call == 0) {
                STAMP(6);
                call = rfl(epv) + 1;
                st_flag(rme, l0, epo, call);
                tbase = (call - 1) * T - NWL;
            }
            n0 = isp ? n0 + NPF * EWG : rfl(t2v) - tbase;
        }
        ctl[sl * 8] = 2;
        return;
    }

    const u32 ordv = P.ord[lane];
    {
        const u32 u = (blockIdx.x - NPWG) + (GRID - NPWG) * wave;
        if (u < 320) {
            const u32 isw = u >= 160, r = u - isw * 160, loc = r / 10, kk = r - loc * 10;
            const u32 du = isw * 1024 + loc * 64 + (rank & 3) * 10 + kk;
            u32x4 v[8];
            fetch_unit<true>(P.x, P.w, isw * 512 + glob_blk(g, isw ? bj : bi, loc) * 16 + kk, lane, wl, v);
#pragma unroll
            for (int i = 0; i < 8; ++i) st128(v[i], rme, lane * 16, OFF_XA + du * 8192 + i * 1024, AUX_SYS);
            WAIT_VM0();
            st_flag(rme, l0, OFF_FX + du * 4, rfl(epv) + 1);
        }
        STAMP(10);
    }

    const u32 call = rfl(epv) + 1;
    const u32 wm = (wave >> 1) & 1, wn = wave & 1;
    const u32 olane = (lane & 15) * (NN * 2) + (lane >> 4) * 16;
    const rsrc_t ro = mk_rsrc(P.out, OUT_BYTES);

    st_flag(rme, l0, epo, call);
    if (blockIdx.x < NPWG + NCWG) {
        const u32 c = blockIdx.x - NPWG;
        const u32 tl2 = P.tiles[c];
        volatile LAS u32* sel = (volatile LAS u32*)(smem + 16);

        for (u32 pass = 0; pass < ((tl2 >> 8) == (tl2 & 255) ? 1u : 2u); ++pass) {
            const u32 tb = tl2 >> (pass * 8);
            const u32 keep = (tb >> 7) & 1, jt = (tb >> 4) & 7, ct = tb & 15;
            const u32 j = perm8(jt);
            const u32 lc = perm16(ct);
            const u32 lr = row_lr(j, keep ? g : 1 - g);

            if (wave >= 4) {
                const u32 lw = wave - 4;
                const u32 xbase = OFF_XA + lr * 32 * BLK + lw * 4096;
                const u32 wbase = OFF_WB + lc * 32 * BLK + lw * 4096;
                const u32 fo = (lane < 32) ? OFF_FX + (lr * 32 + lane) * 8 : OFF_FW + (lc * 32 + lane - 32) * 8;
                u32 pend = 0, taken = 0;
                for (int s = -4; s < 32; ++s) {
                    if (lw == 0 && s < 28) {
                        if (pend == 0) {
                            u64 t0 = 0;
                            for (u32 it = 0;; ++it) {
                                const u64 fv = ld_flag64(myws, fo);
                                const u64 m = __ballot((u32)fv >= call && (u32)(fv >> 32) >= call);
                                pend = (u32)m & (u32)(m >> 32) & ~taken;
                                if (pend) break;
                                if (it == 0) t0 = __builtin_amdgcn_s_memrealtime();
                                if ((it & 63) == 63 && give_up(myws, call, t0)) { pend = ~taken; break; }
                                __builtin_amdgcn_s_sleep(PSLEEP);
                            }
                        }
                        const u32 ks = (u32)__builtin_ctz(pend);
                        pend &= pend - 1;
                        taken |= 1u << ks;
                        sel[s & 1] = ks;
                    }
                    if (s < 29) WAIT_VM(16); else if (s == 29) WAIT_VM(8); else WAIT_VM(0);
                    WAIT_LGKM0();
                    __builtin_amdgcn_s_barrier();
                    CBAR();
                    if (s < 28) {
                        const u32 so = (rfl(sel[s & 1]) & 31) * BLK;
                        LAS char* d = (LAS char*)smem + LDS_CTL + (s & 3) * 32768 + lw * 4096;
#pragma unroll
                        for (int i = 0; i < 4; ++i)
                            __builtin_amdgcn_raw_ptr_buffer_load_lds(rme, d + i * 1024, 16, lane * 16, xbase + so + i * 1024, 0, AUX_SC1);
#pragma unroll
                        for (int i = 0; i < 4; ++i)
                            __builtin_amdgcn_raw_ptr_buffer_load_lds(rme, d + 16384 + i * 1024, 16, lane * 16, wbase + so + i * 1024, 0, AUX_SC1);
                    }
                    CBAR();
                }
                continue;
            }
            f32x4 acc[4][4];
#pragma unroll
            for (int a = 0; a < 4; ++a)
#pragma unroll
                for (int b2 = 0; b2 < 4; ++b2) acc[a][b2] = f32x4{0.f, 0.f, 0.f, 0.f};
            {
                const char* xa = smem + LDS_CTL + wm * 8192 + lane * 16;
                const char* wb = smem + LDS_CTL + 16384 + wn * 8192 + lane * 16;
                bf16x8 xA[8], wA[8], xB[8], wB[8];
                auto rd = [&](bf16x8 (&xf)[8], bf16x8 (&wf)[8], u32 slot) {
                    const u32 so = (slot & 3) * 32768;
#pragma unroll
                    for (int i = 0; i < 8; ++i) {
                        wf[i] = *(const bf16x8*)(wb + so + i * 1024);
                        xf[i] = *(const bf16x8*)(xa + so + i * 1024);
                    }
                };
                auto mm = [&](bf16x8 (&xf)[8], bf16x8 (&wf)[8]) {
#pragma unroll
                    for (int h = 0; h < 2; ++h)
#pragma unroll
                        for (int a = 0; a < 4; ++a)
#pragma unroll
                            for (int b2 = 0; b2 < 4; ++b2)
                                acc[a][b2] = __builtin_amdgcn_mfma_f32_16x16x32_bf16(wf[h * 4 + b2], xf[h * 4 + a], acc[a][b2], 0, 0, 0);
                };
                auto sched = [&]() {
#pragma unroll
                    for (int i = 0; i < 16; ++i) {
                        __builtin_amdgcn_sched_group_barrier(0x100, 1, 0);
                        __builtin_amdgcn_sched_group_barrier(0x008, 2, 0);
                    }
                };
#define BAR() do { __builtin_amdgcn_s_waitcnt(0xC07F); CBAR(); __builtin_amdgcn_s_barrier(); CBAR(); __builtin_amdgcn_sched_barrier(0); } while (0)
                BAR(); BAR(); BAR();
                BAR();
                rd(xA, wA, 0);
                for (u32 s = 0; s < 32; s += 2) {
                    BAR();
                    rd(xB, wB, s + 1);
                    mm(xA, wA);
                    sched();
                    __builtin_amdgcn_sched_barrier(0);
                    BAR();
                    rd(xA, wA, s + 2);
                    mm(xB, wB);
                    sched();
                    __builtin_amdgcn_sched_barrier(0);
                }
                WAIT_LGKM0();
            }
            STAMP(1 + 3 * pass);

            if (!keep) {
                const u32 pu = (((jt & 1) * 2 + (ct & 1)) * 32 + part_idx(jt >> 1, ct >> 1)) * 4 + wave;
                const u32 dst = OFF_SPART + pu * 8192;
#pragma unroll
                for (int a = 0; a < 4; ++a)
#pragma unroll
                    for (int u = 0; u < 2; ++u) {
                        u32x4 o;
                        o[0] = pack2(acc[a][2 * u][0], acc[a][2 * u][1]);
                        o[1] = pack2(acc[a][2 * u][2], acc[a][2 * u][3]);
                        o[2] = pack2(acc[a][2 * u + 1][0], acc[a][2 * u + 1][1]);
                        o[3] = pack2(acc[a][2 * u + 1][2], acc[a][2 * u + 1][3]);
                        st128(o, rme, lane * 16, dst + (a * 2 + u) * 1024, AUX_SYS);
                    }
                WAIT_VM(0);
                st_flag64(rme, l0, OFF_LP + pu * 8, call, 8192);
                STAMP(3 + 3 * pass);
            } else {
                const u32 pu = (j * 16 + lc) * 4 + wave;
                wait_flag64<PSLEEP>(myws, OFF_FP + pu * 8, call);
                STAMP(2 + 3 * pass);
                u32x4 o[8];
                u32 bits[2] = {0u, 0u};
                {
                    u32x4 pv[8];
#pragma unroll
                    for (int i = 0; i < 8; ++i) pv[i] = __builtin_amdgcn_raw_buffer_load_b128(rme, lane * 16, OFF_PART + pu * 8192 + i * 1024, AUX_SC1);
                    const bool bad = rfl(ld_tmo(myws)) == call;
#pragma unroll
                    for (int gi = 0; gi < 8; ++gi) {
                        float y[8];
#pragma unroll
                        for (int k = 0; k < 8; ++k) {
                            const u32 pq = pv[gi][k >> 1];
                            float t = acc[gi >> 1][(gi & 1) * 2 + (k >> 2)][k & 3] + ((k & 1) ? bf_hi(pq) : bf_lo(pq));
                            if (bad) t = __builtin_nanf("");
                            const bool nz = !(t <= 0.f);
                            y[k] = nz ? t : 0.f;
                            bits[gi >> 2] |= nz ? (1u << ((gi & 3) * 8 + k)) : 0u;
                        }
                        o[gi] = u32x4{pack2(y[0], y[1]), pack2(y[2], y[3]), pack2(y[4], y[5]), pack2(y[6], y[7])} & 0x7fff7fffu;
                    }
                }
                u32 tot;
                u32 addr = (u32)(LDS_CTL + wave * 16384) + 2 * wave_excl(__builtin_popcount(bits[0]) + __builtin_popcount(bits[1]), tot);
                const u32 H = tot >> 5, eo = ((2 * tot + 3) & ~3u) - 4 * H;
                u32* wt = (u32*)(wl + 9216);
                wt[lane] = lane < H ? bits[0] : 0u;
                wt[64 + lane] = 64 + lane < H ? bits[1] : 0u;
                wt[128 + lane] = 0u;
                *(u32*)(wl + (lane >= H ? eo + lane * 4 : 10240)) = bits[0];
                *(u32*)(wl + (64 + lane >= H ? eo + 256 + lane * 4 : 10240)) = bits[1];
#pragma unroll
                for (int gi = 0; gi < 8; ++gi)
#pragma unroll
                    for (int k = 0; k < 8; ++k) {
                        const u32 t = (bits[gi >> 2] >> ((gi & 3) * 8 + k)) & 1u;
                        const u32 d = o[gi][k >> 1];
                        *(LAS unsigned short*)((LAS char*)smem + __umul24(addr, t)) = (unsigned short)((k & 1) ? d >> 16 : d);
                        addr += 2 * t;
                    }
                WAIT_LGKM0();
                const u32 len = ((eo + 512 + 63) & ~63u) | tot << 16;
                const u32 tl = jt * 16 + ct;
                const u32 sidx = rank * 512 + ((((u32)__builtin_amdgcn_readlane((int)ordv, (int)(tl >> 2))) >> ((tl & 3) * 8)) & 127) * 4 + wave;
                const u32 so = OFF_CMP + sidx * CSTR;
#pragma unroll
                for (int i = 0; i < 9; ++i)
                    if (i * 1024u < (len & 0xffffu)) {
                        u32x4 v = *(const u32x4*)(wl + i * 1024 + lane * 16);
                        const u32 hw = wt[i * 16 + (lane >> 2)] << (12 - (lane & 3) * 4);
                        v[0] |= (hw << 3) & 0x80008000u; v[1] |= (hw << 2) & 0x80008000u;
                        v[2] |= (hw << 1) & 0x80008000u; v[3] |= hw & 0x80008000u;
                        st128(v, rme, lane * 16, so + i * 1024, AUX_SYS);
                    }
                WAIT_VM(0);
                st_flag64(rme, l0, OFF_FC + sidx * 8, call, len);
                STAMP(3 + 3 * pass);
                const u32 obase = (glob_blk(g, bi, lr) * 128 + wm * 64) * (NN * 2) + (glob_blk(g, bj, lc) * 128 + wn * 64) * 2;
#pragma unroll
                for (int gi = 0; gi < 8; ++gi)
                    st_out(o[gi], ro, olane, obase + (gi >> 1) * 16 * (NN * 2) + (gi & 1) * 64);
                WAIT_LGKM0();
            }
        }
    }

    if (wave >= 8 - XWAVES || blockIdx.x >= NPWG + NCWG) {
        const u32 xi = blockIdx.x < NPWG + NCWG ? (blockIdx.x - NPWG) * XWAVES + wave - (8 - XWAVES)
                                                : NCWG * XWAVES + (blockIdx.x - NPWG - NCWG) * 8 + wave;
        const u32 pi = xi % 7;
        const u32 tbase = (call - 1) * (512 + (NXW - pi + 6) / 7);
        const u32 p = (rank + 1 + pi) & 7;
        STAMP(7);
        u32 tv = ticket(rme, l0, 8 + pi);
        for (;;) {
            const u32 n3 = rfl(tv) - tbase;
            if (n3 >= 512) break;
            const u32 sidx = p * 512 + n3;
            const u32 lw = wait_flag64<XSLEEP>(myws, OFF_FC + sidx * 8, call);
            const u32 len = lw & 0xffffu, nv = (lw >> 16) > 4096 ? 4096 : lw >> 16;
            STAMP(8);
            const u32 so = OFF_CMP + sidx * CSTR;
#pragma unroll
            for (int i = 0; i < 9; ++i)
                if (i * 1024u < len)
                    __builtin_amdgcn_raw_ptr_buffer_load_lds(rme, (LAS char*)wl + i * 1024, 16, lane * 16, so + i * 1024, 0, AUX_SC1);
            const u32 tm = ld_tmo(myws);
            tv = ticket(rme, l0, 8 + pi);
            const u32 to = n3 >> 2, sw = n3 & 3;
            const u32 tl = ((u32)__builtin_amdgcn_readlane((int)ordv, (int)(32 + (to >> 2)))) >> ((to & 3) * 8);
            const u32 tj = perm8((tl >> 4) & 7), tc = perm16(tl & 15);
            const u32 gp = p >> 2;
            const u32 tRB = glob_blk(gp, (p >> 1) & 1, row_lr(tj, gp)), tCB = glob_blk(gp, p & 1, tc);
            const u32 obase = (tRB * 128 + (sw >> 1) * 64) * (NN * 2) + (tCB * 128 + (sw & 1) * 64) * 2;
            WAIT_VM0();
            const bool bad = rfl(tm) == call;
            const u32 H = nv >> 5, eo = ((2 * nv + 3) & ~3u) - 4 * H;
            u32 bits[2];
#pragma unroll
            for (int h = 0; h < 2; ++h) {
                bits[h] = *(const u32*)(wl + eo + (h * 64 + lane) * 4);
                if (H > h * 64) {
                    u32 hw = 0;
#pragma unroll
                    for (int i = 0; i < 4; ++i) {
                        const u32x4 v = *(const u32x4*)(wl + h * 4096 + lane * 64 + i * 16);
#pragma unroll
                        for (int k = 0; k < 4; ++k) hw |= (v[k] >> (15 - i * 4 - k)) & (0x00010001u << (i * 4 + k));
                    }
                    bits[h] = h * 64 + lane < H ? hw : bits[h];
                }
            }
            u32 tot;
            u32 addr = (u32)(LDS_CTL + wave * 16384) + 2 * wave_excl(__builtin_popcount(bits[0]) + __builtin_popcount(bits[1]), tot);
#pragma unroll
            for (int gi = 0; gi < 8; ++gi) {
                u32 e[8];
#pragma unroll
                for (int k = 0; k < 8; ++k) {
                    const u32 t = (bits[gi >> 2] >> ((gi & 3) * 8 + k)) & 1u;
                    e[k] = *(const LAS unsigned short*)((LAS char*)smem + __umul24(addr, t));
                    addr += 2 * t;
                }
                u32x4 r = u32x4{e[0] | (e[1] << 16), e[2] | (e[3] << 16), e[4] | (e[5] << 16), e[6] | (e[7] << 16)} & 0x7fff7fffu;
                if (bad) r = u32x4{0x7fc07fc0u, 0x7fc07fc0u, 0x7fc07fc0u, 0x7fc07fc0u};
                st_out(r, ro, olane, obase + (gi >> 1) * 16 * (NN * 2) + (gi & 1) * 64);
            }
            WAIT_LGKM0();
            STAMP(9);
        }
    }
}

static void build_orders(Params& P) {
    const char* eo = ENTRY_ORDER;
    double t[3][16];
    for (int i = 0; i < 4; ++i) P.entry[i] = 0;
    for (int e = 0; e < 32; ++e) {
        const int kind = eo[2 * e] == 'w' ? 0 : eo[2 * e] == 's' ? 1 : 2;
        const char ch = eo[2 * e + 1];
        const int idx = ch <= '9' ? ch - '0' : ch - 'A' + 10;
        P.entry[e >> 3] |= (u64)(kind << 4 | idx) << ((e & 7) * 8);
        t[kind][idx] = (e + 1) * 1.445;
    }
    auto mx = [](double a, double b) { return a > b ? a : b; };
    double av[128];
    unsigned char* ord = (unsigned char*)P.ord;
    for (int i = 0; i < 128; ++i) {
        const int jt = i >> 4, ct = i & 15;
        av[i] = mx(mx(t[2][jt], t[0][ct]) + LAT_K, mx(mx(t[1][jt & ~1], t[1][jt | 1]), mx(t[0][ct & ~1], t[0][ct | 1])) + LAT_S);
    }
    for (int i = 0; i < 128; ++i) {
        int pos = 0;
        for (int k = 0; k < 128; ++k) pos += av[k] < av[i] || (av[k] == av[i] && k < i);
        ord[i] = (unsigned char)pos;
        ord[128 + pos] = (unsigned char)i;
    }
    const int W = NCWG / 8, D = 32 - W;
    for (int x = 0; x < 8; ++x) {
        int id[32]; double key[32];
        for (int i = 0; i < 32; ++i) {
            const int keep = i >> 4, jt = (x >> 1) + 4 * ((i >> 3) & 1), ct = (i & 7) * 2 + (x & 1);
            id[i] = keep << 7 | jt << 4 | ct;
            key[i] = (keep ? av[jt * 16 + ct] - LAT_K : mx(t[1][jt], t[0][ct]));
        }
        int sorted[32];
        for (int i = 0; i < 32; ++i) {
            int pos = 0;
            for (int k = 0; k < 32; ++k) pos += key[k] < key[i] || (key[k] == key[i] && k < i);
            sorted[pos] = id[i];
        }
        for (int i = 0; i < W; ++i) P.tiles[i * 8 + x] = (unsigned short)(sorted[i] | sorted[i < D ? W + i : i] << 8);
    }
}

extern "C" void kernel_launch(int rank, int nranks, ncclComm_t comm,
                              hipStream_t stream,
                              void* const* d_in, const int* in_sizes, int n_in,
                              void* d_out, int out_size,
                              const int* in_dtypes, int out_dtype,
                              void* const* d_out_all,
                              void* const* d_ws_all, size_t ws_bytes) {
    if (nranks != NR || n_in < 2 || ws_bytes < WS_USED || (unsigned)rank >= (unsigned)NR ||
        in_sizes[0] != MM * KS || in_sizes[1] != KS * NN || out_size != MM * NN) {
        fprintf(stderr, "unexpected problem shape\n");
        return;
    }
    Params P;
    P.x = d_in[0];
    P.w = d_in[1];
    for (int i = 0; i < NR; ++i) P.ws[i] = d_ws_all[i];
    P.out = d_out;
    P.rank = rank;
    build_orders(P);
    HIP_CHECK(hipFuncSetAttribute((const void*)fused_gemm_ar, hipFuncAttributeMaxDynamicSharedMemorySize, LDS_BYTES));
    hipLaunchKernelGGL(fused_gemm_ar, dim3(GRID), dim3(512), LDS_BYTES, stream, P);
    HIP_CHECK(hipGetLastError());
}
```

```cpp
#include <hip/hip_runtime.h>
#include <rccl/rccl.h>
#include <cstdio>
#include <cstdint>

#define HIP_CHECK(expr) do { hipError_t _e = (expr); if (_e != hipSuccess) { \
    fprintf(stderr, "HIP error %s at %s:%d\n", hipGetErrorString(_e), __FILE__, __LINE__); \
    return; } } while (0)

#define NFET 3
#define NPF 2
#define WIN 7
#define EWG 8
#define XWAVES 4
#define DIAG 0
#define LAT_K 12.0
#define LAT_S 19.0
#define ORD 1
#define ENTRY_ORDER "w0s0w1s1w2s2w3s3k0w4w5k1s4k2s5k3w6w7k4k5s6s7w8w9wAwBk6k7wCwDwEwF"
#define OUT_AUX 17
#define XSPLIT 32
#define XSLEEP 4
#define PSLEEP 8

typedef unsigned int u32;
typedef unsigned long long u64;
typedef u32 u32x4 __attribute__((ext_vector_type(4)));
typedef u32 u32x2 __attribute__((ext_vector_type(2)));
typedef float f32x4 __attribute__((ext_vector_type(4)));
typedef short bf16x8 __attribute__((ext_vector_type(8)));
typedef __bf16 bf16v2 __attribute__((ext_vector_type(2)));
typedef float f32x2 __attribute__((ext_vector_type(2)));
#define LAS __attribute__((address_space(3)))

constexpr int NR = 8;
constexpr int MM = 4096, NN = 4096, KS = 512;
constexpr u32 BLK = 16384;
constexpr u32 OFF_XA = 0;
constexpr u32 OFF_WB = 8u << 20;
constexpr u32 OFF_PART = 16u << 20;
constexpr u32 OFF_SPART = 20u << 20;
constexpr u32 CSTR = 9216;
constexpr u32 CMAX = 8704;
constexpr u32 OFF_CMP = 24u << 20;
constexpr u32 OFF_FLAGS = 60u << 20;
constexpr u32 OFF_FX = OFF_FLAGS;
constexpr u32 OFF_FW = OFF_FX + 4096;
constexpr u32 OFF_FP = OFF_FW + 4096;
constexpr u32 OFF_LP = OFF_FP + 4096;
constexpr u32 OFF_FC = OFF_LP + 4096;
constexpr u32 OFF_CTR = OFF_FC + 36864;
constexpr u32 OFF_EP = OFF_CTR + 4096;
constexpr u32 OFF_TMO = OFF_EP + 8192;
constexpr u32 OFF_DIAG = OFF_TMO + 4096;
constexpr u32 WS_USED = OFF_DIAG + (1u << 20);
constexpr u32 OUT_BYTES = (u32)MM * NN * 2;
constexpr u32 IN_BYTES = (u32)MM * KS * 2;

constexpr int NPWG = 7 * EWG;
constexpr int NWL = EWG * NFET;
constexpr int NSL = NFET + NPF;
constexpr int NCWG = EWG == 8 ? 192 : 128;
constexpr int GRID = 256;
constexpr u32 NXW = NCWG * XWAVES + (GRID - NPWG - NCWG) * 8;
constexpr u32 LDS_CTL = DIAG ? 1024 : 512;
constexpr int LDS_BYTES = LDS_CTL + 4 * 32768;
constexpr u64 DEADLINE = 100000000ull;
constexpr int AUX_SYS = 17;
constexpr int AUX_SC1 = 16;
static_assert(GRID * 8 * 4 <= 8192, "epoch words");
static_assert(OFF_CMP + 8u * 512u * CSTR <= OFF_FLAGS, "layout");

struct Orders {
    u64 entry[4];
    unsigned short tiles[NCWG];
    u32 ord[64];
};
struct Params {
    const void* x;
    const void* w;
    void* ws[NR];
    void* out;
    int rank;
    Orders o;
};

typedef __amdgpu_buffer_rsrc_t rsrc_t;
__device__ __forceinline__ rsrc_t mk_rsrc(const void* p, u32 bytes) {
    return __builtin_amdgcn_make_buffer_rsrc((void*)p, (short)0, (int)bytes, 0x00020000);
}
__device__ __forceinline__ u32 rfl(u32 v) { return (u32)__builtin_amdgcn_readfirstlane((int)v); }
__device__ __forceinline__ void* pick(void* const (&t)[NR], u32 idx) {
    void* r = t[0];
#pragma unroll
    for (u32 i = 1; i < NR; ++i) r = (idx == i) ? t[i] : r;
    return r;
}
__device__ __forceinline__ u32 glob_blk(u32 g, u32 b, u32 l) {
    return g == 0 ? b * 16 + l : ((l >> 3) * 2 + b) * 8 + (l & 7);
}
__device__ __forceinline__ u32 perm8(u32 t) { return (t >> 1) + (t & 1) * 4; }
__device__ __forceinline__ u32 perm16(u32 t) { return (t >> 1) + (t & 1) * 8; }
__device__ __forceinline__ u32 row_lr(u32 j, u32 fin) { return (j >> 2) * 8 + fin * 4 + (j & 3); }

__device__ __forceinline__ u32 part_idx(u32 a, u32 b) { return b >= 4 ? b * 4 + a : (b > a ? b * b + a : a * a + a + b); }
__device__ __forceinline__ void part_ab(u32 m, u32& a, u32& b) {
    const u32 sh = m >= 9 ? 3 : m >= 4 ? 2 : m >= 1 ? 1 : 0, o = m - sh * sh;
    a = o < sh ? o : sh;
    b = o < sh ? sh : o - sh;
    if (m >= 16) { a = m & 3; b = m >> 2; }
}
template <bool READ_BACK>
__device__ __forceinline__ void fetch_unit(const void* x, const void* w, u32 su, u32 lane, char* wl, u32x4* v) {
    const u32 isw = su >> 9, GB = (su >> 4) & 31, kc = (su >> 1) & 7, hf = su & 1;
    if (isw) {
        const rsrc_t rw = mk_rsrc(w, IN_BYTES);
        const u32 vo = (lane >> 3) * 65536 + (lane & 7) * 16, so = kc * 64 * 8192 + GB * 256 + hf * 128;
#pragma unroll
        for (int i = 0; i < 8; ++i) v[i] = __builtin_amdgcn_raw_buffer_load_b128(rw, vo, so + i * 8192, 0);
        const u32 wo = (lane >> 5) * 4096 + ((lane & 7) >> 2) * 2048 + (((lane >> 3) & 3) * 16 + (lane & 3) * 4) * 16;
#pragma unroll
        for (int c = 0; c < 8; ++c) {
            u32x4 o;
#pragma unroll
            for (int t = 0; t < 4; ++t)
                o[t] = __builtin_amdgcn_perm(v[2 * t + 1][c >> 1], v[2 * t][c >> 1], (c & 1) ? 0x07060302u : 0x05040100u);
            *(u32x4*)(wl + wo + (c >> 2) * 1024 + (c & 3) * 16) = o;
        }
    } else {
        const rsrc_t rx = mk_rsrc(x, IN_BYTES);
        const u32 vo = (lane >> 3) * 1024 + (lane & 7) * 16, so = GB * 131072 + hf * 65536 + kc * 128;
#pragma unroll
        for (int i = 0; i < 8; ++i) v[i] = __builtin_amdgcn_raw_buffer_load_b128(rx, vo, so + i * 8192, 0);
        const u32 wo = ((lane & 7) >> 2) * 4096 + ((lane & 3) * 16 + (lane >> 3)) * 16;
#pragma unroll
        for (int i = 0; i < 8; ++i) *(u32x4*)(wl + wo + (i >> 1) * 1024 + (i & 1) * 128) = v[i];
    }
    asm volatile("s_waitcnt lgkmcnt(0)" ::: "memory");
    if (READ_BACK) {
#pragma unroll
        for (int i = 0; i < 8; ++i) v[i] = *(const u32x4*)(wl + i * 1024 + lane * 16);
        asm volatile("s_waitcnt lgkmcnt(0)" ::: "memory");
    }
}
__device__ __forceinline__ void st128(u32x4 v, rsrc_t r, u32 voff, u32 soff, int) {
    __builtin_amdgcn_raw_buffer_store_b128(v, r, voff + soff, 0, AUX_SYS);
}
__device__ __forceinline__ void st_out(u32x4 v, rsrc_t r, u32 voff, u32 soff) {
    __builtin_amdgcn_raw_buffer_store_b128(v, r, voff + soff, 0, OUT_AUX);
}
__device__ __forceinline__ void st_flag(rsrc_t r, u32 l0, u32 off, u32 v) {
    __builtin_amdgcn_raw_buffer_store_b32(v, r, l0, off, AUX_SYS);
}
__device__ __forceinline__ void st_flag64(rsrc_t r, u32 l0, u32 off, u32 call, u32 len) {
    __builtin_amdgcn_raw_buffer_store_b64(u32x2{len, call}, r, l0, off, AUX_SYS);
}
__device__ __forceinline__ u64 ld_flag64(const void* ws, u32 off) {
    return __hip_atomic_load((const u64*)((const char*)ws + off), __ATOMIC_RELAXED, __HIP_MEMORY_SCOPE_SYSTEM);
}
#define CBAR() asm volatile("" ::: "memory")
#define WAIT_VM(n) asm volatile("s_waitcnt vmcnt(" #n ")" ::: "memory")
#define WAIT_VM0() do { WAIT_VM(0); __builtin_amdgcn_s_waitcnt(0x0F70); CBAR(); } while (0)
#define WAIT_LGKM0() asm volatile("s_waitcnt lgkmcnt(0)" ::: "memory")

__device__ __forceinline__ bool give_up(void* myws, u32 call, u64 t0) {
    u32* tmo = (u32*)((char*)myws + OFF_TMO);
    bool late = __builtin_amdgcn_s_memrealtime() - t0 > DEADLINE;
    if (late || rfl(__hip_atomic_load(tmo, __ATOMIC_RELAXED, __HIP_MEMORY_SCOPE_AGENT)) == call) {
        __hip_atomic_store(tmo, call, __ATOMIC_RELAXED, __HIP_MEMORY_SCOPE_AGENT);
        return true;
    }
    return false;
}
__device__ __forceinline__ u32 ld_tmo(void* myws) {
    return __hip_atomic_load((u32*)((char*)myws + OFF_TMO), __ATOMIC_RELAXED, __HIP_MEMORY_SCOPE_AGENT);
}
template <int SLP>
__device__ __forceinline__ u32 wait_flag64(void* myws, u32 off, u32 call) {
    u64 t0 = 0;
    for (u32 it = 0;; ++it) {
        u64 v = ld_flag64(myws, off);
        if (rfl((u32)(v >> 32)) >= call) return rfl((u32)v);
        if (it == 0) t0 = __builtin_amdgcn_s_memrealtime();
        if ((it & 63) == 63 && give_up(myws, call, t0)) return 0;
        __builtin_amdgcn_s_sleep(SLP);
    }
}
__device__ __forceinline__ u32 ticket(rsrc_t r, u32 l0, u32 idx) {
    return (u32)__builtin_amdgcn_raw_ptr_buffer_atomic_add_i32(1, r, l0, OFF_CTR + idx * 128, 0);
}
__device__ __forceinline__ u32 mbcnt64(u64 m) {
    return __builtin_amdgcn_mbcnt_hi((u32)(m >> 32), __builtin_amdgcn_mbcnt_lo((u32)m, 0u));
}
__device__ __forceinline__ u32 wave_excl(u32 cnt, u32& tot) {
    u32 pre = 0; tot = 0;
#pragma unroll
    for (int b = 0; b < 7; ++b) {
        const u64 m = __ballot((cnt >> b) & 1);
        pre += mbcnt64(m) << b;
        tot += (u32)__builtin_popcountll(m) << b;
    }
    return pre;
}
__device__ __forceinline__ u32 pack2(float a, float b) {
    f32x2 v = {a, b};
    return __builtin_bit_cast(u32, __builtin_convertvector(v, bf16v2));
}
__device__ __forceinline__ float bf_lo(u32 v) { return __builtin_bit_cast(float, v << 16); }
__device__ __forceinline__ float bf_hi(u32 v) { return __builtin_bit_cast(float, v & 0xffff0000u); }

#define STAMP(slot) do {} while (0)

__global__ void __launch_bounds__(512, 2) fused_gemm_ar(Params P) {
    extern __shared__ __attribute__((aligned(16))) char smem[];
    const u32 lane = threadIdx.x & 63;
    const u32 wave = rfl(threadIdx.x >> 6);
    const u32 rank = (u32)P.rank & 7;
    const u32 g = rank >> 2, bi = (rank >> 1) & 1, bj = rank & 1;
    void* myws = pick(P.ws, rank);
    const rsrc_t rme = mk_rsrc(myws, WS_USED);
    char* wl = smem + LDS_CTL + wave * 16384;

    const u32 l0 = lane == 0 ? 0u : 0x80000000u;
    const u32 epo = OFF_EP + (blockIdx.x * 8 + wave) * 4;
    const u32 epv = __builtin_amdgcn_raw_buffer_load_b32(rme, 0, epo, AUX_SC1);
    *(LAS u32*)smem = 0u;
    STAMP(0);

    if (blockIdx.x < NPWG) {
        const u32 b = blockIdx.x;
        const u32 di = 1 + ((b >> 3) + (b & 7)) % 7;
        const u32 q = (rank + di) & 7;
        volatile LAS u32* ctl = (volatile LAS u32*)(smem + 32);
        if (wave == 0 && lane < 8) ctl[lane * 8] = 0;
        WAIT_LGKM0();
        __builtin_amdgcn_s_barrier();
        CBAR();
        STAMP(5);
        if (wave > NSL) { st_flag(rme, l0, epo, epv + 1); return; }
        if (wave == 0) {
            const rsrc_t rq = mk_rsrc(pick(P.ws, q), WS_USED);
            const u32 call = rfl(epv) + 1;
            st_flag(rme, l0, epo, call);
            WAIT_VM0();
            u32 pk = 3, pfo = 0, pla = 0, plb = 0;
            auto flush = [&]() {
                if (pk == 0) st_flag(rq, l0, pfo, call);
                else __builtin_amdgcn_raw_buffer_store_b64(u32x2{lane == 0 ? pla : plb, call}, rq, lane < pk ? lane * 8 : 0x80000000u, pfo, AUX_SYS);
                pk = 3;
            };
            u32 rr = 0;
            u64 t0 = __builtin_amdgcn_s_memrealtime();
            for (u32 it = 0;; ++it) {
                const u32 stv = ctl[(lane & 7) * 8];
                const u32 rdy = (u32)__ballot(stv == 1) & ((1u << NSL) - 1);
                if (rdy == 0) {
                    if (pk != 3) { WAIT_VM(0); flush(); }
                    if (((u32)__ballot(stv == 2) & ((1u << NSL) - 1)) == (1u << NSL) - 1) break;
                    if ((it & 1023) == 1023 && __builtin_amdgcn_s_memrealtime() - t0 > 2 * DEADLINE) break;
                    __builtin_amdgcn_s_sleep(1);
                    continue;
                }
                u32 sl;
                if (rdy >> NFET) sl = NFET + (u32)__builtin_ctz(rdy >> NFET);
                else {
                    sl = rr + (u32)__builtin_ctz((rdy | rdy << NFET) >> rr);
                    sl = sl >= NFET ? sl - NFET : sl;
                    rr = sl + 1 >= NFET ? 0 : sl + 1;
                }
                const u32 dv = ctl[sl * 8 + (lane & 7)];
                const u32 dA = (u32)__builtin_amdgcn_readlane((int)dv, 1), dB = (u32)__builtin_amdgcn_readlane((int)dv, 2);
                const u32 wA = (u32)__builtin_amdgcn_readlane((int)dv, 3), wB = (u32)__builtin_amdgcn_readlane((int)dv, 4);
                const u32 lA = wA & 0xffffu, lB = wB & 0xffffu;
                const u32 lds_b = (u32)__builtin_amdgcn_readlane((int)dv, 5);
                const u32 fo = (u32)__builtin_amdgcn_readlane((int)dv, 6), kind = (u32)__builtin_amdgcn_readlane((int)dv, 7);
                const char* sp = smem + LDS_CTL + sl * 18432 + lane * 16;
                const u32 nA = (lA + 1023) >> 10, n = nA + ((lB + 1023) >> 10);
                u32x4 cur = *(const u32x4*)(sp + (nA ? 0 : lds_b));
                for (u32 j = 0; j < n; ++j) {
                    const u32 jn = j + 1;
                    const u32x4 nxt = *(const u32x4*)(sp + (jn < nA ? jn * 1024 : lds_b + (jn - nA) * 1024));
                    const u32 o = j < nA ? j * 1024 : (j - nA) * 1024;
                    asm volatile("s_waitcnt vmcnt(%0)" :: "n"(WIN - 1) : "memory");
                    if (j == WIN - 1 && pk != 3) flush();
                    st128(cur, rq, (o + lane * 16 < (j < nA ? lA : lB)) ? lane * 16 : 0x80000000u, (j < nA ? dA : dB) + o, AUX_SYS);
                    cur = nxt;
                }
                WAIT_LGKM0();
                ctl[sl * 8] = 0;
                if (pk != 3) { WAIT_VM(0); flush(); }
                pk = kind; pfo = fo; pla = wA; plb = wB;
            }
            STAMP(3);
            return;
        }
        const u32 sl = wave - 1;
        const u32 gq = q >> 2, qi = (q >> 1) & 1, qj = q & 1;
        const u32 nk2 = (gq == g) ? 10 : 6;
        const u32 kh0 = (gq == g) ? 0 : 10;
        const u32 hs0 = (gq == g) ? (rank & 3) * 10 : 40 + (rank & 3) * 6;
        const u32 T1 = 32 * nk2;
        const u32 T = T1 + 256;
        const u32 lco = OFF_FC + rank * 4096 - T1 * 16;
        const u32 lpo = OFF_LP + (q & 3) * 1024;
        char* sp = smem + LDS_CTL + sl * 18432;
        const bool isp = sl >= NFET;
        const u32 wgi = (b & 7) + (b >= 56 ? 8 : 0);
        u32 n0 = isp ? 1024 + (sl - NFET) * EWG + wgi : sl * EWG + wgi;
        const u32 nend = isp ? ((gq == g) ? 0 : 1024 + 128) : T;
        u32 lA = 4096, lB = 4096, call = 0, tbase = 0;
        if (isp) {
            call = rfl(epv) + 1;
            st_flag(rme, l0, epo, call);
        }
        for (;;) {
            const bool last = n0 >= nend;
            bool ready = last || n0 < T1;
            const u32 o2 = isp ? lpo + ((n0 - 1024) & 127) * 8 : lco + (n0 < T ? n0 : T) * 16 + (lane & 1) * 8;
            u64 t0 = 0;
            bool dead = false;
            for (u32 it = 0;; ++it) {
                if (!ready) {
                    const u32x2 f2 = __builtin_amdgcn_raw_buffer_load_b64(rme, o2, 0, AUX_SYS);
                    ready = rfl(f2[1]) >= call && (u32)__builtin_amdgcn_readlane((int)f2[1], 1) >= call;
                    if (!isp) { lA = rfl(f2[0]); lB = (u32)__builtin_amdgcn_readlane((int)f2[0], 1); }
                }
                if (ready && rfl(ctl[sl * 8]) == 0) break;
                if (it == 0) t0 = __builtin_amdgcn_s_memrealtime();
                if ((it & 63) == 63 && give_up(myws, call, t0)) {
                    if (!ready) { ready = true; if (!isp) lA = lB = 0; }
                    if (__builtin_amdgcn_s_memrealtime() - t0 > 2 * DEADLINE) { dead = true; break; }
                }
                if (ready) __builtin_amdgcn_s_sleep(1); else __builtin_amdgcn_s_sleep(PSLEEP);
            }
            CBAR();
            if (last || dead) break;
            if ((lA & 0xffffu) > CMAX) lA = 0;
            if ((lB & 0xffffu) > CMAX) lB = 0;
            const u32 t2v = ticket(rme, isp ? 0x80000000u : l0, di);
            u32 so, dA, fo, kind;
            if (n0 < T1) {
                const u32 e = (nk2 == 10) ? n0 / 10 : n0 / 6;
                const u32 kk = n0 - e * nk2;
                const u32 eb = (u32)(P.o.entry[(e >> 3) & 3] >> ((e & 7) * 8));
                const u32 ek = (eb >> 4) & 3, idx = eb & 15;
                const bool isw = ek == 0;
                const u32 l = isw ? perm16(idx) : row_lr(perm8(idx & 7), ek == 1 ? 1 - gq : gq);
                const u32 su = (isw ? 512 : 0) + glob_blk(gq, isw ? qj : qi, l) * 16 + kh0 + kk;
                const u32 du = (isw ? 1024 : 0) + l * 64 + hs0 + kk;
                so = 0; dA = OFF_XA + du * 8192; fo = OFF_FX + du * 4; kind = 0;
                u32x4 v[8];
                fetch_unit<false>(P.x, P.w, su, lane, sp, v);
            } else if (n0 < 1024) {
                so = dA = OFF_CMP + (rank * 512 + (n0 - T1) * 2) * CSTR; fo = lco + n0 * 16; kind = 2;
            } else {
                u32 a, bb; part_ab((n0 - 1024) >> 2, a, bb);
                const u32 pu = ((bi * 4 + a) * 16 + bj * 8 + bb) * 4 + (n0 & 3);
                so = OFF_SPART + ((q & 3) * 128 + n0 - 1024) * 8192; dA = OFF_PART + pu * 8192; fo = OFF_FP + pu * 8; kind = 1;
            }
            const u32 ob = kind == 2 ? CSTR : 4096;
            if (kind != 0) {
                const u32 nA = ((lA & 0xffffu) + 1023) >> 10, n = nA + (((lB & 0xffffu) + 1023) >> 10);
#pragma unroll 1
                for (u32 j = 0; j < n; ++j) {
                    const u32 o = j < nA ? j * 1024 : ob + (j - nA) * 1024;
                    __builtin_amdgcn_raw_ptr_buffer_load_lds(rme, (LAS char*)sp + o, 16, lane * 16, so + o, 0, AUX_SC1);
                }
            }
            *(volatile LAS u32x4*)(ctl + sl * 8) = u32x4{0u, dA, dA + ob, lA};
            *(volatile LAS u32x4*)(ctl + sl * 8 + 4) = u32x4{lB, ob, fo, kind};
            WAIT_VM0();
            ctl[sl * 8] = 1;
            CBAR();
            if (call == 0) {
                STAMP(6);
                call = rfl(epv) + 1;
                st_flag(rme, l0, epo, call);
                tbase = (call - 1) * T - NWL;
            }
            n0 = isp ? n0 + NPF * EWG : rfl(t2v) - tbase;
        }
        ctl[sl * 8] = 2;
        return;
    }

    const u32 ordv = P.o.ord[lane];
    {
        const u32 u = (blockIdx.x - NPWG) + (GRID - NPWG) * wave;
        if (u < 320) {
            const u32 isw = u >= 160, r = u - isw * 160, loc = r / 10, kk = r - loc * 10;
            const u32 du = isw * 1024 + loc * 64 + (rank & 3) * 10 + kk;
            u32x4 v[8];
            fetch_unit<true>(P.x, P.w, isw * 512 + glob_blk(g, isw ? bj : bi, loc) * 16 + kk, lane, wl, v);
#pragma unroll
            for (int i = 0; i < 8; ++i) st128(v[i], rme, lane * 16, OFF_XA + du * 8192 + i * 1024, AUX_SYS);
            WAIT_VM0();
            st_flag(rme, l0, OFF_FX + du * 4, rfl(epv) + 1);
        }
        STAMP(10);
    }

    const u32 call = rfl(epv) + 1;
    const u32 wm = (wave >> 1) & 1, wn = wave & 1;
    const u32 olane = (lane & 15) * (NN * 2) + (lane >> 4) * 16;
    const rsrc_t ro = mk_rsrc(P.out, OUT_BYTES);

    st_flag(rme, l0, epo, call);
    if (blockIdx.x < NPWG + NCWG) {
        const u32 c = blockIdx.x - NPWG;
        const u32 tl2 = P.o.tiles[c];
        volatile LAS u32* sel = (volatile LAS u32*)(smem + 16);

        for (u32 pass = 0; pass < ((tl2 >> 8) == (tl2 & 255) ? 1u : 2u); ++pass) {
            const u32 tb = tl2 >> (pass * 8);
            const u32 keep = (tb >> 7) & 1, jt = (tb >> 4) & 7, ct = tb & 15;
            const u32 j = perm8(jt);
            const u32 lc = perm16(ct);
            const u32 lr = row_lr(j, keep ? g : 1 - g);

            if (wave >= 4) {
                const u32 lw = wave - 4;
                const u32 xbase = OFF_XA + lr * 32 * BLK + lw * 4096;
                const u32 wbase = OFF_WB + lc * 32 * BLK + lw * 4096;
                const u32 fo = (lane < 32) ? OFF_FX + (lr * 32 + lane) * 8 : OFF_FW + (lc * 32 + lane - 32) * 8;
                u32 pend = 0, taken = 0;
                for (int s = -4; s < 32; ++s) {
                    const bool refill = pend == 0 && s < 28;
                    if (refill && lw == 0) {
                        u64 t0 = 0;
                        for (u32 it = 0;; ++it) {
                            const u64 fv = ld_flag64(myws, fo);
                            const u64 m = __ballot((u32)fv >= call && (u32)(fv >> 32) >= call);
                            pend = (u32)m & (u32)(m >> 32) & ~taken;
                            if (pend) break;
                            if (it == 0) t0 = __builtin_amdgcn_s_memrealtime();
                            if ((it & 63) == 63 && give_up(myws, call, t0)) { pend = ~taken; break; }
                            __builtin_amdgcn_s_sleep(PSLEEP);
                        }
                        taken |= pend;
                        sel[s & 1] = pend;
                    }
                    if (s < 29) WAIT_VM(16); else if (s == 29) WAIT_VM(8); else WAIT_VM(0);
                    WAIT_LGKM0();
                    __builtin_amdgcn_s_barrier();
                    CBAR();
                    if (s < 28) {
                        if (refill) pend = rfl(sel[s & 1]);
                        const u32 so = ((u32)__builtin_ctz(pend | 0x80000000u)) * BLK;
                        pend &= pend - 1;
                        LAS char* d = (LAS char*)smem + LDS_CTL + (s & 3) * 32768 + lw * 4096;
#pragma unroll
                        for (int i = 0; i < 4; ++i)
                            __builtin_amdgcn_raw_ptr_buffer_load_lds(rme, d + i * 1024, 16, lane * 16, xbase + so + i * 1024, 0, AUX_SC1);
#pragma unroll
                        for (int i = 0; i < 4; ++i)
                            __builtin_amdgcn_raw_ptr_buffer_load_lds(rme, d + 16384 + i * 1024, 16, lane * 16, wbase + so + i * 1024, 0, AUX_SC1);
                    }
                    CBAR();
                }
                continue;
            }
            f32x4 acc[4][4];
#pragma unroll
            for (int a = 0; a < 4; ++a)
#pragma unroll
                for (int b2 = 0; b2 < 4; ++b2) acc[a][b2] = f32x4{0.f, 0.f, 0.f, 0.f};
            {
                const char* xa = smem + LDS_CTL + wm * 8192 + lane * 16;
                const char* wb = smem + LDS_CTL + 16384 + wn * 8192 + lane * 16;
                bf16x8 xA[8], wA[8], xB[8], wB[8];
                auto rd = [&](bf16x8 (&xf)[8], bf16x8 (&wf)[8], u32 slot) {
                    const u32 so = (slot & 3) * 32768;
#pragma unroll
                    for (int i = 0; i < 8; ++i) {
                        wf[i] = *(const bf16x8*)(wb + so + i * 1024);
                        xf[i] = *(const bf16x8*)(xa + so + i * 1024);
                    }
                };
                auto mm = [&](bf16x8 (&xf)[8], bf16x8 (&wf)[8]) {
#pragma unroll
                    for (int h = 0; h < 2; ++h)
#pragma unroll
                        for (int a = 0; a < 4; ++a)
#pragma unroll
                            for (int b2 = 0; b2 < 4; ++b2)
                                acc[a][b2] = __builtin_amdgcn_mfma_f32_16x16x32_bf16(wf[h * 4 + b2], xf[h * 4 + a], acc[a][b2], 0, 0, 0);
                };
                auto sched = [&]() {
#pragma unroll
                    for (int i = 0; i < 16; ++i) {
                        __builtin_amdgcn_sched_group_barrier(0x100, 1, 0);
                        __builtin_amdgcn_sched_group_barrier(0x008, 2, 0);
                    }
                };
#define BAR() do { __builtin_amdgcn_s_waitcnt(0xC07F); CBAR(); __builtin_amdgcn_s_barrier(); CBAR(); __builtin_amdgcn_sched_barrier(0); } while (0)
                BAR(); BAR(); BAR();
                BAR();
                rd(xA, wA, 0);
                for (u32 s = 0; s < 32; s += 2) {
                    BAR();
                    rd(xB, wB, s + 1);
                    mm(xA, wA);
                    sched();
                    __builtin_amdgcn_sched_barrier(0);
                    BAR();
                    rd(xA, wA, s + 2);
                    mm(xB, wB);
                    sched();
                    __builtin_amdgcn_sched_barrier(0);
                }
                WAIT_LGKM0();
            }
            STAMP(1 + 3 * pass);

            if (!keep) {
                const u32 pu = (((jt & 1) * 2 + (ct & 1)) * 32 + part_idx(jt >> 1, ct >> 1)) * 4 + wave;
                const u32 dst = OFF_SPART + pu * 8192;
#pragma unroll
                for (int a = 0; a < 4; ++a)
#pragma unroll
                    for (int u = 0; u < 2; ++u) {
                        u32x4 o;
                        o[0] = pack2(acc[a][2 * u][0], acc[a][2 * u][1]);
                        o[1] = pack2(acc[a][2 * u][2], acc[a][2 * u][3]);
                        o[2] = pack2(acc[a][2 * u + 1][0], acc[a][2 * u + 1][1]);
                        o[3] = pack2(acc[a][2 * u + 1][2], acc[a][2 * u + 1][3]);
                        st128(o, rme, lane * 16, dst + (a * 2 + u) * 1024, AUX_SYS);
                    }
                WAIT_VM(0);
                st_flag64(rme, l0, OFF_LP + pu * 8, call, 8192);
                STAMP(3 + 3 * pass);
            } else {
                const u32 pu = (j * 16 + lc) * 4 + wave;
                wait_flag64<PSLEEP>(myws, OFF_FP + pu * 8, call);
                STAMP(2 + 3 * pass);
                u32x4 o[8];
                u32 bits[2] = {0u, 0u};
                {
                    u32x4 pv[8];
#pragma unroll
                    for (int i = 0; i < 8; ++i) pv[i] = __builtin_amdgcn_raw_buffer_load_b128(rme, lane * 16, OFF_PART + pu * 8192 + i * 1024, AUX_SC1);
                    const bool bad = rfl(ld_tmo(myws)) == call;
#pragma unroll
                    for (int gi = 0; gi < 8; ++gi) {
                        float y[8];
#pragma unroll
                        for (int k = 0; k < 8; ++k) {
                            const u32 pq = pv[gi][k >> 1];
                            float t = acc[gi >> 1][(gi & 1) * 2 + (k >> 2)][k & 3] + ((k & 1) ? bf_hi(pq) : bf_lo(pq));
                            if (bad) t = __builtin_nanf("");
                            const bool nz = !(t <= 0.f);
                            y[k] = nz ? t : 0.f;
                            bits[gi >> 2] |= nz ? (1u << ((gi & 3) * 8 + k)) : 0u;
                        }
                        o[gi] = u32x4{pack2(y[0], y[1]), pack2(y[2], y[3]), pack2(y[4], y[5]), pack2(y[6], y[7])} & 0x7fff7fffu;
                    }
                }
                u32 tot;
                u32 addr = (u32)(LDS_CTL + wave * 16384) + 2 * wave_excl(__builtin_popcount(bits[0]) + __builtin_popcount(bits[1]), tot);
                const u32 H = tot >> 5, eo = ((2 * tot + 3) & ~3u) - 4 * H;
                u32* wt = (u32*)(wl + 9216);
                wt[lane] = lane < H ? bits[0] : 0u;
                wt[64 + lane] = 64 + lane < H ? bits[1] : 0u;
                wt[128 + lane] = 0u;
                *(u32*)(wl + (lane >= H ? eo + lane * 4 : 10240)) = bits[0];
                *(u32*)(wl + (64 + lane >= H ? eo + 256 + lane * 4 : 10240)) = bits[1];
#pragma unroll
                for (int gi = 0; gi < 8; ++gi)
#pragma unroll
                    for (int k = 0; k < 8; ++k) {
                        const u32 t = (bits[gi >> 2] >> ((gi & 3) * 8 + k)) & 1u;
                        const u32 d = o[gi][k >> 1];
                        *(LAS unsigned short*)((LAS char*)smem + __umul24(addr, t)) = (unsigned short)((k & 1) ? d >> 16 : d);
                        addr += 2 * t;
                    }
                WAIT_LGKM0();
                const u32 len = ((eo + 512 + 63) & ~63u) | tot << 16;
                const u32 tl = jt * 16 + ct;
                const u32 sidx = rank * 512 + ((((u32)__builtin_amdgcn_readlane((int)ordv, (int)(tl >> 2))) >> ((tl & 3) * 8)) & 127) * 4 + wave;
                const u32 so = OFF_CMP + sidx * CSTR;
#pragma unroll
                for (int i = 0; i < 9; ++i)
                    if (i * 1024u < (len & 0xffffu)) {
                        u32x4 v = *(const u32x4*)(wl + i * 1024 + lane * 16);
                        const u32 hw = wt[i * 16 + (lane >> 2)] << (12 - (lane & 3) * 4);
                        v[0] |= (hw << 3) & 0x80008000u; v[1] |= (hw << 2) & 0x80008000u;
                        v[2] |= (hw << 1) & 0x80008000u; v[3] |= hw & 0x80008000u;
                        st128(v, rme, lane * 16, so + i * 1024, AUX_SYS);
                    }
                WAIT_VM(0);
                st_flag64(rme, l0, OFF_FC + sidx * 8, call, len);
                STAMP(3 + 3 * pass);
                const u32 obase = (glob_blk(g, bi, lr) * 128 + wm * 64) * (NN * 2) + (glob_blk(g, bj, lc) * 128 + wn * 64) * 2;
#pragma unroll
                for (int gi = 0; gi < 8; ++gi)
                    st_out(o[gi], ro, olane, obase + (gi >> 1) * 16 * (NN * 2) + (gi & 1) * 64);
                WAIT_LGKM0();
            }
        }
    }

    if (wave >= 8 - XWAVES || blockIdx.x >= NPWG + NCWG) {
        const u32 xi = blockIdx.x < NPWG + NCWG ? (blockIdx.x - NPWG) * XWAVES + wave - (8 - XWAVES)
                                                : NCWG * XWAVES + (blockIdx.x - NPWG - NCWG) * 8 + wave;
        const u32 pi = xi % 7;
        const u32 tbase = (call - 1) * (512 + 3 * XSPLIT + (NXW - pi + 6) / 7);
        const u32 p = (rank + 1 + pi) & 7;
        STAMP(7);
        u32 tv = ticket(rme, l0, 8 + pi);
        for (;;) {
            const u32 tk = rfl(tv) - tbase;
            if (tk >= 512 + 3 * XSPLIT) break;
            const bool part = tk >= 512 - XSPLIT;
            const u32 n3 = part ? 512 - XSPLIT + ((tk - (512 - XSPLIT)) >> 2) : tk;
            const u32 q0 = part ? (tk - (512 - XSPLIT)) & 3 : 0, q1 = part ? q0 + 1 : 4;
            const u32 sidx = p * 512 + n3;
            const u32 lw = wait_flag64<XSLEEP>(myws, OFF_FC + sidx * 8, call);
            const u32 len = lw & 0xffffu, nv = (lw >> 16) > 4096 ? 4096 : lw >> 16;
            STAMP(8);
            const u32 so = OFF_CMP + sidx * CSTR;
#pragma unroll
            for (int i = 0; i < 9; ++i)
                if (i * 1024u < len)
                    __builtin_amdgcn_raw_ptr_buffer_load_lds(rme, (LAS char*)wl + i * 1024, 16, lane * 16, so + i * 1024, 0, AUX_SC1);
            const u32 tm = ld_tmo(myws);
            tv = ticket(rme, l0, 8 + pi);
            const u32 to = n3 >> 2, sw = n3 & 3;
            const u32 tl = ((u32)__builtin_amdgcn_readlane((int)ordv, (int)(32 + (to >> 2)))) >> ((to & 3) * 8);
            const u32 tj = perm8((tl >> 4) & 7), tc = perm16(tl & 15);
            const u32 gp = p >> 2;
            const u32 tRB = glob_blk(gp, (p >> 1) & 1, row_lr(tj, gp)), tCB = glob_blk(gp, p & 1, tc);
            const u32 obase = (tRB * 128 + (sw >> 1) * 64) * (NN * 2) + (tCB * 128 + (sw & 1) * 64) * 2;
            WAIT_VM0();
            const bool bad = rfl(tm) == call;
            const u32 H = nv >> 5, eo = ((2 * nv + 3) & ~3u) - 4 * H;
            u32 bits[2];
#pragma unroll
            for (int h = 0; h < 2; ++h) {
                bits[h] = *(const u32*)(wl + eo + (h * 64 + lane) * 4);
                if (H > h * 64) {
                    u32 hw = 0;
#pragma unroll
                    for (int i = 0; i < 4; ++i) {
                        const u32x4 v = *(const u32x4*)(wl + h * 4096 + lane * 64 + i * 16);
#pragma unroll
                        for (int k = 0; k < 4; ++k) hw |= (v[k] >> (15 - i * 4 - k)) & (0x00010001u << (i * 4 + k));
                    }
                    bits[h] = h * 64 + lane < H ? hw : bits[h];
                }
            }
            u32 tot;
            u32 addr = (u32)(LDS_CTL + wave * 16384) + 2 * wave_excl(__builtin_popcount(bits[0]) + __builtin_popcount(bits[1]), tot);
            u64 bq = ((u64)bits[1] << 32 | bits[0]);
            addr += 2 * (u32)__builtin_popcountll(bq & ((1ull << (q0 * 16)) - 1));
            bq >>= q0 * 16;
#pragma unroll 1
            for (u32 qq = q0; qq < q1; ++qq) {
                const u32 b16 = (u32)bq;
                bq >>= 16;
#pragma unroll
                for (int h = 0; h < 2; ++h) {
                    u32 e[8];
#pragma unroll
                    for (int k = 0; k < 8; ++k) {
                        const u32 t = (b16 >> (h * 8 + k)) & 1u;
                        e[k] = *(const LAS unsigned short*)((LAS char*)smem + __umul24(addr, t));
                        addr += 2 * t;
                    }
                    u32x4 r = u32x4{e[0] | (e[1] << 16), e[2] | (e[3] << 16), e[4] | (e[5] << 16), e[6] | (e[7] << 16)} & 0x7fff7fffu;
                    if (bad) r = u32x4{0x7fc07fc0u, 0x7fc07fc0u, 0x7fc07fc0u, 0x7fc07fc0u};
                    st_out(r, ro, olane, obase + qq * 16 * (NN * 2) + h * 64);
                }
            }
            WAIT_LGKM0();
            STAMP(9);
        }
    }
}

constexpr double mx(double a, double b) { return a > b ? a : b; }
constexpr Orders build_orders() {
    Orders P{};
    const char* eo = ENTRY_ORDER;
    double t[3][16] = {};
    for (int e = 0; e < 32; ++e) {
        const int kind = eo[2 * e] == 'w' ? 0 : eo[2 * e] == 's' ? 1 : 2;
        const char ch = eo[2 * e + 1];
        const int idx = ch <= '9' ? ch - '0' : ch - 'A' + 10;
        P.entry[e >> 3] |= (u64)(kind << 4 | idx) << ((e & 7) * 8);
        t[kind][idx] = (e + 1) * 1.445;
    }
    double av[128] = {};
    for (int i = 0; i < 128; ++i) {
        const int jt = i >> 4, ct = i & 15;
        av[i] = mx(mx(t[2][jt], t[0][ct]) + LAT_K, mx(mx(t[1][jt & ~1], t[1][jt | 1]), mx(t[0][ct & ~1], t[0][ct | 1])) + LAT_S);
    }
    for (int i = 0; i < 128; ++i) {
        int pos = 0;
        for (int k = 0; k < 128; ++k) pos += av[k] < av[i] || (av[k] == av[i] && k < i);
        P.ord[i >> 2] |= (u32)pos << ((i & 3) * 8);
        P.ord[32 + (pos >> 2)] |= (u32)i << ((pos & 3) * 8);
    }
    const int W = NCWG / 8, D = 32 - W;
    for (int x = 0; x < 8; ++x) {
        int id[32] = {}, sorted[32] = {};
        double key[32] = {};
        for (int i = 0; i < 32; ++i) {
            const int keep = i >> 4, jt = (x >> 1) + 4 * ((i >> 3) & 1), ct = (i & 7) * 2 + (x & 1);
            id[i] = keep << 7 | jt << 4 | ct;
            key[i] = keep ? av[jt * 16 + ct] - LAT_K : mx(t[1][jt], t[0][ct]);
        }
        for (int i = 0; i < 32; ++i) {
            int pos = 0;
            for (int k = 0; k < 32; ++k) pos += key[k] < key[i] || (key[k] == key[i] && k < i);
            sorted[pos] = id[i];
        }
        for (int i = 0; i < W; ++i) P.tiles[i * 8 + x] = (unsigned short)(sorted[i] | sorted[i < D ? W + i : i] << 8);
    }
    return P;
}
constexpr Orders ORDERS = build_orders();

extern "C" void kernel_launch(int rank, int nranks, ncclComm_t comm,
                              hipStream_t stream,
                              void* const* d_in, const int* in_sizes, int n_in,
                              void* d_out, int out_size,
                              const int* in_dtypes, int out_dtype,
                              void* const* d_out_all,
                              void* const* d_ws_all, size_t ws_bytes) {
    if (nranks != NR || n_in < 2 || ws_bytes < WS_USED || (unsigned)rank >= (unsigned)NR ||
        in_sizes[0] != MM * KS || in_sizes[1] != KS * NN || out_size != MM * NN) {
        fprintf(stderr, "unexpected problem shape\n");
        return;
    }
    Params P;
    P.x = d_in[0];
    P.w = d_in[1];
    for (int i = 0; i < NR; ++i) P.ws[i] = d_ws_all[i];
    P.out = d_out;
    P.rank = rank;
    P.o = ORDERS;
    HIP_CHECK(hipFuncSetAttribute((const void*)fused_gemm_ar, hipFuncAttributeMaxDynamicSharedMemorySize, LDS_BYTES));
    hipLaunchKernelGGL(fused_gemm_ar, dim3(GRID), dim3(512), LDS_BYTES, stream, P);
    HIP_CHECK(hipGetLastError());
}
```

```cpp
#include <hip/hip_runtime.h>
#include <rccl/rccl.h>
#include <cstdio>
#include <cstdint>

#define HIP_CHECK(expr) do { hipError_t _e = (expr); if (_e != hipSuccess) { \
    fprintf(stderr, "HIP error %s at %s:%d\n", hipGetErrorString(_e), __FILE__, __LINE__); \
    return; } } while (0)

#define XS 8
#define XNP 2
#define APAD 128
#define LOOK 4
#define NPUSHW 2
#define SDELAY 32
#define PWIN 3
#define WIN 5
#define WIN2 WIN
#define SELJ 5
#define PRIO 3
#define DIAG 0
#define LAT_K 12.0
#define LAT_S 19.0
#define ORD 1
#define ENTRY_ORDER "w0s0w1s1w2s2w3s3k0w4w5k1s4k2s5k3w6w7k4k5s6s7w8w9wAwBk6k7wCwDwEwF"
#define OUT_AUX 17
#define PSLEEP 8

typedef unsigned int u32;
typedef unsigned long long u64;
typedef u32 u32x4 __attribute__((ext_vector_type(4)));
typedef u32 u32x2 __attribute__((ext_vector_type(2)));
typedef float f32x4 __attribute__((ext_vector_type(4)));
typedef short bf16x8 __attribute__((ext_vector_type(8)));
typedef __bf16 bf16v2 __attribute__((ext_vector_type(2)));
typedef float f32x2 __attribute__((ext_vector_type(2)));
#define LAS __attribute__((address_space(3)))

constexpr int NR = 8;
constexpr int MM = 4096, NN = 4096, KS = 512;
constexpr u32 BLK = 16384;
constexpr u32 OFF_XA = 0;
constexpr u32 OFF_WB = 8u << 20;
constexpr u32 OFF_PART = 16u << 20;
constexpr u32 OFF_SPART = 20u << 20;
constexpr u32 CSTR = 9216;
constexpr u32 CMAX = 8704;
constexpr u32 OFF_CMP = 24u << 20;
constexpr u32 OFF_STG = 30u << 20;
constexpr u32 OFF_FLAGS = 38u << 20;
constexpr u32 OFF_FX = OFF_FLAGS;
constexpr u32 OFF_FW = OFF_FX + 4096;
constexpr u32 OFF_FP = OFF_FW + 4096;
constexpr u32 OFF_NP = OFF_FP + 4096;
constexpr u32 OFF_NT = OFF_NP + 2048;
constexpr u32 OFF_CTR = OFF_NT + 81920;
constexpr u32 OFF_EP = OFF_CTR + 4096;
constexpr u32 OFF_TMO = OFF_EP + 8192;
constexpr u32 OFF_DIAG = OFF_TMO + 4096;
constexpr u32 WS_USED = OFF_DIAG + (1u << 20);
constexpr u32 OUT_BYTES = (u32)MM * NN * 2;
constexpr u32 IN_BYTES = (u32)MM * KS * 2;

constexpr int EWG = 8;
constexpr int NPWG = 7 * EWG;
constexpr int NPUSH = EWG * NPUSHW;
constexpr int NCWG = 192;
constexpr int GRID = 256;
constexpr u32 LDS_CTL = 1024;
constexpr int LDS_BYTES = LDS_CTL + 4 * 32768 + (DIAG ? 1024 : 0);
constexpr u64 DEADLINE = 100000000ull;
constexpr int AUX_SYS = 17;
constexpr int AUX_SC1 = 16;
static_assert(GRID * 8 * 4 <= 8192, "epoch words");
static_assert(OFF_CMP + 512u * CSTR <= OFF_STG, "layout");
static_assert(14 * CSTR <= 4 * 32768, "engine slots");

struct Orders {
    u64 entry[4];
    unsigned short tiles[NCWG];
    u32 ord[64];
};
struct Params {
    const void* x;
    const void* w;
    void* ws[NR];
    void* out;
    int rank;
    Orders o;
};

typedef __amdgpu_buffer_rsrc_t rsrc_t;
__device__ __forceinline__ rsrc_t mk_rsrc(const void* p, u32 bytes) {
    return __builtin_amdgcn_make_buffer_rsrc((void*)p, (short)0, (int)bytes, 0x00020000);
}
__device__ __forceinline__ u32 rfl(u32 v) { return (u32)__builtin_amdgcn_readfirstlane((int)v); }
__device__ __forceinline__ void* pick(void* const (&t)[NR], u32 idx) {
    void* r = t[0];
#pragma unroll
    for (u32 i = 1; i < NR; ++i) r = (idx == i) ? t[i] : r;
    return r;
}
__device__ __forceinline__ u32 glob_blk(u32 g, u32 b, u32 l) {
    return g == 0 ? b * 16 + l : ((l >> 3) * 2 + b) * 8 + (l & 7);
}
__device__ __forceinline__ u32 perm8(u32 t) { return (t >> 1) + (t & 1) * 4; }
__device__ __forceinline__ u32 perm16(u32 t) { return (t >> 1) + (t & 1) * 8; }
__device__ __forceinline__ u32 row_lr(u32 j, u32 fin) { return (j >> 2) * 8 + fin * 4 + (j & 3); }

__device__ __forceinline__ u32 part_idx(u32 a, u32 b) { return b >= 4 ? b * 4 + a : (b > a ? b * b + a : a * a + a + b); }
__device__ __forceinline__ void part_ab(u32 m, u32& a, u32& b) {
    const u32 sh = m >= 9 ? 3 : m >= 4 ? 2 : m >= 1 ? 1 : 0, o = m - sh * sh;
    a = o < sh ? o : sh;
    b = o < sh ? sh : o - sh;
    if (m >= 16) { a = m & 3; b = m >> 2; }
}
__device__ __forceinline__ void unit_desc(const u64 (&entry)[4], u32 s, u32 d, u32 n, u32& su, u32& du) {
    const bool in = (s >> 2) == (d >> 2);
    const u32 gd = d >> 2, e = in ? n / 10 : n / 6, kk = n - e * (in ? 10 : 6);
    const u32 eb = (u32)(entry[(e >> 3) & 3] >> ((e & 7) * 8));
    const u32 ek = (eb >> 4) & 3, idx = eb & 15;
    const bool isw = ek == 0;
    const u32 l = isw ? perm16(idx) : row_lr(perm8(idx & 7), ek == 1 ? 1 - gd : gd);
    su = (isw ? 512 : 0) + glob_blk(gd, isw ? d & 1 : (d >> 1) & 1, l) * 16 + (in ? 0 : 10) + kk;
    du = (isw ? 1024 : 0) + l * 64 + (in ? (s & 3) * 10 : 40 + (s & 3) * 6) + kk;
}
__device__ __forceinline__ void fetch_unit(const void* x, const void* w, u32 su, u32 lane, char* wl, u32x4* v) {
    const u32 isw = su >> 9, GB = (su >> 4) & 31, kc = (su >> 1) & 7, hf = su & 1;
    if (isw) {
        const rsrc_t rw = mk_rsrc(w, IN_BYTES);
        const u32 vo = (lane >> 3) * 65536 + (lane & 7) * 16, so = kc * 64 * 8192 + GB * 256 + hf * 128;
#pragma unroll
        for (int i = 0; i < 8; ++i) v[i] = __builtin_amdgcn_raw_buffer_load_b128(rw, vo, so + i * 8192, 0);
        const u32 wo = (lane >> 5) * 4096 + ((lane & 7) >> 2) * 2048 + (((lane >> 3) & 3) * 16 + (lane & 3) * 4) * 16;
#pragma unroll
        for (int c = 0; c < 8; ++c) {
            u32x4 o;
#pragma unroll
            for (int t = 0; t < 4; ++t)
                o[t] = __builtin_amdgcn_perm(v[2 * t + 1][c >> 1], v[2 * t][c >> 1], (c & 1) ? 0x07060302u : 0x05040100u);
            *(u32x4*)(wl + wo + (c >> 2) * 1024 + (c & 3) * 16) = o;
        }
    } else {
        const rsrc_t rx = mk_rsrc(x, IN_BYTES);
        const u32 vo = (lane >> 3) * 1024 + (lane & 7) * 16, so = GB * 131072 + hf * 65536 + kc * 128;
#pragma unroll
        for (int i = 0; i < 8; ++i) v[i] = __builtin_amdgcn_raw_buffer_load_b128(rx, vo, so + i * 8192, 0);
        const u32 wo = ((lane & 7) >> 2) * 4096 + ((lane & 3) * 16 + (lane >> 3)) * 16;
#pragma unroll
        for (int i = 0; i < 8; ++i) *(u32x4*)(wl + wo + (i >> 1) * 1024 + (i & 1) * 128) = v[i];
    }
    asm volatile("s_waitcnt lgkmcnt(0)" ::: "memory");
#pragma unroll
    for (int i = 0; i < 8; ++i) v[i] = *(const u32x4*)(wl + i * 1024 + lane * 16);
    asm volatile("s_waitcnt lgkmcnt(0)" ::: "memory");
}
__device__ __forceinline__ void st128(u32x4 v, rsrc_t r, u32 voff, u32 soff, int) {
    __builtin_amdgcn_raw_buffer_store_b128(v, r, voff + soff, 0, AUX_SYS);
}
__device__ __forceinline__ void st_out(u32x4 v, rsrc_t r, u32 voff, u32 soff) {
    __builtin_amdgcn_raw_buffer_store_b128(v, r, voff + soff, 0, OUT_AUX);
}
__device__ __forceinline__ void st_flag(rsrc_t r, u32 l0, u32 off, u32 v) {
    __builtin_amdgcn_raw_buffer_store_b32(v, r, l0, off, AUX_SYS);
}
__device__ __forceinline__ void st_flag64(rsrc_t r, u32 l0, u32 off, u32 call, u32 len) {
    __builtin_amdgcn_raw_buffer_store_b64(u32x2{len, call}, r, l0, off, AUX_SYS);
}
__device__ __forceinline__ u64 ld_flag64(const void* ws, u32 off) {
    return __hip_atomic_load((const u64*)((const char*)ws + off), __ATOMIC_RELAXED, __HIP_MEMORY_SCOPE_SYSTEM);
}
#define CBAR() asm volatile("" ::: "memory")
#define WAIT_VM(n) asm volatile("s_waitcnt vmcnt(" #n ")" ::: "memory")
#define WAIT_VM0() do { WAIT_VM(0); __builtin_amdgcn_s_waitcnt(0x0F70); CBAR(); } while (0)
#define WAIT_LGKM0() asm volatile("s_waitcnt lgkmcnt(0)" ::: "memory")

__device__ __forceinline__ bool give_up(void* myws, u32 call, u64 t0) {
    u32* tmo = (u32*)((char*)myws + OFF_TMO);
    bool late = __builtin_amdgcn_s_memrealtime() - t0 > DEADLINE;
    if (late || rfl(__hip_atomic_load(tmo, __ATOMIC_RELAXED, __HIP_MEMORY_SCOPE_AGENT)) == call) {
        __hip_atomic_store(tmo, call, __ATOMIC_RELAXED, __HIP_MEMORY_SCOPE_AGENT);
        return true;
    }
    return false;
}
__device__ __forceinline__ u32 ld_tmo(void* myws) {
    return __hip_atomic_load((u32*)((char*)myws + OFF_TMO), __ATOMIC_RELAXED, __HIP_MEMORY_SCOPE_AGENT);
}
template <int SLP>
__device__ __forceinline__ u32 wait_flag64(void* myws, u32 off, u32 call) {
    u64 t0 = 0;
    for (u32 it = 0;; ++it) {
        u64 v = ld_flag64(myws, off);
        if (rfl((u32)(v >> 32)) >= call) return rfl((u32)v);
        if (it == 0) t0 = __builtin_amdgcn_s_memrealtime();
        if ((it & 63) == 63 && give_up(myws, call, t0)) return 0;
        __builtin_amdgcn_s_sleep(SLP);
    }
}
__device__ __forceinline__ u32 ticket(rsrc_t r, u32 l0, u32 idx) {
    return (u32)__builtin_amdgcn_raw_ptr_buffer_atomic_add_i32(1, r, l0, OFF_CTR + idx * 128, 0);
}
__device__ __forceinline__ u32 mbcnt64(u64 m) {
    return __builtin_amdgcn_mbcnt_hi((u32)(m >> 32), __builtin_amdgcn_mbcnt_lo((u32)m, 0u));
}
__device__ __forceinline__ u32 wave_excl(u32 cnt, u32& tot) {
    u32 pre = 0; tot = 0;
#pragma unroll
    for (int b = 0; b < 7; ++b) {
        const u64 m = __ballot((cnt >> b) & 1);
        pre += mbcnt64(m) << b;
        tot += (u32)__builtin_popcountll(m) << b;
    }
    return pre;
}
__device__ __forceinline__ u32 pack2(float a, float b) {
    f32x2 v = {a, b};
    return __builtin_bit_cast(u32, __builtin_convertvector(v, bf16v2));
}
__device__ __forceinline__ float bf_lo(u32 v) { return __builtin_bit_cast(float, v << 16); }
__device__ __forceinline__ float bf_hi(u32 v) { return __builtin_bit_cast(float, v & 0xffff0000u); }

#define STAMP(slot) do {} while (0)

__global__ void __launch_bounds__(512, 2) fused_gemm_ar(Params P) {
    extern __shared__ __attribute__((aligned(16))) char smem[];
    const u32 lane = threadIdx.x & 63;
    const u32 wave = rfl(threadIdx.x >> 6);
    const u32 rank = (u32)P.rank & 7;
    const u32 g = rank >> 2, bi = (rank >> 1) & 1, bj = rank & 1;
    void* myws = pick(P.ws, rank);
    const rsrc_t rme = mk_rsrc(myws, WS_USED);
    char* wl = smem + LDS_CTL + wave * 16384;

    const u32 l0 = lane == 0 ? 0u : 0x80000000u;
    const u32 epo = OFF_EP + (blockIdx.x * 8 + wave) * 4;
    const u32 epv = __builtin_amdgcn_raw_buffer_load_b32(rme, 0, epo, AUX_SC1);
    *(LAS u32*)smem = 0u;
    STAMP(0);

    const u32 ordv = P.o.ord[lane];
    const u32 olane = (lane & 15) * (NN * 2) + (lane >> 4) * 16;
    const rsrc_t ro = mk_rsrc(P.out, OUT_BYTES);

    if (blockIdx.x < NPWG) {
        const u32 b = blockIdx.x;
        const u32 di = 1 + ((b >> 3) + (b & 7)) % 7;
        const u32 p = (rank + di) & 7;
        const rsrc_t rp = mk_rsrc(pick(P.ws, p), WS_USED);
        volatile LAS u32* ctl = (volatile LAS u32*)(smem + 64);
        const u32 dum = 640 + lane * 4;
        const u32 cra = lane == 0 ? 64 + 15 * 32 + 4 : dum;
        if (wave == 0) *(volatile LAS u32x4*)(ctl + (lane & 15) * 8) = u32x4{0u, (u32)LOOK, 0u, (lane & 15) + NPUSHW >= 14 && (lane & 15) < 14 ? 1u : 0u};
        WAIT_LGKM0();
        __builtin_amdgcn_s_barrier();
        CBAR();
        if (wave == 0) {
            __builtin_amdgcn_s_setprio(PRIO);
            WAIT_VM0();
            u32 prev = 14, pnp = 0, tsl = 0, treq = 0, tsrc = 0;
            u64 t0 = __builtin_amdgcn_s_memrealtime();
            auto select = [&](u32x2 cv) -> bool {
                int mk = (cv[0] & 1) ? (int)((cv[0] >> 16) << 4 | (lane & 15)) : 0x7fffffff;
                mk = min(mk, __builtin_amdgcn_update_dpp(0x7fffffff, mk, 0x111, 0xf, 0xf, false));
                mk = min(mk, __builtin_amdgcn_update_dpp(0x7fffffff, mk, 0x112, 0xf, 0xf, false));
                mk = min(mk, __builtin_amdgcn_update_dpp(0x7fffffff, mk, 0x114, 0xf, 0xf, false));
                mk = min(mk, __builtin_amdgcn_update_dpp(0x7fffffff, mk, 0x118, 0xf, 0xf, false));
                const u32 r = (u32)__builtin_amdgcn_readlane(mk, 15);
                if (r == 0x7fffffffu) return false;
                tsl = r & 15;
                treq = (u32)__builtin_amdgcn_readlane((int)cv[0], (int)tsl);
                tsrc = (u32)__builtin_amdgcn_readlane((int)cv[1], (int)tsl);
                ctl[tsl * 8] = 0;
                asm volatile("ds_add_u32 %0, %1" :: "v"(cra), "v"(tsrc & 1) : "memory");
                return true;
            };
            bool have = false;
            for (;;) {
                if (!have) {
                    bool end = false;
                    for (u32 it = 0;; ++it) {
                        const u32x2 cv = *(volatile LAS u32x2*)(ctl + (lane & 15) * 8);
                        if (select(cv)) { end = false; break; }
                        if (prev != 14) { WAIT_VM(0); ctl[prev * 8 + 2] = pnp; prev = 14; }
                        if (end) break;
                        end = (u32)__builtin_amdgcn_readlane((int)cv[0], 15) != 0;
                        if ((it & 1023) == 1023 && __builtin_amdgcn_s_memrealtime() - t0 > 2 * DEADLINE) end = true;
                        __builtin_amdgcn_s_sleep(1);
                    }
                    if (end) break;
                }
                const u32 sl = tsl, len = ((treq >> 8) & 255) * 64, np = (treq >> 4) & 15, src = tsrc & ~1u;
                const rsrc_t rs = mk_rsrc(pick(P.ws, p), src + len);
                const u32 vo = src + lane * 16;
                LAS char* d = (LAS char*)smem + LDS_CTL + sl * CSTR;
                u32x2 cv = {0u, 0u};
#define PULL_STEP(j, W, ALL) \
                if ((j) == (W) - 1 && (ALL || len > ((j) - 1) * 1024u)) { \
                    asm volatile("s_waitcnt vmcnt(%0)" :: "n"((W) - 1) : "memory"); \
                    ctl[prev * 8 + 2] = pnp; \
                    prev = 14; \
                } \
                if (ALL || (j) * 1024u < len) { \
                    asm volatile("s_waitcnt vmcnt(%0)" :: "n"((W) - 1) : "memory"); \
                    __builtin_amdgcn_raw_ptr_buffer_load_lds(rs, d + ((j) >> 2) * 4096, 16, vo + ((j) >> 2) * 4096, 0, ((j) & 3) * 1024, AUX_SYS); \
                } \
                CBAR(); \
                if ((j) == SELJ - 1) cv = *(volatile LAS u32x2*)(ctl + (lane & 15) * 8); \
                if ((j) == SELJ) have = select(cv); \
                CBAR();
                if (np == 1) {
                    PULL_STEP(0, WIN, 1) PULL_STEP(1, WIN, 1) PULL_STEP(2, WIN, 1) PULL_STEP(3, WIN, 1)
                    PULL_STEP(4, WIN, 1) PULL_STEP(5, WIN, 1) PULL_STEP(6, WIN, 1) PULL_STEP(7, WIN, 1)
                } else {
                    PULL_STEP(0, WIN2, 0) PULL_STEP(1, WIN2, 0) PULL_STEP(2, WIN2, 0) PULL_STEP(3, WIN2, 0) PULL_STEP(4, WIN2, 0)
                    PULL_STEP(5, WIN2, 0) PULL_STEP(6, WIN2, 0) PULL_STEP(7, WIN2, 0) PULL_STEP(8, WIN2, 0)
                }
                if (prev != 14) { WAIT_VM(0); ctl[prev * 8 + 2] = pnp; }
                prev = sl; pnp = np;
            }
            ctl[15 * 8 + 3] = 1;
            STAMP(3);
            __builtin_amdgcn_s_setprio(0);
        }
        const u32 gp = p >> 2;
        if (wave > 7 - NPUSHW) {
            u32 su, du; unit_desc(P.o.entry, rank, p, (7 - wave) * EWG + (b & 7), su, du);
            u32x4 v[8];
            fetch_unit(P.x, P.w, su, lane, smem + LDS_CTL + (6 + wave) * CSTR, v);
            ctl[(6 + wave) * 8 + 3] = 0;
            STAMP(4);
#pragma unroll
            for (int i = 0; i < 8; ++i) {
                asm volatile("s_waitcnt vmcnt(%0)" :: "n"(PWIN - 1) : "memory");
                st128(v[i], rp, lane * 16, OFF_XA + du * 8192 + i * 1024, AUX_SYS);
            }
            WAIT_VM0();
            st_flag(rp, l0, OFF_FX + du * 4, rfl(epv) + 1);
            STAMP(5);
        }
        const u32 call = rfl(epv) + 1;
        st_flag(rme, l0, epo, call);
        if (wave == 1) {
            __builtin_amdgcn_s_setprio(2);
            const u32 T1 = (gp == g) ? 320 : 192;
            const u32 T = T1 + 256;
            const u32 tbase = (call - 1) * (T - NPUSH + 2 * EWG) - NPUSH;
            u32 n = NPUSH, ne = NPUSH;
            u32 pmask = gp == g ? 0u : 0xffffu;
            u32 half = 0;
            bool over = false, force = false;
            u64 t0 = __builtin_amdgcn_s_memrealtime();
            for (u32 it = 0;; ++it) {
                if (over && n == ne && pmask == 0) break;
                u32 k = 0;
                if (!over && n == ne) {
                    k = rfl(ctl[15 * 8 + 1]);
                    k = k > 2 ? 2 : k;
                    if (k == 0 && pmask == 0) { __builtin_amdgcn_s_sleep(1); if ((it & 4095) == 4095 && __builtin_amdgcn_s_memrealtime() - t0 > 2 * DEADLINE) break; continue; }
                    asm volatile("ds_sub_u32 %0, %1" :: "v"(cra), "v"(k) : "memory");
                }
                const u32 tv = (u32)__builtin_amdgcn_raw_ptr_buffer_atomic_add_i32((int)k, rme, k ? l0 : 0x80000000u, OFF_CTR + di * 128, 0);
                const u32 wb = n;
                const u32x4 nv = __builtin_amdgcn_raw_buffer_load_b128(rme, lane < 47 ? OFF_NT + (p * 640 + wb + lane) * 16 :
                    lane == 47 ? OFF_TMO : OFF_NP + ((p & 3) * 32 + (lane - 48) * 2 + ((b >> 2) & 1)) * 16, 0, AUX_SYS);
                WAIT_VM0();
                if (k) {
                    n = rfl(tv) - tbase; ne = n + k;
                    if (ne > T) {
                        over = true;
                        if (ne - (n > T ? n : T) < 2) __builtin_amdgcn_raw_ptr_buffer_atomic_add_i32(1, rme, l0, OFF_CTR + di * 128, 0);
                        ne = T; n = n > T ? T : n;
                    }
                }
                if ((u32)__builtin_amdgcn_readlane((int)nv[0], 47) == call) force = true;
                u64 rm = force ? ~0ull : __ballot(nv[1] >= call && nv[3] >= call);
                u32 fr = (u32)__ballot(ctl[(lane & 15) * 8 + 3] == 0) & 0x3fffu;
                bool progress = false;
                while (fr) {
                    const u32 pr = (u32)(rm >> 48) & pmask;
                    u32 src, dst, aux, aux2 = 0, req;
                    if (pr) {
                        const u32 i = (u32)__builtin_ctz(pr), un = (b & 7) + 8 * i;
                        pmask &= ~(1u << i);
                        u32 a, bb; part_ab(un >> 2, a, bb);
                        const u32 pu = ((((p >> 1) & 1) * 4 + a) * 16 + (p & 1) * 8 + bb) * 4 + (un & 3);
                        src = OFF_SPART + ((rank & 3) * 128 + un) * 8192;
                        dst = OFF_PART + pu * 8192; aux = 0x80000000u | (OFF_FP + pu * 8 + 4);
                        req = 1 | 1 << 4 | 128 << 8;
                    } else if (n < ne && n - wb < 47 && ((rm >> (n - wb)) & 1)) {
                        if (n < T1) {
                            u32 su, du; unit_desc(P.o.entry, p, rank, n, su, du);
                            src = OFF_STG + su * 8192 + 1; dst = OFF_XA + du * 8192; aux = 0x80000000u | (OFF_FX + du * 4);
                            req = 1 | 1 << 4 | 128 << 8 | (n + 1) << 16;
                            ++n;
                        } else {
                            u32 lA = (u32)__builtin_amdgcn_readlane((int)nv[0], (int)(n - wb)), lB = (u32)__builtin_amdgcn_readlane((int)nv[2], (int)(n - wb));
                            if (force || (lA & 0xffffu) > CMAX || (lA >> 16) > 4096 || (lA & 63)) lA = 512;
                            if (force || (lB & 0xffffu) > CMAX || (lB >> 16) > 4096 || (lB & 63)) lB = 512;
                            const u32 n3 = n - T1, to = n3 >> 1;
                            const u32 tl = ((u32)__builtin_amdgcn_readlane((int)ordv, (int)(32 + (to >> 2)))) >> ((to & 3) * 8);
                            const u32 tRB = glob_blk(gp, (p >> 1) & 1, row_lr(perm8((tl >> 4) & 7), gp)), tCB = glob_blk(gp, p & 1, perm16(tl & 15));
                            dst = (tRB * 128 + (n3 & 1) * 64) * (NN * 2) + tCB * 256;
                            if (half) lA = 0;
                            half = (lA & 0xffffu) + (lB & 0xffffu) > CSTR;
                            aux = lA; aux2 = (half ? 0u : lB) | (force ? 0x80000000u : 0u);
                            src = OFF_CMP + (n3 * 2 + 1) * CSTR - (lA & 0xffffu) + (half ? 0 : 1);
                            req = 1 | (n + XS >= T ? 8 : XNP) << 4 | (((aux & 0xffffu) + (aux2 & 0xffffu)) >> 6) << 8 | (n + 1) << 16;
                            n += 1 - half;
                        }
                    } else break;
                    const u32 sl = (u32)__builtin_ctz(fr);
                    fr &= fr - 1;
                    *(volatile LAS u32x4*)(ctl + sl * 8 + 4) = u32x4{dst, aux, aux2, (req >> 4) & 15};
                    ctl[sl * 8 + 3] = (req >> 4) & 15;
                    *(volatile LAS u32x2*)(ctl + sl * 8) = u32x2{req, src};
                    progress = true;
                }
                if (progress || k) t0 = __builtin_amdgcn_s_memrealtime();
                else if ((it & 63) == 63 && give_up(myws, call, t0)) {
                    force = true;
                    if (__builtin_amdgcn_s_memrealtime() - t0 > 2 * DEADLINE) break;
                }
            }
            ctl[15 * 8] = 256;
            __builtin_amdgcn_s_setprio(0);
        }
        {
            bool end = false;
            u64 t0 = __builtin_amdgcn_s_memrealtime();
            for (u32 it = 0;; ++it) {
                const u32x2 cv = *(volatile LAS u32x2*)(ctl + (lane & 15) * 8 + 2);
                const u32 av = (u32)__ballot((int)cv[0] > 0) & 0x3fffu;
                if (av == 0) {
                    if (end) break;
                    end = (u32)__builtin_amdgcn_readlane((int)cv[1], 15) != 0;
                    if ((it & 1023) == 1023 && __builtin_amdgcn_s_memrealtime() - t0 > 3 * DEADLINE) break;
                    __builtin_amdgcn_s_sleep(2);
                    continue;
                }
                end = false;
                const u32 sl = (u32)__builtin_ctz(av);
                u32 old;
                u32x4 dv;
                asm volatile("ds_add_rtn_u32 %0, %2, %3\n\tds_read_b128 %1, %4\n\ts_waitcnt lgkmcnt(0)" : "=&v"(old), "=&v"(dv)
                             : "v"(lane == 0 ? 64 + sl * 32 + 8 : dum), "v"(0xffffffffu), "v"(64 + sl * 32 + 16) : "memory");
                const u32 pc = rfl(old) - 1;
                if ((int)pc < 0) continue;
                const char* sp = smem + LDS_CTL + sl * CSTR;
                const u32 dst = rfl(dv[0]), aux = rfl(dv[1]);
                const u32 dna = lane == 0 ? 64 + sl * 32 + 12 : dum;
                if (aux >> 31) {
                    u32x4 v[8];
#pragma unroll
                    for (int i = 0; i < 8; ++i) v[i] = *(const u32x4*)(sp + i * 1024 + lane * 16);
                    asm volatile("ds_sub_u32 %0, %1" :: "v"(dna), "v"(1u) : "memory");
#pragma unroll
                    for (int i = 0; i < 8; ++i) st128(v[i], rme, lane * 16, dst + i * 1024, AUX_SYS);
                    WAIT_VM0();
                    st_flag(rme, l0, aux & 0x7fffffffu, call);
                    continue;
                }
                const u32 hs = pc & 1, aux2 = rfl(dv[2]), nq = 8 / rfl(dv[3]), q0 = (pc >> 1) * nq;
                const u32 lw = hs ? aux2 & 0x7fffffffu : aux;
                if (lw != 0) {
                    const u32 nv = lw >> 16, sb = hs ? aux & 0xffffu : 0u;
                    const u32 H = nv >> 5, eo = ((2 * nv + 3) & ~3u) - 4 * H;
                    u32 bits[2];
#pragma unroll
                    for (int h = 0; h < 2; ++h) {
                        bits[h] = *(const u32*)(sp + sb + eo + (h * 64 + lane) * 4);
                        if (H > h * 64) {
                            u32 hw = 0;
#pragma unroll
                            for (int i = 0; i < 4; ++i) {
                                const u32x4 v = *(const u32x4*)(sp + sb + (i * H + h * 64 + lane) * 16);
#pragma unroll
                                for (int q = 0; q < 4; ++q) hw |= (v[q] >> (15 - i * 4 - q)) & (0x00010001u << (i * 4 + q));
                            }
                            bits[h] = h * 64 + lane < H ? hw : bits[h];
                        }
                    }
                    u32 tot;
                    u32 addr = LDS_CTL + sl * CSTR + sb + 2 * wave_excl(__builtin_popcount(bits[0]) + __builtin_popcount(bits[1]), tot);
                    u64 bq = ((u64)bits[1] << 32 | bits[0]);
                    addr += 2 * (u32)__builtin_popcountll(bq & ((1ull << (q0 * 16)) - 1));
                    bq >>= q0 * 16;
#pragma unroll 1
                    for (u32 qq = q0; qq < q0 + nq; ++qq) {
                        const u32 b16 = (u32)bq;
                        bq >>= 16;
#pragma unroll
                        for (int h = 0; h < 2; ++h) {
                            u32 e[8];
#pragma unroll
                            for (int q = 0; q < 8; ++q) {
                                const u32 t = (b16 >> (h * 8 + q)) & 1u;
                                e[q] = *(const LAS unsigned short*)((LAS char*)smem + __umul24(addr, t));
                                addr += 2 * t;
                            }
                            u32x4 r = u32x4{e[0] | (e[1] << 16), e[2] | (e[3] << 16), e[4] | (e[5] << 16), e[6] | (e[7] << 16)} & 0x7fff7fffu;
                            if (aux2 >> 31) r = u32x4{0x7fc07fc0u, 0x7fc07fc0u, 0x7fc07fc0u, 0x7fc07fc0u};
                            st_out(r, ro, olane, dst + hs * 128 + qq * 16 * (NN * 2) + h * 64);
                        }
                    }
                }
                asm volatile("ds_sub_u32 %0, %1" :: "v"(dna), "v"(1u) : "memory");
                STAMP(9);
            }
        }
        return;
    }

    {
        const u32 u = (blockIdx.x - NPWG) + (GRID - NPWG) * wave;
        if (wave == 0) ((volatile LAS u32*)(smem + 32))[lane & 7] = 0u;
        if (u < 1024) {
            const u32 e = u >> 5, r = u & 15, other = r >= 10, hf = (u >> 4) & 1;
            const u32 n = other ? e * 6 + r - 10 : e * 10 + r;
            u32 su, du;
            unit_desc(P.o.entry, rank, (g ^ other) * 4 + hf * 3, n, su, du);
            const u32 isw = su >> 9;
            const bool mine = !other && hf == (isw ? bj : bi);
            char* const np = (char*)P.ws[(g ^ other) * 4 + (isw ? hf + 2 * (lane & 1) : 2 * hf + (lane & 1))];
            if (mine || n >= NPUSH) {
                for (u32 i = 0; i < wave; ++i) __builtin_amdgcn_s_sleep(SDELAY);
                u32x4 v[8];
                fetch_unit(P.x, P.w, su, lane, wl, v);
                if (n >= NPUSH) {
#pragma unroll
                    for (int i = 0; i < 8; ++i) st128(v[i], rme, lane * 16, OFF_STG + su * 8192 + i * 1024, AUX_SYS);
                    if (mine) {
#pragma unroll
                        for (int i = 0; i < 8; ++i) st128(v[i], rme, lane * 16, OFF_XA + du * 8192 + i * 1024, AUX_SYS);
                        WAIT_VM(8);
                    } else WAIT_VM(0);
                    if (lane < 4)
                        __hip_atomic_store((u64*)(np + OFF_NT + (rank * 640 + n) * 16 + (lane >> 1) * 8), (u64)(rfl(epv) + 1) << 32, __ATOMIC_RELAXED, __HIP_MEMORY_SCOPE_SYSTEM);
                    WAIT_VM(1);
                } else {
#pragma unroll
                    for (int i = 0; i < 8; ++i) st128(v[i], rme, lane * 16, OFF_XA + du * 8192 + i * 1024, AUX_SYS);
                    WAIT_VM(0);
                }
                st_flag(rme, mine ? l0 : 0x80000000u, OFF_FX + du * 4, rfl(epv) + 1);
            }
        }
        STAMP(10);
    }
    char* const peerp = (char*)P.ws[(rank + 1 + (lane >> 2)) & 7] + OFF_NT + rank * 10240 + ((((rank + 1 + (lane >> 2)) >> 2) & 1) == g ? 5120 : 3072);

    const u32 call = rfl(epv) + 1;
    const u32 wm = (wave >> 1) & 1, wn = wave & 1;

    st_flag(rme, l0, epo, call);
    if (blockIdx.x < NPWG + NCWG) {
        const u32 c = blockIdx.x - NPWG;
        const u32 tl2 = P.o.tiles[c];
        volatile LAS u32* sel = (volatile LAS u32*)(smem + 16);

        for (u32 pass = 0; pass < ((tl2 >> 8) == (tl2 & 255) ? 1u : 2u); ++pass) {
            const u32 tb = tl2 >> (pass * 8);
            const u32 keep = (tb >> 7) & 1, jt = (tb >> 4) & 7, ct = tb & 15;
            const u32 j = perm8(jt);
            const u32 lc = perm16(ct);
            const u32 lr = row_lr(j, keep ? g : 1 - g);

            if (wave >= 4) {
                const u32 lw = wave - 4;
                const u32 xbase = OFF_XA + lr * 32 * BLK + lw * 4096;
                const u32 wbase = OFF_WB + lc * 32 * BLK + lw * 4096;
                const u32 fo = (lane < 32) ? OFF_FX + (lr * 32 + lane) * 8 : OFF_FW + (lc * 32 + lane - 32) * 8;
                u32 pend = 0, taken = 0;
                for (int s = -4; s < 32; ++s) {
                    const bool refill = pend == 0 && s < 28;
                    if (refill && lw == 0) {
                        u64 t0 = 0;
                        for (u32 it = 0;; ++it) {
                            const u64 fv = ld_flag64(myws, fo);
                            const u64 m = __ballot((u32)fv >= call && (u32)(fv >> 32) >= call);
                            pend = (u32)m & (u32)(m >> 32) & ~taken;
                            if (pend) break;
                            if (it == 0) t0 = __builtin_amdgcn_s_memrealtime();
                            if ((it & 63) == 63 && give_up(myws, call, t0)) { pend = ~taken; break; }
                            __builtin_amdgcn_s_sleep(PSLEEP);
                        }
                        taken |= pend;
                        sel[s & 1] = pend;
                    }
                    if (s < 29) WAIT_VM(16); else if (s == 29) WAIT_VM(8); else WAIT_VM(0);
                    WAIT_LGKM0();
                    __builtin_amdgcn_s_barrier();
                    CBAR();
                    if (s < 28) {
                        if (refill) pend = rfl(sel[s & 1]);
                        const u32 so = ((u32)__builtin_ctz(pend | 0x80000000u)) * BLK;
                        pend &= pend - 1;
                        LAS char* d = (LAS char*)smem + LDS_CTL + (s & 3) * 32768 + lw * 4096;
#pragma unroll
                        for (int i = 0; i < 4; ++i)
                            __builtin_amdgcn_raw_ptr_buffer_load_lds(rme, d + i * 1024, 16, lane * 16, xbase + so + i * 1024, 0, AUX_SC1);
#pragma unroll
                        for (int i = 0; i < 4; ++i)
                            __builtin_amdgcn_raw_ptr_buffer_load_lds(rme, d + 16384 + i * 1024, 16, lane * 16, wbase + so + i * 1024, 0, AUX_SC1);
                    }
                    CBAR();
                }
                continue;
            }
            f32x4 acc[4][4];
#pragma unroll
            for (int a = 0; a < 4; ++a)
#pragma unroll
                for (int b2 = 0; b2 < 4; ++b2) acc[a][b2] = f32x4{0.f, 0.f, 0.f, 0.f};
            {
                const char* xa = smem + LDS_CTL + wm * 8192 + lane * 16;
                const char* wb = smem + LDS_CTL + 16384 + wn * 8192 + lane * 16;
                bf16x8 xA[8], wA[8], xB[8], wB[8];
                auto rd = [&](bf16x8 (&xf)[8], bf16x8 (&wf)[8], u32 slot) {
                    const u32 so = (slot & 3) * 32768;
#pragma unroll
                    for (int i = 0; i < 8; ++i) {
                        wf[i] = *(const bf16x8*)(wb + so + i * 1024);
                        xf[i] = *(const bf16x8*)(xa + so + i * 1024);
                    }
                };
                auto mm = [&](bf16x8 (&xf)[8], bf16x8 (&wf)[8]) {
#pragma unroll
                    for (int h = 0; h < 2; ++h)
#pragma unroll
                        for (int a = 0; a < 4; ++a)
#pragma unroll
                            for (int b2 = 0; b2 < 4; ++b2)
                                acc[a][b2] = __builtin_amdgcn_mfma_f32_16x16x32_bf16(wf[h * 4 + b2], xf[h * 4 + a], acc[a][b2], 0, 0, 0);
                };
                auto sched = [&]() {
#pragma unroll
                    for (int i = 0; i < 16; ++i) {
                        __builtin_amdgcn_sched_group_barrier(0x100, 1, 0);
                        __builtin_amdgcn_sched_group_barrier(0x008, 2, 0);
                    }
                };
#define BAR() do { __builtin_amdgcn_s_waitcnt(0xC07F); CBAR(); __builtin_amdgcn_s_barrier(); CBAR(); __builtin_amdgcn_sched_barrier(0); } while (0)
                BAR(); BAR(); BAR();
                BAR();
                rd(xA, wA, 0);
                for (u32 s = 0; s < 32; s += 2) {
                    BAR();
                    rd(xB, wB, s + 1);
                    mm(xA, wA);
                    sched();
                    __builtin_amdgcn_sched_barrier(0);
                    BAR();
                    rd(xA, wA, s + 2);
                    mm(xB, wB);
                    sched();
                    __builtin_amdgcn_sched_barrier(0);
                }
                WAIT_LGKM0();
            }
            STAMP(1 + 3 * pass);

            if (!keep) {
                const u32 pu = (((jt & 1) * 2 + (ct & 1)) * 32 + part_idx(jt >> 1, ct >> 1)) * 4 + wave;
                const u32 dst = OFF_SPART + pu * 8192;
#pragma unroll
                for (int a = 0; a < 4; ++a)
#pragma unroll
                    for (int u = 0; u < 2; ++u) {
                        u32x4 o;
                        o[0] = pack2(acc[a][2 * u][0], acc[a][2 * u][1]);
                        o[1] = pack2(acc[a][2 * u][2], acc[a][2 * u][3]);
                        o[2] = pack2(acc[a][2 * u + 1][0], acc[a][2 * u + 1][1]);
                        o[3] = pack2(acc[a][2 * u + 1][2], acc[a][2 * u + 1][3]);
                        st128(o, rme, lane * 16, dst + (a * 2 + u) * 1024, AUX_SYS);
                    }
                WAIT_VM(0);
                sel[4 + pass * 4 + wave] = 0x80000000u;
                if (__ballot(sel[4 + pass * 4 + (lane & 3)] >> 31) == ~0ull)
                    st_flag64(mk_rsrc(pick(P.ws, (1 - g) * 4 + (pu >> 7)), WS_USED), lane < 2 ? lane * 8 : 0x80000000u, OFF_NP + ((rank & 3) * 32 + ((pu >> 2) & 31)) * 16, call, 0);
                STAMP(3 + 3 * pass);
            } else {
                const u32 pu = (j * 16 + lc) * 4 + wave;
                wait_flag64<PSLEEP>(myws, OFF_FP + pu * 8, call);
                STAMP(2 + 3 * pass);
                u32x4 o[8];
                u32 bits[2] = {0u, 0u};
                {
                    u32x4 pv[8];
#pragma unroll
                    for (int i = 0; i < 8; ++i) pv[i] = __builtin_amdgcn_raw_buffer_load_b128(rme, lane * 16, OFF_PART + pu * 8192 + i * 1024, AUX_SC1);
                    const bool bad = rfl(ld_tmo(myws)) == call;
#pragma unroll
                    for (int gi = 0; gi < 8; ++gi) {
                        float y[8];
#pragma unroll
                        for (int k = 0; k < 8; ++k) {
                            const u32 pq = pv[gi][k >> 1];
                            float t = acc[gi >> 1][(gi & 1) * 2 + (k >> 2)][k & 3] + ((k & 1) ? bf_hi(pq) : bf_lo(pq));
                            if (bad) t = __builtin_nanf("");
                            const bool nz = !(t <= 0.f);
                            y[k] = nz ? t : 0.f;
                            bits[gi >> 2] |= nz ? (1u << ((gi & 3) * 8 + k)) : 0u;
                        }
                        o[gi] = u32x4{pack2(y[0], y[1]), pack2(y[2], y[3]), pack2(y[4], y[5]), pack2(y[6], y[7])} & 0x7fff7fffu;
                    }
                }
                u32 tot;
                u32 addr = (u32)(LDS_CTL + wave * 16384) + 2 * wave_excl(__builtin_popcount(bits[0]) + __builtin_popcount(bits[1]), tot);
                const u32 H = tot >> 5, eo = ((2 * tot + 3) & ~3u) - 4 * H;
                u32* wt = (u32*)(wl + 9216);
                wt[lane] = bits[0];
                wt[64 + lane] = bits[1];
                *(u32*)(wl + (lane >= H ? eo + lane * 4 : 10240)) = bits[0];
                *(u32*)(wl + (64 + lane >= H ? eo + 256 + lane * 4 : 10240)) = bits[1];
#pragma unroll
                for (int gi = 0; gi < 8; ++gi)
#pragma unroll
                    for (int k = 0; k < 8; ++k) {
                        const u32 t = (bits[gi >> 2] >> ((gi & 3) * 8 + k)) & 1u;
                        const u32 d = o[gi][k >> 1];
                        *(LAS unsigned short*)((LAS char*)smem + __umul24(addr, t)) = (unsigned short)((k & 1) ? d >> 16 : d);
                        addr += 2 * t;
                    }
                WAIT_LGKM0();
                const u32 len = ((wave & 1) ? (eo + 512 + 63) & ~63u : (eo + 512 + APAD - 1) & ~(APAD - 1u)) | tot << 16;
                const u32 tl = jt * 16 + ct;
                const u32 sidx = ((((u32)__builtin_amdgcn_readlane((int)ordv, (int)(tl >> 2))) >> ((tl & 3) * 8)) & 127) * 4 + wave;
                const u32 so = OFF_CMP + sidx * CSTR + ((wave & 1) ? 0 : CSTR - (len & 0xffffu));
#pragma unroll
                for (int i = 0; i < 9; ++i)
                    if (i * 1024u < (len & 0xffffu)) {
                        u32x4 v = *(const u32x4*)(wl + i * 1024 + lane * 16);
                        const u32 G = i * 64 + lane, pl = (G >= H) + (G >= 2 * H) + (G >= 3 * H);
                        const u32 hw = G < 4 * H ? wt[(G - pl * H) & 127] << (12 - pl * 4) : 0u;
                        v[0] |= (hw << 3) & 0x80008000u; v[1] |= (hw << 2) & 0x80008000u;
                        v[2] |= (hw << 1) & 0x80008000u; v[3] |= hw & 0x80008000u;
                        st128(v, rme, i * 1024 + lane * 16 < (len & 0xffffu) ? lane * 16 : 0x80000000u, so + i * 1024, AUX_SYS);
                    }
                WAIT_VM(0);
                sel[4 + pass * 4 + wave] = len | 0x80000000u;
                const u32 l4 = sel[4 + pass * 4 + (lane & 3)];
                if (__ballot(l4 >> 31) == ~0ull) {
                    const u64 note = (u64)call << 32 | (l4 & 0x7fffffffu);
                    if (lane < 28)
                        __hip_atomic_store((u64*)(peerp + ((sidx & ~3u) + (lane & 3)) * 8), note, __ATOMIC_RELAXED, __HIP_MEMORY_SCOPE_SYSTEM);
                }
                STAMP(3 + 3 * pass);
                const u32 obase = (glob_blk(g, bi, lr) * 128 + wm * 64) * (NN * 2) + (glob_blk(g, bj, lc) * 128 + wn * 64) * 2;
#pragma unroll
                for (int gi = 0; gi < 8; ++gi)
                    st_out(o[gi], ro, olane, obase + (gi >> 1) * 16 * (NN * 2) + (gi & 1) * 64);
                WAIT_LGKM0();
            }
        }
    }
}

constexpr double mx(double a, double b) { return a > b ? a : b; }
constexpr Orders build_orders() {
    Orders P{};
    const char* eo = ENTRY_ORDER;
    double t[3][16] = {};
    for (int e = 0; e < 32; ++e) {
        const int kind = eo[2 * e] == 'w' ? 0 : eo[2 * e] == 's' ? 1 : 2;
        const char ch = eo[2 * e + 1];
        const int idx = ch <= '9' ? ch - '0' : ch - 'A' + 10;
        P.entry[e >> 3] |= (u64)(kind << 4 | idx) << ((e & 7) * 8);
        t[kind][idx] = (e + 1) * 1.445;
    }
    double av[128] = {};
    for (int i = 0; i < 128; ++i) {
        const int jt = i >> 4, ct = i & 15;
        av[i] = mx(mx(t[2][jt], t[0][ct]) + LAT_K, mx(mx(t[1][jt & ~1], t[1][jt | 1]), mx(t[0][ct & ~1], t[0][ct | 1])) + LAT_S);
    }
    for (int i = 0; i < 128; ++i) {
        int pos = 0;
        for (int k = 0; k < 128; ++k) pos += av[k] < av[i] || (av[k] == av[i] && k < i);
        P.ord[i >> 2] |= (u32)pos << ((i & 3) * 8);
        P.ord[32 + (pos >> 2)] |= (u32)i << ((pos & 3) * 8);
    }
    const int W = NCWG / 8, D = 32 - W;
    for (int x = 0; x < 8; ++x) {
        int id[32] = {}, sorted[32] = {};
        double key[32] = {};
        for (int i = 0; i < 32; ++i) {
            const int keep = i >> 4, jt = (x >> 1) + 4 * ((i >> 3) & 1), ct = (i & 7) * 2 + (x & 1);
            id[i] = keep << 7 | jt << 4 | ct;
            key[i] = keep ? av[jt * 16 + ct] - LAT_K : mx(t[1][jt], t[0][ct]);
        }
        for (int i = 0; i < 32; ++i) {
            int pos = 0;
            for (int k = 0; k < 32; ++k) pos += key[k] < key[i] || (key[k] == key[i] && k < i);
            sorted[pos] = id[i];
        }
        for (int i = 0; i < W; ++i) P.tiles[i * 8 + x] = (unsigned short)(sorted[i] | sorted[i < D ? W + i : i] << 8);
    }
    return P;
}
constexpr Orders ORDERS = build_orders();

extern "C" void kernel_launch(int rank, int nranks, ncclComm_t comm,
                              hipStream_t stream,
                              void* const* d_in, const int* in_sizes, int n_in,
                              void* d_out, int out_size,
                              const int* in_dtypes, int out_dtype,
                              void* const* d_out_all,
                              void* const* d_ws_all, size_t ws_bytes) {
    if (nranks != NR || n_in < 2 || ws_bytes < WS_USED || (unsigned)rank >= (unsigned)NR ||
        in_sizes[0] != MM * KS || in_sizes[1] != KS * NN || out_size != MM * NN) {
        fprintf(stderr, "unexpected problem shape\n");
        return;
    }
    Params P;
    P.x = d_in[0];
    P.w = d_in[1];
    for (int i = 0; i < NR; ++i) P.ws[i] = d_ws_all[i];
    P.out = d_out;
    P.rank = rank;
    P.o = ORDERS;
    HIP_CHECK(hipFuncSetAttribute((const void*)fused_gemm_ar, hipFuncAttributeMaxDynamicSharedMemorySize, LDS_BYTES));
    hipLaunchKernelGGL(fused_gemm_ar, dim3(GRID), dim3(512), LDS_BYTES, stream, P);
    HIP_CHECK(hipGetLastError());
}
```

```cpp
#include <hip/hip_runtime.h>
#include <rccl/rccl.h>
#include <cstdio>
#include <cstdint>

#define HIP_CHECK(expr) do { hipError_t _e = (expr); if (_e != hipSuccess) { \
    fprintf(stderr, "HIP error %s at %s:%d\n", hipGetErrorString(_e), __FILE__, __LINE__); \
    return; } } while (0)

#define XS 16
#define XNP 2
#define KSLEEP 8
#define WSLEEP 1
#define LOOK 6
#define LOOK2 4
#define XE 48
#define XH 16
#define NPUSHW 2
#define SDELAY 20
#define PWIN 3
#define WIN 5
#define SELJ 5
#define DIAG 0
#define LAT_K 12.0
#define LAT_S 19.0
#define ENTRY_ORDER "s0w0w1w2s1w3w4s2k0w5k1s3s4k2s5w6k3w7s6s7w8w9wAwBwCk4wDk5wEk6k7wF"
#define ESCALE 1.34
#define SFIRST 64
#define OUT_AUX 17
#define PSLEEP 32

typedef unsigned int u32;
typedef unsigned long long u64;
typedef u32 u32x4 __attribute__((ext_vector_type(4)));
typedef u32 u32x2 __attribute__((ext_vector_type(2)));
typedef float f32x4 __attribute__((ext_vector_type(4)));
typedef short bf16x8 __attribute__((ext_vector_type(8)));
typedef __bf16 bf16v2 __attribute__((ext_vector_type(2)));
typedef float f32x2 __attribute__((ext_vector_type(2)));
#define LAS __attribute__((address_space(3)))

constexpr int NR = 8;
constexpr int MM = 4096, NN = 4096, KS = 512;
constexpr u32 BLK = 16384;
constexpr u32 OFF_XA = 0;
constexpr u32 OFF_WB = 8u << 20;
constexpr u32 OFF_PART = 16u << 20;
constexpr u32 OFF_SPART = 20u << 20;
constexpr u32 CSTR = 9216;
constexpr u32 CMAX = 8704;
constexpr u32 OFF_CMP = 24u << 20;
constexpr u32 OFF_STG = 30u << 20;
constexpr u32 OFF_FLAGS = 38u << 20;
constexpr u32 OFF_FX = OFF_FLAGS;
constexpr u32 OFF_FW = OFF_FX + 4096;
constexpr u32 OFF_FP = OFF_FW + 4096;
constexpr u32 OFF_NP = OFF_FP + 4096;
constexpr u32 OFF_NT = OFF_NP + 2048;
constexpr u32 OFF_CTR = OFF_NT + 81920;
constexpr u32 OFF_EP = OFF_CTR + 4096;
constexpr u32 OFF_TMO = OFF_EP + 8192;
constexpr u32 OFF_DIAG = OFF_TMO + 4096;
constexpr u32 WS_USED = OFF_DIAG + (1u << 20);
constexpr u32 OUT_BYTES = (u32)MM * NN * 2;
constexpr u32 IN_BYTES = (u32)MM * KS * 2;

constexpr int EWG = 8;
constexpr int NPWG = 7 * EWG;
constexpr int NPUSH = EWG * NPUSHW;
constexpr int NCWG = 192;
constexpr int GRID = 256;
constexpr u32 LDS_CTL = 1024;
constexpr int LDS_BYTES = LDS_CTL + 4 * 32768 + (DIAG ? 1024 : 0);
constexpr u64 DEADLINE = 100000000ull;
constexpr int AUX_SYS = 17;
constexpr int AUX_SC1 = 16;
static_assert(GRID * 8 * 4 <= 8192, "epoch words");
static_assert(OFF_CMP + 512u * CSTR <= OFF_STG, "layout");
static_assert(14 * CSTR <= 4 * 32768, "engine slots");
static_assert((GRID - NPWG - SFIRST) * 8 >= 1024 - 2 * NPUSH && SFIRST >= NPWG, "staging waves");

struct Orders {
    u64 entry[4];
    unsigned short tiles[NCWG];
    u32 ord[64];
};
struct Params {
    const void* x;
    const void* w;
    void* ws[NR];
    void* out;
    int rank;
    Orders o;
};

typedef __amdgpu_buffer_rsrc_t rsrc_t;
__device__ __forceinline__ rsrc_t mk_rsrc(const void* p, u32 bytes) {
    return __builtin_amdgcn_make_buffer_rsrc((void*)p, (short)0, (int)bytes, 0x00020000);
}
__device__ __forceinline__ u32 rfl(u32 v) { return (u32)__builtin_amdgcn_readfirstlane((int)v); }
__device__ __forceinline__ void* pick(void* const (&t)[NR], u32 idx) {
    void* r = t[0];
#pragma unroll
    for (u32 i = 1; i < NR; ++i) r = (idx == i) ? t[i] : r;
    return r;
}
__device__ __forceinline__ u32 glob_blk(u32 g, u32 b, u32 l) {
    return g == 0 ? b * 16 + l : ((l >> 3) * 2 + b) * 8 + (l & 7);
}
__device__ __forceinline__ u32 perm8(u32 t, u32 h) { return (t >> 1) + ((t ^ h) & 1) * 4; }
__device__ __forceinline__ u32 perm16(u32 t, u32 h) { return (t >> 1) + ((t ^ h) & 1) * 8; }
__device__ __forceinline__ u32 row_lr(u32 j, u32 fin) { return (j >> 2) * 8 + fin * 4 + (j & 3); }

__device__ __forceinline__ u32 part_idx(u32 a, u32 b) { return b >= 4 ? b * 4 + a : (b > a ? b * b + a : a * a + a + b); }
__device__ __forceinline__ void part_ab(u32 m, u32& a, u32& b) {
    const u32 sh = m >= 9 ? 3 : m >= 4 ? 2 : m >= 1 ? 1 : 0, o = m - sh * sh;
    a = o < sh ? o : sh;
    b = o < sh ? sh : o - sh;
    if (m >= 16) { a = m & 3; b = m >> 2; }
}
__device__ __forceinline__ void unit_desc(const u64 (&entry)[4], u32 s, u32 d, u32 n, u32& su, u32& du) {
    const bool in = (s >> 2) == (d >> 2);
    const u32 gd = d >> 2, e = in ? n / 10 : n / 6, kk = n - e * (in ? 10 : 6);
    const u32 eb = (u32)(entry[(e >> 3) & 3] >> ((e & 7) * 8));
    const u32 ek = (eb >> 4) & 3, idx = eb & 15;
    const bool isw = ek == 0;
    const u32 l = isw ? perm16(idx, d) : row_lr(perm8(idx & 7, d >> 1), ek == 1 ? 1 - gd : gd);
    su = (isw ? 512 : 0) + glob_blk(gd, isw ? d & 1 : (d >> 1) & 1, l) * 16 + (in ? 0 : 10) + kk;
    du = (isw ? 1024 : 0) + l * 64 + (in ? (s & 3) * 10 : 40 + (s & 3) * 6) + kk;
}
__device__ __forceinline__ void fetch_unit(const void* x, const void* w, u32 su, u32 lane, char* wl, u32x4* v) {
    const u32 isw = su >> 9, GB = (su >> 4) & 31, kc = (su >> 1) & 7, hf = su & 1;
    if (isw) {
        const rsrc_t rw = mk_rsrc(w, IN_BYTES);
        const u32 vo = (lane >> 3) * 65536 + (lane & 7) * 16, so = kc * 64 * 8192 + GB * 256 + hf * 128;
#pragma unroll
        for (int i = 0; i < 8; ++i) v[i] = __builtin_amdgcn_raw_buffer_load_b128(rw, vo, so + i * 8192, 0);
        const u32 wo = (lane >> 5) * 4096 + ((lane & 7) >> 2) * 2048 + (((lane >> 3) & 3) * 16 + (lane & 3) * 4) * 16;
#pragma unroll
        for (int c = 0; c < 8; ++c) {
            u32x4 o;
#pragma unroll
            for (int t = 0; t < 4; ++t)
                o[t] = __builtin_amdgcn_perm(v[2 * t + 1][c >> 1], v[2 * t][c >> 1], (c & 1) ? 0x07060302u : 0x05040100u);
            *(u32x4*)(wl + wo + (c >> 2) * 1024 + (c & 3) * 16) = o;
        }
    } else {
        const rsrc_t rx = mk_rsrc(x, IN_BYTES);
        const u32 vo = (lane >> 3) * 1024 + (lane & 7) * 16, so = GB * 131072 + hf * 65536 + kc * 128;
#pragma unroll
        for (int i = 0; i < 8; ++i) v[i] = __builtin_amdgcn_raw_buffer_load_b128(rx, vo, so + i * 8192, 0);
        const u32 wo = ((lane & 7) >> 2) * 4096 + ((lane & 3) * 16 + (lane >> 3)) * 16;
#pragma unroll
        for (int i = 0; i < 8; ++i) *(u32x4*)(wl + wo + (i >> 1) * 1024 + (i & 1) * 128) = v[i];
    }
    asm volatile("s_waitcnt lgkmcnt(0)" ::: "memory");
#pragma unroll
    for (int i = 0; i < 8; ++i) v[i] = *(const u32x4*)(wl + i * 1024 + lane * 16);
    asm volatile("s_waitcnt lgkmcnt(0)" ::: "memory");
}
__device__ __forceinline__ void st128(u32x4 v, rsrc_t r, u32 voff, u32 soff, int) {
    __builtin_amdgcn_raw_buffer_store_b128(v, r, voff + soff, 0, AUX_SYS);
}
__device__ __forceinline__ void st_out(u32x4 v, rsrc_t r, u32 voff, u32 soff) {
    __builtin_amdgcn_raw_buffer_store_b128(v, r, voff + soff, 0, OUT_AUX);
}
__device__ __forceinline__ void st_flag(rsrc_t r, u32 l0, u32 off, u32 v) {
    __builtin_amdgcn_raw_buffer_store_b32(v, r, l0, off, AUX_SYS);
}
__device__ __forceinline__ void st_flag64(rsrc_t r, u32 l0, u32 off, u32 call, u32 len) {
    __builtin_amdgcn_raw_buffer_store_b64(u32x2{len, call}, r, l0, off, AUX_SYS);
}
__device__ __forceinline__ u64 ld_flag64(const void* ws, u32 off) {
    return __hip_atomic_load((const u64*)((const char*)ws + off), __ATOMIC_RELAXED, __HIP_MEMORY_SCOPE_SYSTEM);
}
#define CBAR() asm volatile("" ::: "memory")
#define WAIT_VM(n) asm volatile("s_waitcnt vmcnt(" #n ")" ::: "memory")
#define WAIT_VM0() do { WAIT_VM(0); __builtin_amdgcn_s_waitcnt(0x0F70); CBAR(); } while (0)
#define WAIT_LGKM0() asm volatile("s_waitcnt lgkmcnt(0)" ::: "memory")

__device__ __forceinline__ bool give_up(void* myws, u32 call, u64 t0) {
    u32* tmo = (u32*)((char*)myws + OFF_TMO);
    bool late = __builtin_amdgcn_s_memrealtime() - t0 > DEADLINE;
    if (late || rfl(__hip_atomic_load(tmo, __ATOMIC_RELAXED, __HIP_MEMORY_SCOPE_AGENT)) == call) {
        __hip_atomic_store(tmo, call, __ATOMIC_RELAXED, __HIP_MEMORY_SCOPE_AGENT);
        return true;
    }
    return false;
}
__device__ __forceinline__ u32 ld_tmo(void* myws) {
    return __hip_atomic_load((u32*)((char*)myws + OFF_TMO), __ATOMIC_RELAXED, __HIP_MEMORY_SCOPE_AGENT);
}
template <int SLP>
__device__ __forceinline__ u32 wait_flag64(void* myws, u32 off, u32 call) {
    u64 t0 = 0;
    for (u32 it = 0;; ++it) {
        u64 v = ld_flag64(myws, off);
        if (rfl((u32)(v >> 32)) >= call) return rfl((u32)v);
        if (it == 0) t0 = __builtin_amdgcn_s_memrealtime();
        if ((it & 63) == 63 && give_up(myws, call, t0)) return 0;
        __builtin_amdgcn_s_sleep(SLP);
    }
}
__device__ __forceinline__ u32 ticket(rsrc_t r, u32 l0, u32 idx) {
    return (u32)__builtin_amdgcn_raw_ptr_buffer_atomic_add_i32(1, r, l0, OFF_CTR + idx * 128, 0);
}
__device__ __forceinline__ u32 wave_excl(u32 cnt, u32& tot) {
    int x = (int)cnt;
    x += __builtin_amdgcn_update_dpp(0, x, 0x111, 0xf, 0xf, false);
    x += __builtin_amdgcn_update_dpp(0, x, 0x112, 0xf, 0xf, false);
    x += __builtin_amdgcn_update_dpp(0, x, 0x114, 0xf, 0xf, false);
    x += __builtin_amdgcn_update_dpp(0, x, 0x118, 0xf, 0xf, false);
    x += __builtin_amdgcn_update_dpp(0, x, 0x142, 0xa, 0xf, false);
    x += __builtin_amdgcn_update_dpp(0, x, 0x143, 0xc, 0xf, false);
    tot = (u32)__builtin_amdgcn_readlane(x, 63);
    return (u32)x - cnt;
}
__device__ __forceinline__ u32 hidden_words(u32 nv) { const u32 h = nv >> 5; return h > 64 && h < 80 ? 64 : h; }
__device__ __forceinline__ u32 pack2(float a, float b) {
    f32x2 v = {a, b};
    return __builtin_bit_cast(u32, __builtin_convertvector(v, bf16v2));
}
__device__ __forceinline__ float bf_lo(u32 v) { return __builtin_bit_cast(float, v << 16); }
__device__ __forceinline__ float bf_hi(u32 v) { return __builtin_bit_cast(float, v & 0xffff0000u); }

#define STAMP(slot) do {} while (0)

__global__ void __launch_bounds__(512, 2) fused_gemm_ar(Params P) {
    extern __shared__ __attribute__((aligned(16))) char smem[];
    const u32 lane = threadIdx.x & 63;
    const u32 wave = rfl(threadIdx.x >> 6);
    const u32 rank = (u32)P.rank & 7;
    const u32 g = rank >> 2, bi = (rank >> 1) & 1, bj = rank & 1;
    void* myws = pick(P.ws, rank);
    const rsrc_t rme = mk_rsrc(myws, WS_USED);
    char* wl = smem + LDS_CTL + wave * 16384;

    const u32 l0 = lane == 0 ? 0u : 0x80000000u;
    const u32 epo = OFF_EP + (blockIdx.x * 8 + wave) * 4;
    const u32 epv = __builtin_amdgcn_raw_buffer_load_b32(rme, 0, epo, AUX_SC1);
    *(LAS u32*)smem = 0u;
    STAMP(0);

    const u32 ordv = P.o.ord[lane];
    const u32 olane = (lane & 15) * (NN * 2) + (lane >> 4) * 16;
    const rsrc_t ro = mk_rsrc(P.out, OUT_BYTES);

    if (blockIdx.x < NPWG) {
        const u32 b = blockIdx.x;
        const u32 di = 1 + ((b >> 3) + (b & 7)) % 7;
        const u32 p = (rank + di) & 7;
        volatile LAS u32* ctl = (volatile LAS u32*)(smem + 64);
        const u32 dum = 640 + lane * 4;
        const u32 cra = lane == 0 ? 64 + 15 * 32 + 4 : dum;
        if (wave == 0) *(volatile LAS u32x4*)(ctl + (lane & 15) * 8) = u32x4{0u, (u32)LOOK, 0u, 0u};
        WAIT_LGKM0();
        __builtin_amdgcn_s_barrier();
        CBAR();
        if (wave == 0) {
            __builtin_amdgcn_s_setprio(3);
            WAIT_VM0();
            u32 prev = 14, pnp = 0, tsl = 0, treq = 0, tsrc = 0;
            u64 t0 = __builtin_amdgcn_s_memrealtime();
            auto select = [&](u32x2 cv) -> bool {
                int mk = (cv[0] & 1) ? (int)((cv[0] >> 16) << 4 | (lane & 15)) : 0x7fffffff;
                mk = min(mk, __builtin_amdgcn_update_dpp(0x7fffffff, mk, 0x111, 0xf, 0xf, false));
                mk = min(mk, __builtin_amdgcn_update_dpp(0x7fffffff, mk, 0x112, 0xf, 0xf, false));
                mk = min(mk, __builtin_amdgcn_update_dpp(0x7fffffff, mk, 0x114, 0xf, 0xf, false));
                mk = min(mk, __builtin_amdgcn_update_dpp(0x7fffffff, mk, 0x118, 0xf, 0xf, false));
                const u32 r = (u32)__builtin_amdgcn_readlane(mk, 15);
                if (r == 0x7fffffffu) return false;
                tsl = r & 15;
                treq = (u32)__builtin_amdgcn_readlane((int)cv[0], (int)tsl);
                tsrc = (u32)__builtin_amdgcn_readlane((int)cv[1], (int)tsl);
                ctl[tsl * 8] = 0;
                asm volatile("ds_add_u32 %0, %1" :: "v"(cra), "v"(tsrc & 1) : "memory");
                return true;
            };
            bool have = false;
            for (;;) {
                if (!have) {
                    bool end = false;
                    for (u32 it = 0;; ++it) {
                        const u32x2 cv = *(volatile LAS u32x2*)(ctl + (lane & 15) * 8);
                        if (select(cv)) { end = false; break; }
                        if (prev != 14) { WAIT_VM(0); ctl[prev * 8 + 2] = pnp; prev = 14; }
                        if (end) break;
                        end = (u32)__builtin_amdgcn_readlane((int)cv[0], 15) != 0;
                        if ((it & 1023) == 1023 && __builtin_amdgcn_s_memrealtime() - t0 > 2 * DEADLINE) end = true;
                        __builtin_amdgcn_s_sleep(1);
                    }
                    if (end) break;
                }
                const u32 sl = tsl, len = ((treq >> 8) & 255) * 64, np = (treq >> 4) & 15, src = tsrc & ~1u, wide = treq & 2, treq0 = treq;
                const rsrc_t rs = mk_rsrc(pick(P.ws, p), src + len);
                const u32 vo = src + lane * 16;
                LAS char* d = (LAS char*)smem + LDS_CTL + sl * CSTR;
                u32x2 cv = {0u, 0u};
#define PULL_STEP(j, W, ALL) \
                if ((j) == (W) - 1 && (ALL || len > ((j) - 1) * 1024u)) { \
                    asm volatile("s_waitcnt vmcnt(%0)" :: "n"((W) - 1) : "memory"); \
                    ctl[prev * 8 + 2] = pnp; \
                    prev = 14; \
                } \
                if (ALL || (j) * 1024u < len) { \
                    asm volatile("s_waitcnt vmcnt(%0)" :: "n"((W) - 1) : "memory"); \
                    __builtin_amdgcn_raw_ptr_buffer_load_lds(rs, d + ((j) >> 2) * 4096, 16, vo + ((j) >> 2) * 4096, 0, ((j) & 3) * 1024, AUX_SYS); \
                } \
                CBAR(); \
                if ((j) == SELJ - 1) cv = *(volatile LAS u32x2*)(ctl + (lane & 15) * 8); \
                if ((j) == SELJ) have = select(cv); \
                CBAR();
#define PULL_UNIT(W) PULL_STEP(0, W, 1) PULL_STEP(1, W, 1) PULL_STEP(2, W, 1) PULL_STEP(3, W, 1) \
                    PULL_STEP(4, W, 1) PULL_STEP(5, W, 1) PULL_STEP(6, W, 1) PULL_STEP(7, W, 1)
#define PULL_PAIR(W) PULL_STEP(0, W, 0) PULL_STEP(1, W, 0) PULL_STEP(2, W, 0) PULL_STEP(3, W, 0) PULL_STEP(4, W, 0) \
                    PULL_STEP(5, W, 0) PULL_STEP(6, W, 0) PULL_STEP(7, W, 0) PULL_STEP(8, W, 0)
                if (!(treq0 & 4)) {
                    if (wide) { PULL_UNIT(WIN + 1) } else { PULL_UNIT(WIN) }
                } else {
                    if (wide) { PULL_PAIR(WIN + 1) } else { PULL_PAIR(WIN) }
                }
                if (prev != 14) { WAIT_VM(0); ctl[prev * 8 + 2] = pnp; }
                prev = sl; pnp = np;
            }
            ctl[15 * 8 + 3] = 1;
            STAMP(3);
            __builtin_amdgcn_s_setprio(0);
        }
        const u32 gp = p >> 2;
        const u32 call = rfl(epv) + 1;
        st_flag(rme, l0, epo, call);
        if (wave == 1) {
            __builtin_amdgcn_s_setprio(2);
            const u32 T1 = (gp == g) ? 320 : 192;
            const u32 TH = T1 + 256 - XH, T = TH + 2 * XH;
            const u32 tbase = (call - 1) * (T - NPUSH + 2 * EWG) - NPUSH;
            u32 n = NPUSH, ne = NPUSH;
            u32 pmask = gp == g ? 0u : 0xffffu;
            u32 half = 0;
            u32 share = NPUSH;
            bool over = false, force = false;
            u64 t0 = __builtin_amdgcn_s_memrealtime();
            for (u32 it = 0;; ++it) {
                if (over && n == ne && pmask == 0) break;
                u32 k = 0;
                if (!over && n == ne) {
                    k = rfl(ctl[15 * 8 + 1]);
                    if (n + XE >= T) k = k > LOOK - LOOK2 ? k - (LOOK - LOOK2) : 0;
                    k = k > 2 ? 2 : k;
                    if (k == 0 && pmask == 0) { __builtin_amdgcn_s_sleep(1); if ((it & 4095) == 4095 && __builtin_amdgcn_s_memrealtime() - t0 > 2 * DEADLINE) break; continue; }
                    asm volatile("ds_sub_u32 %0, %1" :: "v"(cra), "v"(k) : "memory");
                }
                const u32 tv = (u32)__builtin_amdgcn_raw_ptr_buffer_atomic_add_i32((int)k, rme, k ? l0 : 0x80000000u, OFF_CTR + di * 128, 0);
                const u32 wb = n < TH ? n : TH + ((n - TH) >> 1);
                const u32x4 nv = __builtin_amdgcn_raw_buffer_load_b128(rme, lane < 47 ? OFF_NT + (p * 640 + wb + lane) * 16 :
                    lane == 47 ? OFF_TMO : OFF_NP + ((p & 3) * 32 + (lane - 48) * 2 + ((b >> 2) & 1)) * 16, 0, AUX_SYS);
                WAIT_VM0();
                if (k) {
                    n = rfl(tv) - tbase; ne = n + k;
                    share += k * EWG;
                    if (ne > T) {
                        over = true;
                        if (ne - (n > T ? n : T) < 2) __builtin_amdgcn_raw_ptr_buffer_atomic_add_i32(1, rme, l0, OFF_CTR + di * 128, 0);
                        ne = T; n = n > T ? T : n;
                    }
                }
                if ((u32)__builtin_amdgcn_readlane((int)nv[0], 47) == call) force = true;
                u64 rm = force ? ~0ull : __ballot(nv[1] >= call && nv[3] >= call);
                u32 fr = (u32)__ballot(ctl[(lane & 15) * 8 + 3] == 0) & 0x3fffu;
                bool progress = false;
                while (fr) {
                    const u32 pr = (u32)(rm >> 48) & pmask;
                    const u32 m = n < TH ? n : TH + ((n - TH) >> 1);
                    u32 src, dst, aux, aux2 = 0, req, nq = 0;
                    if (pr) {
                        const u32 i = (u32)__builtin_ctz(pr), un = (b & 7) + 8 * i;
                        pmask &= ~(1u << i);
                        u32 a, bb; part_ab(un >> 2, a, bb);
                        const u32 pu = ((((p >> 1) & 1) * 4 + a) * 16 + (p & 1) * 8 + bb) * 4 + (un & 3);
                        src = OFF_SPART + ((rank & 3) * 128 + un) * 8192;
                        dst = OFF_PART + pu * 8192; aux = 0x80000000u | (OFF_FP + pu * 8 + 4);
                        req = 1 | 1 << 4 | 128 << 8 | (15 - __builtin_popcount(pmask)) << 16;
                    } else if (n < ne && m - wb < 47 && ((rm >> (m - wb)) & 1)) {
                        if (n < T1) {
                            u32 su, du; unit_desc(P.o.entry, p, rank, n, su, du);
                            src = OFF_STG + su * 8192 + 1; dst = OFF_XA + du * 8192; aux = 0x80000000u | (OFF_FX + du * 4);
                            req = 1 | 1 << 4 | 128 << 8 | (n + 16) << 16;
                            ++n;
                        } else {
                            u32 lA = (u32)__builtin_amdgcn_readlane((int)nv[0], (int)(m - wb)), lB = (u32)__builtin_amdgcn_readlane((int)nv[2], (int)(m - wb));
                            if (force || (lA & 0xffffu) > CMAX || (lA >> 16) > 4096) lA = 512;
                            if (force || (lB & 0xffffu) > CMAX || (lB >> 16) > 4096) lB = 512;
                            const u32 n3 = m - T1, to = n3 >> 1;
                            const u32 tl = ((u32)__builtin_amdgcn_readlane((int)ordv, (int)(32 + (to >> 2)))) >> ((to & 3) * 8);
                            const u32 tRB = glob_blk(gp, (p >> 1) & 1, row_lr(perm8((tl >> 4) & 7, p >> 1), gp)), tCB = glob_blk(gp, p & 1, perm16(tl & 15, p));
                            dst = (tRB * 128 + (n3 & 1) * 64) * (NN * 2) + tCB * 256;
                            const u32 bA = lA & 0xfff0u, bB = lB & 0xfff0u, oA = (lA & 1) ? bB : 0, oB = (lA & 1) ? 0 : bA;
                            const bool onlyB = half || (n >= TH && ((n - TH) & 1));
                            half = !onlyB && n < TH && bA + bB > CSTR;
                            const bool onlyA = half || (!onlyB && n >= TH);
                            const u32 st = onlyA ? oA & ~127u : onlyB ? oB & ~127u : 0, en = onlyA ? oA + bA : onlyB ? oB + bB : bA + bB;
                            aux = onlyB ? 0u : (lA & 0xffff0000u) | (oA - st) | 1;
                            aux2 = (onlyA ? 0u : (lB & 0xffff0000u) | (oB - st) | 1) | (force ? 0x80000000u : 0u);
                            src = OFF_CMP + n3 * 2 * CSTR + st + (half ? 0 : 1);
                            nq = n + XS >= T ? 1 : 8 / XNP;
                            req = 1 | 4 | ((onlyA || onlyB ? 4 : 8) / nq) << 4 | ((en - st + 63) >> 6) << 8 | (n + 16) << 16;
                            n += 1 - half;
                        }
                    } else break;
                    const u32 sl = (u32)__builtin_ctz(fr);
                    fr &= fr - 1;
                    *(volatile LAS u32x4*)(ctl + sl * 8 + 4) = u32x4{dst, aux, aux2, nq};
                    ctl[sl * 8 + 3] = (req >> 4) & 15;
                    *(volatile LAS u32x2*)(ctl + sl * 8) = u32x2{req | (share < ne ? 2u : 0u), src};
                    progress = true;
                }
                if (progress || k) t0 = __builtin_amdgcn_s_memrealtime();
                else if ((it & 63) == 63 && give_up(myws, call, t0)) {
                    force = true;
                    if (__builtin_amdgcn_s_memrealtime() - t0 > 2 * DEADLINE) break;
                }
            }
            ctl[15 * 8] = 256;
            __builtin_amdgcn_s_setprio(0);
        }
        {
            bool end = false;
            u64 t0 = __builtin_amdgcn_s_memrealtime();
            for (u32 it = 0;; ++it) {
                const u32x2 cv = *(volatile LAS u32x2*)(ctl + (lane & 15) * 8 + 2);
                const u32 av = (u32)__ballot((int)cv[0] > 0) & 0x3fffu;
                if (av == 0) {
                    if (end) break;
                    end = (u32)__builtin_amdgcn_readlane((int)cv[1], 15) != 0;
                    if ((it & 1023) == 1023 && __builtin_amdgcn_s_memrealtime() - t0 > 3 * DEADLINE) break;
                    __builtin_amdgcn_s_sleep(WSLEEP);
                    continue;
                }
                end = false;
                const u32 sl = (u32)__builtin_ctz(av);
                u32 old;
                u32x4 dv;
                asm volatile("ds_add_rtn_u32 %0, %2, %3\n\tds_read_b128 %1, %4\n\ts_waitcnt lgkmcnt(0)" : "=&v"(old), "=&v"(dv)
                             : "v"(lane == 0 ? 64 + sl * 32 + 8 : dum), "v"(0xffffffffu), "v"(64 + sl * 32 + 16) : "memory");
                const u32 pc = rfl(old) - 1;
                if ((int)pc < 0) continue;
                STAMP(8);
                const char* sp = smem + LDS_CTL + sl * CSTR;
                const u32 dst = rfl(dv[0]), aux = rfl(dv[1]);
                const u32 dna = lane == 0 ? 64 + sl * 32 + 12 : dum;
                if (aux >> 31) {
                    u32x4 v[8];
#pragma unroll
                    for (int i = 0; i < 8; ++i) v[i] = *(const u32x4*)(sp + i * 1024 + lane * 16);
                    asm volatile("ds_sub_u32 %0, %1" :: "v"(dna), "v"(1u) : "memory");
#pragma unroll
                    for (int i = 0; i < 8; ++i) st128(v[i], rme, lane * 16, dst + i * 1024, AUX_SYS);
                    WAIT_VM0();
                    st_flag(rme, l0, aux & 0x7fffffffu, call);
                    continue;
                }
                const u32 aux2 = rfl(dv[2]), nq = rfl(dv[3]);
                const bool two = aux & aux2 & 1;
                const u32 hs = two ? pc & 1 : aux2 & 1, q0 = (two ? pc >> 1 : pc) * nq;
                const u32 lw = hs ? aux2 : aux;
                const u32 ob = dst + hs * 128 + q0 * 16 * (NN * 2);
                if (aux2 >> 31) {
#pragma unroll 1
                    for (u32 i = 0; i < 2 * nq; ++i)
                        st_out(u32x4{0x7fc07fc0u, 0x7fc07fc0u, 0x7fc07fc0u, 0x7fc07fc0u}, ro, olane, ob + (i >> 1) * 16 * (NN * 2) + (i & 1) * 64);
                } else {
                    const u32 nv = lw >> 16, sb = lw & 0xfff0u;
                    const u32 H = hidden_words(nv), eo = ((2 * nv + 3) & ~3u) - 4 * H;
                    u32 bits[2];
                    auto hidden = [&](u32 w) {
                        u32x4 v[4];
#pragma unroll
                        for (int i = 0; i < 4; ++i) v[i] = *(const u32x4*)(sp + sb + (i * H + w) * 16);
                        __builtin_amdgcn_sched_barrier(0);
                        u32 hw = 0;
#pragma unroll
                        for (int i = 0; i < 4; ++i) {
                            hw = (__builtin_amdgcn_perm(v[i][1], v[i][0], 0x07050301u) & 0x80808080u) | ((hw >> 1) & 0x7f7f7f7fu);
                            hw = (__builtin_amdgcn_perm(v[i][3], v[i][2], 0x07050301u) & 0x80808080u) | ((hw >> 1) & 0x7f7f7f7fu);
                        }
                        return hw;
                    };
                    bits[0] = *(const u32*)(sp + sb + eo + lane * 4);
                    bits[1] = *(const u32*)(sp + sb + eo + 256 + lane * 4);
                    const u32 h0 = hidden(lane);
                    bits[0] = lane < H ? h0 : bits[0];
                    if (H > 64) { const u32 h1 = hidden(64 + lane); bits[1] = 64 + lane < H ? h1 : bits[1]; }
                    u32 tot;
                    u32 addr = LDS_CTL + sl * CSTR + sb + 2 * wave_excl(__builtin_popcount(bits[0]) + __builtin_popcount(bits[1]), tot);
                    u64 bq = ((u64)bits[1] << 32 | bits[0]);
                    addr += 2 * (u32)__builtin_popcountll(bq & ((1ull << (q0 * 16)) - 1));
                    bq >>= q0 * 16;
#pragma unroll 1
                    for (u32 qq = 0; qq < nq; ++qq) {
                        const u32 b16 = (u32)bq;
                        bq >>= 16;
                        u32 e[16];
#pragma unroll
                        for (int q = 0; q < 16; ++q) {
                            const u32 t = (b16 >> q) & 1u;
                            e[q] = *(const LAS unsigned short*)((LAS char*)smem + __umul24(addr, t));
                            asm("v_lshl_add_u32 %0, %1, 1, %0" : "+v"(addr) : "v"(t));
                        }
#pragma unroll
                        for (int h = 0; h < 2; ++h)
                            st_out(u32x4{e[h * 8] | (e[h * 8 + 1] << 16), e[h * 8 + 2] | (e[h * 8 + 3] << 16),
                                         e[h * 8 + 4] | (e[h * 8 + 5] << 16), e[h * 8 + 6] | (e[h * 8 + 7] << 16)} & 0x7fff7fffu,
                                   ro, olane, ob + qq * 16 * (NN * 2) + h * 64);
                    }
                }
                asm volatile("ds_sub_u32 %0, %1" :: "v"(dna), "v"(1u) : "memory");
                STAMP(9);
            }
        }
        return;
    }

    {
        if (wave == 0) ((volatile LAS u32*)(smem + 32))[lane < 12 ? lane : 0] = 0u;
        if (blockIdx.x < 2 * NPWG && wave > 7 - NPUSHW) {
            const u32 b = blockIdx.x - NPWG, q = (rank + 1 + ((b >> 3) + (b & 7)) % 7) & 7;
            const rsrc_t rq = mk_rsrc(pick(P.ws, q), WS_USED);
            u32 su, du; unit_desc(P.o.entry, rank, q, (7 - wave) * EWG + (b & 7), su, du);
            u32x4 v[8];
            fetch_unit(P.x, P.w, su, lane, wl, v);
            STAMP(11);
#pragma unroll
            for (int i = 0; i < 8; ++i) {
                asm volatile("s_waitcnt vmcnt(%0)" :: "n"(PWIN - 1) : "memory");
                st128(v[i], rq, lane * 16, OFF_XA + du * 8192 + i * 1024, AUX_SYS);
            }
            WAIT_VM0();
            st_flag(rq, l0, OFF_FX + du * 4, rfl(epv) + 1);
            STAMP(12);
        }
        const u32 u = (blockIdx.x - NPWG - SFIRST) + (GRID - NPWG - SFIRST) * wave;
        if (blockIdx.x >= NPWG + SFIRST && u < 1024 - 2 * NPUSH) {
            const bool four = u >= 2 * NPUSH && u < 768 - 2 * NPUSH;
            const u32 hf = u & 1, other = four ? (u >> 1) & 1 : 0;
            const u32 n = u < 2 * NPUSH ? u >> 1 : four ? (u + 2 * NPUSH) >> 2 : (u >> 1) - 192 + NPUSH;
            u32 su, du;
            unit_desc(P.o.entry, rank, (g ^ other) * 4 + hf * 3, n, su, du);
            const u32 isw = su >> 9;
            const bool mine = !other && hf == (isw ? bj : bi);
            const u32 nd = (g ^ other) * 4 + (isw ? hf + 2 * (lane & 1) : 2 * hf + (lane & 1));
            char* const np = (char*)P.ws[nd];
            if (mine || n >= NPUSH) {
                for (u32 i = 0; i < wave; ++i) __builtin_amdgcn_s_sleep(SDELAY);
                u32x4 v[8];
                fetch_unit(P.x, P.w, su, lane, wl, v);
                if (n >= NPUSH) {
#pragma unroll
                    for (int i = 0; i < 8; ++i) st128(v[i], rme, lane * 16, OFF_STG + su * 8192 + i * 1024, AUX_SYS);
                    if (mine) {
#pragma unroll
                        for (int i = 0; i < 8; ++i) st128(v[i], rme, lane * 16, OFF_XA + du * 8192 + i * 1024, AUX_SYS);
                        WAIT_VM(8);
                    } else WAIT_VM(0);
                    if (lane < 4)
                        __hip_atomic_store((u64*)(np + OFF_NT + (rank * 640 + n) * 16 + (lane >> 1) * 8), (u64)(rfl(epv) + 1) << 32, __ATOMIC_RELAXED, __HIP_MEMORY_SCOPE_SYSTEM);
                    WAIT_VM(1);
                } else {
#pragma unroll
                    for (int i = 0; i < 8; ++i) st128(v[i], rme, lane * 16, OFF_XA + du * 8192 + i * 1024, AUX_SYS);
                    WAIT_VM(0);
                }
                st_flag(rme, mine ? l0 : 0x80000000u, OFF_FX + du * 4, rfl(epv) + 1);
            }
        }
        STAMP(10);
    }
    char* const peerp = (char*)P.ws[(rank + 1 + (lane >> 2)) & 7] + OFF_NT + rank * 10240 + ((((rank + 1 + (lane >> 2)) >> 2) & 1) == g ? 5120 : 3072);

    const u32 call = rfl(epv) + 1;
    const u32 wm = (wave >> 1) & 1, wn = wave & 1;

    st_flag(rme, l0, epo, call);
    if (blockIdx.x < NPWG + NCWG) {
        const u32 c = blockIdx.x - NPWG;
        const u32 tl2 = P.o.tiles[c];
        volatile LAS u32* sel = (volatile LAS u32*)(smem + 16);

        for (u32 pass = 0; pass < ((tl2 >> 8) == (tl2 & 255) ? 1u : 2u); ++pass) {
            const u32 tb = tl2 >> (pass * 8);
            const u32 keep = (tb >> 7) & 1, jt = (tb >> 4) & 7, ct = tb & 15;
            const u32 j = perm8(jt, bi);
            const u32 lc = perm16(ct, bj);
            const u32 lr = row_lr(j, keep ? g : 1 - g);

            if (wave >= 4) {
                const u32 lw = wave - 4;
                const u32 xbase = OFF_XA + lr * 32 * BLK + lw * 4096;
                const u32 wbase = OFF_WB + lc * 32 * BLK + lw * 4096;
                const u32 fo = (lane < 32) ? OFF_FX + (lr * 32 + lane) * 8 : OFF_FW + (lc * 32 + lane - 32) * 8;
                u32 pend = 0, taken = 0;
                for (int s = -4; s < 32; ++s) {
                    const bool refill = pend == 0 && s < 28;
                    if (refill && lw == 0) {
                        u64 t0 = 0;
                        for (u32 it = 0;; ++it) {
                            const u64 fv = ld_flag64(myws, fo);
                            const u64 m = __ballot((u32)fv >= call && (u32)(fv >> 32) >= call);
                            pend = (u32)m & (u32)(m >> 32) & ~taken;
                            if (pend) break;
                            if (it == 0) t0 = __builtin_amdgcn_s_memrealtime();
                            if ((it & 63) == 63 && give_up(myws, call, t0)) { pend = ~taken; break; }
                            __builtin_amdgcn_s_sleep(PSLEEP);
                        }
                        taken |= pend;
                        sel[s & 1] = pend;
                    }
                    if (s < 29) WAIT_VM(16); else if (s == 29) WAIT_VM(8); else WAIT_VM(0);
                    WAIT_LGKM0();
                    __builtin_amdgcn_s_barrier();
                    CBAR();
                    if (s < 28) {
                        if (refill) pend = rfl(sel[s & 1]);
                        const u32 so = ((u32)__builtin_ctz(pend | 0x80000000u)) * BLK;
                        pend &= pend - 1;
                        LAS char* d = (LAS char*)smem + LDS_CTL + (s & 3) * 32768 + lw * 4096;
#pragma unroll
                        for (int i = 0; i < 4; ++i)
                            __builtin_amdgcn_raw_ptr_buffer_load_lds(rme, d + i * 1024, 16, lane * 16, xbase + so + i * 1024, 0, AUX_SC1);
#pragma unroll
                        for (int i = 0; i < 4; ++i)
                            __builtin_amdgcn_raw_ptr_buffer_load_lds(rme, d + 16384 + i * 1024, 16, lane * 16, wbase + so + i * 1024, 0, AUX_SC1);
                    }
                    CBAR();
                }
                continue;
            }
            f32x4 acc[4][4];
#pragma unroll
            for (int a = 0; a < 4; ++a)
#pragma unroll
                for (int b2 = 0; b2 < 4; ++b2) acc[a][b2] = f32x4{0.f, 0.f, 0.f, 0.f};
            {
                const char* xa = smem + LDS_CTL + wm * 8192 + lane * 16;
                const char* wb = smem + LDS_CTL + 16384 + wn * 8192 + lane * 16;
                bf16x8 xA[8], wA[8], xB[8], wB[8];
                auto rd = [&](bf16x8 (&xf)[8], bf16x8 (&wf)[8], u32 slot) {
                    const u32 so = (slot & 3) * 32768;
#pragma unroll
                    for (int i = 0; i < 8; ++i) {
                        wf[i] = *(const bf16x8*)(wb + so + i * 1024);
                        xf[i] = *(const bf16x8*)(xa + so + i * 1024);
                    }
                };
                auto mm = [&](bf16x8 (&xf)[8], bf16x8 (&wf)[8]) {
#pragma unroll
                    for (int h = 0; h < 2; ++h)
#pragma unroll
                        for (int a = 0; a < 4; ++a)
#pragma unroll
                            for (int b2 = 0; b2 < 4; ++b2)
                                acc[a][b2] = __builtin_amdgcn_mfma_f32_16x16x32_bf16(wf[h * 4 + b2], xf[h * 4 + a], acc[a][b2], 0, 0, 0);
                };
                auto sched = [&]() {
#pragma unroll
                    for (int i = 0; i < 16; ++i) {
                        __builtin_amdgcn_sched_group_barrier(0x100, 1, 0);
                        __builtin_amdgcn_sched_group_barrier(0x008, 2, 0);
                    }
                };
#define BAR() do { __builtin_amdgcn_s_waitcnt(0xC07F); CBAR(); __builtin_amdgcn_s_barrier(); CBAR(); __builtin_amdgcn_sched_barrier(0); } while (0)
                BAR(); BAR(); BAR();
                BAR();
                rd(xA, wA, 0);
                for (u32 s = 0; s < 32; s += 2) {
                    BAR();
                    rd(xB, wB, s + 1);
                    mm(xA, wA);
                    sched();
                    __builtin_amdgcn_sched_barrier(0);
                    BAR();
                    rd(xA, wA, s + 2);
                    mm(xB, wB);
                    sched();
                    __builtin_amdgcn_sched_barrier(0);
                }
                WAIT_LGKM0();
            }
            STAMP(1 + 3 * pass);

            if (!keep) {
                const u32 pu = (((j >> 2) * 2 + (lc >> 3)) * 32 + part_idx(jt >> 1, ct >> 1)) * 4 + wave;
                const u32 dst = OFF_SPART + pu * 8192;
#pragma unroll
                for (int a = 0; a < 4; ++a)
#pragma unroll
                    for (int u = 0; u < 2; ++u) {
                        u32x4 o;
                        o[0] = pack2(acc[a][2 * u][0], acc[a][2 * u][1]);
                        o[1] = pack2(acc[a][2 * u][2], acc[a][2 * u][3]);
                        o[2] = pack2(acc[a][2 * u + 1][0], acc[a][2 * u + 1][1]);
                        o[3] = pack2(acc[a][2 * u + 1][2], acc[a][2 * u + 1][3]);
                        st128(o, rme, lane * 16, dst + (a * 2 + u) * 1024, AUX_SYS);
                    }
                WAIT_VM(0);
                sel[4 + pass * 4 + wave] = 0x80000000u;
                if (__ballot(sel[4 + pass * 4 + (lane & 3)] >> 31) == ~0ull)
                    st_flag64(mk_rsrc(pick(P.ws, (1 - g) * 4 + (pu >> 7)), WS_USED), lane < 2 ? lane * 8 : 0x80000000u, OFF_NP + ((rank & 3) * 32 + ((pu >> 2) & 31)) * 16, call, 0);
                STAMP(3 + 3 * pass);
            } else {
                const u32 pu = (j * 16 + lc) * 4 + wave;
                wait_flag64<KSLEEP>(myws, OFF_FP + pu * 8, call);
                STAMP(2 + 3 * pass);
                u32x4 o[8];
                u32 bits[2] = {0u, 0u};
                {
                    u32x4 pv[8];
#pragma unroll
                    for (int i = 0; i < 8; ++i) pv[i] = __builtin_amdgcn_raw_buffer_load_b128(rme, lane * 16, OFF_PART + pu * 8192 + i * 1024, AUX_SC1);
                    const bool bad = rfl(ld_tmo(myws)) == call;
#pragma unroll
                    for (int gi = 0; gi < 8; ++gi) {
                        float y[8];
#pragma unroll
                        for (int k = 0; k < 8; ++k) {
                            const u32 pq = pv[gi][k >> 1];
                            float t = acc[gi >> 1][(gi & 1) * 2 + (k >> 2)][k & 3] + ((k & 1) ? bf_hi(pq) : bf_lo(pq));
                            if (bad) t = __builtin_nanf("");
                            const bool nz = !(t <= 0.f);
                            y[k] = nz ? t : 0.f;
                            bits[gi >> 2] |= nz ? (1u << ((gi & 3) * 8 + k)) : 0u;
                        }
                        o[gi] = u32x4{pack2(y[0], y[1]), pack2(y[2], y[3]), pack2(y[4], y[5]), pack2(y[6], y[7])} & 0x7fff7fffu;
                    }
                }
                u32 tot;
                u32 addr = (u32)(LDS_CTL + wave * 16384) + 2 * wave_excl(__builtin_popcount(bits[0]) + __builtin_popcount(bits[1]), tot);
                const u32 H = hidden_words(tot), eo = ((2 * tot + 3) & ~3u) - 4 * H;
                u32 len = (eo + 512 + 15) & ~15u, off;
                asm volatile("ds_add_rtn_u32 %0, %1, %2" : "=v"(off) : "v"(lane == 0 ? 64 + pass * 8 + (wave >> 1) * 4 : LDS_CTL + wave * 16384 + 10496 + lane * 4), "v"(len) : "memory");
                u32* wt = (u32*)(wl + 9216);
                wt[lane] = bits[0];
                wt[64 + lane] = bits[1];
                *(u32*)(wl + (lane >= H ? eo + lane * 4 : 10240)) = bits[0];
                *(u32*)(wl + (64 + lane >= H ? eo + 256 + lane * 4 : 10240)) = bits[1];
#pragma unroll
                for (int gi = 0; gi < 8; ++gi)
#pragma unroll
                    for (int k = 0; k < 8; ++k) {
                        const u32 t = (bits[gi >> 2] >> ((gi & 3) * 8 + k)) & 1u;
                        const u32 d = o[gi][k >> 1];
                        *(LAS unsigned short*)((LAS char*)smem + __umul24(addr, t)) = (unsigned short)((k & 1) ? d >> 16 : d);
                        asm("v_lshl_add_u32 %0, %1, 1, %0" : "+v"(addr) : "v"(t));
                    }
                asm volatile("s_waitcnt lgkmcnt(0)" : "+v"(off) :: "memory");
                off = rfl(off);
                const u32 tl = jt * 16 + ct;
                const u32 sidx = ((((u32)__builtin_amdgcn_readlane((int)ordv, (int)(tl >> 2))) >> ((tl & 3) * 8)) & 127) * 4 + wave;
                const u32 so = OFF_CMP + (sidx & ~1u) * CSTR + off;
#pragma unroll
                for (int i = 0; i < 9; ++i)
                    if (i * 1024u < len) {
                        u32x4 v = *(const u32x4*)(wl + i * 1024 + lane * 16);
                        const u32 G = i * 64 + lane, pl = (G >= H) + (G >= 2 * H) + (G >= 3 * H);
                        const u32 hw = G < 4 * H ? wt[(G - pl * H) & 127] << (6 - pl * 2) : 0u;
                        v[0] |= __builtin_amdgcn_perm(0u, hw << 1, 0x010c000cu) & 0x80008000u;
                        v[1] |= __builtin_amdgcn_perm(0u, hw << 1, 0x030c020cu) & 0x80008000u;
                        v[2] |= __builtin_amdgcn_perm(0u, hw, 0x010c000cu) & 0x80008000u;
                        v[3] |= __builtin_amdgcn_perm(0u, hw, 0x030c020cu) & 0x80008000u;
                        st128(v, rme, i * 1024 + lane * 16 < len ? lane * 16 : 0x80000000u, so + i * 1024, AUX_SYS);
                    }
                WAIT_VM(0);
                sel[4 + pass * 4 + wave] = len | (off != 0) | tot << 16 | 0x80000000u;
                const u32 l4 = sel[4 + pass * 4 + (lane & 3)];
                if (__ballot(l4 >> 31) == ~0ull) {
                    const u64 note = (u64)call << 32 | (l4 & 0x7fffffffu);
                    if (lane < 28)
                        __hip_atomic_store((u64*)(peerp + ((sidx & ~3u) + (lane & 3)) * 8), note, __ATOMIC_RELAXED, __HIP_MEMORY_SCOPE_SYSTEM);
                }
                STAMP(3 + 3 * pass);
                const u32 obase = (glob_blk(g, bi, lr) * 128 + wm * 64) * (NN * 2) + (glob_blk(g, bj, lc) * 128 + wn * 64) * 2;
#pragma unroll
                for (int gi = 0; gi < 8; ++gi)
                    st_out(o[gi], ro, olane, obase + (gi >> 1) * 16 * (NN * 2) + (gi & 1) * 64);
                WAIT_LGKM0();
            }
        }
    }
}

constexpr double mx(double a, double b) { return a > b ? a : b; }
constexpr Orders build_orders() {
    Orders P{};
    const char* eo = ENTRY_ORDER;
    double t[3][16] = {};
    for (int e = 0; e < 32; ++e) {
        const int kind = eo[2 * e] == 'w' ? 0 : eo[2 * e] == 's' ? 1 : 2;
        const char ch = eo[2 * e + 1];
        const int idx = ch <= '9' ? ch - '0' : ch - 'A' + 10;
        P.entry[e >> 3] |= (u64)(kind << 4 | idx) << ((e & 7) * 8);
        t[kind][idx] = (e + 1) * ESCALE;
    }
    double av[128] = {};
    for (int i = 0; i < 128; ++i) {
        const int jt = i >> 4, ct = i & 15;
        av[i] = mx(mx(t[2][jt], t[0][ct]) + LAT_K, mx(t[1][jt], t[0][ct]) + LAT_S);
    }
    for (int i = 0; i < 128; ++i) {
        int pos = 0;
        for (int k = 0; k < 128; ++k) pos += av[k] < av[i] || (av[k] == av[i] && k < i);
        P.ord[i >> 2] |= (u32)pos << ((i & 3) * 8);
        P.ord[32 + (pos >> 2)] |= (u32)i << ((pos & 3) * 8);
    }
    const int W = NCWG / 8, D = 32 - W;
    for (int x = 0; x < 8; ++x) {
        int id[32] = {}, sorted[32] = {};
        double key[32] = {};
        for (int i = 0; i < 32; ++i) {
            const int keep = i >> 4, jt = (x >> 1) + 4 * ((i >> 3) & 1), ct = (i & 7) * 2 + (x & 1);
            id[i] = keep << 7 | jt << 4 | ct;
            key[i] = keep ? av[jt * 16 + ct] - LAT_K : mx(t[1][jt], t[0][ct]);
        }
        for (int i = 0; i < 32; ++i) {
            int pos = 0;
            for (int k = 0; k < 32; ++k) pos += key[k] < key[i] || (key[k] == key[i] && k < i);
            sorted[pos] = id[i];
        }
        for (int i = 0; i < W; ++i) P.tiles[i * 8 + x] = (unsigned short)(sorted[i] | sorted[i < D ? W + i : i] << 8);
    }
    return P;
}
constexpr Orders ORDERS = build_orders();

extern "C" void kernel_launch(int rank, int nranks, ncclComm_t comm,
                              hipStream_t stream,
                              void* const* d_in, const int* in_sizes, int n_in,
                              void* d_out, int out_size,
                              const int* in_dtypes, int out_dtype,
                              void* const* d_out_all,
                              void* const* d_ws_all, size_t ws_bytes) {
    if (nranks != NR || n_in < 2 || ws_bytes < WS_USED || (unsigned)rank >= (unsigned)NR ||
        in_sizes[0] != MM * KS || in_sizes[1] != KS * NN || out_size != MM * NN) {
        fprintf(stderr, "unexpected problem shape\n");
        return;
    }
    Params P;
    P.x = d_in[0];
    P.w = d_in[1];
    for (int i = 0; i < NR; ++i) P.ws[i] = d_ws_all[i];
    P.out = d_out;
    P.rank = rank;
    P.o = ORDERS;
    HIP_CHECK(hipFuncSetAttribute((const void*)fused_gemm_ar, hipFuncAttributeMaxDynamicSharedMemorySize, LDS_BYTES));
    hipLaunchKernelGGL(fused_gemm_ar, dim3(GRID), dim3(512), LDS_BYTES, stream, P);
    HIP_CHECK(hipGetLastError());
}
```

```cpp
#include <hip/hip_runtime.h>
#include <rccl/rccl.h>
#include <cstdio>
#include <cstdint>

#define HIP_CHECK(expr) do { hipError_t _e = (expr); if (_e != hipSuccess) { \
    fprintf(stderr, "HIP error %s at %s:%d\n", hipGetErrorString(_e), __FILE__, __LINE__); \
    return; } } while (0)

#define XS 16
#define XNP 2
#define KSLEEP 8
#define WSLEEP 1
#define LOOK 6
#define LOOK2 4
#define XE 48
#define XH 16
#define NPUSHW 2
#define SDELAY 20
#define PWIN 3
#define WIN 5
#define SELJ 5
#define DIAG 0
#define LAT_K 12.0
#define LAT_S 50.0
#define ENTRY_ORDER "s0w0s1w1w2w3w4w5s2k0k1s3w6w7k2s4s5k3s6s7k4k5w8w9k6k7wAwBwCwDwEwF"
#define ESCALE 1.34
#define SFIRST 72
#define OUT_AUX 17
#define PSLEEP 32

typedef unsigned int u32;
typedef unsigned long long u64;
typedef u32 u32x4 __attribute__((ext_vector_type(4)));
typedef u32 u32x2 __attribute__((ext_vector_type(2)));
typedef float f32x4 __attribute__((ext_vector_type(4)));
typedef short bf16x8 __attribute__((ext_vector_type(8)));
typedef __bf16 bf16v2 __attribute__((ext_vector_type(2)));
typedef float f32x2 __attribute__((ext_vector_type(2)));
#define LAS __attribute__((address_space(3)))

constexpr int NR = 8;
constexpr int MM = 4096, NN = 4096, KS = 512;
constexpr u32 BLK = 16384;
constexpr u32 OFF_XA = 0;
constexpr u32 OFF_WB = 8u << 20;
constexpr u32 OFF_PART = 16u << 20;
constexpr u32 OFF_SPART = 20u << 20;
constexpr u32 CSTR = 9216;
constexpr u32 CMAX = 8704;
constexpr u32 OFF_CMP = 24u << 20;
constexpr u32 OFF_STG = 30u << 20;
constexpr u32 OFF_FLAGS = 38u << 20;
constexpr u32 OFF_FX = OFF_FLAGS;
constexpr u32 OFF_FW = OFF_FX + 4096;
constexpr u32 OFF_FP = OFF_FW + 4096;
constexpr u32 OFF_NP = OFF_FP + 4096;
constexpr u32 OFF_NT = OFF_NP + 2048;
constexpr u32 OFF_CTR = OFF_NT + 81920;
constexpr u32 OFF_EP = OFF_CTR + 4096;
constexpr u32 OFF_TMO = OFF_EP + 8192;
constexpr u32 OFF_DIAG = OFF_TMO + 4096;
constexpr u32 WS_USED = OFF_DIAG + (1u << 20);
constexpr u32 OUT_BYTES = (u32)MM * NN * 2;
constexpr u32 IN_BYTES = (u32)MM * KS * 2;

constexpr int EWG = 8;
constexpr int NPWG = 7 * EWG;
constexpr int NPUSH = EWG * NPUSHW;
constexpr int NCWG = 192;
constexpr int GRID = 256;
constexpr u32 LDS_CTL = 1024;
constexpr int LDS_BYTES = LDS_CTL + 4 * 32768 + (DIAG ? 1024 : 0);
constexpr u64 DEADLINE = 100000000ull;
constexpr int AUX_SYS = 17;
constexpr int AUX_SC1 = 16;
static_assert(GRID * 8 * 4 <= 8192, "epoch words");
static_assert(OFF_CMP + 512u * CSTR <= OFF_STG, "layout");
static_assert(14 * CSTR <= 4 * 32768, "engine slots");
static_assert(NPUSHW <= 2, "first loads");
static_assert((GRID - NPWG - SFIRST) * 8 >= 1024 - 2 * NPUSH && SFIRST >= NPWG, "staging waves");

struct Orders {
    u64 entry[4];
    unsigned short tiles[NCWG];
    u32 ord[64];
};
struct Params {
    const void* x;
    const void* w;
    void* me;
    int rank;
    void* out;
    void* ws[NR];
    u32 first[GRID];
    Orders o;
};

constexpr double mx(double a, double b) { return a > b ? a : b; }
constexpr Orders build_orders() {
    Orders P{};
    const char* eo = ENTRY_ORDER;
    double t[3][16] = {};
    for (int e = 0; e < 32; ++e) {
        const int kind = eo[2 * e] == 'w' ? 0 : eo[2 * e] == 's' ? 1 : 2;
        const char ch = eo[2 * e + 1];
        const int idx = ch <= '9' ? ch - '0' : ch - 'A' + 10;
        P.entry[e >> 3] |= (u64)(kind << 4 | idx) << ((e & 7) * 8);
        t[kind][idx] = (e + 1) * ESCALE;
    }
    double av[128] = {};
    for (int i = 0; i < 128; ++i) {
        const int jt = i >> 4, ct = i & 15;
        av[i] = mx(mx(t[2][jt], t[0][ct]) + LAT_K, mx(t[1][jt], t[0][ct]) + LAT_S);
    }
    for (int i = 0; i < 128; ++i) {
        int pos = 0;
        for (int k = 0; k < 128; ++k) pos += av[k] < av[i] || (av[k] == av[i] && k < i);
        P.ord[i >> 2] |= (u32)pos << ((i & 3) * 8);
        P.ord[32 + (pos >> 2)] |= (u32)i << ((pos & 3) * 8);
    }
    const int W = NCWG / 8, D = 32 - W;
    for (int x = 0; x < 8; ++x) {
        int id[32] = {}, sorted[32] = {};
        double key[32] = {};
        for (int i = 0; i < 32; ++i) {
            const int keep = i >> 4, jt = (x >> 1) + 4 * ((i >> 3) & 1), ct = (i & 7) * 2 + (x & 1);
            id[i] = keep << 7 | jt << 4 | ct;
            key[i] = keep ? av[jt * 16 + ct] - LAT_K : mx(t[1][jt], t[0][ct]);
        }
        for (int i = 0; i < 32; ++i) {
            int pos = 0;
            for (int k = 0; k < 32; ++k) pos += key[k] < key[i] || (key[k] == key[i] && k < i);
            sorted[pos] = id[i];
        }
        for (int i = 0; i < W; ++i) P.tiles[i * 8 + x] = (unsigned short)(sorted[i] | sorted[i < D ? W + i : i] << 8);
    }
    return P;
}
constexpr Orders ORDERS = build_orders();

typedef __amdgpu_buffer_rsrc_t rsrc_t;
__device__ __forceinline__ rsrc_t mk_rsrc(const void* p, u32 bytes) {
    return __builtin_amdgcn_make_buffer_rsrc((void*)p, (short)0, (int)bytes, 0x00020000);
}
__device__ __forceinline__ u32 rfl(u32 v) { return (u32)__builtin_amdgcn_readfirstlane((int)v); }
__device__ __forceinline__ void* pick(void* const (&t)[NR], u32 idx) {
    void* r = t[0];
#pragma unroll
    for (u32 i = 1; i < NR; ++i) r = (idx == i) ? t[i] : r;
    return r;
}
__host__ __device__ __forceinline__ u32 glob_blk(u32 g, u32 b, u32 l) {
    return g == 0 ? b * 16 + l : ((l >> 3) * 2 + b) * 8 + (l & 7);
}
__host__ __device__ __forceinline__ u32 perm8(u32 t, u32 h) { return (t >> 1) + ((t ^ h) & 1) * 4; }
__host__ __device__ __forceinline__ u32 perm16(u32 t, u32 h) { return (t >> 1) + ((t ^ h) & 1) * 8; }
__host__ __device__ __forceinline__ u32 row_lr(u32 j, u32 fin) { return (j >> 2) * 8 + fin * 4 + (j & 3); }

__device__ __forceinline__ u32 part_idx(u32 a, u32 b) { return b >= 4 ? b * 4 + a : (b > a ? b * b + a : a * a + a + b); }
__device__ __forceinline__ void part_ab(u32 m, u32& a, u32& b) {
    const u32 sh = m >= 9 ? 3 : m >= 4 ? 2 : m >= 1 ? 1 : 0, o = m - sh * sh;
    a = o < sh ? o : sh;
    b = o < sh ? sh : o - sh;
    if (m >= 16) { a = m & 3; b = m >> 2; }
}
__host__ __device__ __forceinline__ void unit_desc(u32 s, u32 d, u32 n, u32& su, u32& du) {
    constexpr u64 e0 = ORDERS.entry[0], e1 = ORDERS.entry[1], e2 = ORDERS.entry[2], e3 = ORDERS.entry[3];
    const bool in = (s >> 2) == (d >> 2);
    const u32 gd = d >> 2, e = in ? n / 10 : n / 6, kk = n - e * (in ? 10 : 6);
    const u32 eb = (u32)(((e & 16) ? ((e & 8) ? e3 : e2) : ((e & 8) ? e1 : e0)) >> ((e & 7) * 8));
    const u32 ek = (eb >> 4) & 3, idx = eb & 15;
    const bool isw = ek == 0;
    const u32 l = isw ? perm16(idx, d) : row_lr(perm8(idx & 7, d >> 1), ek == 1 ? 1 - gd : gd);
    su = (isw ? 512 : 0) + glob_blk(gd, isw ? d & 1 : (d >> 1) & 1, l) * 16 + (in ? 0 : 10) + kk;
    du = (isw ? 1024 : 0) + l * 64 + (in ? (s & 3) * 10 : 40 + (s & 3) * 6) + kk;
}
__host__ __device__ __forceinline__ void stage_desc(u32 u, u32& n, u32& other, u32& hf) {
    const bool four = u >= 2 * NPUSH && u < 768 - 2 * NPUSH;
    hf = u & 1; other = four ? (u >> 1) & 1 : 0;
    n = u < 2 * NPUSH ? u >> 1 : four ? (u + 2 * NPUSH) >> 2 : (u >> 1) - 192 + NPUSH;
}
__device__ __forceinline__ void fetch_issue(const void* x, const void* w, u32 su, u32 lane, u32x4* v) {
    const u32 isw = su >> 9, GB = (su >> 4) & 31, kc = (su >> 1) & 7, hf = su & 1;
    const rsrc_t r = mk_rsrc(isw ? w : x, IN_BYTES);
    const u32 vo = isw ? (lane >> 3) * 65536 + (lane & 7) * 16 : (lane & 15) * 1024 + (lane >> 4) * 16;
    const u32 so = isw ? kc * 64 * 8192 + GB * 256 + hf * 128 : GB * 131072 + hf * 65536 + kc * 128;
#pragma unroll
    for (int i = 0; i < 8; ++i) v[i] = __builtin_amdgcn_raw_buffer_load_b128(r, vo, so + (isw ? i * 8192 : (i & 3) * 16384 + (i >> 2) * 64), 0);
}
__device__ __forceinline__ void fetch_finish(u32 su, u32 lane, char* wl, u32x4* v) {
    if (su >> 9) {
        const u32 wo = (lane >> 5) * 4096 + ((lane & 7) >> 2) * 2048 + (((lane >> 3) & 3) * 16 + (lane & 3) * 4) * 16;
#pragma unroll
        for (int c = 0; c < 8; ++c) {
            u32x4 o;
#pragma unroll
            for (int t = 0; t < 4; ++t)
                o[t] = __builtin_amdgcn_perm(v[2 * t + 1][c >> 1], v[2 * t][c >> 1], (c & 1) ? 0x07060302u : 0x05040100u);
            *(u32x4*)(wl + wo + (c >> 2) * 1024 + (c & 3) * 16) = o;
        }
        asm volatile("s_waitcnt lgkmcnt(0)" ::: "memory");
#pragma unroll
        for (int i = 0; i < 8; ++i) v[i] = *(const u32x4*)(wl + i * 1024 + lane * 16);
        asm volatile("s_waitcnt lgkmcnt(0)" ::: "memory");
    }
}
__device__ __forceinline__ void st128(u32x4 v, rsrc_t r, u32 voff, u32 soff, int) {
    __builtin_amdgcn_raw_buffer_store_b128(v, r, voff + soff, 0, AUX_SYS);
}
__device__ __forceinline__ void st_out(u32x4 v, rsrc_t r, u32 voff, u32 soff) {
    __builtin_amdgcn_raw_buffer_store_b128(v, r, voff + soff, 0, OUT_AUX);
}
__device__ __forceinline__ void st_flag(rsrc_t r, u32 l0, u32 off, u32 v) {
    __builtin_amdgcn_raw_buffer_store_b32(v, r, l0, off, AUX_SYS);
}
__device__ __forceinline__ void st_flag64(rsrc_t r, u32 l0, u32 off, u32 call, u32 len) {
    __builtin_amdgcn_raw_buffer_store_b64(u32x2{len, call}, r, l0, off, AUX_SYS);
}
__device__ __forceinline__ u64 ld_flag64(const void* ws, u32 off) {
    return __hip_atomic_load((const u64*)((const char*)ws + off), __ATOMIC_RELAXED, __HIP_MEMORY_SCOPE_SYSTEM);
}
#define CBAR() asm volatile("" ::: "memory")
#define WAIT_VM(n) asm volatile("s_waitcnt vmcnt(" #n ")" ::: "memory")
#define WAIT_VM0() do { WAIT_VM(0); __builtin_amdgcn_s_waitcnt(0x0F70); CBAR(); } while (0)
#define WAIT_LGKM0() asm volatile("s_waitcnt lgkmcnt(0)" ::: "memory")

__device__ __forceinline__ bool give_up(void* myws, u32 call, u64 t0) {
    u32* tmo = (u32*)((char*)myws + OFF_TMO);
    bool late = __builtin_amdgcn_s_memrealtime() - t0 > DEADLINE;
    if (late || rfl(__hip_atomic_load(tmo, __ATOMIC_RELAXED, __HIP_MEMORY_SCOPE_AGENT)) == call) {
        __hip_atomic_store(tmo, call, __ATOMIC_RELAXED, __HIP_MEMORY_SCOPE_AGENT);
        return true;
    }
    return false;
}
__device__ __forceinline__ u32 ld_tmo(void* myws) {
    return __hip_atomic_load((u32*)((char*)myws + OFF_TMO), __ATOMIC_RELAXED, __HIP_MEMORY_SCOPE_AGENT);
}
template <int SLP>
__device__ __forceinline__ u32 wait_flag64(void* myws, u32 off, u32 call) {
    u64 t0 = 0;
    for (u32 it = 0;; ++it) {
        u64 v = ld_flag64(myws, off);
        if (rfl((u32)(v >> 32)) >= call) return rfl((u32)v);
        if (it == 0) t0 = __builtin_amdgcn_s_memrealtime();
        if ((it & 63) == 63 && give_up(myws, call, t0)) return 0;
        __builtin_amdgcn_s_sleep(SLP);
    }
}
__device__ __forceinline__ u32 ticket(rsrc_t r, u32 l0, u32 idx) {
    return (u32)__builtin_amdgcn_raw_ptr_buffer_atomic_add_i32(1, r, l0, OFF_CTR + idx * 128, 0);
}
__device__ __forceinline__ u32 wave_excl(u32 cnt, u32& tot) {
    int x = (int)cnt;
    x += __builtin_amdgcn_update_dpp(0, x, 0x111, 0xf, 0xf, false);
    x += __builtin_amdgcn_update_dpp(0, x, 0x112, 0xf, 0xf, false);
    x += __builtin_amdgcn_update_dpp(0, x, 0x114, 0xf, 0xf, false);
    x += __builtin_amdgcn_update_dpp(0, x, 0x118, 0xf, 0xf, false);
    x += __builtin_amdgcn_update_dpp(0, x, 0x142, 0xa, 0xf, false);
    x += __builtin_amdgcn_update_dpp(0, x, 0x143, 0xc, 0xf, false);
    tot = (u32)__builtin_amdgcn_readlane(x, 63);
    return (u32)x - cnt;
}
__device__ __forceinline__ u32 hidden_words(u32 nv) { const u32 h = nv >> 5; return h > 64 && h < 80 ? 64 : h; }
__device__ __forceinline__ u32 pack2(float a, float b) {
    f32x2 v = {a, b};
    return __builtin_bit_cast(u32, __builtin_convertvector(v, bf16v2));
}
__device__ __forceinline__ float bf_lo(u32 v) { return __builtin_bit_cast(float, v << 16); }
__device__ __forceinline__ float bf_hi(u32 v) { return __builtin_bit_cast(float, v & 0xffff0000u); }

#define STAMP(slot) do {} while (0)

__global__ void __launch_bounds__(512, 2) fused_gemm_ar(Params P) {
    extern __shared__ __attribute__((aligned(16))) char smem[];
    const u32 lane = threadIdx.x & 63;
    const u32 wave = rfl(threadIdx.x >> 6);
    const void* px = P.x;
    const void* pw = P.w;
    void* myws = P.me;
    u32 rk = (u32)P.rank, fw = P.first[blockIdx.x];
    asm volatile("" : "+s"(px), "+s"(pw), "+s"(myws), "+s"(rk), "+s"(fw));
    u32x4 v[8];
    const u32 su0 = wave < 2 ? (fw >> (wave * 16)) & 0xffffu : 0xffffu;
    if (su0 != 0xffffu) fetch_issue(px, pw, su0, lane, v);
    __builtin_amdgcn_sched_barrier(0);
    const u32 rank = rk & 7;
    const u32 g = rank >> 2, bi = (rank >> 1) & 1, bj = rank & 1;
    const u32 lb = (blockIdx.x + NPWG) & (GRID - 1);
    const rsrc_t rme = mk_rsrc(myws, WS_USED);
    char* wl = smem + LDS_CTL + wave * 16384;

    const u32 l0 = lane == 0 ? 0u : 0x80000000u;
    const u32 epo = OFF_EP + (lb * 8 + wave) * 4;
    const u32 epv = __builtin_amdgcn_raw_buffer_load_b32(rme, 0, epo, AUX_SC1);
    *(LAS u32*)smem = 0u;
    STAMP(0);

    const u32 olane = (lane & 15) * (NN * 2) + (lane >> 4) * 16;
    const rsrc_t ro = mk_rsrc(P.out, OUT_BYTES);

    if (lb < NPWG) {
        const u32 b = lb;
        const u32 di = 1 + ((b >> 3) + (b & 7)) % 7;
        const u32 p = (rank + di) & 7;
        volatile LAS u32* ctl = (volatile LAS u32*)(smem + 64);
        const u32 dum = 640 + lane * 4;
        const u32 cra = lane == 0 ? 64 + 15 * 32 + 4 : dum;
        if (wave == 0) *(volatile LAS u32x4*)(ctl + (lane & 15) * 8) = u32x4{0u, (u32)LOOK, 0u, 0u};
        WAIT_LGKM0();
        __builtin_amdgcn_s_barrier();
        CBAR();
        if (wave == 0) {
            __builtin_amdgcn_s_setprio(3);
            WAIT_VM0();
            u32 prev = 14, pnp = 0, tsl = 0, treq = 0, tsrc = 0;
            u64 t0 = __builtin_amdgcn_s_memrealtime();
            auto select = [&](u32x2 cv) -> bool {
                int mk = (cv[0] & 1) ? (int)((cv[0] >> 16) << 4 | (lane & 15)) : 0x7fffffff;
                mk = min(mk, __builtin_amdgcn_update_dpp(0x7fffffff, mk, 0x111, 0xf, 0xf, false));
                mk = min(mk, __builtin_amdgcn_update_dpp(0x7fffffff, mk, 0x112, 0xf, 0xf, false));
                mk = min(mk, __builtin_amdgcn_update_dpp(0x7fffffff, mk, 0x114, 0xf, 0xf, false));
                mk = min(mk, __builtin_amdgcn_update_dpp(0x7fffffff, mk, 0x118, 0xf, 0xf, false));
                const u32 r = (u32)__builtin_amdgcn_readlane(mk, 15);
                if (r == 0x7fffffffu) return false;
                tsl = r & 15;
                treq = (u32)__builtin_amdgcn_readlane((int)cv[0], (int)tsl);
                tsrc = (u32)__builtin_amdgcn_readlane((int)cv[1], (int)tsl);
                ctl[tsl * 8] = 0;
                asm volatile("ds_add_u32 %0, %1" :: "v"(cra), "v"(tsrc & 1) : "memory");
                return true;
            };
            bool have = false;
            for (;;) {
                if (!have) {
                    bool end = false;
                    for (u32 it = 0;; ++it) {
                        const u32x2 cv = *(volatile LAS u32x2*)(ctl + (lane & 15) * 8);
                        if (select(cv)) { end = false; break; }
                        if (prev != 14) { WAIT_VM(0); ctl[prev * 8 + 2] = pnp; prev = 14; }
                        if (end) break;
                        end = (u32)__builtin_amdgcn_readlane((int)cv[0], 15) != 0;
                        if ((it & 1023) == 1023 && __builtin_amdgcn_s_memrealtime() - t0 > 2 * DEADLINE) end = true;
                        __builtin_amdgcn_s_sleep(1);
                    }
                    if (end) break;
                }
                const u32 sl = tsl, len = ((treq >> 8) & 255) * 64, np = (treq >> 4) & 15, src = tsrc & ~1u, wide = treq & 2, treq0 = treq;
                const rsrc_t rs = mk_rsrc(pick(P.ws, p), src + len);
                const u32 vo = src + lane * 16;
                LAS char* d = (LAS char*)smem + LDS_CTL + sl * CSTR;
                u32x2 cv = {0u, 0u};
#define PULL_STEP(j, W, ALL) \
                if ((j) == (W) - 1 && (ALL || len > ((j) - 1) * 1024u)) { \
                    asm volatile("s_waitcnt vmcnt(%0)" :: "n"((W) - 1) : "memory"); \
                    ctl[prev * 8 + 2] = pnp; \
                    prev = 14; \
                } \
                if (ALL || (j) * 1024u < len) { \
                    asm volatile("s_waitcnt vmcnt(%0)" :: "n"((W) - 1) : "memory"); \
                    __builtin_amdgcn_raw_ptr_buffer_load_lds(rs, d + ((j) >> 2) * 4096, 16, vo + ((j) >> 2) * 4096, 0, ((j) & 3) * 1024, AUX_SYS); \
                } \
                CBAR(); \
                if ((j) == SELJ - 1) cv = *(volatile LAS u32x2*)(ctl + (lane & 15) * 8); \
                if ((j) == SELJ) have = select(cv); \
                CBAR();
#define PULL_UNIT(W) PULL_STEP(0, W, 1) PULL_STEP(1, W, 1) PULL_STEP(2, W, 1) PULL_STEP(3, W, 1) \
                    PULL_STEP(4, W, 1) PULL_STEP(5, W, 1) PULL_STEP(6, W, 1) PULL_STEP(7, W, 1)
#define PULL_PAIR(W) PULL_STEP(0, W, 0) PULL_STEP(1, W, 0) PULL_STEP(2, W, 0) PULL_STEP(3, W, 0) PULL_STEP(4, W, 0) \
                    PULL_STEP(5, W, 0) PULL_STEP(6, W, 0) PULL_STEP(7, W, 0) PULL_STEP(8, W, 0)
                if (!(treq0 & 4)) {
                    if (wide) { PULL_UNIT(WIN + 1) } else { PULL_UNIT(WIN) }
                } else {
                    if (wide) { PULL_PAIR(WIN + 1) } else { PULL_PAIR(WIN) }
                }
                if (prev != 14) { WAIT_VM(0); ctl[prev * 8 + 2] = pnp; }
                prev = sl; pnp = np;
            }
            ctl[15 * 8 + 3] = 1;
            STAMP(3);
            __builtin_amdgcn_s_setprio(0);
        }
        const u32 gp = p >> 2;
        const u32 call = rfl(epv) + 1;
        st_flag(rme, l0, epo, call);
        if (wave == 1) {
            __builtin_amdgcn_s_setprio(2);
            const u32 ordv = P.o.ord[lane];
            const u32 T1 = (gp == g) ? 320 : 192;
            const u32 TH = T1 + 256 - XH, T = TH + 2 * XH;
            const u32 tbase = (call - 1) * (T - NPUSH + 2 * EWG) - NPUSH;
            u32 n = NPUSH, ne = NPUSH;
            u32 pmask = gp == g ? 0u : 0xffffu;
            u32 half = 0;
            u32 share = NPUSH;
            bool over = false, force = false;
            u64 t0 = __builtin_amdgcn_s_memrealtime();
            for (u32 it = 0;; ++it) {
                if (over && n == ne && pmask == 0) break;
                u32 k = 0;
                if (!over && n == ne) {
                    k = rfl(ctl[15 * 8 + 1]);
                    if (n + XE >= T) k = k > LOOK - LOOK2 ? k - (LOOK - LOOK2) : 0;
                    k = k > 2 ? 2 : k;
                    if (k == 0 && pmask == 0) { __builtin_amdgcn_s_sleep(1); if ((it & 4095) == 4095 && __builtin_amdgcn_s_memrealtime() - t0 > 2 * DEADLINE) break; continue; }
                    asm volatile("ds_sub_u32 %0, %1" :: "v"(cra), "v"(k) : "memory");
                }
                const u32 tv = (u32)__builtin_amdgcn_raw_ptr_buffer_atomic_add_i32((int)k, rme, k ? l0 : 0x80000000u, OFF_CTR + di * 128, 0);
                const u32 wb = n < TH ? n : TH + ((n - TH) >> 1);
                const u32x4 nv = __builtin_amdgcn_raw_buffer_load_b128(rme, lane < 47 ? OFF_NT + (p * 640 + wb + lane) * 16 :
                    lane == 47 ? OFF_TMO : OFF_NP + ((p & 3) * 32 + (lane - 48) * 2 + ((b >> 2) & 1)) * 16, 0, AUX_SYS);
                WAIT_VM0();
                if (k) {
                    n = rfl(tv) - tbase; ne = n + k;
                    share += k * EWG;
                    if (ne > T) {
                        over = true;
                        if (ne - (n > T ? n : T) < 2) __builtin_amdgcn_raw_ptr_buffer_atomic_add_i32(1, rme, l0, OFF_CTR + di * 128, 0);
                        ne = T; n = n > T ? T : n;
                    }
                }
                if ((u32)__builtin_amdgcn_readlane((int)nv[0], 47) == call) force = true;
                u64 rm = force ? ~0ull : __ballot(nv[1] >= call && nv[3] >= call);
                u32 fr = (u32)__ballot(ctl[(lane & 15) * 8 + 3] == 0) & 0x3fffu;
                bool progress = false;
                while (fr) {
                    const u32 pr = (u32)(rm >> 48) & pmask;
                    const u32 m = n < TH ? n : TH + ((n - TH) >> 1);
                    u32 src, dst, aux, aux2 = 0, req, nq = 0;
                    if (pr) {
                        const u32 i = (u32)__builtin_ctz(pr), un = (b & 7) + 8 * i;
                        pmask &= ~(1u << i);
                        u32 a, bb; part_ab(un >> 2, a, bb);
                        const u32 pu = ((((p >> 1) & 1) * 4 + a) * 16 + (p & 1) * 8 + bb) * 4 + (un & 3);
                        src = OFF_SPART + ((rank & 3) * 128 + un) * 8192;
                        dst = OFF_PART + pu * 8192; aux = 0x80000000u | (OFF_FP + pu * 8 + 4);
                        req = 1 | 1 << 4 | 128 << 8 | (15 - __builtin_popcount(pmask)) << 16;
                    } else if (n < ne && m - wb < 47 && ((rm >> (m - wb)) & 1)) {
                        if (n < T1) {
                            u32 su, du; unit_desc(p, rank, n, su, du);
                            src = OFF_STG + su * 8192 + 1; dst = OFF_XA + du * 8192; aux = 0x80000000u | (OFF_FX + du * 4);
                            req = 1 | 1 << 4 | 128 << 8 | (n + 16) << 16;
                            ++n;
                        } else {
                            u32 lA = (u32)__builtin_amdgcn_readlane((int)nv[0], (int)(m - wb)), lB = (u32)__builtin_amdgcn_readlane((int)nv[2], (int)(m - wb));
                            if (force || (lA & 0xffffu) > CMAX || (lA >> 16) > 4096) lA = 512;
                            if (force || (lB & 0xffffu) > CMAX || (lB >> 16) > 4096) lB = 512;
                            const u32 n3 = m - T1, to = n3 >> 1;
                            const u32 tl = ((u32)__builtin_amdgcn_readlane((int)ordv, (int)(32 + (to >> 2)))) >> ((to & 3) * 8);
                            const u32 tRB = glob_blk(gp, (p >> 1) & 1, row_lr(perm8((tl >> 4) & 7, p >> 1), gp)), tCB = glob_blk(gp, p & 1, perm16(tl & 15, p));
                            dst = (tRB * 128 + (n3 & 1) * 64) * (NN * 2) + tCB * 256;
                            const u32 bA = lA & 0xfff0u, bB = lB & 0xfff0u, oA = (lA & 1) ? bB : 0, oB = (lA & 1) ? 0 : bA;
                            const bool onlyB = half || (n >= TH && ((n - TH) & 1));
                            half = !onlyB && n < TH && bA + bB > CSTR;
                            const bool onlyA = half || (!onlyB && n >= TH);
                            const u32 st = onlyA ? oA & ~127u : onlyB ? oB & ~127u : 0, en = onlyA ? oA + bA : onlyB ? oB + bB : bA + bB;
                            aux = onlyB ? 0u : (lA & 0xffff0000u) | (oA - st) | 1;
                            aux2 = (onlyA ? 0u : (lB & 0xffff0000u) | (oB - st) | 1) | (force ? 0x80000000u : 0u);
                            src = OFF_CMP + n3 * 2 * CSTR + st + (half ? 0 : 1);
                            nq = n + XS >= T ? 1 : 8 / XNP;
                            req = 1 | 4 | ((onlyA || onlyB ? 4 : 8) / nq) << 4 | ((en - st + 63) >> 6) << 8 | (n + 16) << 16;
                            n += 1 - half;
                        }
                    } else break;
                    const u32 sl = (u32)__builtin_ctz(fr);
                    fr &= fr - 1;
                    *(volatile LAS u32x4*)(ctl + sl * 8 + 4) = u32x4{dst, aux, aux2, nq};
                    ctl[sl * 8 + 3] = (req >> 4) & 15;
                    *(volatile LAS u32x2*)(ctl + sl * 8) = u32x2{req | (share < ne ? 2u : 0u), src};
                    progress = true;
                }
                if (progress || k) t0 = __builtin_amdgcn_s_memrealtime();
                else if ((it & 63) == 63 && give_up(myws, call, t0)) {
                    force = true;
                    if (__builtin_amdgcn_s_memrealtime() - t0 > 2 * DEADLINE) break;
                }
            }
            ctl[15 * 8] = 256;
            __builtin_amdgcn_s_setprio(0);
        }
        {
            bool end = false;
            u64 t0 = __builtin_amdgcn_s_memrealtime();
            for (u32 it = 0;; ++it) {
                const u32x2 cv = *(volatile LAS u32x2*)(ctl + (lane & 15) * 8 + 2);
                const u32 av = (u32)__ballot((int)cv[0] > 0) & 0x3fffu;
                if (av == 0) {
                    if (end) break;
                    end = (u32)__builtin_amdgcn_readlane((int)cv[1], 15) != 0;
                    if ((it & 1023) == 1023 && __builtin_amdgcn_s_memrealtime() - t0 > 3 * DEADLINE) break;
                    __builtin_amdgcn_s_sleep(WSLEEP);
                    continue;
                }
                end = false;
                const u32 sl = (u32)__builtin_ctz(av);
                u32 old;
                u32x4 dv;
                asm volatile("ds_add_rtn_u32 %0, %2, %3\n\tds_read_b128 %1, %4\n\ts_waitcnt lgkmcnt(0)" : "=&v"(old), "=&v"(dv)
                             : "v"(lane == 0 ? 64 + sl * 32 + 8 : dum), "v"(0xffffffffu), "v"(64 + sl * 32 + 16) : "memory");
                const u32 pc = rfl(old) - 1;
                if ((int)pc < 0) continue;
                STAMP(8);
                const char* sp = smem + LDS_CTL + sl * CSTR;
                const u32 dst = rfl(dv[0]), aux = rfl(dv[1]);
                const u32 dna = lane == 0 ? 64 + sl * 32 + 12 : dum;
                if (aux >> 31) {
                    u32x4 v[8];
#pragma unroll
                    for (int i = 0; i < 8; ++i) v[i] = *(const u32x4*)(sp + i * 1024 + lane * 16);
                    asm volatile("ds_sub_u32 %0, %1" :: "v"(dna), "v"(1u) : "memory");
#pragma unroll
                    for (int i = 0; i < 8; ++i) st128(v[i], rme, lane * 16, dst + i * 1024, AUX_SYS);
                    WAIT_VM0();
                    st_flag(rme, l0, aux & 0x7fffffffu, call);
                    continue;
                }
                const u32 aux2 = rfl(dv[2]), nq = rfl(dv[3]);
                const bool two = aux & aux2 & 1;
                const u32 hs = two ? pc & 1 : aux2 & 1, q0 = (two ? pc >> 1 : pc) * nq;
                const u32 lw = hs ? aux2 : aux;
                const u32 ob = dst + hs * 128 + q0 * 16 * (NN * 2);
                if (aux2 >> 31) {
#pragma unroll 1
                    for (u32 i = 0; i < 2 * nq; ++i)
                        st_out(u32x4{0x7fc07fc0u, 0x7fc07fc0u, 0x7fc07fc0u, 0x7fc07fc0u}, ro, olane, ob + (i >> 1) * 16 * (NN * 2) + (i & 1) * 64);
                } else {
                    const u32 nv = lw >> 16, sb = lw & 0xfff0u;
                    const u32 H = hidden_words(nv), eo = ((2 * nv + 3) & ~3u) - 4 * H;
                    u32 bits[2];
                    auto hidden = [&](u32 w) {
                        u32x4 v[4];
#pragma unroll
                        for (int i = 0; i < 4; ++i) v[i] = *(const u32x4*)(sp + sb + (i * H + w) * 16);
                        __builtin_amdgcn_sched_barrier(0);
                        u32 hw = 0;
#pragma unroll
                        for (int i = 0; i < 4; ++i) {
                            hw = (__builtin_amdgcn_perm(v[i][1], v[i][0], 0x07050301u) & 0x80808080u) | ((hw >> 1) & 0x7f7f7f7fu);
                            hw = (__builtin_amdgcn_perm(v[i][3], v[i][2], 0x07050301u) & 0x80808080u) | ((hw >> 1) & 0x7f7f7f7fu);
                        }
                        return hw;
                    };
                    bits[0] = *(const u32*)(sp + sb + eo + lane * 4);
                    bits[1] = *(const u32*)(sp + sb + eo + 256 + lane * 4);
                    const u32 h0 = hidden(lane);
                    bits[0] = lane < H ? h0 : bits[0];
                    if (H > 64) { const u32 h1 = hidden(64 + lane); bits[1] = 64 + lane < H ? h1 : bits[1]; }
                    u32 tot;
                    u32 addr = LDS_CTL + sl * CSTR + sb + 2 * wave_excl(__builtin_popcount(bits[0]) + __builtin_popcount(bits[1]), tot);
                    u64 bq = ((u64)bits[1] << 32 | bits[0]);
                    addr += 2 * (u32)__builtin_popcountll(bq & ((1ull << (q0 * 16)) - 1));
                    bq >>= q0 * 16;
#pragma unroll 1
                    for (u32 qq = 0; qq < nq; ++qq) {
                        const u32 b16 = (u32)bq;
                        bq >>= 16;
                        u32 e[16];
#pragma unroll
                        for (int q = 0; q < 16; ++q) {
                            const u32 t = (b16 >> q) & 1u;
                            e[q] = *(const LAS unsigned short*)((LAS char*)smem + __umul24(addr, t));
                            asm("v_lshl_add_u32 %0, %1, 1, %0" : "+v"(addr) : "v"(t));
                        }
#pragma unroll
                        for (int h = 0; h < 2; ++h)
                            st_out(u32x4{e[h * 8] | (e[h * 8 + 1] << 16), e[h * 8 + 2] | (e[h * 8 + 3] << 16),
                                         e[h * 8 + 4] | (e[h * 8 + 5] << 16), e[h * 8 + 6] | (e[h * 8 + 7] << 16)} & 0x7fff7fffu,
                                   ro, olane, ob + qq * 16 * (NN * 2) + h * 64);
                    }
                }
                asm volatile("ds_sub_u32 %0, %1" :: "v"(dna), "v"(1u) : "memory");
                STAMP(9);
            }
        }
        return;
    }

    {
        if (wave == 0) ((volatile LAS u32*)(smem + 32))[lane < 12 ? lane : 0] = 0u;
        if (lb < 2 * NPWG && wave < NPUSHW) {
            const u32 b = lb - NPWG, q = (rank + 1 + ((b >> 3) + (b & 7)) % 7) & 7;
            const rsrc_t rq = mk_rsrc(pick(P.ws, q), WS_USED);
            u32 su, du; unit_desc(rank, q, wave * EWG + (b & 7), su, du);
            fetch_finish(su0, lane, wl, v);
            STAMP(11);
#pragma unroll
            for (int i = 0; i < 8; ++i) {
                asm volatile("s_waitcnt vmcnt(%0)" :: "n"(PWIN - 1) : "memory");
                st128(v[i], rq, lane * 16, OFF_XA + du * 8192 + i * 1024, AUX_SYS);
            }
            WAIT_VM0();
            st_flag(rq, l0, OFF_FX + du * 4, rfl(epv) + 1);
            STAMP(12);
        }
        const u32 u = (lb - NPWG - SFIRST) + (GRID - NPWG - SFIRST) * wave;
        if (lb >= NPWG + SFIRST && u < 1024 - 2 * NPUSH) {
            u32 n, other, hf, su, du;
            stage_desc(u, n, other, hf);
            unit_desc(rank, (g ^ other) * 4 + hf * 3, n, su, du);
            const u32 isw = su >> 9;
            const bool mine = !other && hf == (isw ? bj : bi);
            const u32 nd = (g ^ other) * 4 + (isw ? hf + 2 * (lane & 1) : 2 * hf + (lane & 1));
            char* const np = (char*)P.ws[nd];
            if (mine || n >= NPUSH) {
                if (wave) {
                    for (u32 i = 0; i < wave; ++i) __builtin_amdgcn_s_sleep(SDELAY);
                    fetch_issue(px, pw, su, lane, v);
                }
                fetch_finish(su, lane, wl, v);
                if (n >= NPUSH) {
#pragma unroll
                    for (int i = 0; i < 8; ++i) st128(v[i], rme, lane * 16, OFF_STG + su * 8192 + i * 1024, AUX_SYS);
                    if (mine) {
#pragma unroll
                        for (int i = 0; i < 8; ++i) st128(v[i], rme, lane * 16, OFF_XA + du * 8192 + i * 1024, AUX_SYS);
                        WAIT_VM(8);
                    } else WAIT_VM(0);
                    if (lane < 4)
                        __hip_atomic_store((u64*)(np + OFF_NT + (rank * 640 + n) * 16 + (lane >> 1) * 8), (u64)(rfl(epv) + 1) << 32, __ATOMIC_RELAXED, __HIP_MEMORY_SCOPE_SYSTEM);
                    WAIT_VM(1);
                } else {
#pragma unroll
                    for (int i = 0; i < 8; ++i) st128(v[i], rme, lane * 16, OFF_XA + du * 8192 + i * 1024, AUX_SYS);
                    WAIT_VM(0);
                }
                st_flag(rme, mine ? l0 : 0x80000000u, OFF_FX + du * 4, rfl(epv) + 1);
            }
        }
        STAMP(10);
    }

    const u32 call = rfl(epv) + 1;
    const u32 wm = (wave >> 1) & 1, wn = wave & 1;

    st_flag(rme, l0, epo, call);
    if (lb < NPWG + NCWG) {
        const u32 c = lb - NPWG;
        const u32 tl2 = P.o.tiles[c];
        volatile LAS u32* sel = (volatile LAS u32*)(smem + 16);

        for (u32 pass = 0; pass < ((tl2 >> 8) == (tl2 & 255) ? 1u : 2u); ++pass) {
            const u32 tb = tl2 >> (pass * 8);
            const u32 keep = (tb >> 7) & 1, jt = (tb >> 4) & 7, ct = tb & 15;
            const u32 j = perm8(jt, bi);
            const u32 lc = perm16(ct, bj);
            const u32 lr = row_lr(j, keep ? g : 1 - g);

            if (wave >= 4) {
                const u32 lw = wave - 4;
                const u32 xbase = OFF_XA + lr * 32 * BLK + lw * 4096;
                const u32 wbase = OFF_WB + lc * 32 * BLK + lw * 4096;
                const u32 fo = (lane < 32) ? OFF_FX + (lr * 32 + lane) * 8 : OFF_FW + (lc * 32 + lane - 32) * 8;
                u32 pend = 0, taken = 0;
                for (int s = -4; s < 32; ++s) {
                    const bool refill = pend == 0 && s < 28;
                    if (refill && lw == 0) {
                        u64 t0 = 0;
                        for (u32 it = 0;; ++it) {
                            const u64 fv = ld_flag64(myws, fo);
                            const u64 m = __ballot((u32)fv >= call && (u32)(fv >> 32) >= call);
                            pend = (u32)m & (u32)(m >> 32) & ~taken;
                            if (pend) break;
                            if (it == 0) t0 = __builtin_amdgcn_s_memrealtime();
                            if ((it & 63) == 63 && give_up(myws, call, t0)) { pend = ~taken; break; }
                            __builtin_amdgcn_s_sleep(PSLEEP);
                        }
                        taken |= pend;
                        sel[s & 1] = pend;
                    }
                    if (s < 29) WAIT_VM(16); else if (s == 29) WAIT_VM(8); else WAIT_VM(0);
                    WAIT_LGKM0();
                    __builtin_amdgcn_s_barrier();
                    CBAR();
                    if (s < 28) {
                        if (refill) pend = rfl(sel[s & 1]);
                        const u32 so = ((u32)__builtin_ctz(pend | 0x80000000u)) * BLK;
                        pend &= pend - 1;
                        LAS char* d = (LAS char*)smem + LDS_CTL + (s & 3) * 32768 + lw * 4096;
#pragma unroll
                        for (int i = 0; i < 4; ++i)
                            __builtin_amdgcn_raw_ptr_buffer_load_lds(rme, d + i * 1024, 16, lane * 16, xbase + so + i * 1024, 0, AUX_SC1);
#pragma unroll
                        for (int i = 0; i < 4; ++i)
                            __builtin_amdgcn_raw_ptr_buffer_load_lds(rme, d + 16384 + i * 1024, 16, lane * 16, wbase + so + i * 1024, 0, AUX_SC1);
                    }
                    CBAR();
                }
                continue;
            }
            f32x4 acc[4][4];
#pragma unroll
            for (int a = 0; a < 4; ++a)
#pragma unroll
                for (int b2 = 0; b2 < 4; ++b2) acc[a][b2] = f32x4{0.f, 0.f, 0.f, 0.f};
            {
                const char* xa = smem + LDS_CTL + wm * 8192 + lane * 16;
                const char* wb = smem + LDS_CTL + 16384 + wn * 8192 + lane * 16;
                bf16x8 xA[8], wA[8], xB[8], wB[8];
                auto rd = [&](bf16x8 (&xf)[8], bf16x8 (&wf)[8], u32 slot) {
                    const u32 so = (slot & 3) * 32768;
#pragma unroll
                    for (int i = 0; i < 8; ++i) {
                        wf[i] = *(const bf16x8*)(wb + so + i * 1024);
                        xf[i] = *(const bf16x8*)(xa + so + i * 1024);
                    }
                };
                auto mm = [&](bf16x8 (&xf)[8], bf16x8 (&wf)[8]) {
#pragma unroll
                    for (int h = 0; h < 2; ++h)
#pragma unroll
                        for (int a = 0; a < 4; ++a)
#pragma unroll
                            for (int b2 = 0; b2 < 4; ++b2)
                                acc[a][b2] = __builtin_amdgcn_mfma_f32_16x16x32_bf16(wf[h * 4 + b2], xf[h * 4 + a], acc[a][b2], 0, 0, 0);
                };
                auto sched = [&]() {
#pragma unroll
                    for (int i = 0; i < 16; ++i) {
                        __builtin_amdgcn_sched_group_barrier(0x100, 1, 0);
                        __builtin_amdgcn_sched_group_barrier(0x008, 2, 0);
                    }
                };
#define BAR() do { __builtin_amdgcn_s_waitcnt(0xC07F); CBAR(); __builtin_amdgcn_s_barrier(); CBAR(); __builtin_amdgcn_sched_barrier(0); } while (0)
                BAR(); BAR(); BAR();
                BAR();
                rd(xA, wA, 0);
                for (u32 s = 0; s < 32; s += 2) {
                    BAR();
                    rd(xB, wB, s + 1);
                    mm(xA, wA);
                    sched();
                    __builtin_amdgcn_sched_barrier(0);
                    BAR();
                    rd(xA, wA, s + 2);
                    mm(xB, wB);
                    sched();
                    __builtin_amdgcn_sched_barrier(0);
                }
                WAIT_LGKM0();
            }
            STAMP(1 + 3 * pass);
            u32 lane_ = threadIdx.x & 63;
            asm volatile("" : "+v"(lane_));
            const u32 lane = lane_;

            if (!keep) {
                const u32 pu = (((j >> 2) * 2 + (lc >> 3)) * 32 + part_idx(jt >> 1, ct >> 1)) * 4 + wave;
                const u32 dst = OFF_SPART + pu * 8192;
#pragma unroll
                for (int a = 0; a < 4; ++a)
#pragma unroll
                    for (int u = 0; u < 2; ++u) {
                        u32x4 o;
                        o[0] = pack2(acc[a][2 * u][0], acc[a][2 * u][1]);
                        o[1] = pack2(acc[a][2 * u][2], acc[a][2 * u][3]);
                        o[2] = pack2(acc[a][2 * u + 1][0], acc[a][2 * u + 1][1]);
                        o[3] = pack2(acc[a][2 * u + 1][2], acc[a][2 * u + 1][3]);
                        st128(o, rme, lane * 16, dst + (a * 2 + u) * 1024, AUX_SYS);
                    }
                WAIT_VM(0);
                sel[4 + pass * 4 + wave] = 0x80000000u;
                if (__ballot(sel[4 + pass * 4 + (lane & 3)] >> 31) == ~0ull)
                    st_flag64(mk_rsrc(pick(P.ws, (1 - g) * 4 + (pu >> 7)), WS_USED), lane < 2 ? lane * 8 : 0x80000000u, OFF_NP + ((rank & 3) * 32 + ((pu >> 2) & 31)) * 16, call, 0);
                STAMP(3 + 3 * pass);
            } else {
                const u32 pu = (j * 16 + lc) * 4 + wave;
                CBAR();
                char* const peerp = (char*)P.ws[(rank + 1 + (lane >> 2)) & 7] + OFF_NT + rank * 10240 + ((((rank + 1 + (lane >> 2)) >> 2) & 1) == g ? 5120 : 3072);
                wait_flag64<KSLEEP>(myws, OFF_FP + pu * 8, call);
                STAMP(2 + 3 * pass);
                u32x4 o[8];
                u32 bits[2] = {0u, 0u};
                {
                    u32x4 pv[8];
#pragma unroll
                    for (int i = 0; i < 8; ++i) pv[i] = __builtin_amdgcn_raw_buffer_load_b128(rme, lane * 16, OFF_PART + pu * 8192 + i * 1024, AUX_SC1);
                    const bool bad = rfl(ld_tmo(myws)) == call;
#pragma unroll
                    for (int gi = 0; gi < 8; ++gi) {
                        float y[8];
#pragma unroll
                        for (int k = 0; k < 8; ++k) {
                            const u32 pq = pv[gi][k >> 1];
                            float t = acc[gi >> 1][(gi & 1) * 2 + (k >> 2)][k & 3] + ((k & 1) ? bf_hi(pq) : bf_lo(pq));
                            if (bad) t = __builtin_nanf("");
                            const bool nz = !(t <= 0.f);
                            y[k] = nz ? t : 0.f;
                            bits[gi >> 2] |= nz ? (1u << ((gi & 3) * 8 + k)) : 0u;
                        }
                        o[gi] = u32x4{pack2(y[0], y[1]), pack2(y[2], y[3]), pack2(y[4], y[5]), pack2(y[6], y[7])} & 0x7fff7fffu;
                    }
                }
                u32 tot;
                u32 addr = (u32)(LDS_CTL + wave * 16384) + 2 * wave_excl(__builtin_popcount(bits[0]) + __builtin_popcount(bits[1]), tot);
                const u32 H = hidden_words(tot), eo = ((2 * tot + 3) & ~3u) - 4 * H;
                u32 len = (eo + 512 + 15) & ~15u, off;
                asm volatile("ds_add_rtn_u32 %0, %1, %2" : "=v"(off) : "v"(lane == 0 ? 64 + pass * 8 + (wave >> 1) * 4 : LDS_CTL + wave * 16384 + 10496 + lane * 4), "v"(len) : "memory");
                u32* wt = (u32*)(wl + 9216);
                wt[lane] = bits[0];
                wt[64 + lane] = bits[1];
                *(u32*)(wl + (lane >= H ? eo + lane * 4 : 10240)) = bits[0];
                *(u32*)(wl + (64 + lane >= H ? eo + 256 + lane * 4 : 10240)) = bits[1];
#pragma unroll
                for (int gi = 0; gi < 8; ++gi)
#pragma unroll
                    for (int k = 0; k < 8; ++k) {
                        const u32 t = (bits[gi >> 2] >> ((gi & 3) * 8 + k)) & 1u;
                        const u32 d = o[gi][k >> 1];
                        *(LAS unsigned short*)((LAS char*)smem + __umul24(addr, t)) = (unsigned short)((k & 1) ? d >> 16 : d);
                        asm("v_lshl_add_u32 %0, %1, 1, %0" : "+v"(addr) : "v"(t));
                    }
                asm volatile("s_waitcnt lgkmcnt(0)" : "+v"(off) :: "memory");
                off = rfl(off);
                const u32 tl = jt * 16 + ct;
                const u32 sidx = ((P.o.ord[tl >> 2] >> ((tl & 3) * 8)) & 127) * 4 + wave;
                const u32 so = OFF_CMP + (sidx & ~1u) * CSTR + off;
#pragma unroll
                for (int i = 0; i < 9; ++i)
                    if (i * 1024u < len) {
                        u32x4 v = *(const u32x4*)(wl + i * 1024 + lane * 16);
                        const u32 G = i * 64 + lane, pl = (G >= H) + (G >= 2 * H) + (G >= 3 * H);
                        const u32 hw = G < 4 * H ? wt[(G - pl * H) & 127] << (6 - pl * 2) : 0u;
                        v[0] |= __builtin_amdgcn_perm(0u, hw << 1, 0x010c000cu) & 0x80008000u;
                        v[1] |= __builtin_amdgcn_perm(0u, hw << 1, 0x030c020cu) & 0x80008000u;
                        v[2] |= __builtin_amdgcn_perm(0u, hw, 0x010c000cu) & 0x80008000u;
                        v[3] |= __builtin_amdgcn_perm(0u, hw, 0x030c020cu) & 0x80008000u;
                        st128(v, rme, i * 1024 + lane * 16 < len ? lane * 16 : 0x80000000u, so + i * 1024, AUX_SYS);
                    }
                WAIT_VM(0);
                sel[4 + pass * 4 + wave] = len | (off != 0) | tot << 16 | 0x80000000u;
                const u32 l4 = sel[4 + pass * 4 + (lane & 3)];
                if (__ballot(l4 >> 31) == ~0ull) {
                    const u64 note = (u64)call << 32 | (l4 & 0x7fffffffu);
                    if (lane < 28)
                        __hip_atomic_store((u64*)(peerp + ((sidx & ~3u) + (lane & 3)) * 8), note, __ATOMIC_RELAXED, __HIP_MEMORY_SCOPE_SYSTEM);
                }
                STAMP(3 + 3 * pass);
                const u32 obase = (glob_blk(g, bi, lr) * 128 + wm * 64) * (NN * 2) + (glob_blk(g, bj, lc) * 128 + wn * 64) * 2;
#pragma unroll
                for (int gi = 0; gi < 8; ++gi)
                    st_out(o[gi], ro, olane, obase + (gi >> 1) * 16 * (NN * 2) + (gi & 1) * 64);
                WAIT_LGKM0();
            }
        }
    }
}

extern "C" void kernel_launch(int rank, int nranks, ncclComm_t comm,
                              hipStream_t stream,
                              void* const* d_in, const int* in_sizes, int n_in,
                              void* d_out, int out_size,
                              const int* in_dtypes, int out_dtype,
                              void* const* d_out_all,
                              void* const* d_ws_all, size_t ws_bytes) {
    if (nranks != NR || n_in < 2 || ws_bytes < WS_USED || (unsigned)rank >= (unsigned)NR ||
        in_sizes[0] != MM * KS || in_sizes[1] != KS * NN || out_size != MM * NN) {
        fprintf(stderr, "unexpected problem shape\n");
        return;
    }
    Params P;
    P.x = d_in[0];
    P.w = d_in[1];
    for (int i = 0; i < NR; ++i) P.ws[i] = d_ws_all[i];
    P.me = d_ws_all[rank];
    P.out = d_out;
    P.rank = rank;
    P.o = ORDERS;
    for (u32 b = 0; b < (u32)GRID; ++b) {
        u32 f[2] = {0xffffu, 0xffffu}, du;
        if (b < (u32)NPWG) {
            for (u32 w = 0; w < (u32)NPUSHW; ++w) unit_desc(rank, (rank + 1 + ((b >> 3) + (b & 7)) % 7) & 7, w * EWG + (b & 7), f[w], du);
        } else if (b >= (u32)SFIRST && b < (u32)(GRID - NPWG)) {
            u32 n, other, hf;
            stage_desc(b - SFIRST, n, other, hf);
            unit_desc(rank, (((u32)rank >> 2) ^ other) * 4 + hf * 3, n, f[0], du);
        }
        P.first[b] = f[0] | f[1] << 16;
    }
    HIP_CHECK(hipFuncSetAttribute((const void*)fused_gemm_ar, hipFuncAttributeMaxDynamicSharedMemorySize, LDS_BYTES));
    hipLaunchKernelGGL(fused_gemm_ar, dim3(GRID), dim3(512), LDS_BYTES, stream, P);
    HIP_CHECK(hipGetLastError());
}
```

```cpp
#include <hip/hip_runtime.h>
#include <rccl/rccl.h>
#include <cstdio>
#include <cstdint>

#define HIP_CHECK(expr) do { hipError_t _e = (expr); if (_e != hipSuccess) { \
    fprintf(stderr, "HIP error %s at %s:%d\n", hipGetErrorString(_e), __FILE__, __LINE__); \
    return; } } while (0)

#define XS 16
#define XNP 2
#define KSLEEP 8
#define WSLEEP 1
#define LOOK 6
#define LOOK2 4
#define XE 48
#define XH 16
#define NPUSHW 2
#define SDELAY 20
#define PWIN 3
#define WIN 5
#define SELJ 5
#define DIAG 0
#define LAT_K 12.0
#define LAT_S 50.0
#define ENTRY_ORDER "s0w0s1w1w2w3w4w5s2k0k1s3w6w7k2s4s5k3s6s7k4k5w8w9k6k7wAwBwCwDwEwF"
#define ESCALE 1.34
#define SFIRST 72
#define OUT_AUX 17
#define PSLEEP 32

typedef unsigned int u32;
typedef unsigned long long u64;
typedef u32 u32x4 __attribute__((ext_vector_type(4)));
typedef u32 u32x2 __attribute__((ext_vector_type(2)));
typedef float f32x4 __attribute__((ext_vector_type(4)));
typedef short bf16x8 __attribute__((ext_vector_type(8)));
typedef __bf16 bf16v2 __attribute__((ext_vector_type(2)));
typedef float f32x2 __attribute__((ext_vector_type(2)));
#define LAS __attribute__((address_space(3)))

constexpr int NR = 8;
constexpr int MM = 4096, NN = 4096, KS = 512;
constexpr u32 BLK = 16384;
constexpr u32 OFF_XA = 0;
constexpr u32 OFF_WB = 8u << 20;
constexpr u32 OFF_PART = 16u << 20;
constexpr u32 OFF_SPART = 20u << 20;
constexpr u32 CSTR = 9216;
constexpr u32 CMAX = 8704;
constexpr u32 OFF_CMP = 24u << 20;
constexpr u32 OFF_STG = 30u << 20;
constexpr u32 OFF_FLAGS = 38u << 20;
constexpr u32 OFF_FX = OFF_FLAGS;
constexpr u32 OFF_FW = OFF_FX + 4096;
constexpr u32 OFF_FP = OFF_FW + 4096;
constexpr u32 OFF_NP = OFF_FP + 4096;
constexpr u32 OFF_NT = OFF_NP + 2048;
constexpr u32 OFF_CTR = OFF_NT + 81920;
constexpr u32 OFF_EP = OFF_CTR + 4096;
constexpr u32 OFF_TMO = OFF_EP + 8192;
constexpr u32 OFF_DIAG = OFF_TMO + 4096;
constexpr u32 WS_USED = OFF_DIAG + (1u << 20);
constexpr u32 OUT_BYTES = (u32)MM * NN * 2;
constexpr u32 IN_BYTES = (u32)MM * KS * 2;

constexpr int EWG = 8;
constexpr int NPWG = 7 * EWG;
constexpr int NPUSH = EWG * NPUSHW;
constexpr int NCWG = 192;
constexpr int GRID = 256;
constexpr u32 LDS_CTL = 1024;
constexpr int LDS_BYTES = LDS_CTL + 4 * 32768 + (DIAG ? 1024 : 0);
constexpr u64 DEADLINE = 100000000ull;
constexpr int AUX_SYS = 17;
constexpr int AUX_SC1 = 16;
static_assert(GRID * 8 * 4 <= 8192, "epoch words");
static_assert(OFF_CMP + 512u * CSTR <= OFF_STG, "layout");
static_assert(14 * CSTR <= 4 * 32768, "engine slots");
static_assert(NPUSHW <= 2, "first loads");
static_assert((GRID - NPWG - SFIRST) * 8 >= 1024 - 2 * NPUSH && SFIRST >= NPWG, "staging waves");

struct Orders {
    u64 entry[4];
    unsigned short tiles[NCWG];
    u32 ord[64];
};
struct Params {
    const void* x;
    const void* w;
    void* me;
    int rank;
    void* out;
    void* ws[NR];
    u32 first[GRID];
    Orders o;
};

constexpr double mx(double a, double b) { return a > b ? a : b; }
constexpr Orders build_orders() {
    Orders P{};
    const char* eo = ENTRY_ORDER;
    double t[3][16] = {};
    for (int e = 0; e < 32; ++e) {
        const int kind = eo[2 * e] == 'w' ? 0 : eo[2 * e] == 's' ? 1 : 2;
        const char ch = eo[2 * e + 1];
        const int idx = ch <= '9' ? ch - '0' : ch - 'A' + 10;
        P.entry[e >> 3] |= (u64)(kind << 4 | idx) << ((e & 7) * 8);
        t[kind][idx] = (e + 1) * ESCALE;
    }
    double av[128] = {};
    for (int i = 0; i < 128; ++i) {
        const int jt = i >> 4, ct = i & 15;
        av[i] = mx(mx(t[2][jt], t[0][ct]) + LAT_K, mx(t[1][jt], t[0][ct]) + LAT_S);
    }
    for (int i = 0; i < 128; ++i) {
        int pos = 0;
        for (int k = 0; k < 128; ++k) pos += av[k] < av[i] || (av[k] == av[i] && k < i);
        P.ord[i >> 2] |= (u32)pos << ((i & 3) * 8);
        P.ord[32 + (pos >> 2)] |= (u32)i << ((pos & 3) * 8);
    }
    const int W = NCWG / 8, D = 32 - W;
    for (int x = 0; x < 8; ++x) {
        int id[32] = {}, sorted[32] = {};
        double key[32] = {};
        for (int i = 0; i < 32; ++i) {
            const int keep = i >> 4, jt = (x >> 1) + 4 * ((i >> 3) & 1), ct = (i & 7) * 2 + (x & 1);
            id[i] = keep << 7 | jt << 4 | ct;
            key[i] = keep ? av[jt * 16 + ct] - LAT_K : mx(t[1][jt], t[0][ct]);
        }
        for (int i = 0; i < 32; ++i) {
            int pos = 0;
            for (int k = 0; k < 32; ++k) pos += key[k] < key[i] || (key[k] == key[i] && k < i);
            sorted[pos] = id[i];
        }
        for (int i = 0; i < W; ++i) P.tiles[i * 8 + x] = (unsigned short)(sorted[i] | sorted[i < D ? W + i : i] << 8);
    }
    return P;
}
constexpr Orders ORDERS = build_orders();

typedef __amdgpu_buffer_rsrc_t rsrc_t;
__device__ __forceinline__ rsrc_t mk_rsrc(const void* p, u32 bytes) {
    return __builtin_amdgcn_make_buffer_rsrc((void*)p, (short)0, (int)bytes, 0x00020000);
}
__device__ __forceinline__ u32 rfl(u32 v) { return (u32)__builtin_amdgcn_readfirstlane((int)v); }
__device__ __forceinline__ void* pick(void* const (&t)[NR], u32 idx) {
    void* r = t[0];
#pragma unroll
    for (u32 i = 1; i < NR; ++i) r = (idx == i) ? t[i] : r;
    return r;
}
__host__ __device__ __forceinline__ u32 glob_blk(u32 g, u32 b, u32 l) {
    return g == 0 ? b * 16 + l : ((l >> 3) * 2 + b) * 8 + (l & 7);
}
__host__ __device__ __forceinline__ u32 perm8(u32 t, u32 h) { return (t >> 1) + ((t ^ h) & 1) * 4; }
__host__ __device__ __forceinline__ u32 perm16(u32 t, u32 h) { return (t >> 1) + ((t ^ h) & 1) * 8; }
__host__ __device__ __forceinline__ u32 row_lr(u32 j, u32 fin) { return (j >> 2) * 8 + fin * 4 + (j & 3); }

__device__ __forceinline__ u32 part_idx(u32 a, u32 b) { return b >= 4 ? b * 4 + a : (b > a ? b * b + a : a * a + a + b); }
__device__ __forceinline__ void part_ab(u32 m, u32& a, u32& b) {
    const u32 sh = m >= 9 ? 3 : m >= 4 ? 2 : m >= 1 ? 1 : 0, o = m - sh * sh;
    a = o < sh ? o : sh;
    b = o < sh ? sh : o - sh;
    if (m >= 16) { a = m & 3; b = m >> 2; }
}
__host__ __device__ __forceinline__ void unit_desc(u32 s, u32 d, u32 n, u32& su, u32& du) {
    constexpr u64 e0 = ORDERS.entry[0], e1 = ORDERS.entry[1], e2 = ORDERS.entry[2], e3 = ORDERS.entry[3];
    const bool in = (s >> 2) == (d >> 2);
    const u32 gd = d >> 2, e = in ? n / 10 : n / 6, kk = n - e * (in ? 10 : 6);
    const u32 eb = (u32)(((e & 16) ? ((e & 8) ? e3 : e2) : ((e & 8) ? e1 : e0)) >> ((e & 7) * 8));
    const u32 ek = (eb >> 4) & 3, idx = eb & 15;
    const bool isw = ek == 0;
    const u32 l = isw ? perm16(idx, d) : row_lr(perm8(idx & 7, d >> 1), ek == 1 ? 1 - gd : gd);
    su = (isw ? 512 : 0) + glob_blk(gd, isw ? d & 1 : (d >> 1) & 1, l) * 16 + (in ? 0 : 10) + kk;
    du = (isw ? 1024 : 0) + l * 64 + (in ? (s & 3) * 10 : 40 + (s & 3) * 6) + kk;
}
__host__ __device__ __forceinline__ void stage_desc(u32 u, u32& n, u32& other, u32& hf) {
    const bool four = u >= 2 * NPUSH && u < 768 - 2 * NPUSH;
    hf = u & 1; other = four ? (u >> 1) & 1 : 0;
    n = u < 2 * NPUSH ? u >> 1 : four ? (u + 2 * NPUSH) >> 2 : (u >> 1) - 192 + NPUSH;
}
__device__ __forceinline__ void fetch_issue(const void* x, const void* w, u32 su, u32 lane, u32x4* v) {
    const u32 isw = su >> 9, GB = (su >> 4) & 31, kc = (su >> 1) & 7, hf = su & 1;
    const rsrc_t r = mk_rsrc(isw ? w : x, IN_BYTES);
    const u32 vo = isw ? (lane >> 3) * 65536 + (lane & 7) * 16 : (lane & 15) * 1024 + (lane >> 4) * 16;
    const u32 so = isw ? kc * 64 * 8192 + GB * 256 + hf * 128 : GB * 131072 + hf * 65536 + kc * 128;
#pragma unroll
    for (int i = 0; i < 8; ++i) v[i] = __builtin_amdgcn_raw_buffer_load_b128(r, vo, so + (isw ? i * 8192 : (i & 3) * 16384 + (i >> 2) * 64), 0);
}
__device__ __forceinline__ void fetch_finish(u32 su, u32 lane, char* wl, u32x4* v) {
    if (su >> 9) {
        const u32 wo = (lane >> 5) * 4096 + ((lane & 7) >> 2) * 2048 + (((lane >> 3) & 3) * 16 + (lane & 3) * 4) * 16;
#pragma unroll
        for (int c = 0; c < 8; ++c) {
            u32x4 o;
#pragma unroll
            for (int t = 0; t < 4; ++t)
                o[t] = __builtin_amdgcn_perm(v[2 * t + 1][c >> 1], v[2 * t][c >> 1], (c & 1) ? 0x07060302u : 0x05040100u);
            *(u32x4*)(wl + wo + (c >> 2) * 1024 + (c & 3) * 16) = o;
        }
        asm volatile("s_waitcnt lgkmcnt(0)" ::: "memory");
#pragma unroll
        for (int i = 0; i < 8; ++i) v[i] = *(const u32x4*)(wl + i * 1024 + lane * 16);
        asm volatile("s_waitcnt lgkmcnt(0)" ::: "memory");
    }
}
__device__ __forceinline__ void st128(u32x4 v, rsrc_t r, u32 voff, u32 soff, int) {
    __builtin_amdgcn_raw_buffer_store_b128(v, r, voff + soff, 0, AUX_SYS);
}
__device__ __forceinline__ void st_out(u32x4 v, rsrc_t r, u32 voff, u32 soff) {
    __builtin_amdgcn_raw_buffer_store_b128(v, r, voff + soff, 0, OUT_AUX);
}
__device__ __forceinline__ void st_flag(rsrc_t r, u32 l0, u32 off, u32 v) {
    __builtin_amdgcn_raw_buffer_store_b32(v, r, l0, off, AUX_SYS);
}
__device__ __forceinline__ void st_flag64(rsrc_t r, u32 l0, u32 off, u32 call, u32 len) {
    __builtin_amdgcn_raw_buffer_store_b64(u32x2{len, call}, r, l0, off, AUX_SYS);
}
__device__ __forceinline__ u64 ld_flag64(const void* ws, u32 off) {
    return __hip_atomic_load((const u64*)((const char*)ws + off), __ATOMIC_RELAXED, __HIP_MEMORY_SCOPE_SYSTEM);
}
#define CBAR() asm volatile("" ::: "memory")
#define WAIT_VM(n) asm volatile("s_waitcnt vmcnt(" #n ")" ::: "memory")
#define WAIT_VM0() do { WAIT_VM(0); __builtin_amdgcn_s_waitcnt(0x0F70); CBAR(); } while (0)
#define WAIT_LGKM0() asm volatile("s_waitcnt lgkmcnt(0)" ::: "memory")

__device__ __forceinline__ bool give_up(void* myws, u32 call, u64 t0) {
    u32* tmo = (u32*)((char*)myws + OFF_TMO);
    bool late = __builtin_amdgcn_s_memrealtime() - t0 > DEADLINE;
    if (late || rfl(__hip_atomic_load(tmo, __ATOMIC_RELAXED, __HIP_MEMORY_SCOPE_AGENT)) == call) {
        __hip_atomic_store(tmo, call, __ATOMIC_RELAXED, __HIP_MEMORY_SCOPE_AGENT);
        return true;
    }
    return false;
}
__device__ __forceinline__ u32 ld_tmo(void* myws) {
    return __hip_atomic_load((u32*)((char*)myws + OFF_TMO), __ATOMIC_RELAXED, __HIP_MEMORY_SCOPE_AGENT);
}
template <int SLP>
__device__ __forceinline__ u32 wait_flag64(void* myws, u32 off, u32 call) {
    u64 t0 = 0;
    for (u32 it = 0;; ++it) {
        u64 v = ld_flag64(myws, off);
        if (rfl((u32)(v >> 32)) >= call) return rfl((u32)v);
        if (it == 0) t0 = __builtin_amdgcn_s_memrealtime();
        if ((it & 63) == 63 && give_up(myws, call, t0)) return 0;
        __builtin_amdgcn_s_sleep(SLP);
    }
}
__device__ __forceinline__ u32 wave_excl(u32 cnt, u32& tot) {
    int x = (int)cnt;
    x += __builtin_amdgcn_update_dpp(0, x, 0x111, 0xf, 0xf, false);
    x += __builtin_amdgcn_update_dpp(0, x, 0x112, 0xf, 0xf, false);
    x += __builtin_amdgcn_update_dpp(0, x, 0x114, 0xf, 0xf, false);
    x += __builtin_amdgcn_update_dpp(0, x, 0x118, 0xf, 0xf, false);
    x += __builtin_amdgcn_update_dpp(0, x, 0x142, 0xa, 0xf, false);
    x += __builtin_amdgcn_update_dpp(0, x, 0x143, 0xc, 0xf, false);
    tot = (u32)__builtin_amdgcn_readlane(x, 63);
    return (u32)x - cnt;
}
__device__ __forceinline__ u32 hidden_words(u32 nv) { const u32 h = nv >> 5; return h > 64 && h < 80 ? 64 : h; }
__device__ __forceinline__ u32 pack2(float a, float b) {
    f32x2 v = {a, b};
    return __builtin_bit_cast(u32, __builtin_convertvector(v, bf16v2));
}
__device__ __forceinline__ float bf_lo(u32 v) { return __builtin_bit_cast(float, v << 16); }
__device__ __forceinline__ float bf_hi(u32 v) { return __builtin_bit_cast(float, v & 0xffff0000u); }

#define STAMP(slot) do {} while (0)

__global__ void __launch_bounds__(512, 2) fused_gemm_ar(Params P) {
    extern __shared__ __attribute__((aligned(16))) char smem[];
    const u32 lane = threadIdx.x & 63;
    const u32 wave = rfl(threadIdx.x >> 6);
    const void* px = P.x;
    const void* pw = P.w;
    u64 pme = (u64)P.me;
    u32 rk = (u32)P.rank, fw = P.first[blockIdx.x];
    asm volatile("" : "+s"(px), "+s"(pw), "+s"(pme), "+s"(rk), "+s"(fw));
    void* myws = (void*)(__attribute__((address_space(1))) void*)pme;
    u32x4 v[8];
    const u32 su0 = wave < 2 ? (fw >> (wave * 16)) & 0xffffu : 0xffffu;
    if (su0 != 0xffffu) fetch_issue(px, pw, su0, lane, v);
    __builtin_amdgcn_sched_barrier(0);
    const u32 rank = rk & 7;
    const u32 g = rank >> 2, bi = (rank >> 1) & 1, bj = rank & 1;
    const u32 lb = (blockIdx.x + NPWG) & (GRID - 1);
    const rsrc_t rme = mk_rsrc(myws, WS_USED);
    char* wl = smem + LDS_CTL + wave * 16384;

    const u32 l0 = lane == 0 ? 0u : 0x80000000u;
    const u32 epo = OFF_EP + (lb * 8 + wave) * 4;
    const u32 epv = __builtin_amdgcn_raw_buffer_load_b32(rme, 0, epo, AUX_SC1);
    *(LAS u32*)smem = 0u;
    STAMP(0);

    const u32 olane = (lane & 15) * (NN * 2) + (lane >> 4) * 16;
    const rsrc_t ro = mk_rsrc(P.out, OUT_BYTES);

    if (lb < NPWG) {
        const u32 b = lb;
        const u32 di = 1 + ((b >> 3) + (b & 7)) % 7;
        const u32 p = (rank + di) & 7;
        volatile LAS u32* ctl = (volatile LAS u32*)(smem + 64);
        const u32 dum = 640 + lane * 4;
        const u32 cra = lane == 0 ? 64 + 15 * 32 + 4 : dum;
        if (wave == 0) *(volatile LAS u32x4*)(ctl + (lane & 15) * 8) = u32x4{0u, (u32)LOOK, 0u, 0u};
        WAIT_LGKM0();
        __builtin_amdgcn_s_barrier();
        CBAR();
        if (wave == 0) {
            __builtin_amdgcn_s_setprio(3);
            WAIT_VM0();
            u32 prev = 14, pnp = 0, tsl = 0, treq = 0, tsrc = 0;
            u64 t0 = __builtin_amdgcn_s_memrealtime();
            auto select = [&](u32x2 cv) -> bool {
                int mk = (cv[0] & 1) ? (int)((cv[0] >> 16) << 4 | (lane & 15)) : 0x7fffffff;
                mk = min(mk, __builtin_amdgcn_update_dpp(0x7fffffff, mk, 0x111, 0xf, 0xf, false));
                mk = min(mk, __builtin_amdgcn_update_dpp(0x7fffffff, mk, 0x112, 0xf, 0xf, false));
                mk = min(mk, __builtin_amdgcn_update_dpp(0x7fffffff, mk, 0x114, 0xf, 0xf, false));
                mk = min(mk, __builtin_amdgcn_update_dpp(0x7fffffff, mk, 0x118, 0xf, 0xf, false));
                const u32 r = (u32)__builtin_amdgcn_readlane(mk, 15);
                if (r == 0x7fffffffu) return false;
                tsl = r & 15;
                treq = (u32)__builtin_amdgcn_readlane((int)cv[0], (int)tsl);
                tsrc = (u32)__builtin_amdgcn_readlane((int)cv[1], (int)tsl);
                ctl[tsl * 8] = 0;
                asm volatile("ds_add_u32 %0, %1" :: "v"(cra), "v"(tsrc & 1) : "memory");
                return true;
            };
            bool have = false;
            for (;;) {
                if (!have) {
                    bool end = false;
                    for (u32 it = 0;; ++it) {
                        const u32x2 cv = *(volatile LAS u32x2*)(ctl + (lane & 15) * 8);
                        if (select(cv)) { end = false; break; }
                        if (prev != 14) { WAIT_VM(0); ctl[prev * 8 + 2] = pnp; prev = 14; }
                        if (end) break;
                        end = (u32)__builtin_amdgcn_readlane((int)cv[0], 15) != 0;
                        if ((it & 1023) == 1023 && __builtin_amdgcn_s_memrealtime() - t0 > 2 * DEADLINE) end = true;
                        __builtin_amdgcn_s_sleep(1);
                    }
                    if (end) break;
                }
                const u32 sl = tsl, len = ((treq >> 8) & 255) * 64, np = (treq >> 4) & 15, src = tsrc & ~1u, wide = treq & 2, treq0 = treq;
                const rsrc_t rs = mk_rsrc(pick(P.ws, p), src + len);
                const u32 vo = src + lane * 16;
                LAS char* d = (LAS char*)smem + LDS_CTL + sl * CSTR;
                u32x2 cv = {0u, 0u};
#define PULL_STEP(j, W, ALL) \
                if ((j) == (W) - 1 && (ALL || len > ((j) - 1) * 1024u)) { \
                    asm volatile("s_waitcnt vmcnt(%0)" :: "n"((W) - 1) : "memory"); \
                    ctl[prev * 8 + 2] = pnp; \
                    prev = 14; \
                } \
                if (ALL || (j) * 1024u < len) { \
                    asm volatile("s_waitcnt vmcnt(%0)" :: "n"((W) - 1) : "memory"); \
                    __builtin_amdgcn_raw_ptr_buffer_load_lds(rs, d + ((j) >> 2) * 4096, 16, vo + ((j) >> 2) * 4096, 0, ((j) & 3) * 1024, AUX_SYS); \
                } \
                CBAR(); \
                if ((j) == SELJ - 1) cv = *(volatile LAS u32x2*)(ctl + (lane & 15) * 8); \
                if ((j) == SELJ) have = select(cv); \
                CBAR();
#define PULL_UNIT(W) PULL_STEP(0, W, 1) PULL_STEP(1, W, 1) PULL_STEP(2, W, 1) PULL_STEP(3, W, 1) \
                    PULL_STEP(4, W, 1) PULL_STEP(5, W, 1) PULL_STEP(6, W, 1) PULL_STEP(7, W, 1)
#define PULL_PAIR(W) PULL_STEP(0, W, 0) PULL_STEP(1, W, 0) PULL_STEP(2, W, 0) PULL_STEP(3, W, 0) PULL_STEP(4, W, 0) \
                    PULL_STEP(5, W, 0) PULL_STEP(6, W, 0) PULL_STEP(7, W, 0) PULL_STEP(8, W, 0)
                if (!(treq0 & 4)) {
                    if (wide) { PULL_UNIT(WIN + 1) } else { PULL_UNIT(WIN) }
                } else {
                    if (wide) { PULL_PAIR(WIN + 1) } else { PULL_PAIR(WIN) }
                }
                if (prev != 14) { WAIT_VM(0); ctl[prev * 8 + 2] = pnp; }
                prev = sl; pnp = np;
            }
            ctl[15 * 8 + 3] = 1;
            STAMP(3);
            __builtin_amdgcn_s_setprio(0);
        }
        const u32 gp = p >> 2;
        const u32 call = rfl(epv) + 1;
        st_flag(rme, l0, epo, call);
        if (wave == 1) {
            __builtin_amdgcn_s_setprio(2);
            const u32 ordv = P.o.ord[lane];
            const u32 T1 = (gp == g) ? 320 : 192;
            const u32 TH = T1 + 256 - XH, T = TH + 2 * XH;
            const u32 tbase = (call - 1) * (T - NPUSH + 2 * EWG) - NPUSH;
            u32 n = NPUSH, ne = NPUSH;
            u32 pmask = gp == g ? 0u : 0xffffu;
            u32 half = 0;
            u32 share = NPUSH;
            bool over = false, force = false;
            u64 t0 = __builtin_amdgcn_s_memrealtime();
            for (u32 it = 0;; ++it) {
                if (over && n == ne && pmask == 0) break;
                u32 k = 0;
                if (!over && n == ne) {
                    k = rfl(ctl[15 * 8 + 1]);
                    if (n + XE >= T) k = k > LOOK - LOOK2 ? k - (LOOK - LOOK2) : 0;
                    k = k > 2 ? 2 : k;
                    if (k == 0 && pmask == 0) { __builtin_amdgcn_s_sleep(1); if ((it & 4095) == 4095 && __builtin_amdgcn_s_memrealtime() - t0 > 2 * DEADLINE) break; continue; }
                    asm volatile("ds_sub_u32 %0, %1" :: "v"(cra), "v"(k) : "memory");
                }
                const u32 tv = (u32)__builtin_amdgcn_raw_ptr_buffer_atomic_add_i32((int)k, rme, k ? l0 : 0x80000000u, OFF_CTR + di * 128, 0);
                const u32 wb = n < TH ? n : TH + ((n - TH) >> 1);
                const u32x4 nv = __builtin_amdgcn_raw_buffer_load_b128(rme, lane < 47 ? OFF_NT + (p * 640 + wb + lane) * 16 :
                    lane == 47 ? OFF_TMO : OFF_NP + ((p & 3) * 32 + (lane - 48) * 2 + ((b >> 2) & 1)) * 16, 0, AUX_SYS);
                WAIT_VM0();
                if (k) {
                    n = rfl(tv) - tbase; ne = n + k;
                    share += k * EWG;
                    if (ne > T) {
                        over = true;
                        if (ne - (n > T ? n : T) < 2) __builtin_amdgcn_raw_ptr_buffer_atomic_add_i32(1, rme, l0, OFF_CTR + di * 128, 0);
                        ne = T; n = n > T ? T : n;
                    }
                }
                if ((u32)__builtin_amdgcn_readlane((int)nv[0], 47) == call) force = true;
                u64 rm = force ? ~0ull : __ballot(nv[1] >= call && nv[3] >= call);
                u32 fr = (u32)__ballot(ctl[(lane & 15) * 8 + 3] == 0) & 0x3fffu;
                bool progress = false;
                while (fr) {
                    const u32 pr = (u32)(rm >> 48) & pmask;
                    const u32 m = n < TH ? n : TH + ((n - TH) >> 1);
                    u32 src, dst, aux, aux2 = 0, req, nq = 0;
                    if (pr) {
                        const u32 i = (u32)__builtin_ctz(pr), un = (b & 7) + 8 * i;
                        pmask &= ~(1u << i);
                        u32 a, bb; part_ab(un >> 2, a, bb);
                        const u32 pu = ((((p >> 1) & 1) * 4 + a) * 16 + (p & 1) * 8 + bb) * 4 + (un & 3);
                        src = OFF_SPART + ((rank & 3) * 128 + un) * 8192;
                        dst = OFF_PART + pu * 8192; aux = 0x80000000u | (OFF_FP + pu * 8 + 4);
                        req = 1 | 1 << 4 | 128 << 8 | (15 - __builtin_popcount(pmask)) << 16;
                    } else if (n < ne && m - wb < 47 && ((rm >> (m - wb)) & 1)) {
                        if (n < T1) {
                            u32 su, du; unit_desc(p, rank, n, su, du);
                            src = OFF_STG + su * 8192 + 1; dst = OFF_XA + du * 8192; aux = 0x80000000u | (OFF_FX + du * 4);
                            req = 1 | 1 << 4 | 128 << 8 | (n + 16) << 16;
                            ++n;
                        } else {
                            u32 lA = (u32)__builtin_amdgcn_readlane((int)nv[0], (int)(m - wb)), lB = (u32)__builtin_amdgcn_readlane((int)nv[2], (int)(m - wb));
                            if (force || (lA & 0xffffu) > CMAX || (lA >> 16) > 4096) lA = 512;
                            if (force || (lB & 0xffffu) > CMAX || (lB >> 16) > 4096) lB = 512;
                            const u32 n3 = m - T1, to = n3 >> 1;
                            const u32 tl = ((u32)__builtin_amdgcn_readlane((int)ordv, (int)(32 + (to >> 2)))) >> ((to & 3) * 8);
                            const u32 tRB = glob_blk(gp, (p >> 1) & 1, row_lr(perm8((tl >> 4) & 7, p >> 1), gp)), tCB = glob_blk(gp, p & 1, perm16(tl & 15, p));
                            dst = (tRB * 128 + (n3 & 1) * 64) * (NN * 2) + tCB * 256;
                            const u32 bA = lA & 0xfff0u, bB = lB & 0xfff0u, oA = (lA & 1) ? bB : 0, oB = (lA & 1) ? 0 : bA;
                            const bool onlyB = half || (n >= TH && ((n - TH) & 1));
                            half = !onlyB && n < TH && bA + bB > CSTR;
                            const bool onlyA = half || (!onlyB && n >= TH);
                            const u32 st = onlyA ? oA & ~127u : onlyB ? oB & ~127u : 0, en = onlyA ? oA + bA : onlyB ? oB + bB : bA + bB;
                            aux = onlyB ? 0u : (lA & 0xffff0000u) | (oA - st) | 1;
                            aux2 = (onlyA ? 0u : (lB & 0xffff0000u) | (oB - st) | 1) | (force ? 0x80000000u : 0u);
                            src = OFF_CMP + n3 * 2 * CSTR + st + (half ? 0 : 1);
                            nq = n + XS >= T ? 1 : 8 / XNP;
                            req = 1 | 4 | ((onlyA || onlyB ? 4 : 8) / nq) << 4 | ((en - st + 63) >> 6) << 8 | (n + 16) << 16;
                            n += 1 - half;
                        }
                    } else break;
                    const u32 sl = (u32)__builtin_ctz(fr);
                    fr &= fr - 1;
                    *(volatile LAS u32x4*)(ctl + sl * 8 + 4) = u32x4{dst, aux, aux2, nq};
                    ctl[sl * 8 + 3] = (req >> 4) & 15;
                    *(volatile LAS u32x2*)(ctl + sl * 8) = u32x2{req | (share < ne ? 2u : 0u), src};
                    progress = true;
                }
                if (progress || k) t0 = __builtin_amdgcn_s_memrealtime();
                else if ((it & 63) == 63 && give_up(myws, call, t0)) {
                    force = true;
                    if (__builtin_amdgcn_s_memrealtime() - t0 > 2 * DEADLINE) break;
                }
            }
            ctl[15 * 8] = 256;
            __builtin_amdgcn_s_setprio(0);
        }
        {
            bool end = false;
            u64 t0 = __builtin_amdgcn_s_memrealtime();
            for (u32 it = 0;; ++it) {
                const u32x2 cv = *(volatile LAS u32x2*)(ctl + (lane & 15) * 8 + 2);
                const u32 av = (u32)__ballot((int)cv[0] > 0) & 0x3fffu;
                if (av == 0) {
                    if (end) break;
                    end = (u32)__builtin_amdgcn_readlane((int)cv[1], 15) != 0;
                    if ((it & 1023) == 1023 && __builtin_amdgcn_s_memrealtime() - t0 > 3 * DEADLINE) break;
                    __builtin_amdgcn_s_sleep(WSLEEP);
                    continue;
                }
                end = false;
                const u32 sl = (u32)__builtin_ctz(av);
                u32 old;
                u32x4 dv;
                asm volatile("ds_add_rtn_u32 %0, %2, %3\n\tds_read_b128 %1, %4\n\ts_waitcnt lgkmcnt(0)" : "=&v"(old), "=&v"(dv)
                             : "v"(lane == 0 ? 64 + sl * 32 + 8 : dum), "v"(0xffffffffu), "v"(64 + sl * 32 + 16) : "memory");
                const u32 pc = rfl(old) - 1;
                if ((int)pc < 0) continue;
                STAMP(8);
                const char* sp = smem + LDS_CTL + sl * CSTR;
                const u32 dst = rfl(dv[0]), aux = rfl(dv[1]);
                const u32 dna = lane == 0 ? 64 + sl * 32 + 12 : dum;
                if (aux >> 31) {
                    u32x4 v[8];
#pragma unroll
                    for (int i = 0; i < 8; ++i) v[i] = *(const u32x4*)(sp + i * 1024 + lane * 16);
                    asm volatile("ds_sub_u32 %0, %1" :: "v"(dna), "v"(1u) : "memory");
#pragma unroll
                    for (int i = 0; i < 8; ++i) st128(v[i], rme, lane * 16, dst + i * 1024, AUX_SYS);
                    WAIT_VM0();
                    st_flag(rme, l0, aux & 0x7fffffffu, call);
                    continue;
                }
                const u32 aux2 = rfl(dv[2]), nq = rfl(dv[3]);
                const bool two = aux & aux2 & 1;
                const u32 hs = two ? pc & 1 : aux2 & 1, q0 = (two ? pc >> 1 : pc) * nq;
                const u32 lw = hs ? aux2 : aux;
                const u32 ob = dst + hs * 128 + q0 * 16 * (NN * 2);
                if (aux2 >> 31) {
#pragma unroll 1
                    for (u32 i = 0; i < 2 * nq; ++i)
                        st_out(u32x4{0x7fc07fc0u, 0x7fc07fc0u, 0x7fc07fc0u, 0x7fc07fc0u}, ro, olane, ob + (i >> 1) * 16 * (NN * 2) + (i & 1) * 64);
                } else {
                    const u32 nv = lw >> 16, sb = lw & 0xfff0u;
                    const u32 H = hidden_words(nv), eo = ((2 * nv + 3) & ~3u) - 4 * H;
                    u32 bits[2];
                    auto hidden = [&](u32 w) {
                        u32x4 v[4];
#pragma unroll
                        for (int i = 0; i < 4; ++i) v[i] = *(const u32x4*)(sp + sb + (i * H + w) * 16);
                        __builtin_amdgcn_sched_barrier(0);
                        u32 hw = 0;
#pragma unroll
                        for (int i = 0; i < 4; ++i) {
                            hw = (__builtin_amdgcn_perm(v[i][1], v[i][0], 0x07050301u) & 0x80808080u) | ((hw >> 1) & 0x7f7f7f7fu);
                            hw = (__builtin_amdgcn_perm(v[i][3], v[i][2], 0x07050301u) & 0x80808080u) | ((hw >> 1) & 0x7f7f7f7fu);
                        }
                        return hw;
                    };
                    bits[0] = *(const u32*)(sp + sb + eo + lane * 4);
                    bits[1] = *(const u32*)(sp + sb + eo + 256 + lane * 4);
                    const u32 h0 = hidden(lane);
                    bits[0] = lane < H ? h0 : bits[0];
                    if (H > 64) { const u32 h1 = hidden(64 + lane); bits[1] = 64 + lane < H ? h1 : bits[1]; }
                    u32 tot;
                    u32 addr = LDS_CTL + sl * CSTR + sb + 2 * wave_excl(__builtin_popcount(bits[0]) + __builtin_popcount(bits[1]), tot);
                    u64 bq = ((u64)bits[1] << 32 | bits[0]);
                    addr += 2 * (u32)__builtin_popcountll(bq & ((1ull << (q0 * 16)) - 1));
                    bq >>= q0 * 16;
#pragma unroll 1
                    for (u32 qq = 0; qq < nq; ++qq) {
                        const u32 b16 = (u32)bq;
                        bq >>= 16;
                        u32 e[16];
#pragma unroll
                        for (int q = 0; q < 16; ++q) {
                            const u32 t = (b16 >> q) & 1u;
                            e[q] = *(const LAS unsigned short*)((LAS char*)smem + __umul24(addr, t));
                            asm("v_lshl_add_u32 %0, %1, 1, %0" : "+v"(addr) : "v"(t));
                        }
#pragma unroll
                        for (int h = 0; h < 2; ++h)
                            st_out(u32x4{e[h * 8] | (e[h * 8 + 1] << 16), e[h * 8 + 2] | (e[h * 8 + 3] << 16),
                                         e[h * 8 + 4] | (e[h * 8 + 5] << 16), e[h * 8 + 6] | (e[h * 8 + 7] << 16)} & 0x7fff7fffu,
                                   ro, olane, ob + qq * 16 * (NN * 2) + h * 64);
                    }
                }
                asm volatile("ds_sub_u32 %0, %1" :: "v"(dna), "v"(1u) : "memory");
                STAMP(9);
            }
        }
        return;
    }

    {
        if (wave == 0) ((volatile LAS u32*)(smem + 32))[lane < 12 ? lane : 0] = 0u;
        if (lb < 2 * NPWG && wave < NPUSHW) {
            const u32 b = lb - NPWG, q = (rank + 1 + ((b >> 3) + (b & 7)) % 7) & 7;
            const rsrc_t rq = mk_rsrc(pick(P.ws, q), WS_USED);
            u32 su, du; unit_desc(rank, q, wave * EWG + (b & 7), su, du);
            fetch_finish(su0, lane, wl, v);
            STAMP(11);
#pragma unroll
            for (int i = 0; i < 8; ++i) {
                asm volatile("s_waitcnt vmcnt(%0)" :: "n"(PWIN - 1) : "memory");
                st128(v[i], rq, lane * 16, OFF_XA + du * 8192 + i * 1024, AUX_SYS);
            }
            WAIT_VM0();
            st_flag(rq, l0, OFF_FX + du * 4, rfl(epv) + 1);
            STAMP(12);
        }
        const u32 u = (lb - NPWG - SFIRST) + (GRID - NPWG - SFIRST) * wave;
        if (lb >= NPWG + SFIRST && u < 1024 - 2 * NPUSH) {
            u32 n, other, hf, su, du;
            stage_desc(u, n, other, hf);
            unit_desc(rank, (g ^ other) * 4 + hf * 3, n, su, du);
            const u32 isw = su >> 9;
            const bool mine = !other && hf == (isw ? bj : bi);
            const u32 nd = (g ^ other) * 4 + (isw ? hf + 2 * (lane & 1) : 2 * hf + (lane & 1));
            char* const np = (char*)P.ws[nd];
            if (mine || n >= NPUSH) {
                if (wave) {
                    for (u32 i = 0; i < wave; ++i) __builtin_amdgcn_s_sleep(SDELAY);
                    fetch_issue(px, pw, su, lane, v);
                }
                fetch_finish(su, lane, wl, v);
                if (n >= NPUSH) {
#pragma unroll
                    for (int i = 0; i < 8; ++i) st128(v[i], rme, lane * 16, OFF_STG + su * 8192 + i * 1024, AUX_SYS);
                    if (mine) {
#pragma unroll
                        for (int i = 0; i < 8; ++i) st128(v[i], rme, lane * 16, OFF_XA + du * 8192 + i * 1024, AUX_SYS);
                        WAIT_VM(8);
                    } else WAIT_VM(0);
                    if (lane < 4)
                        __hip_atomic_store((u64*)(np + OFF_NT + (rank * 640 + n) * 16 + (lane >> 1) * 8), (u64)(rfl(epv) + 1) << 32, __ATOMIC_RELAXED, __HIP_MEMORY_SCOPE_SYSTEM);
                    WAIT_VM(1);
                } else {
#pragma unroll
                    for (int i = 0; i < 8; ++i) st128(v[i], rme, lane * 16, OFF_XA + du * 8192 + i * 1024, AUX_SYS);
                    WAIT_VM(0);
                }
                st_flag(rme, mine ? l0 : 0x80000000u, OFF_FX + du * 4, rfl(epv) + 1);
            }
        }
        STAMP(10);
    }

    const u32 call = rfl(epv) + 1;
    const u32 wm = (wave >> 1) & 1, wn = wave & 1;

    st_flag(rme, l0, epo, call);
    if (lb < NPWG + NCWG) {
        const u32 c = lb - NPWG;
        const u32 tl2 = P.o.tiles[c];
        volatile LAS u32* sel = (volatile LAS u32*)(smem + 16);

        for (u32 pass = 0; pass < ((tl2 >> 8) == (tl2 & 255) ? 1u : 2u); ++pass) {
            const u32 tb = tl2 >> (pass * 8);
            const u32 keep = (tb >> 7) & 1, jt = (tb >> 4) & 7, ct = tb & 15;
            const u32 j = perm8(jt, bi);
            const u32 lc = perm16(ct, bj);
            const u32 lr = row_lr(j, keep ? g : 1 - g);

            if (wave >= 4) {
                const u32 lw = wave - 4;
                const u32 xbase = OFF_XA + lr * 32 * BLK + lw * 4096;
                const u32 wbase = OFF_WB + lc * 32 * BLK + lw * 4096;
                const u32 fo = (lane < 32) ? OFF_FX + (lr * 32 + lane) * 8 : OFF_FW + (lc * 32 + lane - 32) * 8;
                u32 pend = 0, taken = 0;
                for (int s = -4; s < 32; ++s) {
                    const bool refill = pend == 0 && s < 28;
                    if (refill && lw == 0) {
                        u64 t0 = 0;
                        for (u32 it = 0;; ++it) {
                            const u64 fv = ld_flag64(myws, fo);
                            const u64 m = __ballot((u32)fv >= call && (u32)(fv >> 32) >= call);
                            pend = (u32)m & (u32)(m >> 32) & ~taken;
                            if (pend) break;
                            if (it == 0) t0 = __builtin_amdgcn_s_memrealtime();
                            if ((it & 63) == 63 && give_up(myws, call, t0)) { pend = ~taken; break; }
                            __builtin_amdgcn_s_sleep(PSLEEP);
                        }
                        taken |= pend;
                        sel[s & 1] = pend;
                    }
                    if (s < 29) WAIT_VM(16); else if (s == 29) WAIT_VM(8); else WAIT_VM(0);
                    WAIT_LGKM0();
                    __builtin_amdgcn_s_barrier();
                    CBAR();
                    if (s < 28) {
                        if (refill) pend = rfl(sel[s & 1]);
                        const u32 so = ((u32)__builtin_ctz(pend | 0x80000000u)) * BLK;
                        pend &= pend - 1;
                        LAS char* d = (LAS char*)smem + LDS_CTL + (s & 3) * 32768 + lw * 4096;
#pragma unroll
                        for (int i = 0; i < 4; ++i)
                            __builtin_amdgcn_raw_ptr_buffer_load_lds(rme, d + i * 1024, 16, lane * 16, xbase + so + i * 1024, 0, AUX_SC1);
#pragma unroll
                        for (int i = 0; i < 4; ++i)
                            __builtin_amdgcn_raw_ptr_buffer_load_lds(rme, d + 16384 + i * 1024, 16, lane * 16, wbase + so + i * 1024, 0, AUX_SC1);
                    }
                    CBAR();
                }
                continue;
            }
            f32x4 acc[4][4];
#pragma unroll
            for (int a = 0; a < 4; ++a)
#pragma unroll
                for (int b2 = 0; b2 < 4; ++b2) acc[a][b2] = f32x4{0.f, 0.f, 0.f, 0.f};
            {
                const char* xa = smem + LDS_CTL + wm * 8192 + lane * 16;
                const char* wb = smem + LDS_CTL + 16384 + wn * 8192 + lane * 16;
                bf16x8 xA[8], wA[8], xB[8], wB[8];
                auto rd = [&](bf16x8 (&xf)[8], bf16x8 (&wf)[8], u32 slot) {
                    const u32 so = (slot & 3) * 32768;
#pragma unroll
                    for (int i = 0; i < 8; ++i) {
                        wf[i] = *(const bf16x8*)(wb + so + i * 1024);
                        xf[i] = *(const bf16x8*)(xa + so + i * 1024);
                    }
                };
                auto mm = [&](bf16x8 (&xf)[8], bf16x8 (&wf)[8]) {
#pragma unroll
                    for (int h = 0; h < 2; ++h)
#pragma unroll
                        for (int a = 0; a < 4; ++a)
#pragma unroll
                            for (int b2 = 0; b2 < 4; ++b2)
                                acc[a][b2] = __builtin_amdgcn_mfma_f32_16x16x32_bf16(wf[h * 4 + b2], xf[h * 4 + a], acc[a][b2], 0, 0, 0);
                };
                auto sched = [&]() {
#pragma unroll
                    for (int i = 0; i < 16; ++i) {
                        __builtin_amdgcn_sched_group_barrier(0x100, 1, 0);
                        __builtin_amdgcn_sched_group_barrier(0x008, 2, 0);
                    }
                };
#define BAR() do { __builtin_amdgcn_s_waitcnt(0xC07F); CBAR(); __builtin_amdgcn_s_barrier(); CBAR(); __builtin_amdgcn_sched_barrier(0); } while (0)
                BAR(); BAR(); BAR();
                BAR();
                rd(xA, wA, 0);
                for (u32 s = 0; s < 32; s += 2) {
                    BAR();
                    rd(xB, wB, s + 1);
                    mm(xA, wA);
                    sched();
                    __builtin_amdgcn_sched_barrier(0);
                    BAR();
                    rd(xA, wA, s + 2);
                    mm(xB, wB);
                    sched();
                    __builtin_amdgcn_sched_barrier(0);
                }
                WAIT_LGKM0();
            }
            STAMP(1 + 3 * pass);
            u32 lane_ = threadIdx.x & 63;
            asm volatile("" : "+v"(lane_));
            const u32 lane = lane_;

            if (!keep) {
                const u32 pu = (((j >> 2) * 2 + (lc >> 3)) * 32 + part_idx(jt >> 1, ct >> 1)) * 4 + wave;
                const u32 dst = OFF_SPART + pu * 8192;
#pragma unroll
                for (int a = 0; a < 4; ++a)
#pragma unroll
                    for (int u = 0; u < 2; ++u) {
                        u32x4 o;
                        o[0] = pack2(acc[a][2 * u][0], acc[a][2 * u][1]);
                        o[1] = pack2(acc[a][2 * u][2], acc[a][2 * u][3]);
                        o[2] = pack2(acc[a][2 * u + 1][0], acc[a][2 * u + 1][1]);
                        o[3] = pack2(acc[a][2 * u + 1][2], acc[a][2 * u + 1][3]);
                        st128(o, rme, lane * 16, dst + (a * 2 + u) * 1024, AUX_SYS);
                    }
                WAIT_VM(0);
                sel[4 + pass * 4 + wave] = 0x80000000u;
                if (__ballot(sel[4 + pass * 4 + (lane & 3)] >> 31) == ~0ull)
                    st_flag64(mk_rsrc(pick(P.ws, (1 - g) * 4 + (pu >> 7)), WS_USED), lane < 2 ? lane * 8 : 0x80000000u, OFF_NP + ((rank & 3) * 32 + ((pu >> 2) & 31)) * 16, call, 0);
                STAMP(3 + 3 * pass);
            } else {
                const u32 pu = (j * 16 + lc) * 4 + wave;
                CBAR();
                char* const peerp = (char*)P.ws[(rank + 1 + (lane >> 2)) & 7] + OFF_NT + rank * 10240 + ((((rank + 1 + (lane >> 2)) >> 2) & 1) == g ? 5120 : 3072);
                wait_flag64<KSLEEP>(myws, OFF_FP + pu * 8, call);
                STAMP(2 + 3 * pass);
                u32x4 o[8];
                u32 bits[2] = {0u, 0u};
                {
                    u32x4 pv[8];
#pragma unroll
                    for (int i = 0; i < 8; ++i) pv[i] = __builtin_amdgcn_raw_buffer_load_b128(rme, lane * 16, OFF_PART + pu * 8192 + i * 1024, AUX_SC1);
                    const bool bad = rfl(ld_tmo(myws)) == call;
#pragma unroll
                    for (int gi = 0; gi < 8; ++gi) {
                        float y[8];
#pragma unroll
                        for (int k = 0; k < 8; ++k) {
                            const u32 pq = pv[gi][k >> 1];
                            float t = acc[gi >> 1][(gi & 1) * 2 + (k >> 2)][k & 3] + ((k & 1) ? bf_hi(pq) : bf_lo(pq));
                            if (bad) t = __builtin_nanf("");
                            const bool nz = !(t <= 0.f);
                            y[k] = nz ? t : 0.f;
                            bits[gi >> 2] |= nz ? (1u << ((gi & 3) * 8 + k)) : 0u;
                        }
                        o[gi] = u32x4{pack2(y[0], y[1]), pack2(y[2], y[3]), pack2(y[4], y[5]), pack2(y[6], y[7])} & 0x7fff7fffu;
                    }
                }
                u32 tot;
                u32 addr = (u32)(LDS_CTL + wave * 16384) + 2 * wave_excl(__builtin_popcount(bits[0]) + __builtin_popcount(bits[1]), tot);
                const u32 H = hidden_words(tot), eo = ((2 * tot + 3) & ~3u) - 4 * H;
                u32 len = (eo + 512 + 15) & ~15u, off;
                asm volatile("ds_add_rtn_u32 %0, %1, %2" : "=v"(off) : "v"(lane == 0 ? 64 + pass * 8 + (wave >> 1) * 4 : LDS_CTL + wave * 16384 + 10496 + lane * 4), "v"(len) : "memory");
                u32* wt = (u32*)(wl + 9216);
                wt[lane] = bits[0];
                wt[64 + lane] = bits[1];
                *(u32*)(wl + (lane >= H ? eo + lane * 4 : 10240)) = bits[0];
                *(u32*)(wl + (64 + lane >= H ? eo + 256 + lane * 4 : 10240)) = bits[1];
#pragma unroll
                for (int gi = 0; gi < 8; ++gi)
#pragma unroll
                    for (int k = 0; k < 8; ++k) {
                        const u32 t = (bits[gi >> 2] >> ((gi & 3) * 8 + k)) & 1u;
                        const u32 d = o[gi][k >> 1];
                        *(LAS unsigned short*)((LAS char*)smem + __umul24(addr, t)) = (unsigned short)((k & 1) ? d >> 16 : d);
                        asm("v_lshl_add_u32 %0, %1, 1, %0" : "+v"(addr) : "v"(t));
                    }
                asm volatile("s_waitcnt lgkmcnt(0)" : "+v"(off) :: "memory");
                off = rfl(off);
                const u32 tl = jt * 16 + ct;
                const u32 sidx = ((P.o.ord[tl >> 2] >> ((tl & 3) * 8)) & 127) * 4 + wave;
                const u32 so = OFF_CMP + (sidx & ~1u) * CSTR + off;
#pragma unroll
                for (int i = 0; i < 9; ++i)
                    if (i * 1024u < len) {
                        u32x4 v = *(const u32x4*)(wl + i * 1024 + lane * 16);
                        const u32 G = i * 64 + lane, pl = (G >= H) + (G >= 2 * H) + (G >= 3 * H);
                        const u32 hw = G < 4 * H ? wt[(G - pl * H) & 127] << (6 - pl * 2) : 0u;
                        v[0] |= __builtin_amdgcn_perm(0u, hw << 1, 0x010c000cu) & 0x80008000u;
                        v[1] |= __builtin_amdgcn_perm(0u, hw << 1, 0x030c020cu) & 0x80008000u;
                        v[2] |= __builtin_amdgcn_perm(0u, hw, 0x010c000cu) & 0x80008000u;
                        v[3] |= __builtin_amdgcn_perm(0u, hw, 0x030c020cu) & 0x80008000u;
                        st128(v, rme, i * 1024 + lane * 16 < len ? lane * 16 : 0x80000000u, so + i * 1024, AUX_SYS);
                    }
                WAIT_VM(0);
                sel[4 + pass * 4 + wave] = len | (off != 0) | tot << 16 | 0x80000000u;
                const u32 l4 = sel[4 + pass * 4 + (lane & 3)];
                if (__ballot(l4 >> 31) == ~0ull) {
                    const u64 note = (u64)call << 32 | (l4 & 0x7fffffffu);
                    if (lane < 28)
                        __hip_atomic_store((u64*)(peerp + ((sidx & ~3u) + (lane & 3)) * 8), note, __ATOMIC_RELAXED, __HIP_MEMORY_SCOPE_SYSTEM);
                }
                STAMP(3 + 3 * pass);
                const u32 obase = (glob_blk(g, bi, lr) * 128 + wm * 64) * (NN * 2) + (glob_blk(g, bj, lc) * 128 + wn * 64) * 2;
#pragma unroll
                for (int gi = 0; gi < 8; ++gi)
                    st_out(o[gi], ro, olane, obase + (gi >> 1) * 16 * (NN * 2) + (gi & 1) * 64);
                WAIT_LGKM0();
            }
        }
    }
}

extern "C" void kernel_launch(int rank, int nranks, ncclComm_t comm,
                              hipStream_t stream,
                              void* const* d_in, const int* in_sizes, int n_in,
                              void* d_out, int out_size,
                              const int* in_dtypes, int out_dtype,
                              void* const* d_out_all,
                              void* const* d_ws_all, size_t ws_bytes) {
    if (nranks != NR || n_in < 2 || ws_bytes < WS_USED || (unsigned)rank >= (unsigned)NR ||
        in_sizes[0] != MM * KS || in_sizes[1] != KS * NN || out_size != MM * NN) {
        fprintf(stderr, "unexpected problem shape\n");
        return;
    }
    Params P;
    P.x = d_in[0];
    P.w = d_in[1];
    for (int i = 0; i < NR; ++i) P.ws[i] = d_ws_all[i];
    P.me = d_ws_all[rank];
    P.out = d_out;
    P.rank = rank;
    P.o = ORDERS;
    for (u32 b = 0; b < (u32)GRID; ++b) {
        u32 f[2] = {0xffffu, 0xffffu}, du;
        if (b < (u32)NPWG) {
            for (u32 w = 0; w < (u32)NPUSHW; ++w) unit_desc(rank, (rank + 1 + ((b >> 3) + (b & 7)) % 7) & 7, w * EWG + (b & 7), f[w], du);
        } else if (b >= (u32)SFIRST && b < (u32)(GRID - NPWG)) {
            u32 n, other, hf;
            stage_desc(b - SFIRST, n, other, hf);
            unit_desc(rank, (((u32)rank >> 2) ^ other) * 4 + hf * 3, n, f[0], du);
        }
        P.first[b] = f[0] | f[1] << 16;
    }
    HIP_CHECK(hipFuncSetAttribute((const void*)fused_gemm_ar, hipFuncAttributeMaxDynamicSharedMemorySize, LDS_BYTES));
    hipLaunchKernelGGL(fused_gemm_ar, dim3(GRID), dim3(512), LDS_BYTES, stream, P);
    HIP_CHECK(hipGetLastError());
}
```

```cpp
#include <hip/hip_runtime.h>
#include <rccl/rccl.h>
#include <cstdio>
#include <cstdint>

#define HIP_CHECK(expr) do { hipError_t _e = (expr); if (_e != hipSuccess) { \
    fprintf(stderr, "HIP error %s at %s:%d\n", hipGetErrorString(_e), __FILE__, __LINE__); \
    return; } } while (0)

#define XS 16
#define XNP 2
#define KSLEEP 8
#define TWARM 1
#define TSTEP 23
#define WSLEEP 1
#define LOOK 6
#define LOOK2 4
#define XE 48
#define XH 16
#define NPUSHW 2
#define SDELAY 20
#define PWIN 3
#define WIN 5
#define SELJ 5
#define DIAG 0
#define LAT_K 12.0
#define LAT_S 50.0
#define ENTRY_ORDER "s0w0s1w1w2w3w4w5s2k0k1s3w6w7k2s4s5k3s6s7k4k5w8w9k6k7wAwBwCwDwEwF"
#define ESCALE 1.34
#define SFIRST 72
#define OUT_AUX 17
#define PSLEEP 32

typedef unsigned int u32;
typedef unsigned long long u64;
typedef u32 u32x4 __attribute__((ext_vector_type(4)));
typedef u32 u32x2 __attribute__((ext_vector_type(2)));
typedef float f32x4 __attribute__((ext_vector_type(4)));
typedef short bf16x8 __attribute__((ext_vector_type(8)));
typedef __bf16 bf16v2 __attribute__((ext_vector_type(2)));
typedef float f32x2 __attribute__((ext_vector_type(2)));
#define LAS __attribute__((address_space(3)))

constexpr int NR = 8;
constexpr int MM = 4096, NN = 4096, KS = 512;
constexpr u32 BLK = 16384;
constexpr u32 OFF_XA = 0;
constexpr u32 OFF_WB = 8u << 20;
constexpr u32 OFF_PART = 16u << 20;
constexpr u32 OFF_SPART = 20u << 20;
constexpr u32 CSTR = 9216;
constexpr u32 CMAX = 8704;
constexpr u32 OFF_CMP = 24u << 20;
constexpr u32 OFF_STG = 30u << 20;
constexpr u32 OFF_FLAGS = 38u << 20;
constexpr u32 OFF_FX = OFF_FLAGS;
constexpr u32 OFF_FW = OFF_FX + 4096;
constexpr u32 OFF_FP = OFF_FW + 4096;
constexpr u32 OFF_NP = OFF_FP + 4096;
constexpr u32 OFF_NT = OFF_NP + 2048;
constexpr u32 OFF_CTR = OFF_NT + 81920;
constexpr u32 OFF_EP = OFF_CTR + 4096;
constexpr u32 OFF_TMO = OFF_EP + 8192;
constexpr u32 OFF_DIAG = OFF_TMO + 4096;
constexpr u32 WS_USED = OFF_DIAG + (1u << 20);
constexpr u32 OUT_BYTES = (u32)MM * NN * 2;
constexpr u32 IN_BYTES = (u32)MM * KS * 2;

constexpr int EWG = 8;
constexpr int NPWG = 7 * EWG;
constexpr int NPUSH = EWG * NPUSHW;
constexpr int NCWG = 192;
constexpr int GRID = 256;
constexpr u32 LDS_CTL = 1024;
constexpr int LDS_BYTES = LDS_CTL + 4 * 32768 + (DIAG ? 1024 : 0);
constexpr u64 DEADLINE = 300000000ull;
constexpr int AUX_SYS = 17;
constexpr int AUX_SC1 = 16;
static_assert(GRID * 8 * 4 <= 8192, "epoch words");
static_assert(OFF_CMP + 512u * CSTR <= OFF_STG, "layout");
static_assert(14 * CSTR <= 4 * 32768, "engine slots");
static_assert(NPUSHW <= 2 && SFIRST >= NPWG + 16, "first loads");
static_assert((GRID - NPWG - SFIRST) * 8 >= 1024 - 2 * NPUSH && SFIRST >= NPWG, "staging waves");

struct Orders {
    u64 entry[4];
    unsigned short tiles[NCWG];
    u32 ord[64];
};
struct Params {
    const void* x;
    const void* w;
    void* me;
    int rank;
    void* out;
    void* ws[NR];
    u32 first[GRID];
    Orders o;
};

constexpr double mx(double a, double b) { return a > b ? a : b; }
constexpr Orders build_orders() {
    Orders P{};
    const char* eo = ENTRY_ORDER;
    double t[3][16] = {};
    for (int e = 0; e < 32; ++e) {
        const int kind = eo[2 * e] == 'w' ? 0 : eo[2 * e] == 's' ? 1 : 2;
        const char ch = eo[2 * e + 1];
        const int idx = ch <= '9' ? ch - '0' : ch - 'A' + 10;
        P.entry[e >> 3] |= (u64)(kind << 4 | idx) << ((e & 7) * 8);
        t[kind][idx] = (e + 1) * ESCALE;
    }
    double av[128] = {};
    for (int i = 0; i < 128; ++i) {
        const int jt = i >> 4, ct = i & 15;
        av[i] = mx(mx(t[2][jt], t[0][ct]) + LAT_K, mx(t[1][jt], t[0][ct]) + LAT_S);
    }
    for (int i = 0; i < 128; ++i) {
        int pos = 0;
        for (int k = 0; k < 128; ++k) pos += av[k] < av[i] || (av[k] == av[i] && k < i);
        P.ord[i >> 2] |= (u32)pos << ((i & 3) * 8);
        P.ord[32 + (pos >> 2)] |= (u32)i << ((pos & 3) * 8);
    }
    const int W = NCWG / 8, D = 32 - W;
    for (int x = 0; x < 8; ++x) {
        int id[32] = {}, sorted[32] = {};
        double key[32] = {};
        for (int i = 0; i < 32; ++i) {
            const int keep = i >> 4, jt = (x >> 1) + 4 * ((i >> 3) & 1), ct = (i & 7) * 2 + (x & 1);
            id[i] = keep << 7 | jt << 4 | ct;
            key[i] = keep ? av[jt * 16 + ct] - LAT_K : mx(t[1][jt], t[0][ct]);
        }
        for (int i = 0; i < 32; ++i) {
            int pos = 0;
            for (int k = 0; k < 32; ++k) pos += key[k] < key[i] || (key[k] == key[i] && k < i);
            sorted[pos] = id[i];
        }
        for (int i = 0; i < W; ++i) P.tiles[i * 8 + x] = (unsigned short)(sorted[i] | sorted[i < D ? W + i : i] << 8);
    }
    return P;
}
constexpr Orders ORDERS = build_orders();

typedef __amdgpu_buffer_rsrc_t rsrc_t;
__device__ __forceinline__ rsrc_t mk_rsrc(const void* p, u32 bytes) {
    return __builtin_amdgcn_make_buffer_rsrc((void*)p, (short)0, (int)bytes, 0x00020000);
}
__device__ __forceinline__ u32 rfl(u32 v) { return (u32)__builtin_amdgcn_readfirstlane((int)v); }
__device__ __forceinline__ void* pick(void* const (&t)[NR], u32 idx) {
    void* r = t[0];
#pragma unroll
    for (u32 i = 1; i < NR; ++i) r = (idx == i) ? t[i] : r;
    return r;
}
__host__ __device__ __forceinline__ u32 glob_blk(u32 g, u32 b, u32 l) {
    return g == 0 ? b * 16 + l : ((l >> 3) * 2 + b) * 8 + (l & 7);
}
__host__ __device__ __forceinline__ u32 perm8(u32 t, u32 h) { return (t >> 1) + ((t ^ h) & 1) * 4; }
__host__ __device__ __forceinline__ u32 perm16(u32 t, u32 h) { return (t >> 1) + ((t ^ h) & 1) * 8; }
__host__ __device__ __forceinline__ u32 row_lr(u32 j, u32 fin) { return (j >> 2) * 8 + fin * 4 + (j & 3); }

__device__ __forceinline__ u32 part_idx(u32 a, u32 b) { return b >= 4 ? b * 4 + a : (b > a ? b * b + a : a * a + a + b); }
__device__ __forceinline__ void part_ab(u32 m, u32& a, u32& b) {
    const u32 sh = m >= 9 ? 3 : m >= 4 ? 2 : m >= 1 ? 1 : 0, o = m - sh * sh;
    a = o < sh ? o : sh;
    b = o < sh ? sh : o - sh;
    if (m >= 16) { a = m & 3; b = m >> 2; }
}
__host__ __device__ __forceinline__ void unit_desc(u32 s, u32 d, u32 n, u32& su, u32& du) {
    constexpr u64 e0 = ORDERS.entry[0], e1 = ORDERS.entry[1], e2 = ORDERS.entry[2], e3 = ORDERS.entry[3];
    const bool in = (s >> 2) == (d >> 2);
    const u32 gd = d >> 2, e = in ? n / 10 : n / 6, kk = n - e * (in ? 10 : 6);
    const u32 eb = (u32)(((e & 16) ? ((e & 8) ? e3 : e2) : ((e & 8) ? e1 : e0)) >> ((e & 7) * 8));
    const u32 ek = (eb >> 4) & 3, idx = eb & 15;
    const bool isw = ek == 0;
    const u32 l = isw ? perm16(idx, d) : row_lr(perm8(idx & 7, d >> 1), ek == 1 ? 1 - gd : gd);
    su = (isw ? 512 : 0) + glob_blk(gd, isw ? d & 1 : (d >> 1) & 1, l) * 16 + (in ? 0 : 10) + kk;
    du = (isw ? 1024 : 0) + l * 64 + (in ? (s & 3) * 10 : 40 + (s & 3) * 6) + kk;
}
__host__ __device__ __forceinline__ void stage_desc(u32 u, u32& n, u32& other, u32& hf) {
    const bool four = u >= 2 * NPUSH && u < 768 - 2 * NPUSH;
    hf = u & 1; other = four ? (u >> 1) & 1 : 0;
    n = u < 2 * NPUSH ? u >> 1 : four ? (u + 2 * NPUSH) >> 2 : (u >> 1) - 192 + NPUSH;
}
__device__ __forceinline__ void fetch_issue(const void* x, const void* w, u32 su, u32 lane, u32x4* v) {
    const u32 isw = su >> 9, GB = (su >> 4) & 31, kc = (su >> 1) & 7, hf = su & 1;
    const rsrc_t r = mk_rsrc(isw ? w : x, IN_BYTES);
    const u32 vo = isw ? (lane >> 3) * 65536 + (lane & 7) * 16 : (lane & 15) * 1024 + (lane >> 4) * 16;
    const u32 so = isw ? kc * 64 * 8192 + GB * 256 + hf * 128 : GB * 131072 + hf * 65536 + kc * 128;
#pragma unroll
    for (int i = 0; i < 8; ++i) v[i] = __builtin_amdgcn_raw_buffer_load_b128(r, vo, so + (isw ? i * 8192 : (i & 3) * 16384 + (i >> 2) * 64), 0);
}
__device__ __forceinline__ void fetch_finish(u32 su, u32 lane, char* wl, u32x4* v) {
    if (su >> 9) {
        const u32 wo = (lane >> 5) * 4096 + ((lane & 7) >> 2) * 2048 + (((lane >> 3) & 3) * 16 + (lane & 3) * 4) * 16;
#pragma unroll
        for (int c = 0; c < 8; ++c) {
            u32x4 o;
#pragma unroll
            for (int t = 0; t < 4; ++t)
                o[t] = __builtin_amdgcn_perm(v[2 * t + 1][c >> 1], v[2 * t][c >> 1], (c & 1) ? 0x07060302u : 0x05040100u);
            *(u32x4*)(wl + wo + (c >> 2) * 1024 + (c & 3) * 16) = o;
        }
        asm volatile("s_waitcnt lgkmcnt(0)" ::: "memory");
#pragma unroll
        for (int i = 0; i < 8; ++i) v[i] = *(const u32x4*)(wl + i * 1024 + lane * 16);
        asm volatile("s_waitcnt lgkmcnt(0)" ::: "memory");
    }
}
__device__ __forceinline__ void st128(u32x4 v, rsrc_t r, u32 voff, u32 soff, int) {
    __builtin_amdgcn_raw_buffer_store_b128(v, r, voff + soff, 0, AUX_SYS);
}
__device__ __forceinline__ void st_out(u32x4 v, rsrc_t r, u32 voff, u32 soff) {
    __builtin_amdgcn_raw_buffer_store_b128(v, r, voff + soff, 0, OUT_AUX);
}
__device__ __forceinline__ void st_flag(rsrc_t r, u32 l0, u32 off, u32 v) {
    __builtin_amdgcn_raw_buffer_store_b32(v, r, l0, off, AUX_SYS);
}
__device__ __forceinline__ void st_flag64(rsrc_t r, u32 l0, u32 off, u32 call, u32 len) {
    __builtin_amdgcn_raw_buffer_store_b64(u32x2{len, call}, r, l0, off, AUX_SYS);
}
__device__ __forceinline__ u64 ld_flag64(const void* ws, u32 off) {
    return __hip_atomic_load((const u64*)((const char*)ws + off), __ATOMIC_RELAXED, __HIP_MEMORY_SCOPE_SYSTEM);
}
#define CBAR() asm volatile("" ::: "memory")
#define WAIT_VM(n) asm volatile("s_waitcnt vmcnt(" #n ")" ::: "memory")
#define WAIT_VM0() do { WAIT_VM(0); __builtin_amdgcn_s_waitcnt(0x0F70); CBAR(); } while (0)
#define WAIT_LGKM0() asm volatile("s_waitcnt lgkmcnt(0)" ::: "memory")

__device__ __forceinline__ bool give_up(void* myws, u32 call, u64 t0) {
    u32* tmo = (u32*)((char*)myws + OFF_TMO);
    bool late = __builtin_amdgcn_s_memrealtime() - t0 > DEADLINE;
    if (late || rfl(__hip_atomic_load(tmo, __ATOMIC_RELAXED, __HIP_MEMORY_SCOPE_AGENT)) == call) {
        __hip_atomic_store(tmo, call, __ATOMIC_RELAXED, __HIP_MEMORY_SCOPE_AGENT);
        return true;
    }
    return false;
}
__device__ __forceinline__ u32 ld_tmo(void* myws) {
    return __hip_atomic_load((u32*)((char*)myws + OFF_TMO), __ATOMIC_RELAXED, __HIP_MEMORY_SCOPE_AGENT);
}
template <int SLP>
__device__ __forceinline__ u32 wait_flag64(void* myws, u32 off, u32 call) {
    u64 t0 = 0;
    for (u32 it = 0;; ++it) {
        u64 v = ld_flag64(myws, off);
        if (rfl((u32)(v >> 32)) >= call) return rfl((u32)v);
        if (it == 0) t0 = __builtin_amdgcn_s_memrealtime();
        if ((it & 63) == 63 && give_up(myws, call, t0)) return 0;
        __builtin_amdgcn_s_sleep(SLP);
    }
}
__device__ __forceinline__ u32 wave_excl(u32 cnt, u32& tot) {
    int x = (int)cnt;
    x += __builtin_amdgcn_update_dpp(0, x, 0x111, 0xf, 0xf, false);
    x += __builtin_amdgcn_update_dpp(0, x, 0x112, 0xf, 0xf, false);
    x += __builtin_amdgcn_update_dpp(0, x, 0x114, 0xf, 0xf, false);
    x += __builtin_amdgcn_update_dpp(0, x, 0x118, 0xf, 0xf, false);
    x += __builtin_amdgcn_update_dpp(0, x, 0x142, 0xa, 0xf, false);
    x += __builtin_amdgcn_update_dpp(0, x, 0x143, 0xc, 0xf, false);
    tot = (u32)__builtin_amdgcn_readlane(x, 63);
    return (u32)x - cnt;
}
__device__ __forceinline__ u32 hidden_words(u32 nv) { const u32 h = nv >> 5; return h > 64 && h < 80 ? 64 : h; }
__device__ __forceinline__ u32 pack2(float a, float b) {
    f32x2 v = {a, b};
    return __builtin_bit_cast(u32, __builtin_convertvector(v, bf16v2));
}
__device__ __forceinline__ float bf_lo(u32 v) { return __builtin_bit_cast(float, v << 16); }
__device__ __forceinline__ float bf_hi(u32 v) { return __builtin_bit_cast(float, v & 0xffff0000u); }

#define STAMP(slot) do {} while (0)

__global__ void __launch_bounds__(512, 2) fused_gemm_ar(Params P) {
    extern __shared__ __attribute__((aligned(16))) char smem[];
    const u32 lane = threadIdx.x & 63;
    const u32 wave = rfl(threadIdx.x >> 6);
    const void* px = P.x;
    const void* pw = P.w;
    u64 pme = (u64)P.me;
    u32 rk = (u32)P.rank, fw = P.first[blockIdx.x];
    void* pt = P.ws[wave];
    void* po = P.out;
    asm volatile("" : "+s"(px), "+s"(pw), "+s"(pme), "+s"(rk), "+s"(fw), "+s"(pt), "+s"(po));
    if (TWARM && blockIdx.x - NPWG < 16) {
        const bool wsw = blockIdx.x < NPWG + 8;
        const rsrc_t rt = mk_rsrc(wsw ? pt : po, wsw ? WS_USED : wave == 0 ? OUT_BYTES : 0);
        const u32 t = __builtin_amdgcn_raw_buffer_load_b32(rt, lane < 32 ? (lane << TSTEP) + (blockIdx.x & 7) * 4096 + lane * 256 : 0x80000000u, 0, AUX_SYS);
        asm volatile("" :: "v"(t));
    }
    void* myws = (void*)(__attribute__((address_space(1))) void*)pme;
    u32x4 v[8];
    const u32 su0 = wave < 2 ? (fw >> (wave * 16)) & 0xffffu : 0xffffu;
    if (su0 != 0xffffu) fetch_issue(px, pw, su0, lane, v);
    __builtin_amdgcn_sched_barrier(0);
    const u32 rank = rk & 7;
    const u32 g = rank >> 2, bi = (rank >> 1) & 1, bj = rank & 1;
    const u32 lb = (blockIdx.x + NPWG) & (GRID - 1);
    const rsrc_t rme = mk_rsrc(myws, WS_USED);
    char* wl = smem + LDS_CTL + wave * 16384;

    const u32 l0 = lane == 0 ? 0u : 0x80000000u;
    const u32 epo = OFF_EP + (lb * 8 + wave) * 4;
    const u32 epv = __builtin_amdgcn_raw_buffer_load_b32(rme, 0, epo, AUX_SC1);
    *(LAS u32*)smem = 0u;
    STAMP(0);

    const u32 olane = (lane & 15) * (NN * 2) + (lane >> 4) * 16;
    const rsrc_t ro = mk_rsrc(po, OUT_BYTES);

    if (lb < NPWG) {
        const u32 b = lb;
        const u32 di = 1 + ((b >> 3) + (b & 7)) % 7;
        const u32 p = (rank + di) & 7;
        volatile LAS u32* ctl = (volatile LAS u32*)(smem + 64);
        const u32 dum = 640 + lane * 4;
        const u32 cra = lane == 0 ? 64 + 15 * 32 + 4 : dum;
        if (wave == 0) *(volatile LAS u32x4*)(ctl + (lane & 15) * 8) = u32x4{0u, (u32)LOOK, 0u, 0u};
        WAIT_LGKM0();
        __builtin_amdgcn_s_barrier();
        CBAR();
        if (wave == 0) {
            __builtin_amdgcn_s_setprio(3);
            WAIT_VM0();
            u32 prev = 14, pnp = 0, tsl = 0, treq = 0, tsrc = 0;
            u64 t0 = __builtin_amdgcn_s_memrealtime();
            auto select = [&](u32x2 cv) -> bool {
                int mk = (cv[0] & 1) ? (int)((cv[0] >> 16) << 4 | (lane & 15)) : 0x7fffffff;
                mk = min(mk, __builtin_amdgcn_update_dpp(0x7fffffff, mk, 0x111, 0xf, 0xf, false));
                mk = min(mk, __builtin_amdgcn_update_dpp(0x7fffffff, mk, 0x112, 0xf, 0xf, false));
                mk = min(mk, __builtin_amdgcn_update_dpp(0x7fffffff, mk, 0x114, 0xf, 0xf, false));
                mk = min(mk, __builtin_amdgcn_update_dpp(0x7fffffff, mk, 0x118, 0xf, 0xf, false));
                const u32 r = (u32)__builtin_amdgcn_readlane(mk, 15);
                if (r == 0x7fffffffu) return false;
                tsl = r & 15;
                treq = (u32)__builtin_amdgcn_readlane((int)cv[0], (int)tsl);
                tsrc = (u32)__builtin_amdgcn_readlane((int)cv[1], (int)tsl);
                ctl[tsl * 8] = 0;
                asm volatile("ds_add_u32 %0, %1" :: "v"(cra), "v"(tsrc & 1) : "memory");
                return true;
            };
            bool have = false;
            for (;;) {
                if (!have) {
                    bool end = false;
                    for (u32 it = 0;; ++it) {
                        const u32x2 cv = *(volatile LAS u32x2*)(ctl + (lane & 15) * 8);
                        if (select(cv)) { end = false; break; }
                        if (prev != 14) { WAIT_VM(0); ctl[prev * 8 + 2] = pnp; prev = 14; }
                        if (end) break;
                        end = (u32)__builtin_amdgcn_readlane((int)cv[0], 15) != 0;
                        if ((it & 1023) == 1023 && __builtin_amdgcn_s_memrealtime() - t0 > 2 * DEADLINE) end = true;
                        __builtin_amdgcn_s_sleep(1);
                    }
                    if (end) break;
                }
                const u32 sl = tsl, len = ((treq >> 8) & 255) * 64, np = (treq >> 4) & 15, src = tsrc & ~1u, wide = treq & 2, treq0 = treq;
                const rsrc_t rs = mk_rsrc(pick(P.ws, p), src + len);
                const u32 vo = src + lane * 16;
                LAS char* d = (LAS char*)smem + LDS_CTL + sl * CSTR;
                u32x2 cv = {0u, 0u};
#define PULL_STEP(j, W, ALL) \
                if ((j) == (W) - 1 && (ALL || len > ((j) - 1) * 1024u)) { \
                    asm volatile("s_waitcnt vmcnt(%0)" :: "n"((W) - 1) : "memory"); \
                    ctl[prev * 8 + 2] = pnp; \
                    prev = 14; \
                } \
                if (ALL || (j) * 1024u < len) { \
                    asm volatile("s_waitcnt vmcnt(%0)" :: "n"((W) - 1) : "memory"); \
                    __builtin_amdgcn_raw_ptr_buffer_load_lds(rs, d + ((j) >> 2) * 4096, 16, vo + ((j) >> 2) * 4096, 0, ((j) & 3) * 1024, AUX_SYS); \
                } \
                CBAR(); \
                if ((j) == SELJ - 1) cv = *(volatile LAS u32x2*)(ctl + (lane & 15) * 8); \
                if ((j) == SELJ) have = select(cv); \
                CBAR();
#define PULL_UNIT(W) PULL_STEP(0, W, 1) PULL_STEP(1, W, 1) PULL_STEP(2, W, 1) PULL_STEP(3, W, 1) \
                    PULL_STEP(4, W, 1) PULL_STEP(5, W, 1) PULL_STEP(6, W, 1) PULL_STEP(7, W, 1)
#define PULL_PAIR(W) PULL_STEP(0, W, 0) PULL_STEP(1, W, 0) PULL_STEP(2, W, 0) PULL_STEP(3, W, 0) PULL_STEP(4, W, 0) \
                    PULL_STEP(5, W, 0) PULL_STEP(6, W, 0) PULL_STEP(7, W, 0) PULL_STEP(8, W, 0)
                if (!(treq0 & 4)) {
                    if (wide) { PULL_UNIT(WIN + 1) } else { PULL_UNIT(WIN) }
                } else {
                    if (wide) { PULL_PAIR(WIN + 1) } else { PULL_PAIR(WIN) }
                }
                if (prev != 14) { WAIT_VM(0); ctl[prev * 8 + 2] = pnp; }
                prev = sl; pnp = np;
            }
            ctl[15 * 8 + 3] = 1;
            STAMP(3);
            __builtin_amdgcn_s_setprio(0);
        }
        const u32 gp = p >> 2;
        const u32 call = rfl(epv) + 1;
        st_flag(rme, l0, epo, call);
        if (wave == 1) {
            __builtin_amdgcn_s_setprio(2);
            const u32 ordv = P.o.ord[lane];
            const u32 T1 = (gp == g) ? 320 : 192;
            const u32 TH = T1 + 256 - XH, T = TH + 2 * XH;
            const u32 tbase = (call - 1) * (T - NPUSH + 2 * EWG) - NPUSH;
            u32 n = NPUSH, ne = NPUSH;
            u32 pmask = gp == g ? 0u : 0xffffu;
            u32 half = 0;
            u32 share = NPUSH;
            bool over = false, force = false;
            u64 t0 = __builtin_amdgcn_s_memrealtime();
            for (u32 it = 0;; ++it) {
                if (over && n == ne && pmask == 0) break;
                u32 k = 0;
                if (!over && n == ne) {
                    k = rfl(ctl[15 * 8 + 1]);
                    if (n + XE >= T) k = k > LOOK - LOOK2 ? k - (LOOK - LOOK2) : 0;
                    k = k > 2 ? 2 : k;
                    if (k == 0 && pmask == 0) { __builtin_amdgcn_s_sleep(1); if ((it & 4095) == 4095 && __builtin_amdgcn_s_memrealtime() - t0 > 2 * DEADLINE) break; continue; }
                    asm volatile("ds_sub_u32 %0, %1" :: "v"(cra), "v"(k) : "memory");
                }
                const u32 tv = (u32)__builtin_amdgcn_raw_ptr_buffer_atomic_add_i32((int)k, rme, k ? l0 : 0x80000000u, OFF_CTR + di * 128, 0);
                const u32 wb = n < TH ? n : TH + ((n - TH) >> 1);
                const u32x4 nv = __builtin_amdgcn_raw_buffer_load_b128(rme, lane < 47 ? OFF_NT + (p * 640 + wb + lane) * 16 :
                    lane == 47 ? OFF_TMO : OFF_NP + ((p & 3) * 32 + (lane - 48) * 2 + ((b >> 2) & 1)) * 16, 0, AUX_SYS);
                WAIT_VM0();
                if (k) {
                    n = rfl(tv) - tbase; ne = n + k;
                    share += k * EWG;
                    if (ne > T) {
                        over = true;
                        if (ne - (n > T ? n : T) < 2) __builtin_amdgcn_raw_ptr_buffer_atomic_add_i32(1, rme, l0, OFF_CTR + di * 128, 0);
                        ne = T; n = n > T ? T : n;
                    }
                }
                if ((u32)__builtin_amdgcn_readlane((int)nv[0], 47) == call) force = true;
                u64 rm = force ? ~0ull : __ballot(nv[1] >= call && nv[3] >= call);
                u32 fr = (u32)__ballot(ctl[(lane & 15) * 8 + 3] == 0) & 0x3fffu;
                bool progress = false;
                while (fr) {
                    const u32 pr = (u32)(rm >> 48) & pmask;
                    const u32 m = n < TH ? n : TH + ((n - TH) >> 1);
                    u32 src, dst, aux, aux2 = 0, req, nq = 0;
                    if (pr) {
                        const u32 i = (u32)__builtin_ctz(pr), un = (b & 7) + 8 * i;
                        pmask &= ~(1u << i);
                        u32 a, bb; part_ab(un >> 2, a, bb);
                        const u32 pu = ((((p >> 1) & 1) * 4 + a) * 16 + (p & 1) * 8 + bb) * 4 + (un & 3);
                        src = OFF_SPART + ((rank & 3) * 128 + un) * 8192;
                        dst = OFF_PART + pu * 8192; aux = 0x80000000u | (OFF_FP + pu * 8 + 4);
                        req = 1 | 1 << 4 | 128 << 8 | (15 - __builtin_popcount(pmask)) << 16;
                    } else if (n < ne && m - wb < 47 && ((rm >> (m - wb)) & 1)) {
                        if (n < T1) {
                            u32 su, du; unit_desc(p, rank, n, su, du);
                            src = OFF_STG + su * 8192 + 1; dst = OFF_XA + du * 8192; aux = 0x80000000u | (OFF_FX + du * 4);
                            req = 1 | 1 << 4 | 128 << 8 | (n + 16) << 16;
                            ++n;
                        } else {
                            u32 lA = (u32)__builtin_amdgcn_readlane((int)nv[0], (int)(m - wb)), lB = (u32)__builtin_amdgcn_readlane((int)nv[2], (int)(m - wb));
                            if (force || (lA & 0xffffu) > CMAX || (lA >> 16) > 4096) lA = 512;
                            if (force || (lB & 0xffffu) > CMAX || (lB >> 16) > 4096) lB = 512;
                            const u32 n3 = m - T1, to = n3 >> 1;
                            const u32 tl = ((u32)__builtin_amdgcn_readlane((int)ordv, (int)(32 + (to >> 2)))) >> ((to & 3) * 8);
                            const u32 tRB = glob_blk(gp, (p >> 1) & 1, row_lr(perm8((tl >> 4) & 7, p >> 1), gp)), tCB = glob_blk(gp, p & 1, perm16(tl & 15, p));
                            dst = (tRB * 128 + (n3 & 1) * 64) * (NN * 2) + tCB * 256;
                            const u32 bA = lA & 0xfff0u, bB = lB & 0xfff0u, oA = (lA & 1) ? bB : 0, oB = (lA & 1) ? 0 : bA;
                            const bool onlyB = half || (n >= TH && ((n - TH) & 1));
                            half = !onlyB && n < TH && bA + bB > CSTR;
                            const bool onlyA = half || (!onlyB && n >= TH);
                            const u32 st = onlyA ? oA & ~127u : onlyB ? oB & ~127u : 0, en = onlyA ? oA + bA : onlyB ? oB + bB : bA + bB;
                            aux = onlyB ? 0u : (lA & 0xffff0000u) | (oA - st) | 1;
                            aux2 = (onlyA ? 0u : (lB & 0xffff0000u) | (oB - st) | 1) | (force ? 0x80000000u : 0u);
                            src = OFF_CMP + n3 * 2 * CSTR + st + (half ? 0 : 1);
                            nq = n + XS >= T ? 1 : 8 / XNP;
                            req = 1 | 4 | ((onlyA || onlyB ? 4 : 8) / nq) << 4 | ((en - st + 63) >> 6) << 8 | (n + 16) << 16;
                            n += 1 - half;
                        }
                    } else break;
                    const u32 sl = (u32)__builtin_ctz(fr);
                    fr &= fr - 1;
                    *(volatile LAS u32x4*)(ctl + sl * 8 + 4) = u32x4{dst, aux, aux2, nq};
                    ctl[sl * 8 + 3] = (req >> 4) & 15;
                    *(volatile LAS u32x2*)(ctl + sl * 8) = u32x2{req | (share < ne ? 2u : 0u), src};
                    progress = true;
                }
                if (progress || k) t0 = __builtin_amdgcn_s_memrealtime();
                else if ((it & 63) == 63 && give_up(myws, call, t0)) {
                    force = true;
                    if (__builtin_amdgcn_s_memrealtime() - t0 > 2 * DEADLINE) break;
                }
            }
            ctl[15 * 8] = 256;
            __builtin_amdgcn_s_setprio(0);
        }
        {
            bool end = false;
            u64 t0 = __builtin_amdgcn_s_memrealtime();
            for (u32 it = 0;; ++it) {
                const u32x2 cv = *(volatile LAS u32x2*)(ctl + (lane & 15) * 8 + 2);
                const u32 av = (u32)__ballot((int)cv[0] > 0) & 0x3fffu;
                if (av == 0) {
                    if (end) break;
                    end = (u32)__builtin_amdgcn_readlane((int)cv[1], 15) != 0;
                    if ((it & 1023) == 1023 && __builtin_amdgcn_s_memrealtime() - t0 > 3 * DEADLINE) break;
                    __builtin_amdgcn_s_sleep(WSLEEP);
                    continue;
                }
                end = false;
                const u32 sl = (u32)__builtin_ctz(av);
                u32 old;
                u32x4 dv;
                asm volatile("ds_add_rtn_u32 %0, %2, %3\n\tds_read_b128 %1, %4\n\ts_waitcnt lgkmcnt(0)" : "=&v"(old), "=&v"(dv)
                             : "v"(lane == 0 ? 64 + sl * 32 + 8 : dum), "v"(0xffffffffu), "v"(64 + sl * 32 + 16) : "memory");
                const u32 pc = rfl(old) - 1;
                if ((int)pc < 0) continue;
                STAMP(8);
                const char* sp = smem + LDS_CTL + sl * CSTR;
                const u32 dst = rfl(dv[0]), aux = rfl(dv[1]);
                const u32 dna = lane == 0 ? 64 + sl * 32 + 12 : dum;
                if (aux >> 31) {
                    u32x4 v[8];
#pragma unroll
                    for (int i = 0; i < 8; ++i) v[i] = *(const u32x4*)(sp + i * 1024 + lane * 16);
                    asm volatile("ds_sub_u32 %0, %1" :: "v"(dna), "v"(1u) : "memory");
#pragma unroll
                    for (int i = 0; i < 8; ++i) st128(v[i], rme, lane * 16, dst + i * 1024, AUX_SYS);
                    WAIT_VM0();
                    st_flag(rme, l0, aux & 0x7fffffffu, call);
                    continue;
                }
                const u32 aux2 = rfl(dv[2]), nq = rfl(dv[3]);
                const bool two = aux & aux2 & 1;
                const u32 hs = two ? pc & 1 : aux2 & 1, q0 = (two ? pc >> 1 : pc) * nq;
                const u32 lw = hs ? aux2 : aux;
                const u32 ob = dst + hs * 128 + q0 * 16 * (NN * 2);
                if (aux2 >> 31) {
#pragma unroll 1
                    for (u32 i = 0; i < 2 * nq; ++i)
                        st_out(u32x4{0x7fc07fc0u, 0x7fc07fc0u, 0x7fc07fc0u, 0x7fc07fc0u}, ro, olane, ob + (i >> 1) * 16 * (NN * 2) + (i & 1) * 64);
                } else {
                    const u32 nv = lw >> 16, sb = lw & 0xfff0u;
                    const u32 H = hidden_words(nv), eo = ((2 * nv + 3) & ~3u) - 4 * H;
                    u32 bits[2];
                    auto hidden = [&](u32 w) {
                        u32x4 v[4];
#pragma unroll
                        for (int i = 0; i < 4; ++i) v[i] = *(const u32x4*)(sp + sb + (i * H + w) * 16);
                        __builtin_amdgcn_sched_barrier(0);
                        u32 hw = 0;
#pragma unroll
                        for (int i = 0; i < 4; ++i) {
                            hw = (__builtin_amdgcn_perm(v[i][1], v[i][0], 0x07050301u) & 0x80808080u) | ((hw >> 1) & 0x7f7f7f7fu);
                            hw = (__builtin_amdgcn_perm(v[i][3], v[i][2], 0x07050301u) & 0x80808080u) | ((hw >> 1) & 0x7f7f7f7fu);
                        }
                        return hw;
                    };
                    bits[0] = *(const u32*)(sp + sb + eo + lane * 4);
                    bits[1] = *(const u32*)(sp + sb + eo + 256 + lane * 4);
                    const u32 h0 = hidden(lane);
                    bits[0] = lane < H ? h0 : bits[0];
                    if (H > 64) { const u32 h1 = hidden(64 + lane); bits[1] = 64 + lane < H ? h1 : bits[1]; }
                    u32 tot;
                    u32 addr = LDS_CTL + sl * CSTR + sb + 2 * wave_excl(__builtin_popcount(bits[0]) + __builtin_popcount(bits[1]), tot);
                    u64 bq = ((u64)bits[1] << 32 | bits[0]);
                    addr += 2 * (u32)__builtin_popcountll(bq & ((1ull << (q0 * 16)) - 1));
                    bq >>= q0 * 16;
#pragma unroll 1
                    for (u32 qq = 0; qq < nq; ++qq) {
                        const u32 b16 = (u32)bq;
                        bq >>= 16;
                        u32 e[16];
#pragma unroll
                        for (int q = 0; q < 16; ++q) {
                            const u32 t = (b16 >> q) & 1u;
                            e[q] = *(const LAS unsigned short*)((LAS char*)smem + __umul24(addr, t));
                            asm("v_lshl_add_u32 %0, %1, 1, %0" : "+v"(addr) : "v"(t));
                        }
#pragma unroll
                        for (int h = 0; h < 2; ++h)
                            st_out(u32x4{e[h * 8] | (e[h * 8 + 1] << 16), e[h * 8 + 2] | (e[h * 8 + 3] << 16),
                                         e[h * 8 + 4] | (e[h * 8 + 5] << 16), e[h * 8 + 6] | (e[h * 8 + 7] << 16)} & 0x7fff7fffu,
                                   ro, olane, ob + qq * 16 * (NN * 2) + h * 64);
                    }
                }
                asm volatile("ds_sub_u32 %0, %1" :: "v"(dna), "v"(1u) : "memory");
                STAMP(9);
            }
        }
        return;
    }

    {
        if (wave == 0) ((volatile LAS u32*)(smem + 32))[lane < 12 ? lane : 0] = 0u;
        if (lb < 2 * NPWG && wave < NPUSHW) {
            const u32 b = lb - NPWG, q = (rank + 1 + ((b >> 3) + (b & 7)) % 7) & 7;
            const rsrc_t rq = mk_rsrc(pick(P.ws, q), WS_USED);
            u32 su, du; unit_desc(rank, q, wave * EWG + (b & 7), su, du);
            fetch_finish(su0, lane, wl, v);
            STAMP(11);
#pragma unroll
            for (int i = 0; i < 8; ++i) {
                asm volatile("s_waitcnt vmcnt(%0)" :: "n"(PWIN - 1) : "memory");
                st128(v[i], rq, lane * 16, OFF_XA + du * 8192 + i * 1024, AUX_SYS);
            }
            WAIT_VM0();
            st_flag(rq, l0, OFF_FX + du * 4, rfl(epv) + 1);
            STAMP(12);
        }
        const u32 u = (lb - NPWG - SFIRST) + (GRID - NPWG - SFIRST) * wave;
        if (lb >= NPWG + SFIRST && u < 1024 - 2 * NPUSH) {
            u32 n, other, hf, su, du;
            stage_desc(u, n, other, hf);
            unit_desc(rank, (g ^ other) * 4 + hf * 3, n, su, du);
            const u32 isw = su >> 9;
            const bool mine = !other && hf == (isw ? bj : bi);
            const u32 nd = (g ^ other) * 4 + (isw ? hf + 2 * (lane & 1) : 2 * hf + (lane & 1));
            char* const np = (char*)P.ws[nd];
            if (mine || n >= NPUSH) {
                if (wave) {
                    for (u32 i = 0; i < wave; ++i) __builtin_amdgcn_s_sleep(SDELAY);
                    fetch_issue(px, pw, su, lane, v);
                }
                fetch_finish(su, lane, wl, v);
                if (n >= NPUSH) {
#pragma unroll
                    for (int i = 0; i < 8; ++i) st128(v[i], rme, lane * 16, OFF_STG + su * 8192 + i * 1024, AUX_SYS);
                    if (mine) {
#pragma unroll
                        for (int i = 0; i < 8; ++i) st128(v[i], rme, lane * 16, OFF_XA + du * 8192 + i * 1024, AUX_SYS);
                        WAIT_VM(8);
                    } else WAIT_VM(0);
                    if (lane < 4)
                        __hip_atomic_store((u64*)(np + OFF_NT + (rank * 640 + n) * 16 + (lane >> 1) * 8), (u64)(rfl(epv) + 1) << 32, __ATOMIC_RELAXED, __HIP_MEMORY_SCOPE_SYSTEM);
                    WAIT_VM(1);
                } else {
#pragma unroll
                    for (int i = 0; i < 8; ++i) st128(v[i], rme, lane * 16, OFF_XA + du * 8192 + i * 1024, AUX_SYS);
                    WAIT_VM(0);
                }
                st_flag(rme, mine ? l0 : 0x80000000u, OFF_FX + du * 4, rfl(epv) + 1);
            }
        }
        STAMP(10);
    }

    const u32 call = rfl(epv) + 1;
    const u32 wm = (wave >> 1) & 1, wn = wave & 1;

    st_flag(rme, l0, epo, call);
    if (lb < NPWG + NCWG) {
        const u32 c = lb - NPWG;
        const u32 tl2 = P.o.tiles[c];
        volatile LAS u32* sel = (volatile LAS u32*)(smem + 16);

        for (u32 pass = 0; pass < ((tl2 >> 8) == (tl2 & 255) ? 1u : 2u); ++pass) {
            const u32 tb = tl2 >> (pass * 8);
            const u32 keep = (tb >> 7) & 1, jt = (tb >> 4) & 7, ct = tb & 15;
            const u32 j = perm8(jt, bi);
            const u32 lc = perm16(ct, bj);
            const u32 lr = row_lr(j, keep ? g : 1 - g);

            if (wave >= 4) {
                const u32 lw = wave - 4;
                const u32 xbase = OFF_XA + lr * 32 * BLK + lw * 4096;
                const u32 wbase = OFF_WB + lc * 32 * BLK + lw * 4096;
                const u32 fo = (lane < 32) ? OFF_FX + (lr * 32 + lane) * 8 : OFF_FW + (lc * 32 + lane - 32) * 8;
                u32 pend = 0, taken = 0;
                for (int s = -4; s < 32; ++s) {
                    const bool refill = pend == 0 && s < 28;
                    if (refill && lw == 0) {
                        u64 t0 = 0;
                        for (u32 it = 0;; ++it) {
                            const u64 fv = ld_flag64(myws, fo);
                            const u64 m = __ballot((u32)fv >= call && (u32)(fv >> 32) >= call);
                            pend = (u32)m & (u32)(m >> 32) & ~taken;
                            if (pend) break;
                            if (it == 0) t0 = __builtin_amdgcn_s_memrealtime();
                            if ((it & 63) == 63 && give_up(myws, call, t0)) { pend = ~taken; break; }
                            __builtin_amdgcn_s_sleep(PSLEEP);
                        }
                        taken |= pend;
                        sel[s & 1] = pend;
                    }
                    if (s < 29) WAIT_VM(16); else if (s == 29) WAIT_VM(8); else WAIT_VM(0);
                    WAIT_LGKM0();
                    __builtin_amdgcn_s_barrier();
                    CBAR();
                    if (s < 28) {
                        if (refill) pend = rfl(sel[s & 1]);
                        const u32 so = ((u32)__builtin_ctz(pend | 0x80000000u)) * BLK;
                        pend &= pend - 1;
                        LAS char* d = (LAS char*)smem + LDS_CTL + (s & 3) * 32768 + lw * 4096;
#pragma unroll
                        for (int i = 0; i < 4; ++i)
                            __builtin_amdgcn_raw_ptr_buffer_load_lds(rme, d + i * 1024, 16, lane * 16, xbase + so + i * 1024, 0, AUX_SC1);
#pragma unroll
                        for (int i = 0; i < 4; ++i)
                            __builtin_amdgcn_raw_ptr_buffer_load_lds(rme, d + 16384 + i * 1024, 16, lane * 16, wbase + so + i * 1024, 0, AUX_SC1);
                    }
                    CBAR();
                }
                continue;
            }
            f32x4 acc[4][4];
#pragma unroll
            for (int a = 0; a < 4; ++a)
#pragma unroll
                for (int b2 = 0; b2 < 4; ++b2) acc[a][b2] = f32x4{0.f, 0.f, 0.f, 0.f};
            {
                const char* xa = smem + LDS_CTL + wm * 8192 + lane * 16;
                const char* wb = smem + LDS_CTL + 16384 + wn * 8192 + lane * 16;
                bf16x8 xA[8], wA[8], xB[8], wB[8];
                auto rd = [&](bf16x8 (&xf)[8], bf16x8 (&wf)[8], u32 slot) {
                    const u32 so = (slot & 3) * 32768;
#pragma unroll
                    for (int i = 0; i < 8; ++i) {
                        wf[i] = *(const bf16x8*)(wb + so + i * 1024);
                        xf[i] = *(const bf16x8*)(xa + so + i * 1024);
                    }
                };
                auto mm = [&](bf16x8 (&xf)[8], bf16x8 (&wf)[8]) {
#pragma unroll
                    for (int h = 0; h < 2; ++h)
#pragma unroll
                        for (int a = 0; a < 4; ++a)
#pragma unroll
                            for (int b2 = 0; b2 < 4; ++b2)
                                acc[a][b2] = __builtin_amdgcn_mfma_f32_16x16x32_bf16(wf[h * 4 + b2], xf[h * 4 + a], acc[a][b2], 0, 0, 0);
                };
                auto sched = [&]() {
#pragma unroll
                    for (int i = 0; i < 16; ++i) {
                        __builtin_amdgcn_sched_group_barrier(0x100, 1, 0);
                        __builtin_amdgcn_sched_group_barrier(0x008, 2, 0);
                    }
                };
#define BAR() do { __builtin_amdgcn_s_waitcnt(0xC07F); CBAR(); __builtin_amdgcn_s_barrier(); CBAR(); __builtin_amdgcn_sched_barrier(0); } while (0)
                BAR(); BAR(); BAR();
                BAR();
                rd(xA, wA, 0);
                for (u32 s = 0; s < 32; s += 2) {
                    BAR();
                    rd(xB, wB, s + 1);
                    mm(xA, wA);
                    sched();
                    __builtin_amdgcn_sched_barrier(0);
                    BAR();
                    rd(xA, wA, s + 2);
                    mm(xB, wB);
                    sched();
                    __builtin_amdgcn_sched_barrier(0);
                }
                WAIT_LGKM0();
            }
            STAMP(1 + 3 * pass);
            u32 lane_ = threadIdx.x & 63;
            asm volatile("" : "+v"(lane_));
            const u32 lane = lane_;

            if (!keep) {
                const u32 pu = (((j >> 2) * 2 + (lc >> 3)) * 32 + part_idx(jt >> 1, ct >> 1)) * 4 + wave;
                const u32 dst = OFF_SPART + pu * 8192;
#pragma unroll
                for (int a = 0; a < 4; ++a)
#pragma unroll
                    for (int u = 0; u < 2; ++u) {
                        u32x4 o;
                        o[0] = pack2(acc[a][2 * u][0], acc[a][2 * u][1]);
                        o[1] = pack2(acc[a][2 * u][2], acc[a][2 * u][3]);
                        o[2] = pack2(acc[a][2 * u + 1][0], acc[a][2 * u + 1][1]);
                        o[3] = pack2(acc[a][2 * u + 1][2], acc[a][2 * u + 1][3]);
                        st128(o, rme, lane * 16, dst + (a * 2 + u) * 1024, AUX_SYS);
                    }
                WAIT_VM(0);
                sel[4 + pass * 4 + wave] = 0x80000000u;
                if (__ballot(sel[4 + pass * 4 + (lane & 3)] >> 31) == ~0ull)
                    st_flag64(mk_rsrc(pick(P.ws, (1 - g) * 4 + (pu >> 7)), WS_USED), lane < 2 ? lane * 8 : 0x80000000u, OFF_NP + ((rank & 3) * 32 + ((pu >> 2) & 31)) * 16, call, 0);
                STAMP(3 + 3 * pass);
            } else {
                const u32 pu = (j * 16 + lc) * 4 + wave;
                CBAR();
                char* const peerp = (char*)P.ws[(rank + 1 + (lane >> 2)) & 7] + OFF_NT + rank * 10240 + ((((rank + 1 + (lane >> 2)) >> 2) & 1) == g ? 5120 : 3072);
                wait_flag64<KSLEEP>(myws, OFF_FP + pu * 8, call);
                STAMP(2 + 3 * pass);
                u32x4 o[8];
                u32 bits[2] = {0u, 0u};
                {
                    u32x4 pv[8];
#pragma unroll
                    for (int i = 0; i < 8; ++i) pv[i] = __builtin_amdgcn_raw_buffer_load_b128(rme, lane * 16, OFF_PART + pu * 8192 + i * 1024, AUX_SC1);
                    const bool bad = rfl(ld_tmo(myws)) == call;
#pragma unroll
                    for (int gi = 0; gi < 8; ++gi) {
                        float y[8];
#pragma unroll
                        for (int k = 0; k < 8; ++k) {
                            const u32 pq = pv[gi][k >> 1];
                            float t = acc[gi >> 1][(gi & 1) * 2 + (k >> 2)][k & 3] + ((k & 1) ? bf_hi(pq) : bf_lo(pq));
                            if (bad) t = __builtin_nanf("");
                            const bool nz = !(t <= 0.f);
                            y[k] = nz ? t : 0.f;
                            bits[gi >> 2] |= nz ? (1u << ((gi & 3) * 8 + k)) : 0u;
                        }
                        o[gi] = u32x4{pack2(y[0], y[1]), pack2(y[2], y[3]), pack2(y[4], y[5]), pack2(y[6], y[7])} & 0x7fff7fffu;
                    }
                }
                u32 tot;
                u32 addr = (u32)(LDS_CTL + wave * 16384) + 2 * wave_excl(__builtin_popcount(bits[0]) + __builtin_popcount(bits[1]), tot);
                const u32 H = hidden_words(tot), eo = ((2 * tot + 3) & ~3u) - 4 * H;
                u32 len = (eo + 512 + 15) & ~15u, off;
                asm volatile("ds_add_rtn_u32 %0, %1, %2" : "=v"(off) : "v"(lane == 0 ? 64 + pass * 8 + (wave >> 1) * 4 : LDS_CTL + wave * 16384 + 10496 + lane * 4), "v"(len) : "memory");
                u32* wt = (u32*)(wl + 9216);
                wt[lane] = bits[0];
                wt[64 + lane] = bits[1];
                *(u32*)(wl + (lane >= H ? eo + lane * 4 : 10240)) = bits[0];
                *(u32*)(wl + (64 + lane >= H ? eo + 256 + lane * 4 : 10240)) = bits[1];
#pragma unroll
                for (int gi = 0; gi < 8; ++gi)
#pragma unroll
                    for (int k = 0; k < 8; ++k) {
                        const u32 t = (bits[gi >> 2] >> ((gi & 3) * 8 + k)) & 1u;
                        const u32 d = o[gi][k >> 1];
                        *(LAS unsigned short*)((LAS char*)smem + __umul24(addr, t)) = (unsigned short)((k & 1) ? d >> 16 : d);
                        asm("v_lshl_add_u32 %0, %1, 1, %0" : "+v"(addr) : "v"(t));
                    }
                asm volatile("s_waitcnt lgkmcnt(0)" : "+v"(off) :: "memory");
                off = rfl(off);
                const u32 tl = jt * 16 + ct;
                const u32 sidx = ((P.o.ord[tl >> 2] >> ((tl & 3) * 8)) & 127) * 4 + wave;
                const u32 so = OFF_CMP + (sidx & ~1u) * CSTR + off;
#pragma unroll
                for (int i = 0; i < 9; ++i)
                    if (i * 1024u < len) {
                        u32x4 v = *(const u32x4*)(wl + i * 1024 + lane * 16);
                        const u32 G = i * 64 + lane, pl = (G >= H) + (G >= 2 * H) + (G >= 3 * H);
                        const u32 hw = G < 4 * H ? wt[(G - pl * H) & 127] << (6 - pl * 2) : 0u;
                        v[0] |= __builtin_amdgcn_perm(0u, hw << 1, 0x010c000cu) & 0x80008000u;
                        v[1] |= __builtin_amdgcn_perm(0u, hw << 1, 0x030c020cu) & 0x80008000u;
                        v[2] |= __builtin_amdgcn_perm(0u, hw, 0x010c000cu) & 0x80008000u;
                        v[3] |= __builtin_amdgcn_perm(0u, hw, 0x030c020cu) & 0x80008000u;
                        st128(v, rme, i * 1024 + lane * 16 < len ? lane * 16 : 0x80000000u, so + i * 1024, AUX_SYS);
                    }
                WAIT_VM(0);
                sel[4 + pass * 4 + wave] = len | (off != 0) | tot << 16 | 0x80000000u;
                const u32 l4 = sel[4 + pass * 4 + (lane & 3)];
                if (__ballot(l4 >> 31) == ~0ull) {
                    const u64 note = (u64)call << 32 | (l4 & 0x7fffffffu);
                    if (lane < 28)
                        __hip_atomic_store((u64*)(peerp + ((sidx & ~3u) + (lane & 3)) * 8), note, __ATOMIC_RELAXED, __HIP_MEMORY_SCOPE_SYSTEM);
                }
                STAMP(3 + 3 * pass);
                const u32 obase = (glob_blk(g, bi, lr) * 128 + wm * 64) * (NN * 2) + (glob_blk(g, bj, lc) * 128 + wn * 64) * 2;
#pragma unroll
                for (int gi = 0; gi < 8; ++gi)
                    st_out(o[gi], ro, olane, obase + (gi >> 1) * 16 * (NN * 2) + (gi & 1) * 64);
                WAIT_LGKM0();
            }
        }
    }
}

extern "C" void kernel_launch(int rank, int nranks, ncclComm_t comm,
                              hipStream_t stream,
                              void* const* d_in, const int* in_sizes, int n_in,
                              void* d_out, int out_size,
                              const int* in_dtypes, int out_dtype,
                              void* const* d_out_all,
                              void* const* d_ws_all, size_t ws_bytes) {
    if (nranks != NR || n_in < 2 || ws_bytes < WS_USED || (unsigned)rank >= (unsigned)NR ||
        in_sizes[0] != MM * KS || in_sizes[1] != KS * NN || out_size != MM * NN) {
        fprintf(stderr, "unexpected problem shape\n");
        return;
    }
    Params P;
    P.x = d_in[0];
    P.w = d_in[1];
    for (int i = 0; i < NR; ++i) P.ws[i] = d_ws_all[i];
    P.me = d_ws_all[rank];
    P.out = d_out;
    P.rank = rank;
    P.o = ORDERS;
    for (u32 b = 0; b < (u32)GRID; ++b) {
        u32 f[2] = {0xffffu, 0xffffu}, du;
        if (b < (u32)NPWG) {
            for (u32 w = 0; w < (u32)NPUSHW; ++w) unit_desc(rank, (rank + 1 + ((b >> 3) + (b & 7)) % 7) & 7, w * EWG + (b & 7), f[w], du);
        } else if (b >= (u32)SFIRST && b < (u32)(GRID - NPWG)) {
            u32 n, other, hf;
            stage_desc(b - SFIRST, n, other, hf);
            unit_desc(rank, (((u32)rank >> 2) ^ other) * 4 + hf * 3, n, f[0], du);
        }
        P.first[b] = f[0] | f[1] << 16;
    }
    HIP_CHECK(hipFuncSetAttribute((const void*)fused_gemm_ar, hipFuncAttributeMaxDynamicSharedMemorySize, LDS_BYTES));
    hipLaunchKernelGGL(fused_gemm_ar, dim3(GRID), dim3(512), LDS_BYTES, stream, P);
    HIP_CHECK(hipGetLastError());
}
```
